```python
import numpy as np

kernel_name = "gpu_gemm_rs_m4096_k4096_n16384_e5m2e5m2_dequant"


ABSMAX_RTOL = 0.05

COMPUTE_DTYPE = "float32"
EPILOGUE = "dequant"
IN_QUANT = None
OUT_QUANT = None
IN_FP8 = "e5m2e5m2"
IN_DTYPES = {"x": "e5m2", "w_mat": "e5m2",
             "scale_x": "float32", "scale_w": "float32"}
WORLD_SIZE = 8

def _snap_fp8(x, mant_bits, max_abs, min_normal):
    a = np.abs(x.astype(np.float64))
    _m, _e = np.frexp(a)
    q = np.where(a >= min_normal, np.ldexp(1.0, _e - 1 - mant_bits),
                 min_normal * 2.0 ** -mant_bits)
    snapped = np.minimum(np.round(a / q) * q, max_abs)
    return (np.sign(x) * snapped).astype(np.float32)

def setup_inputs():

    x = _snap_fp8(np.random.randn(4096, 4096).astype(np.float32), 2, 57344.0, 2.0 ** -14)
    w_mat = _snap_fp8(np.random.randn(4096, 16384).astype(np.float32), 2, 57344.0, 2.0 ** -14)
    scale_x = (0.02 * (1.0 + np.random.rand(1))).astype(np.float32)
    scale_w = (0.02 * (1.0 + np.random.rand(1))).astype(np.float32)
    return {"x": x, "w_mat": w_mat, "scale_x": scale_x, "scale_w": scale_w}

def reference(x, w_mat, scale_x, scale_w):
    acc = (x @ w_mat).astype(np.float32)
    y = (acc * (scale_x * scale_w)).astype(np.float32)
    return y
```

```cpp
#include <hip/hip_runtime.h>
#include <rccl/rccl.h>
#include <cstdio>
#include <cstdint>

#define HIP_CHECK(expr) do { hipError_t _e = (expr); if (_e != hipSuccess) { \
    fprintf(stderr, "HIP error %s at %s:%d\n", hipGetErrorString(_e), __FILE__, __LINE__); \
    return; } } while (0)

typedef unsigned u32x4 __attribute__((ext_vector_type(4)));
typedef int i32x4 __attribute__((ext_vector_type(4)));
typedef int i32x8 __attribute__((ext_vector_type(8)));
typedef float f32x4 __attribute__((ext_vector_type(4)));
typedef float f32x16 __attribute__((ext_vector_type(16)));

namespace {

constexpr int NR = 8;
constexpr int GN = 16384;
constexpr int KS = 512;
constexpr int NB = 2048;
constexpr int NWG = 256;
constexpr int NROUND = 16;
constexpr int KSTEPS = 64;
constexpr int STAGE = 24576;
constexpr int NSTAGE = 6;
constexpr int LDS_CTRL = STAGE * NSTAGE;
constexpr int LDS_BYTES = LDS_CTRL + 64;

constexpr size_t WS_CALL = 0;
constexpr size_t WS_PROG = 32768;
constexpr size_t WS_WPROG = WS_PROG + 8192;
constexpr size_t WS_DONE = WS_WPROG + 1024;
constexpr size_t WS_TMO = WS_DONE + 1024;
constexpr size_t WS_DBG = 65536;
constexpr size_t WS_DATA = 1u << 20;
constexpr size_t XS_BYTES = 16u << 20;
constexpr size_t WT_BYTES = 8u << 20;
constexpr size_t PAR_BYTES = XS_BYTES + WT_BYTES;
constexpr size_t WS_NEED = WS_DATA + 2 * PAR_BYTES;

constexpr unsigned long long DEADLINE_TICKS = 50000000ull;

#define HOLD_ROUND 15
#define PUSH_RT0 20

struct Params {
  const uint8_t* x;
  const uint8_t* w;
  const float* sx;
  const float* sw;
  uint8_t* ws[NR];
  float* out[NR];
  int rank;
  int pad;
};

#define STAMP(slot) do {} while (0)

__device__ __forceinline__ unsigned ld_sys(const void* ptr) {
  return __hip_atomic_load((const unsigned*)ptr, __ATOMIC_RELAXED, __HIP_MEMORY_SCOPE_SYSTEM);
}
__device__ __forceinline__ unsigned ld_agent(const void* ptr) {
  return __hip_atomic_load((const unsigned*)ptr, __ATOMIC_RELAXED, __HIP_MEMORY_SCOPE_AGENT);
}
__device__ __forceinline__ void st_sys(void* ptr, unsigned v) {
  __hip_atomic_store((unsigned*)ptr, v, __ATOMIC_RELAXED, __HIP_MEMORY_SCOPE_SYSTEM);
}
__device__ __forceinline__ void st_agent(void* ptr, unsigned v) {
  __hip_atomic_store((unsigned*)ptr, v, __ATOMIC_RELAXED, __HIP_MEMORY_SCOPE_AGENT);
}

__device__ __forceinline__ bool wave_wait_ge(const void* flag, unsigned want, unsigned* tmo, unsigned call) {
  const unsigned long long t0 = __builtin_amdgcn_s_memrealtime();
  for (;;) {
    const unsigned v = ld_sys(flag);
    if (__all(v >= want)) return true;
    if (ld_agent(tmo) == call || __builtin_amdgcn_s_memrealtime() - t0 > DEADLINE_TICKS) {
      st_agent(tmo, call);
      return false;
    }
    __builtin_amdgcn_s_sleep(8);
  }
}

__device__ __forceinline__ void st16_sys(void* base, unsigned off, u32x4 v) {
  asm volatile("global_store_dwordx4 %0, %1, %2 sc0 sc1\n\ts_nop 1" :: "v"(off), "v"(v), "s"(base) : "memory");
}

__device__ __forceinline__ void st1_agent(void* base, unsigned v) {
  asm volatile("s_mov_b64 exec, 1\n\tglobal_store_dword %0, %1, %2 sc1\n\ts_mov_b64 exec, -1"
               :: "v"(0u), "v"(v), "s"(base) : "memory");
}

template <int N> __device__ __forceinline__ void wait_vm() {
  asm volatile("s_waitcnt vmcnt(%0)" :: "n"(N) : "memory");
}

constexpr bool is_w(int r) { return r >= 2 && r < 10; }
constexpr int n_ld(int r) { return r >= NROUND ? 0 : (is_w(r) ? 16 : 1); }
constexpr int n_st(int r) { return r < 0 || r >= NROUND ? 0 : (is_w(r) ? 4 : 8); }
constexpr int x_idx(int r) { return r < 2 ? r : r - 8; }
constexpr int PF = 3;

struct PushRegs {
  unsigned w[PF + 1][16];
  u32x4 x[PF + 1];
};

template <int R> __device__ __forceinline__ void push_load(const Params& p, PushRegs& g, unsigned xoff, unsigned woff) {
  if constexpr (R < NROUND) {
    constexpr int s = R % (PF + 1);
    if constexpr (is_w(R)) {
      const unsigned o = woff + (R - 2) * 256;
      const uint8_t* wb = p.w;
#define WLD(i) asm volatile("global_load_dword %0, %1, %2" : "=v"(g.w[s][i]) : "v"(o), "s"(wb + (size_t)(i) * GN) : "memory")
      WLD(0); WLD(1); WLD(2); WLD(3); WLD(4); WLD(5); WLD(6); WLD(7);
      WLD(8); WLD(9); WLD(10); WLD(11); WLD(12); WLD(13); WLD(14); WLD(15);
#undef WLD
    } else {
      constexpr int i = x_idx(R);
      const unsigned o = xoff + (unsigned)((4 * (i & 1)) * 512 + (i >> 1) * 128) * KS;
      asm volatile("global_load_dwordx4 %0, %1, %2" : "=v"(g.x[s]) : "v"(o), "s"(p.x) : "memory");
    }
  }
}

template <int R> __device__ __forceinline__ void push_pin(PushRegs& g) {
  constexpr int s = R % (PF + 1);
  if constexpr (is_w(R)) {
    asm volatile("" : "+v"(g.w[s][0]), "+v"(g.w[s][1]), "+v"(g.w[s][2]), "+v"(g.w[s][3]), "+v"(g.w[s][4]), "+v"(g.w[s][5]),
                      "+v"(g.w[s][6]), "+v"(g.w[s][7]), "+v"(g.w[s][8]), "+v"(g.w[s][9]), "+v"(g.w[s][10]), "+v"(g.w[s][11]),
                      "+v"(g.w[s][12]), "+v"(g.w[s][13]), "+v"(g.w[s][14]), "+v"(g.w[s][15]) :: "memory");
  } else {
    asm volatile("" : "+v"(g.x[s]) :: "memory");
  }
}

template <int R> __device__ __forceinline__ void push_store(const Params& p, PushRegs& g, uint8_t* wdst, unsigned xdoff, unsigned wdoff) {
  constexpr int s = R % (PF + 1);
  if constexpr (is_w(R)) {
    u32x4 o[4];
#pragma unroll
    for (int q = 0; q < 4; ++q) {
      const unsigned d0 = g.w[s][4 * q], d1 = g.w[s][4 * q + 1], d2 = g.w[s][4 * q + 2], d3 = g.w[s][4 * q + 3];
      const unsigned t0 = __builtin_amdgcn_perm(d1, d0, 0x05010400u), t1 = __builtin_amdgcn_perm(d1, d0, 0x07030602u);
      const unsigned t2 = __builtin_amdgcn_perm(d3, d2, 0x05010400u), t3 = __builtin_amdgcn_perm(d3, d2, 0x07030602u);
      o[0][q] = __builtin_amdgcn_perm(t2, t0, 0x05040100u);
      o[1][q] = __builtin_amdgcn_perm(t2, t0, 0x07060302u);
      o[2][q] = __builtin_amdgcn_perm(t3, t1, 0x05040100u);
      o[3][q] = __builtin_amdgcn_perm(t3, t1, 0x07060302u);
    }
    const unsigned od = wdoff + (R - 2) * (1u << 20);
#pragma unroll
    for (int j = 0; j < 4; ++j) st16_sys(wdst, od + j * 2048, o[j]);
  } else {
    constexpr int i = x_idx(R);
    const unsigned od = xdoff + (unsigned)(4 * i) * (512u << 10);
#pragma unroll
    for (int d = 0; d < NR; ++d) st16_sys(p.ws[d], od, g.x[s]);
  }
}

template <int R> __device__ __forceinline__ void push_round(const Params& p, PushRegs& g, uint8_t* wdst, unsigned xoff, unsigned woff,
                                                            unsigned xdoff, unsigned wdoff, unsigned* wprog, unsigned base, int lane) {
  if constexpr (R >= 2) {
    wait_vm<n_ld(R + 1) + n_st(R - 1) + n_ld(R + 2)>();
    st1_agent(wprog, base + R - 1);
  } else if constexpr (R == 1) {
    wait_vm<n_ld(2) + n_st(0) + n_ld(3)>();
  } else {
    wait_vm<n_ld(1) + n_ld(2)>();
  }
  push_pin<R>(g);
  push_store<R>(p, g, wdst, xdoff, wdoff);
  push_load<R + PF>(p, g, xoff, woff);
  STAMP(8 + R);
}

__device__ __forceinline__ void glds2(const uint8_t* gbase, unsigned voff, unsigned lds) {
  unsigned keep;
  asm volatile("s_mov_b32 %0, m0\n\ts_mov_b32 m0, %3\n\ts_nop 0\n\t"
               "global_load_lds_dwordx4 %1, %2\n\t"
               "global_load_lds_dwordx4 %1, %2 offset:1024\n\t"
               "s_mov_b32 m0, %0"
               : "=&s"(keep) : "v"(voff), "s"(gbase), "s"(lds));
}

struct Frags { i32x8 a[2]; i32x8 b[4]; };

__device__ __forceinline__ i32x8 ld_frag(const uint8_t* q) {
  const i32x4 lo = *(const i32x4*)q, hi = *(const i32x4*)(q + 1024);
  return __builtin_shufflevector(lo, hi, 0, 1, 2, 3, 4, 5, 6, 7);
}

#define MFMA(mi, j) acc[mi][j] = __builtin_amdgcn_mfma_scale_f32_32x32x64_f8f6f4(cur.a[mi], cur.b[j], acc[mi][j], 1, 1, 0, 0, 0, 0)
#define SB() __builtin_amdgcn_sched_barrier(0)

template <int VM, bool DMA, bool RD>
__device__ __forceinline__ void kstep(f32x16 (&acc)[2][4], const Frags& cur, Frags& nxt, const uint8_t* lds, unsigned ldsb,
                                      int& slot, const uint8_t*& ga, const uint8_t*& gb, unsigned va, unsigned vb,
                                      unsigned ra, unsigned rb, unsigned wa, unsigned wb) {
  asm volatile("s_waitcnt vmcnt(%0) lgkmcnt(0)\n\ts_barrier" :: "n"(VM) : "memory");
  const unsigned wslot = ldsb + slot * STAGE;
  slot = slot == NSTAGE - 1 ? 0 : slot + 1;
  const uint8_t* qa = lds + slot * STAGE + ra;
  const uint8_t* qb = lds + slot * STAGE + rb;
  SB();
  MFMA(0, 0); SB();
  if constexpr (DMA) glds2(ga, va, wslot + wa);
  SB();
  MFMA(0, 1); SB();
  if constexpr (RD) { nxt.a[0] = ld_frag(qa); nxt.b[0] = ld_frag(qb); }
  SB();
  MFMA(0, 2); SB();
  if constexpr (DMA) glds2(gb, vb, wslot + wb);
  SB();
  MFMA(0, 3); SB();
  if constexpr (RD) { nxt.b[1] = ld_frag(qb + 2048); nxt.b[2] = ld_frag(qb + 4096); }
  SB();
  MFMA(1, 0); SB();
  if constexpr (DMA) glds2(gb + 2048, vb, wslot + wb + 2048);
  SB();
  MFMA(1, 1); SB();
  if constexpr (RD) { nxt.b[3] = ld_frag(qb + 6144); nxt.a[1] = ld_frag(qa + 2048); }
  SB();
  MFMA(1, 2); SB();
  MFMA(1, 3); SB();
  if constexpr (DMA) { ga += 8192; gb += 16384; }
}

}

__global__ void __launch_bounds__(256, 1) gemm_rs_fused(const Params p) {
  __shared__ __attribute__((aligned(16))) uint8_t lds[LDS_BYTES];
  const int b = blockIdx.x;
  const int lane = threadIdx.x & 63;
  const int wave = __builtin_amdgcn_readfirstlane(threadIdx.x >> 6);
  const int rank = p.rank;
  uint8_t* const myws = p.ws[rank];
  STAMP(0);

  const int cb = b & 7;
  int rt = (b >> 3) + PUSH_RT0;
  rt = rt >= 32 ? rt - 32 : rt;
  const bool pusher = b < 64;
  unsigned* const callp = (unsigned*)(myws + WS_CALL + (size_t)b * 128);
  unsigned* const tmo = (unsigned*)(myws + WS_TMO);

  unsigned call;
  if (pusher) {
    const int pw = b * 4 + wave;
    const int bi = pw & 63, xrb = bi >> 4, xks = (bi >> 1) & 7, xh = bi & 1;
    const unsigned xoff = (unsigned)(((pw >> 6) * 512 + xrb * 32 + (lane & 31)) * KS + xks * 64 + (lane >> 5) * 32 + xh * 16);
    const int wd = pw & 7, wt = pw >> 3, wks = wt >> 2, wnb = (wt >> 1) & 1, wh = wt & 1;
    const unsigned woff = (unsigned)((wks * 64 + (lane >> 5) * 32 + wh * 16) * GN + wd * NB + wnb * 128 + (lane & 31) * 4);
    PushRegs g;
    unsigned cv;
    asm volatile("global_load_dword %0, %1, %2 sc1" : "=v"(cv) : "v"(0u), "s"(callp) : "memory");
    push_load<0>(p, g, xoff, woff);
    push_load<1>(p, g, xoff, woff);
    push_load<2>(p, g, xoff, woff);
    asm volatile("s_waitcnt vmcnt(%1)" : "+v"(cv) : "n"(n_ld(0) + n_ld(1) + n_ld(2)) : "memory");
    call = __builtin_amdgcn_readfirstlane(cv) + 1;
    const unsigned par = (unsigned)(WS_DATA + (call & 1) * PAR_BYTES);
    const unsigned xdoff = par + (unsigned)((pw >> 6) * (512 << 10) + (rank * 8 + xks) * 8192 + xrb * 2048 + xh * 1024 + lane * 16);
    const unsigned wdoff = par + (unsigned)XS_BYTES + (unsigned)((rank * 8 + wks) * 16384 + wnb * 8192 + wh * 1024 + lane * 16);
    uint8_t* const wdst = p.ws[wd];
    unsigned* const wprog = (unsigned*)(myws + WS_WPROG) + pw;
    const unsigned base = (call - 1) * NROUND;
#define PR(r) push_round<r>(p, g, wdst, xoff, woff, xdoff, wdoff, wprog, base, lane)
    PR(0); PR(1); PR(2); PR(3); PR(4); PR(5); PR(6); PR(7);
    PR(8); PR(9); PR(10); PR(11); PR(12); PR(13); PR(14); PR(15);
#undef PR
    wait_vm<n_st(NROUND - 1)>();
    st1_agent(wprog, base + NROUND - 1);
    wait_vm<0>();
    st1_agent(wprog, base + NROUND);
    STAMP(24);
  } else {
    call = __builtin_amdgcn_readfirstlane(ld_agent(callp)) + 1;
  }
  const unsigned base = (call - 1) * NROUND;

  if (rt == 31 && cb == 0 && wave == 1) {
    uint8_t* const fl = p.ws[lane >> 3] + WS_PROG + (size_t)(rank * 8 + (lane & 7)) * 128;
    const unsigned* wp = (const unsigned*)(myws + WS_WPROG) + lane * 4;
    const unsigned long long t0 = __builtin_amdgcn_s_memrealtime();
    unsigned pub = base;
    while (pub < base + NROUND) {
      const unsigned v0 = ld_agent(wp), v1 = ld_agent(wp + 1), v2 = ld_agent(wp + 2), v3 = ld_agent(wp + 3);
      const unsigned mn = min(min(v0, v1), min(v2, v3));
      unsigned np = pub;
      while (np < base + NROUND && __all(mn > np)) ++np;
      if (np != pub) {
        pub = np;
        st_sys(fl, pub);
        STAMP(8 + (pub - base - 1));
      } else {
        if (ld_agent(tmo) == call || __builtin_amdgcn_s_memrealtime() - t0 > DEADLINE_TICKS) {
          st_agent(tmo, call);
          break;
        }
        __builtin_amdgcn_s_sleep(2);
      }
    }
  }

  unsigned* const ctrl = (unsigned*)(lds + LDS_CTRL);
  const uint8_t* const progl = myws + WS_PROG + (size_t)((lane & 7) * 8 + cb) * 128;
  if (wave == 0) {
    const int xr = rt < 8 ? (rt >> 2) : 8 + (rt >> 2);
    const int need = max(xr, 2 + cb) + 1;
    wave_wait_ge(progl, base + need, tmo, call);
    __builtin_amdgcn_fence(__ATOMIC_ACQUIRE, "");
    asm volatile("s_waitcnt vmcnt(0)" ::: "memory");
    const unsigned t = ld_agent(tmo);
    if (lane == 0) ctrl[0] = t == call;
  }
  __syncthreads();
  const bool dead0 = __builtin_amdgcn_readfirstlane(ctrl[0]) != 0;
  STAMP(1);

  const int wm = wave >> 1, wn = wave & 1;
  f32x16 acc[2][4];
#pragma unroll
  for (int i = 0; i < 2; ++i)
#pragma unroll
    for (int j = 0; j < 4; ++j)
#pragma unroll
      for (int r = 0; r < 16; ++r) acc[i][j][r] = 0.f;

  if (!dead0) {
    const uint8_t* par = myws + WS_DATA + (size_t)(call & 1) * PAR_BYTES;
    const uint8_t* ga = par + (size_t)rt * (512 << 10);
    const uint8_t* gb = par + XS_BYTES + (size_t)cb * (1 << 20);
    const unsigned wa = wave * 2048, wb = 8192 + wave * 4096;
    const unsigned va = lane * 16 + wave * 2048, vb = lane * 16 + wave * 4096;
    const unsigned ra = lane * 16 + wm * 4096, rb = 8192 + lane * 16 + wn * 8192;
    const unsigned ldsb = (unsigned)(uintptr_t)lds;
#pragma unroll
    for (int s = 0; s < NSTAGE; ++s) {
      glds2(ga, va, ldsb + s * STAGE + wa);
      glds2(gb, vb, ldsb + s * STAGE + wb);
      glds2(gb + 2048, vb, ldsb + s * STAGE + wb + 2048);
      ga += 8192; gb += 16384;
    }
    asm volatile("s_waitcnt vmcnt(30)\n\ts_barrier" ::: "memory");
    Frags f0, f1;
    f0.a[0] = ld_frag(lds + ra); f0.a[1] = ld_frag(lds + ra + 2048);
    f0.b[0] = ld_frag(lds + rb); f0.b[1] = ld_frag(lds + rb + 2048);
    f0.b[2] = ld_frag(lds + rb + 4096); f0.b[3] = ld_frag(lds + rb + 6144);
    int slot = 0;
#pragma unroll 1
    for (int s = 0; s < KSTEPS - NSTAGE; s += 2) {
      kstep<24, true, true>(acc, f0, f1, lds, ldsb, slot, ga, gb, va, vb, ra, rb, wa, wb);
      kstep<24, true, true>(acc, f1, f0, lds, ldsb, slot, ga, gb, va, vb, ra, rb, wa, wb);
    }
    kstep<24, false, true>(acc, f0, f1, lds, ldsb, slot, ga, gb, va, vb, ra, rb, wa, wb);
    kstep<18, false, true>(acc, f1, f0, lds, ldsb, slot, ga, gb, va, vb, ra, rb, wa, wb);
    kstep<12, false, true>(acc, f0, f1, lds, ldsb, slot, ga, gb, va, vb, ra, rb, wa, wb);
    kstep<6, false, true>(acc, f1, f0, lds, ldsb, slot, ga, gb, va, vb, ra, rb, wa, wb);
    kstep<0, false, true>(acc, f0, f1, lds, ldsb, slot, ga, gb, va, vb, ra, rb, wa, wb);
    kstep<0, false, false>(acc, f1, f0, lds, ldsb, slot, ga, gb, va, vb, ra, rb, wa, wb);
  }
  STAMP(2);

  const int dst = rt & 7;
  if (wave == 0) {
    if (dst != rank) wave_wait_ge(myws + WS_PROG + (size_t)(rank * 8 + cb) * 128, base + HOLD_ROUND, tmo, call);
    const unsigned t = ld_agent(tmo);
    if (lane == 0) ctrl[1] = t == call;
  }
  __syncthreads();
  const bool dead = __builtin_amdgcn_readfirstlane(ctrl[1]) != 0;
  STAMP(3);

  {
    float sc = p.sx[0] * p.sw[0];
    if (dead) sc = __builtin_nanf("");
    float* const ob = p.out[dst];
    unsigned off = (unsigned)((((rt >> 3) * 128 + wm * 64 + 4 * (lane >> 5)) * GN + rank * NB + cb * 256 + wn * 128 + 4 * (lane & 31)) * 4);
#pragma unroll
    for (int mi = 0; mi < 2; ++mi)
#pragma unroll
      for (int r = 0; r < 16; ++r) {
        const f32x4 v = {acc[mi][0][r] * sc, acc[mi][1][r] * sc, acc[mi][2][r] * sc, acc[mi][3][r] * sc};
        st16_sys(ob, off + (unsigned)(mi * 32 + (r & 3) + 8 * (r >> 2)) * (GN * 4), __builtin_bit_cast(u32x4, v));
      }
  }
  STAMP(4);
  asm volatile("s_waitcnt vmcnt(0)" ::: "memory");
  __syncthreads();
  if (threadIdx.x == 0) {
    if (dst != rank) st_sys(p.ws[dst] + WS_DONE + (size_t)(rank * 32 + (rt >> 3) * 8 + cb) * 4, call);
    st_agent(callp, call);
  }
  STAMP(5);

  if (rt == 0 && cb == 0) {
    const int src = threadIdx.x >> 5;
    const bool ok = wave_wait_ge(myws + WS_DONE + (size_t)threadIdx.x * 4, src == rank ? 0u : call, tmo, call);
    __builtin_amdgcn_fence(__ATOMIC_ACQUIRE, "");
    asm volatile("s_waitcnt vmcnt(0)" ::: "memory");
    if (!ok && lane == 0) p.out[rank][(size_t)wave * GN + rank * NB] = __builtin_nanf("");
    STAMP(6);
  }
}

extern "C" void kernel_launch(int rank, int nranks, ncclComm_t comm, hipStream_t stream,
                              void* const* d_in, const int* in_sizes, int n_in,
                              void* d_out, int out_size, const int* in_dtypes, int out_dtype,
                              void* const* d_out_all, void* const* d_ws_all, size_t ws_bytes) {
  (void)comm; (void)in_sizes; (void)in_dtypes; (void)out_dtype; (void)d_out;
  if (nranks != NR || (unsigned)rank >= (unsigned)NR || n_in != 4 || ws_bytes < WS_NEED || out_size != 512 * GN) {
    fprintf(stderr, "kernel_launch: unexpected problem shape\n");
    return;
  }
  Params p;
  p.x = (const uint8_t*)d_in[0];
  p.w = (const uint8_t*)d_in[1];
  p.sx = (const float*)d_in[2];
  p.sw = (const float*)d_in[3];
  for (int i = 0; i < NR; ++i) {
    p.ws[i] = (uint8_t*)d_ws_all[i];
    p.out[i] = (float*)d_out_all[i];
  }
  p.rank = rank;
  p.pad = 0;
  hipLaunchKernelGGL(gemm_rs_fused, dim3(NWG), dim3(256), 0, stream, p);
  HIP_CHECK(hipGetLastError());
}
```

```cpp
#include <hip/hip_runtime.h>
#include <rccl/rccl.h>
#include <cstdio>
#include <cstdint>

#define HIP_CHECK(expr) do { hipError_t _e = (expr); if (_e != hipSuccess) { \
    fprintf(stderr, "HIP error %s at %s:%d\n", hipGetErrorString(_e), __FILE__, __LINE__); \
    return; } } while (0)

typedef unsigned u32x4 __attribute__((ext_vector_type(4)));
typedef int i32x4 __attribute__((ext_vector_type(4)));
typedef int i32x8 __attribute__((ext_vector_type(8)));
typedef float f32x4 __attribute__((ext_vector_type(4)));
typedef float f32x16 __attribute__((ext_vector_type(16)));

namespace {

constexpr int NR = 8;
constexpr int GN = 16384;
constexpr int KS = 512;
constexpr int NB = 2048;
constexpr int NWG = 256;
constexpr int NROUND = 24;
constexpr int KSTEPS = 64;
constexpr int STAGE = 24576;
constexpr int NSTAGE = 6;
constexpr int LDS_CTRL = STAGE * NSTAGE;
constexpr int LDS_BYTES = LDS_CTRL + 64;

constexpr size_t WS_CALL = 0;
constexpr size_t WS_PROG = 32768;
constexpr size_t WS_WPROG = WS_PROG + 8192;
constexpr size_t WS_TMO = WS_WPROG + 1024;
constexpr size_t WS_DONE = WS_TMO + 1024;
constexpr size_t WS_REL = WS_DONE + 1024;
constexpr size_t WS_DBG = 65536;
constexpr size_t WS_DATA = 1u << 20;
constexpr size_t XS_BYTES = 16u << 20;
constexpr size_t WT_BYTES = 8u << 20;
constexpr size_t PAR_BYTES = XS_BYTES + WT_BYTES;
constexpr size_t WS_NEED = WS_DATA + 2 * PAR_BYTES;

constexpr unsigned long long DEADLINE_TICKS = 50000000ull;

#define OUT_D 3
#define PUSH_REL 6
#define OUT_REL 6
#define REL_K 24

struct Params {
  const uint8_t* x;
  const uint8_t* w;
  const float* sx;
  const float* sw;
  uint8_t* ws[NR];
  float* out[NR];
  int rank;
  int pad;
};

#define STAMP(slot) do {} while (0)

__device__ __forceinline__ unsigned ld_sys(const void* ptr) {
  return __hip_atomic_load((const unsigned*)ptr, __ATOMIC_RELAXED, __HIP_MEMORY_SCOPE_SYSTEM);
}
__device__ __forceinline__ unsigned ld_agent(const void* ptr) {
  return __hip_atomic_load((const unsigned*)ptr, __ATOMIC_RELAXED, __HIP_MEMORY_SCOPE_AGENT);
}
__device__ __forceinline__ void st_sys(void* ptr, unsigned v) {
  __hip_atomic_store((unsigned*)ptr, v, __ATOMIC_RELAXED, __HIP_MEMORY_SCOPE_SYSTEM);
}
__device__ __forceinline__ void st_agent(void* ptr, unsigned v) {
  __hip_atomic_store((unsigned*)ptr, v, __ATOMIC_RELAXED, __HIP_MEMORY_SCOPE_AGENT);
}

__device__ __forceinline__ bool wave_wait_ge(const void* flag, unsigned want, unsigned* tmo, unsigned call) {
  const unsigned long long t0 = __builtin_amdgcn_s_memrealtime();
  for (;;) {
    const unsigned v = ld_sys(flag);
    if (__all(v >= want)) return true;
    if (ld_agent(tmo) == call || __builtin_amdgcn_s_memrealtime() - t0 > DEADLINE_TICKS) {
      st_agent(tmo, call);
      return false;
    }
    __builtin_amdgcn_s_sleep(8);
  }
}

__device__ __forceinline__ void wave_wait_k(const void* flag, unsigned want, int k, unsigned* tmo, unsigned call) {
  const unsigned long long t0 = __builtin_amdgcn_s_memrealtime();
  for (;;) {
    const unsigned v = ld_agent(flag);
    if (__popcll(__ballot(v >= want)) >= k) return;
    if (ld_agent(tmo) == call || __builtin_amdgcn_s_memrealtime() - t0 > DEADLINE_TICKS) {
      st_agent(tmo, call);
      return;
    }
    __builtin_amdgcn_s_sleep(4);
  }
}

__device__ __forceinline__ void st16_sys(void* base, unsigned off, u32x4 v) {
  asm volatile("global_store_dwordx4 %0, %1, %2 sc0 sc1\n\ts_nop 1" :: "v"(off), "v"(v), "s"(base) : "memory");
}

__device__ __forceinline__ void st1_agent(void* base, unsigned v) {
  asm volatile("s_mov_b64 exec, 1\n\tglobal_store_dword %0, %1, %2 sc1\n\ts_mov_b64 exec, -1"
               :: "v"(0u), "v"(v), "s"(base) : "memory");
}

template <int N> __device__ __forceinline__ void wait_vm() {
  asm volatile("s_waitcnt vmcnt(%0)" :: "n"(N) : "memory");
}

#define PUSH_D 2
constexpr bool is_w(int r) { return r >= 4 && r < 12; }
constexpr int x_idx(int r) { return r < 4 ? r : r - 8; }
constexpr int NSLOT = 4 * NROUND;
constexpr int ld_at(int n) { return n / 4 + 2 >= NROUND ? 0 : (is_w(n / 4 + 2) ? 4 : 1); }
constexpr int younger(int n) { int c = 0; for (int m = n - PUSH_D + 1; m < n; ++m) c += m < 0 ? 0 : 1 + ld_at(m); return c; }
static_assert(PUSH_D >= 1 && PUSH_D <= 4, "a round's loads must have landed one round after their issue");

struct PushCtx {
  const uint8_t* x;
  const uint8_t* w;
  uint8_t* dst;
  unsigned* wprog;
  unsigned* rel;
  unsigned xoff, woff, xdoff, wdoff, base, call;
};

struct PushRegs {
  unsigned w[3][16];
  u32x4 x[3][4];
};

template <int R, int I> __device__ __forceinline__ void push_load(const PushCtx& c, PushRegs& g) {
  constexpr int s = R % 3;
  if constexpr (is_w(R)) {
    const unsigned o = c.woff + (R - 4) * 256;
#pragma unroll
    for (int i = 4 * I; i < 4 * I + 4; ++i)
      asm volatile("global_load_dword %0, %1, %2" : "=v"(g.w[s][i]) : "v"(o), "s"(c.w + (size_t)i * GN) : "memory");
  } else {
    constexpr int xi = x_idx(R);
    asm volatile("global_load_dwordx4 %0, %1, %2 offset:%3" : "=v"(g.x[s][I])
                 : "v"(c.xoff + (unsigned)(((xi & 3) * 1024 + (xi >> 2) * 128 + (I >> 1) * 32) * KS)), "s"(c.x), "n"((I & 1) * 16) : "memory");
  }
}

template <int N> __device__ __forceinline__ void push_slot(const Params& p, const PushCtx& c, PushRegs& g, u32x4 (&o)[4]) {
  constexpr int R = N / 4, I = N % 4, s = R % 3;
  if constexpr (N >= PUSH_D) {
    wait_vm<younger(N)>();
    if constexpr ((N - PUSH_D) % 4 == 3) { st1_agent(c.wprog, c.base + (N - PUSH_D) / 4 + 1); STAMP(8 + (N - PUSH_D) / 4); }
  }
  if constexpr (N == NSLOT - PUSH_REL) st1_agent(c.rel, c.call);
  if constexpr (I == 0) {
    if constexpr (is_w(R)) {
      unsigned (&w)[16] = g.w[s];
      asm volatile("" : "+v"(w[0]), "+v"(w[1]), "+v"(w[2]), "+v"(w[3]), "+v"(w[4]), "+v"(w[5]), "+v"(w[6]), "+v"(w[7]),
                        "+v"(w[8]), "+v"(w[9]), "+v"(w[10]), "+v"(w[11]), "+v"(w[12]), "+v"(w[13]), "+v"(w[14]), "+v"(w[15]) :: "memory");
#pragma unroll
      for (int q = 0; q < 4; ++q) {
        const unsigned d0 = w[4 * q], d1 = w[4 * q + 1], d2 = w[4 * q + 2], d3 = w[4 * q + 3];
        const unsigned t0 = __builtin_amdgcn_perm(d1, d0, 0x05010400u), t1 = __builtin_amdgcn_perm(d1, d0, 0x07030602u);
        const unsigned t2 = __builtin_amdgcn_perm(d3, d2, 0x05010400u), t3 = __builtin_amdgcn_perm(d3, d2, 0x07030602u);
        o[0][q] = __builtin_amdgcn_perm(t2, t0, 0x05040100u);
        o[1][q] = __builtin_amdgcn_perm(t2, t0, 0x07060302u);
        o[2][q] = __builtin_amdgcn_perm(t3, t1, 0x05040100u);
        o[3][q] = __builtin_amdgcn_perm(t3, t1, 0x07060302u);
      }
    } else {
      asm volatile("" : "+v"(g.x[s][0]), "+v"(g.x[s][1]), "+v"(g.x[s][2]), "+v"(g.x[s][3]) :: "memory");
#pragma unroll
      for (int q = 0; q < 4; ++q) o[q] = g.x[s][q];
    }
  }
  if constexpr (R + 2 < NROUND) push_load<R + 2, I>(c, g);
  if constexpr (is_w(R)) st16_sys(c.dst, c.wdoff + (R - 4) * (1u << 20) + I * 2048, o[I]);
  else st16_sys(c.dst, c.xdoff + (unsigned)((x_idx(R) >> 2) * 8 + (x_idx(R) & 3) * 2) * (512u << 10) + I * 1024, o[I]);
  if constexpr (N + 1 < NSLOT) push_slot<N + 1>(p, c, g, o);
}

__device__ __forceinline__ void glds2(const uint8_t* gbase, unsigned voff, unsigned lds) {
  unsigned keep;
  asm volatile("s_mov_b32 %0, m0\n\ts_mov_b32 m0, %3\n\ts_nop 0\n\t"
               "global_load_lds_dwordx4 %1, %2\n\t"
               "global_load_lds_dwordx4 %1, %2 offset:1024\n\t"
               "s_mov_b32 m0, %0"
               : "=&s"(keep) : "v"(voff), "s"(gbase), "s"(lds));
}

struct Frags { i32x8 a[2]; i32x8 b[4]; };

__device__ __forceinline__ i32x8 ld_frag(const uint8_t* q) {
  const i32x4 lo = *(const i32x4*)q, hi = *(const i32x4*)(q + 1024);
  return __builtin_shufflevector(lo, hi, 0, 1, 2, 3, 4, 5, 6, 7);
}

#define MFMA(mi, j) acc[mi][j] = __builtin_amdgcn_mfma_scale_f32_32x32x64_f8f6f4(cur.a[mi], cur.b[j], acc[mi][j], 1, 1, 0, 0, 0, 0)
#define SB() __builtin_amdgcn_sched_barrier(0)

template <int VM, bool DMA, bool RD>
__device__ __forceinline__ void kstep(f32x16 (&acc)[2][4], const Frags& cur, Frags& nxt, const uint8_t* lds, unsigned ldsb,
                                      int& slot, const uint8_t*& ga, const uint8_t*& gb, unsigned va, unsigned vb,
                                      unsigned ra, unsigned rb, unsigned wa, unsigned wb) {
  asm volatile("s_waitcnt vmcnt(%0) lgkmcnt(0)\n\ts_barrier" :: "n"(VM) : "memory");
  const unsigned wslot = ldsb + slot * STAGE;
  slot = slot == NSTAGE - 1 ? 0 : slot + 1;
  const uint8_t* qa = lds + slot * STAGE + ra;
  const uint8_t* qb = lds + slot * STAGE + rb;
  SB();
  MFMA(0, 0); SB();
  if constexpr (DMA) glds2(ga, va, wslot + wa);
  SB();
  MFMA(0, 1); SB();
  if constexpr (RD) { nxt.a[0] = ld_frag(qa); nxt.b[0] = ld_frag(qb); }
  SB();
  MFMA(0, 2); SB();
  if constexpr (DMA) glds2(gb, vb, wslot + wb);
  SB();
  MFMA(0, 3); SB();
  if constexpr (RD) { nxt.b[1] = ld_frag(qb + 2048); nxt.b[2] = ld_frag(qb + 4096); }
  SB();
  MFMA(1, 0); SB();
  if constexpr (DMA) glds2(gb + 2048, vb, wslot + wb + 2048);
  SB();
  MFMA(1, 1); SB();
  if constexpr (RD) { nxt.b[3] = ld_frag(qb + 6144); nxt.a[1] = ld_frag(qa + 2048); }
  SB();
  MFMA(1, 2); SB();
  MFMA(1, 3); SB();
  if constexpr (DMA) { ga += 8192; gb += 16384; }
}

}

__global__ void __launch_bounds__(256, 1) gemm_rs_fused(const Params p) {
  __shared__ __attribute__((aligned(16))) uint8_t lds[LDS_BYTES];
  const int b = blockIdx.x;
  const int lane = threadIdx.x & 63;
  const int wave = __builtin_amdgcn_readfirstlane(threadIdx.x >> 6);
  const int rank = p.rank;
  uint8_t* const myws = p.ws[rank];
  STAMP(0);

  const int grp = b & 7, dst = grp ^ rank;
  const int qc = 3 - (b >> 6), cb = (b >> 3) & 7;
  const int rt = qc * 8 + dst;
  unsigned* const callp = (unsigned*)(myws + WS_CALL + (size_t)b * 128);
  unsigned* const tmo = (unsigned*)(myws + WS_TMO);
  unsigned* const relp = (unsigned*)(myws + WS_REL) + grp * 128;

  unsigned call;
  if (qc == 3) {
    const int w = cb * 4 + wave;
    PushCtx c;
    c.x = p.x; c.w = p.w; c.dst = p.ws[dst];
    const int xks = (w & 15) >> 1, xrb = (w & 1) * 2;
    c.xoff = (unsigned)(((w >> 4) * 512 + xrb * 32 + (lane & 31)) * KS + xks * 64 + (lane >> 5) * 32);
    const int wks = w >> 2, wnb = (w >> 1) & 1, wh = w & 1;
    c.woff = (unsigned)((wks * 64 + (lane >> 5) * 32 + wh * 16) * GN + dst * NB + wnb * 128 + (lane & 31) * 4);
    PushRegs g;
    u32x4 o[4];
    unsigned cv;
    asm volatile("global_load_dword %0, %1, %2 sc1" : "=v"(cv) : "v"(0u), "s"(callp) : "memory");
    push_load<0, 0>(c, g); push_load<0, 1>(c, g); push_load<0, 2>(c, g); push_load<0, 3>(c, g);
    push_load<1, 0>(c, g); push_load<1, 1>(c, g); push_load<1, 2>(c, g); push_load<1, 3>(c, g);
    c.wprog = (unsigned*)(myws + WS_WPROG) + grp * 32 + w;
    c.rel = relp + w;
    asm volatile("s_waitcnt vmcnt(4)" : "+v"(cv) :: "memory");
    call = __builtin_amdgcn_readfirstlane(cv) + 1;
    const unsigned par = (unsigned)(WS_DATA + (call & 1) * PAR_BYTES);
    c.xdoff = par + (unsigned)((w >> 4) * (512 << 10) + (rank * 8 + xks) * 8192 + xrb * 2048 + lane * 16);
    c.wdoff = par + (unsigned)XS_BYTES + (unsigned)((rank * 8 + wks) * 16384 + wnb * 8192 + wh * 1024 + lane * 16);
    c.base = (call - 1) * NROUND;
    c.call = call;
    STAMP(7);
    push_slot<0>(p, c, g, o);
    wait_vm<0>();
    st1_agent(c.wprog, c.base + NROUND);
    STAMP(6);
  } else {
    call = __builtin_amdgcn_readfirstlane(ld_agent(callp)) + 1;
  }
  const unsigned base = (call - 1) * NROUND;

  if (b == 64 && wave == 1) {
    uint8_t* const fl = p.ws[(lane >> 3) ^ rank] + WS_PROG + (size_t)(rank * 8 + (lane & 7)) * 128;
    const unsigned* wp = (const unsigned*)(myws + WS_WPROG) + lane * 4;
    const unsigned long long t0 = __builtin_amdgcn_s_memrealtime();
    unsigned pub = base;
    for (;;) {
      const unsigned v0 = ld_agent(wp), v1 = ld_agent(wp + 1), v2 = ld_agent(wp + 2), v3 = ld_agent(wp + 3);
      unsigned mn = min(min(v0, v1), min(v2, v3));
      mn = min(mn, (unsigned)__shfl_xor((int)mn, 1));
      mn = min(mn, (unsigned)__shfl_xor((int)mn, 2));
      mn = min(mn, (unsigned)__shfl_xor((int)mn, 4));
      if (mn > pub) { pub = mn; st_sys(fl, pub); }
      if (__all(pub >= base + NROUND)) break;
      if (ld_agent(tmo) == call || __builtin_amdgcn_s_memrealtime() - t0 > DEADLINE_TICKS) {
        st_agent(tmo, call);
        break;
      }
    }
  }

  unsigned* const ctrl = (unsigned*)(lds + LDS_CTRL);
  const uint8_t* const progl = myws + WS_PROG + (size_t)((lane & 7) * 8 + grp) * 128;
  if (wave == 0) {
    const int need = qc == 0 ? 4 + cb + 1 : 8 + 4 * qc + (dst >> 1) + 1;
    wave_wait_ge(progl, base + need, tmo, call);
    __builtin_amdgcn_fence(__ATOMIC_ACQUIRE, "");
    asm volatile("s_waitcnt vmcnt(0)" ::: "memory");
    const unsigned t = ld_agent(tmo);
    if (lane == 0) ctrl[0] = t == call;
  }
  __syncthreads();
  const bool dead0 = __builtin_amdgcn_readfirstlane(ctrl[0]) != 0;
  STAMP(1);

  const int wm = wave >> 1, wn = wave & 1;
  f32x16 acc[2][4];
#pragma unroll
  for (int i = 0; i < 2; ++i)
#pragma unroll
    for (int j = 0; j < 4; ++j)
#pragma unroll
      for (int r = 0; r < 16; ++r) acc[i][j][r] = 0.f;

  if (!dead0) {
    const uint8_t* par = myws + WS_DATA + (size_t)(call & 1) * PAR_BYTES;
    const uint8_t* ga = par + (size_t)rt * (512 << 10);
    const uint8_t* gb = par + XS_BYTES + (size_t)cb * (1 << 20);
    const unsigned wa = wave * 2048, wb = 8192 + wave * 4096;
    const unsigned va = lane * 16 + wave * 2048, vb = lane * 16 + wave * 4096;
    const unsigned ra = lane * 16 + wm * 4096, rb = 8192 + lane * 16 + wn * 8192;
    const unsigned ldsb = (unsigned)(uintptr_t)lds;
#pragma unroll
    for (int s = 0; s < NSTAGE; ++s) {
      glds2(ga, va, ldsb + s * STAGE + wa);
      glds2(gb, vb, ldsb + s * STAGE + wb);
      glds2(gb + 2048, vb, ldsb + s * STAGE + wb + 2048);
      ga += 8192; gb += 16384;
    }
    asm volatile("s_waitcnt vmcnt(30)\n\ts_barrier" ::: "memory");
    Frags f0, f1;
    f0.a[0] = ld_frag(lds + ra); f0.a[1] = ld_frag(lds + ra + 2048);
    f0.b[0] = ld_frag(lds + rb); f0.b[1] = ld_frag(lds + rb + 2048);
    f0.b[2] = ld_frag(lds + rb + 4096); f0.b[3] = ld_frag(lds + rb + 6144);
    int slot = 0;
#pragma unroll 1
    for (int s = 0; s < KSTEPS - NSTAGE; s += 2) {
      kstep<24, true, true>(acc, f0, f1, lds, ldsb, slot, ga, gb, va, vb, ra, rb, wa, wb);
      kstep<24, true, true>(acc, f1, f0, lds, ldsb, slot, ga, gb, va, vb, ra, rb, wa, wb);
    }
    kstep<24, false, true>(acc, f0, f1, lds, ldsb, slot, ga, gb, va, vb, ra, rb, wa, wb);
    kstep<18, false, true>(acc, f1, f0, lds, ldsb, slot, ga, gb, va, vb, ra, rb, wa, wb);
    kstep<12, false, true>(acc, f0, f1, lds, ldsb, slot, ga, gb, va, vb, ra, rb, wa, wb);
    kstep<6, false, true>(acc, f1, f0, lds, ldsb, slot, ga, gb, va, vb, ra, rb, wa, wb);
    kstep<0, false, true>(acc, f0, f1, lds, ldsb, slot, ga, gb, va, vb, ra, rb, wa, wb);
    kstep<0, false, false>(acc, f1, f0, lds, ldsb, slot, ga, gb, va, vb, ra, rb, wa, wb);
  }
  STAMP(2);

  if (wave == 0) {
    if (grp != 0) wave_wait_k(relp + qc * 32 + (lane & 31), call, 2 * REL_K, tmo, call);
    const unsigned t = ld_agent(tmo);
    if (lane == 0) ctrl[1] = t == call;
  }
  __syncthreads();
  const bool dead = __builtin_amdgcn_readfirstlane(ctrl[1]) != 0;
  STAMP(3);

  {
    float sc = p.sx[0] * p.sw[0];
    if (dead) sc = __builtin_nanf("");
    float* const ob = p.out[dst];
    unsigned* const reln = relp + ((qc + 1) & 3) * 32 + cb * 4 + wave;
    const unsigned off = (unsigned)(((qc * 128 + wm * 64 + 4 * (lane >> 5)) * GN + rank * NB + cb * 256 + wn * 128 + 4 * (lane & 31)) * 4);
#pragma unroll
    for (int t = 0; t < 32; ++t) {
      const int mi = t >> 4, r = t & 15;
      if (t == 32 - OUT_REL && qc != 3) st1_agent(reln, call);
      const f32x4 v = {acc[mi][0][r] * sc, acc[mi][1][r] * sc, acc[mi][2][r] * sc, acc[mi][3][r] * sc};
      st16_sys(ob, off + (unsigned)(mi * 32 + (r & 3) + 8 * (r >> 2)) * (GN * 4), __builtin_bit_cast(u32x4, v));
      wait_vm<OUT_D - 1>();
    }
  }
  STAMP(4);
  asm volatile("s_waitcnt vmcnt(0)" ::: "memory");
  __syncthreads();
  if (threadIdx.x == 0) {
    if (grp != 0) st_sys(p.ws[dst] + WS_DONE + (size_t)(rank * 32 + qc * 8 + cb) * 4, call);
    st_agent(callp, call);
  }
  STAMP(5);

  if (b == 192) {
    const bool ok = wave_wait_ge(myws + WS_DONE + (size_t)threadIdx.x * 4, (int)(threadIdx.x >> 5) == rank ? 0u : call, tmo, call);
    __builtin_amdgcn_fence(__ATOMIC_ACQUIRE, "");
    asm volatile("s_waitcnt vmcnt(0)" ::: "memory");
    if (!ok && lane == 0) p.out[rank][(size_t)wave * GN + rank * NB] = __builtin_nanf("");
    STAMP(6);
  }
}

extern "C" void kernel_launch(int rank, int nranks, ncclComm_t comm, hipStream_t stream,
                              void* const* d_in, const int* in_sizes, int n_in,
                              void* d_out, int out_size, const int* in_dtypes, int out_dtype,
                              void* const* d_out_all, void* const* d_ws_all, size_t ws_bytes) {
  (void)comm; (void)in_sizes; (void)in_dtypes; (void)out_dtype; (void)d_out;
  if (nranks != NR || (unsigned)rank >= (unsigned)NR || n_in != 4 || ws_bytes < WS_NEED || out_size != 512 * GN) {
    fprintf(stderr, "kernel_launch: unexpected problem shape\n");
    return;
  }
  Params p;
  p.x = (const uint8_t*)d_in[0];
  p.w = (const uint8_t*)d_in[1];
  p.sx = (const float*)d_in[2];
  p.sw = (const float*)d_in[3];
  for (int i = 0; i < NR; ++i) {
    p.ws[i] = (uint8_t*)d_ws_all[i];
    p.out[i] = (float*)d_out_all[i];
  }
  p.rank = rank;
  p.pad = 0;
  hipLaunchKernelGGL(gemm_rs_fused, dim3(NWG), dim3(256), 0, stream, p);
  HIP_CHECK(hipGetLastError());
}
```

```cpp
#include <hip/hip_runtime.h>
#include <rccl/rccl.h>
#include <cstdio>
#include <cstdint>

#define HIP_CHECK(expr) do { hipError_t _e = (expr); if (_e != hipSuccess) { \
    fprintf(stderr, "HIP error %s at %s:%d\n", hipGetErrorString(_e), __FILE__, __LINE__); \
    return; } } while (0)

typedef unsigned u32x4 __attribute__((ext_vector_type(4)));
typedef int i32x4 __attribute__((ext_vector_type(4)));
typedef int i32x8 __attribute__((ext_vector_type(8)));
typedef float f32x4 __attribute__((ext_vector_type(4)));
typedef float f32x16 __attribute__((ext_vector_type(16)));

namespace {

constexpr int NR = 8;
constexpr int GN = 16384;
constexpr int KS = 512;
constexpr int NB = 2048;
constexpr int NWG = 256;
constexpr int NROUND = 24;
constexpr int KSTEPS = 64;
constexpr int STAGE = 24576;
constexpr int NSTAGE = 6;
constexpr int LDS_CTRL = STAGE * NSTAGE;
constexpr int LDS_PK = LDS_CTRL + 64;
constexpr int LDS_BYTES = LDS_PK + 4 * 2048;

constexpr size_t WS_CALL = 0;
constexpr size_t WS_PROG = 32768;
constexpr size_t WS_WPROG = WS_PROG + 8192;
constexpr size_t WS_TMO = WS_WPROG + 1024;
constexpr size_t WS_DONE = WS_TMO + 1024;
constexpr size_t WS_REL = WS_DONE + 4096;
constexpr size_t WS_DBG = 65536;
constexpr size_t WS_DATA = 1u << 20;
constexpr size_t XS_BYTES = 16u << 20;
constexpr size_t WT_BYTES = 8u << 20;
constexpr size_t PAR_BYTES = XS_BYTES + WT_BYTES;
constexpr size_t OBLK = 13 * 1024;
constexpr size_t WS_OS = WS_DATA + 2 * PAR_BYTES;
constexpr size_t WS_NEED = WS_OS + 8 * 256 * OBLK;

constexpr unsigned long long DEADLINE_TICKS = 50000000ull;

#define OUT_D 2
#define REL_ROUND 21
#define OUT_REL 3
#define NCRED 64

struct Params {
  const uint8_t* x;
  const uint8_t* w;
  const float* sx;
  const float* sw;
  uint8_t* ws[NR];
  float* out[NR];
  int rank;
  int pad;
};

#define STAMP(slot) do {} while (0)

__device__ __forceinline__ unsigned ld_sys(const void* ptr) {
  return __hip_atomic_load((const unsigned*)ptr, __ATOMIC_RELAXED, __HIP_MEMORY_SCOPE_SYSTEM);
}
__device__ __forceinline__ unsigned ld_agent(const void* ptr) {
  return __hip_atomic_load((const unsigned*)ptr, __ATOMIC_RELAXED, __HIP_MEMORY_SCOPE_AGENT);
}
__device__ __forceinline__ void st_sys(void* ptr, unsigned v) {
  __hip_atomic_store((unsigned*)ptr, v, __ATOMIC_RELAXED, __HIP_MEMORY_SCOPE_SYSTEM);
}
__device__ __forceinline__ void st_agent(void* ptr, unsigned v) {
  __hip_atomic_store((unsigned*)ptr, v, __ATOMIC_RELAXED, __HIP_MEMORY_SCOPE_AGENT);
}

__device__ __forceinline__ bool wave_wait_ge(const void* flag, unsigned want, unsigned* tmo, unsigned call) {
  const unsigned long long t0 = __builtin_amdgcn_s_memrealtime();
  for (;;) {
    const unsigned v = ld_sys(flag);
    if (__all(v >= want)) return true;
    if (ld_agent(tmo) == call || __builtin_amdgcn_s_memrealtime() - t0 > DEADLINE_TICKS) {
      st_agent(tmo, call);
      return false;
    }
    __builtin_amdgcn_s_sleep(8);
  }
}

__device__ __forceinline__ void link_acquire(unsigned* rel, unsigned* tmo, unsigned call, int lane) {
  unsigned t = 0;
  if (lane == 0) t = __hip_atomic_fetch_add(rel, 1u, __ATOMIC_RELAXED, __HIP_MEMORY_SCOPE_AGENT);
  const unsigned want = __builtin_amdgcn_readfirstlane(t) + (call - 1) * NCRED;
  const unsigned long long t0 = __builtin_amdgcn_s_memrealtime();
  while (__builtin_amdgcn_readfirstlane(ld_agent(rel + 32)) <= want) {
    if (__builtin_amdgcn_readfirstlane(ld_agent(tmo)) == call || __builtin_amdgcn_s_memrealtime() - t0 > DEADLINE_TICKS) {
      st_agent(tmo, call);
      return;
    }
    __builtin_amdgcn_s_sleep(4);
  }
}

__device__ __forceinline__ void st16_sys(void* base, unsigned off, u32x4 v) {
  asm volatile("global_store_dwordx4 %0, %1, %2 sc0 sc1\n\ts_nop 1" :: "v"(off), "v"(v), "s"(base) : "memory");
}

__device__ __forceinline__ void st1_agent(void* base, unsigned v) {
  asm volatile("s_mov_b64 exec, 1\n\tglobal_store_dword %0, %1, %2 sc1\n\ts_mov_b64 exec, -1"
               :: "v"(0u), "v"(v), "s"(base) : "memory");
}

__device__ __forceinline__ void add1_agent(void* base, unsigned v) {
  asm volatile("s_mov_b64 exec, 1\n\tglobal_atomic_add %0, %1, %2\n\ts_mov_b64 exec, -1"
               :: "v"(0u), "v"(v), "s"(base) : "memory");
}
__device__ __forceinline__ void st1_sys(void* base, unsigned v) {
  asm volatile("s_mov_b64 exec, 1\n\tglobal_store_dword %0, %1, %2 sc0 sc1\n\ts_mov_b64 exec, -1"
               :: "v"(0u), "v"(v), "s"(base) : "memory");
}
__device__ __forceinline__ void st16_sys_m(void* base, unsigned off, u32x4 v, unsigned long long mask) {
  asm volatile("s_mov_b64 exec, %3\n\tglobal_store_dwordx4 %0, %1, %2 sc0 sc1\n\ts_mov_b64 exec, -1\n\ts_nop 0"
               :: "v"(off), "v"(v), "s"(base), "s"(mask) : "memory");
}

template <int N> __device__ __forceinline__ void wait_vm() {
  asm volatile("s_waitcnt vmcnt(%0)" :: "n"(N) : "memory");
}

#define PUSH_D 2
constexpr bool is_w(int r) { return r >= 4 && r < 12; }
constexpr int w_idx(int r) { return r - 4; }
constexpr int x_idx(int r) { return r < 4 ? r : r - 8; }
constexpr int NSLOT = 4 * NROUND;
constexpr int ld_at(int n) { return n / 4 + 2 >= NROUND ? 0 : (is_w(n / 4 + 2) ? 4 : 1); }
constexpr int younger(int n) { int c = 0; for (int m = n - PUSH_D + 1; m < n; ++m) c += m < 0 ? 0 : 1 + ld_at(m); return c; }
static_assert(PUSH_D >= 1 && PUSH_D <= 4, "a round's loads must have landed one round after their issue");

struct PushCtx {
  const uint8_t* x;
  const uint8_t* w;
  uint8_t* dst;
  unsigned* wprog;
  unsigned xoff, woff, xdoff, wdoff, base;
};

struct PushRegs {
  unsigned w[3][16];
  u32x4 x[3][4];
};

template <int R, int I> __device__ __forceinline__ void push_load(const PushCtx& c, PushRegs& g) {
  constexpr int s = R % 3;
  if constexpr (is_w(R)) {
    const unsigned o = c.woff + w_idx(R) * 256;
#pragma unroll
    for (int i = 4 * I; i < 4 * I + 4; ++i)
      asm volatile("global_load_dword %0, %1, %2" : "=v"(g.w[s][i]) : "v"(o), "s"(c.w + (size_t)i * GN) : "memory");
  } else {
    constexpr int xi = x_idx(R);
    asm volatile("global_load_dwordx4 %0, %1, %2 offset:%3" : "=v"(g.x[s][I])
                 : "v"(c.xoff + (unsigned)(((xi & 3) * 1024 + (xi >> 2) * 128 + (I >> 1) * 32) * KS)), "s"(c.x), "n"((I & 1) * 16) : "memory");
  }
}

template <int N> __device__ __forceinline__ void push_slot(const Params& p, const PushCtx& c, PushRegs& g, u32x4 (&o)[4]) {
  constexpr int R = N / 4, I = N % 4, s = R % 3;
  if constexpr (N >= PUSH_D) {
    wait_vm<younger(N)>();
    if constexpr ((N - PUSH_D) % 4 == 3) { st1_agent(c.wprog, c.base + (N - PUSH_D) / 4 + 1); STAMP(8 + (N - PUSH_D) / 4); }
  }
  if constexpr (I == 0) {
    if constexpr (is_w(R)) {
      unsigned (&w)[16] = g.w[s];
      asm volatile("" : "+v"(w[0]), "+v"(w[1]), "+v"(w[2]), "+v"(w[3]), "+v"(w[4]), "+v"(w[5]), "+v"(w[6]), "+v"(w[7]),
                        "+v"(w[8]), "+v"(w[9]), "+v"(w[10]), "+v"(w[11]), "+v"(w[12]), "+v"(w[13]), "+v"(w[14]), "+v"(w[15]) :: "memory");
#pragma unroll
      for (int q = 0; q < 4; ++q) {
        const unsigned d0 = w[4 * q], d1 = w[4 * q + 1], d2 = w[4 * q + 2], d3 = w[4 * q + 3];
        const unsigned t0 = __builtin_amdgcn_perm(d1, d0, 0x05010400u), t1 = __builtin_amdgcn_perm(d1, d0, 0x07030602u);
        const unsigned t2 = __builtin_amdgcn_perm(d3, d2, 0x05010400u), t3 = __builtin_amdgcn_perm(d3, d2, 0x07030602u);
        o[0][q] = __builtin_amdgcn_perm(t2, t0, 0x05040100u);
        o[1][q] = __builtin_amdgcn_perm(t2, t0, 0x07060302u);
        o[2][q] = __builtin_amdgcn_perm(t3, t1, 0x05040100u);
        o[3][q] = __builtin_amdgcn_perm(t3, t1, 0x07060302u);
      }
    } else {
      asm volatile("" : "+v"(g.x[s][0]), "+v"(g.x[s][1]), "+v"(g.x[s][2]), "+v"(g.x[s][3]) :: "memory");
#pragma unroll
      for (int q = 0; q < 4; ++q) o[q] = g.x[s][q];
    }
  }
  if constexpr (R + 2 < NROUND) push_load<R + 2, I>(c, g);
  if constexpr (is_w(R)) st16_sys(c.dst, c.wdoff + w_idx(R) * (1u << 20) + I * 2048, o[I]);
  else st16_sys(c.dst, c.xdoff + (unsigned)((x_idx(R) >> 2) * 8 + (x_idx(R) & 3) * 2) * (512u << 10) + I * 1024, o[I]);
  if constexpr (N + 1 < NSLOT) push_slot<N + 1>(p, c, g, o);
}

__device__ __forceinline__ void glds2(const uint8_t* gbase, unsigned voff, unsigned lds) {
  unsigned keep;
  asm volatile("s_mov_b32 %0, m0\n\ts_mov_b32 m0, %3\n\ts_nop 0\n\t"
               "global_load_lds_dwordx4 %1, %2\n\t"
               "global_load_lds_dwordx4 %1, %2 offset:1024\n\t"
               "s_mov_b32 m0, %0"
               : "=&s"(keep) : "v"(voff), "s"(gbase), "s"(lds));
}

struct Frags { i32x8 a[2]; i32x8 b[4]; };

__device__ __forceinline__ i32x8 ld_frag(const uint8_t* q) {
  const i32x4 lo = *(const i32x4*)q, hi = *(const i32x4*)(q + 1024);
  return __builtin_shufflevector(lo, hi, 0, 1, 2, 3, 4, 5, 6, 7);
}

#define MFMA(mi, j) acc[mi][j] = __builtin_amdgcn_mfma_scale_f32_32x32x64_f8f6f4(cur.a[mi], cur.b[j], acc[mi][j], 1, 1, 0, 0, 0, 0)
#define SB() __builtin_amdgcn_sched_barrier(0)

template <int VM, bool DMA, bool RD>
__device__ __forceinline__ void kstep(f32x16 (&acc)[2][4], const Frags& cur, Frags& nxt, const uint8_t* lds, unsigned ldsb,
                                      int& slot, const uint8_t*& ga, const uint8_t*& gb, unsigned va, unsigned vb,
                                      unsigned ra, unsigned rb, unsigned wa, unsigned wb) {
  asm volatile("s_waitcnt vmcnt(%0) lgkmcnt(0)\n\ts_barrier" :: "n"(VM) : "memory");
  const unsigned wslot = ldsb + slot * STAGE;
  slot = slot == NSTAGE - 1 ? 0 : slot + 1;
  const uint8_t* qa = lds + slot * STAGE + ra;
  const uint8_t* qb = lds + slot * STAGE + rb;
  SB();
  MFMA(0, 0); SB();
  if constexpr (DMA) glds2(ga, va, wslot + wa);
  SB();
  MFMA(0, 1); SB();
  if constexpr (RD) { nxt.a[0] = ld_frag(qa); nxt.b[0] = ld_frag(qb); }
  SB();
  MFMA(0, 2); SB();
  if constexpr (DMA) glds2(gb, vb, wslot + wb);
  SB();
  MFMA(0, 3); SB();
  if constexpr (RD) { nxt.b[1] = ld_frag(qb + 2048); nxt.b[2] = ld_frag(qb + 4096); }
  SB();
  MFMA(1, 0); SB();
  if constexpr (DMA) glds2(gb + 2048, vb, wslot + wb + 2048);
  SB();
  MFMA(1, 1); SB();
  if constexpr (RD) { nxt.b[3] = ld_frag(qb + 6144); nxt.a[1] = ld_frag(qa + 2048); }
  SB();
  MFMA(1, 2); SB();
  MFMA(1, 3); SB();
  if constexpr (DMA) { ga += 8192; gb += 16384; }
}

constexpr unsigned REC_CAP = 510, HDR_RAW = 0x80000000u, HDR_B = 0x40000000u;

struct OutBlk {
  u32x4 pay[12];
  u32x4 rec;
  unsigned long long mask;
  bool raw;
};

__device__ __forceinline__ unsigned wave_max_u32(unsigned v) {
  v = max(v, (unsigned)__builtin_amdgcn_update_dpp(0, (int)v, 0x111, 0xf, 0xf, false));
  v = max(v, (unsigned)__builtin_amdgcn_update_dpp(0, (int)v, 0x112, 0xf, 0xf, false));
  v = max(v, (unsigned)__builtin_amdgcn_update_dpp(0, (int)v, 0x114, 0xf, 0xf, false));
  v = max(v, (unsigned)__builtin_amdgcn_update_dpp(0, (int)v, 0x118, 0xf, 0xf, false));
  return max(max((unsigned)__builtin_amdgcn_readlane((int)v, 15), (unsigned)__builtin_amdgcn_readlane((int)v, 31)),
             max((unsigned)__builtin_amdgcn_readlane((int)v, 47), (unsigned)__builtin_amdgcn_readlane((int)v, 63)));
}

template <int BITS>
__device__ __forceinline__ void put_records(unsigned q, unsigned tag, unsigned& cnt, unsigned cap, uint8_t* scr, unsigned dump) {
  for (unsigned long long act = __ballot(q != 0); act != 0; act = __ballot(q != 0)) {
    const unsigned sh = (unsigned)__builtin_ctz(q | 0x80000000u) & (32u - BITS);
    const unsigned idx = cnt + __builtin_amdgcn_mbcnt_hi((unsigned)(act >> 32), __builtin_amdgcn_mbcnt_lo((unsigned)act, 0u));
    const unsigned at = (q != 0 && idx < cap) ? 4 + 2 * idx : dump;
    *(unsigned short*)(scr + at) = (unsigned short)(tag | (sh << (BITS == 4 ? 2 : 1)) | ((q >> sh) & ((1u << BITS) - 1)));
    q &= ~(((1u << BITS) - 1) << sh);
    cnt += __popcll(act);
  }
}

__device__ __forceinline__ void enc_out(const f32x16 (&a)[4], OutBlk& o, uint8_t* scr, int lane) {
  float v[64];
  float m = 0.f;
#pragma unroll
  for (int i = 0; i < 64; i += 2) {
    v[i] = a[i & 3][i >> 2]; v[i + 1] = a[(i + 1) & 3][i >> 2];
    asm("v_max3_f32 %0, |%1|, |%2|, %0" : "+v"(m) : "v"(v[i]), "v"(v[i + 1]));
  }
  const int e = max((int)(wave_max_u32(__builtin_bit_cast(unsigned, m)) >> 23) - 157, -126);
  const float up = __builtin_bit_cast(float, (unsigned)(127 - min(e, 0)) << 23);
  unsigned G[64];
  unsigned bad = 0, low = 0;
  int ntop = 0, nlow = 0;
#pragma unroll
  for (int i = 0; i < 64; ++i) {
    const float x = v[i] * up;
    G[i] = (unsigned)(int)x;
    bad = __builtin_amdgcn_bitop3_b32(__builtin_bit_cast(unsigned, (float)(int)G[i]), __builtin_bit_cast(unsigned, x), bad, 0xbe);
    low |= G[i];
    if ((i & 7) == 0) {
      ntop += __popcll(__ballot((int)(G[i] ^ (G[i] << 1)) < 0));
      nlow += __popcll(__ballot((G[i] & 128u) != 0));
    }
  }
  const bool modeb = ntop < nlow;
  bad |= low & 15u;
  if (modeb) {
#pragma unroll
    for (int i = 0; i < 64; ++i) {
      const unsigned t = G[i] << 1;
      G[i] = t | (((G[i] ^ t) >> 27) & 16u);
    }
  }
  unsigned cnt = 0;
#pragma unroll
  for (int w = 0; w < 8; ++w) {
    const unsigned* g = G + 8 * w;
    const unsigned b0 = __builtin_amdgcn_perm(__builtin_amdgcn_perm(g[3], g[2], 0x0c0c0400u), __builtin_amdgcn_perm(g[1], g[0], 0x0c0c0400u), 0x05040100u);
    const unsigned b1 = __builtin_amdgcn_perm(__builtin_amdgcn_perm(g[7], g[6], 0x0c0c0400u), __builtin_amdgcn_perm(g[5], g[4], 0x0c0c0400u), 0x05040100u);
    put_records<4>((b0 >> 4) | b1,
                   (unsigned)(lane << 10 | w << 7), cnt, REC_CAP, scr, 1024 + 2 * lane);
  }
#pragma unroll
  for (int g = 0; g < 16; ++g) {
    o.pay[(3 * g) >> 2][(3 * g) & 3] = __builtin_amdgcn_perm(G[4 * g + 1], G[4 * g], 0x05030201u);
    o.pay[(3 * g + 1) >> 2][(3 * g + 1) & 3] = __builtin_amdgcn_perm(G[4 * g + 2], G[4 * g + 1], 0x06050302u);
    o.pay[(3 * g + 2) >> 2][(3 * g + 2) & 3] = __builtin_amdgcn_perm(G[4 * g + 3], G[4 * g + 2], 0x07060503u);
  }
  o.raw = __ballot(bad != 0) != 0 || cnt > REC_CAP || e > 0;
  *(unsigned*)(scr + (lane == 0 ? 0 : 1024 + 128 + 4 * lane)) = cnt | (unsigned)(e + 127) << 16 | (modeb ? HDR_B : 0u);
  __builtin_amdgcn_fence(__ATOMIC_ACQ_REL, "wavefront");
  o.rec = *(const u32x4*)(scr + lane * 16);
  __builtin_amdgcn_fence(__ATOMIC_ACQ_REL, "wavefront");
  const unsigned nl = ((4 + 2 * cnt + 63) >> 6) * 4;
  o.mask = nl >= 64 ? ~0ull : (1ull << nl) - 1;
}

#define LOC_ST 1
__device__ __forceinline__ void st16_out(void* base, unsigned off, f32x4 v) {
  asm volatile("global_store_dwordx4 %0, %1, %2 sc1\n\ts_nop 1" :: "v"(off), "v"(v), "s"(base) : "memory");
}

__device__ __forceinline__ constexpr unsigned row_off(int r) { return (unsigned)((r & 3) + 8 * (r >> 2)) * (GN * 4); }

template <bool LAST>
__device__ __forceinline__ void send_out(const OutBlk& o, const f32x16 (&a)[4], bool raw, float sc, uint8_t* slot, float* ob, unsigned off,
                                         int lane, unsigned* flag, unsigned fval, unsigned* rel) {
  if (raw) {
    st16_sys_m(slot, lane * 16, u32x4{HDR_RAW, 0u, 0u, 0u}, 15ull);
#pragma unroll
    for (int r = 0; r < 16; ++r) {
      if (LAST && r == 16 - OUT_REL) add1_agent(rel, 1);
      const f32x4 v = {a[0][r] * sc, a[1][r] * sc, a[2][r] * sc, a[3][r] * sc};
      st16_sys(ob, off + row_off(r), __builtin_bit_cast(u32x4, v));
      wait_vm<OUT_D - 1>();
      if (LAST && r == OUT_D - 1) st1_sys(flag, fval);
    }
  } else {
    st16_sys_m(slot, lane * 16, o.rec, o.mask);
#pragma unroll
    for (int i = 0; i < 12; ++i) {
      if (LAST && i == 12 - OUT_REL) add1_agent(rel, 1);
      st16_sys(slot, 1024 + i * 1024 + lane * 16, o.pay[i]);
      wait_vm<OUT_D - 1>();
      if (LAST && i == OUT_D - 1) st1_sys(flag, fval);
    }
  }
}

__device__ __forceinline__ void dec_out(const uint8_t* slot, float* ob, unsigned off, float sc, bool dead, uint8_t* scr, int lane) {
  u32x4 rec, y[12];
  asm volatile("global_load_dwordx4 %0, %13, %15 offset:-4096 sc0 sc1\n\tglobal_load_dwordx4 %1, %13, %15 offset:-3072 sc0 sc1\n\t"
               "global_load_dwordx4 %2, %13, %15 offset:-2048 sc0 sc1\n\tglobal_load_dwordx4 %3, %13, %15 offset:-1024 sc0 sc1\n\t"
               "global_load_dwordx4 %4, %13, %15 sc0 sc1\n\tglobal_load_dwordx4 %5, %13, %15 offset:1024 sc0 sc1\n\t"
               "global_load_dwordx4 %6, %13, %15 offset:2048 sc0 sc1\n\tglobal_load_dwordx4 %7, %13, %15 offset:3072 sc0 sc1\n\t"
               "global_load_dwordx4 %8, %14, %15 offset:-4096 sc0 sc1\n\tglobal_load_dwordx4 %9, %14, %15 offset:-3072 sc0 sc1\n\t"
               "global_load_dwordx4 %10, %14, %15 offset:-2048 sc0 sc1\n\tglobal_load_dwordx4 %11, %14, %15 offset:-1024 sc0 sc1\n\t"
               "global_load_dwordx4 %12, %14, %15 sc0 sc1\n\ts_waitcnt vmcnt(0)"
               : "=&v"(rec), "=&v"(y[0]), "=&v"(y[1]), "=&v"(y[2]), "=&v"(y[3]), "=&v"(y[4]), "=&v"(y[5]), "=&v"(y[6]),
                 "=&v"(y[7]), "=&v"(y[8]), "=&v"(y[9]), "=&v"(y[10]), "=&v"(y[11])
               : "v"(4096 + lane * 16), "v"(12288 + lane * 16), "s"(slot) : "memory");
  const unsigned hdr = __builtin_amdgcn_readfirstlane(rec[0]);
  if (dead) {
    const float n = __builtin_nanf("");
#pragma unroll
    for (int r = 0; r < 16; ++r) st16_out(ob, off + row_off(r), f32x4{n, n, n, n});
    return;
  }
  if (hdr & HDR_RAW) return;
  const unsigned cnt = hdr & 0xffffu;
  const float down = __builtin_bit_cast(float, ((hdr >> 16) & 255u) << 23);
  *(u32x4*)(scr + lane * 32) = u32x4{0u, 0u, 0u, 0u};
  *(u32x4*)(scr + lane * 32 + 16) = u32x4{0u, 0u, 0u, 0u};
  __builtin_amdgcn_fence(__ATOMIC_ACQ_REL, "wavefront");
#pragma unroll
  for (int i = 0; i < 8; ++i) {
    const unsigned r16 = (rec[i >> 1] >> (16 * (i & 1))) & 0xffffu;
    const unsigned val = (unsigned)(lane * 8 + i - 2) < cnt ? (r16 & 15u) << ((r16 >> 2) & 28u) : 0u;
    __hip_atomic_fetch_or((unsigned*)(scr + (r16 >> 7) * 4), val, __ATOMIC_RELAXED, __HIP_MEMORY_SCOPE_WORKGROUP);
  }
  __builtin_amdgcn_fence(__ATOMIC_ACQ_REL, "wavefront");
  const u32x4 q0 = *(const u32x4*)(scr + lane * 32), q1 = *(const u32x4*)(scr + lane * 32 + 16);
  __builtin_amdgcn_fence(__ATOMIC_ACQ_REL, "wavefront");
  const bool modeb = (hdr & HDR_B) != 0;
#pragma unroll
  for (int g = 0; g < 16; ++g) {
    const unsigned d0 = y[(3 * g) >> 2][(3 * g) & 3], d1 = y[(3 * g + 1) >> 2][(3 * g + 1) & 3], d2 = y[(3 * g + 2) >> 2][(3 * g + 2) & 3];
    const unsigned qw = g < 8 ? q0[(g >> 1) & 3] : q1[(g >> 1) & 3];
    const unsigned q = ((g & 1) ? qw : qw << 4) & 0xf0f0f0f0u;
    unsigned G[4] = {__builtin_amdgcn_perm(d0, q, 0x06050400u), __builtin_amdgcn_perm(d1, d0, 0x0504030cu) | ((q >> 8) & 255u),
                     __builtin_amdgcn_perm(d2, d1, 0x0403020cu) | ((q >> 16) & 255u), __builtin_amdgcn_perm(d2, q, 0x07060503u)};
    f32x4 v;
#pragma unroll
    for (int j = 0; j < 4; ++j) {
      if (modeb) G[j] = ((unsigned)((int)G[j] >> 1) & ~8u) ^ (G[j] << 27);
      v[j] = ((float)(int)G[j] * down) * sc;
    }
    st16_out(ob, off + row_off(g), v);
  }
}

__device__ __forceinline__ void recv_tile(uint8_t* myws, float* out, int src, int stg, int si, int wave, int lane, unsigned call,
                                          unsigned* tmo, float sc, uint8_t* scr) {
  const int t = (src * 4 + stg) * 32 + si * 4 + wave;
  const int q = stg, cb = si;
  const unsigned off = (unsigned)(((q * 128 + (wave >> 1) * 64 + 4 * (lane >> 5)) * GN + src * NB + cb * 256 + (wave & 1) * 128 + 4 * (lane & 31)) * 4);
#pragma unroll 1
  for (int mi = 0; mi < 2; ++mi) {
    wave_wait_ge(myws + WS_DONE + (size_t)t * 4, 2 * call - 1 + mi, tmo, call);
    const bool dead = __builtin_amdgcn_readfirstlane(ld_agent(tmo)) == call;
    dec_out(myws + WS_OS + (size_t)(t * 2 + mi) * OBLK, out, off + (unsigned)mi * 32 * (GN * 4), sc, dead, scr, lane);
  }
}

}

__global__ void __launch_bounds__(256, 1) gemm_rs_fused(const Params p) {
  __shared__ __attribute__((aligned(16))) uint8_t lds[LDS_BYTES];
  const int b = blockIdx.x;
  const int lane = threadIdx.x & 63;
  const int wave = __builtin_amdgcn_readfirstlane(threadIdx.x >> 6);
  const int rank = p.rank;
  uint8_t* const myws = p.ws[rank];
  STAMP(0);

  const int grp = b & 7, dst = grp ^ rank;
  const int stg = 3 - (b >> 6), si = (b >> 3) & 7;
  const int qc = stg, cb = si;
  const int rt = qc * 8 + dst;
  unsigned* const callp = (unsigned*)(myws + WS_CALL + (size_t)b * 128);
  unsigned* const tmo = (unsigned*)(myws + WS_TMO);
  unsigned* const relp = (unsigned*)(myws + WS_REL) + grp * 64;

  unsigned call;
  if (stg == 3) {
    const int w = si * 4 + wave;
    PushCtx c;
    c.x = p.x; c.w = p.w; c.dst = p.ws[dst];
    const int xks = (w & 15) >> 1, xrb = (w & 1) * 2;
    c.xoff = (unsigned)(((w >> 4) * 512 + xrb * 32 + (lane & 31)) * KS + xks * 64 + (lane >> 5) * 32);
    const int wks = w >> 2, wnb = (w >> 1) & 1, wh = w & 1;
    c.woff = (unsigned)((wks * 64 + (lane >> 5) * 32 + wh * 16) * GN + dst * NB + wnb * 128 + (lane & 31) * 4);
    PushRegs g;
    u32x4 o[4];
    unsigned cv;
    asm volatile("global_load_dword %0, %1, %2 sc1" : "=v"(cv) : "v"(0u), "s"(callp) : "memory");
    push_load<0, 0>(c, g); push_load<0, 1>(c, g); push_load<0, 2>(c, g); push_load<0, 3>(c, g);
    push_load<1, 0>(c, g); push_load<1, 1>(c, g); push_load<1, 2>(c, g); push_load<1, 3>(c, g);
    c.wprog = (unsigned*)(myws + WS_WPROG) + grp * 32 + w;
    asm volatile("s_waitcnt vmcnt(4)" : "+v"(cv) :: "memory");
    call = __builtin_amdgcn_readfirstlane(cv) + 1;
    const unsigned par = (unsigned)(WS_DATA + (call & 1) * PAR_BYTES);
    c.xdoff = par + (unsigned)((w >> 4) * (512 << 10) + (rank * 8 + xks) * 8192 + xrb * 2048 + lane * 16);
    c.wdoff = par + (unsigned)XS_BYTES + (unsigned)((rank * 8 + wks) * 16384 + wnb * 8192 + wh * 1024 + lane * 16);
    c.base = (call - 1) * NROUND;
    STAMP(7);
    push_slot<0>(p, c, g, o);
    wait_vm<0>();
    st1_agent(c.wprog, c.base + NROUND);
    STAMP(6);
  } else {
    call = __builtin_amdgcn_readfirstlane(ld_agent(callp)) + 1;
  }
  const unsigned base = (call - 1) * NROUND;

  if (b == 64 && wave == 1) {
    uint8_t* const fl = p.ws[(lane >> 3) ^ rank] + WS_PROG + (size_t)(rank * 8 + (lane & 7)) * 128;
    const unsigned* wp = (const unsigned*)(myws + WS_WPROG) + lane * 4;
    const unsigned long long t0 = __builtin_amdgcn_s_memrealtime();
    unsigned pub = base;
    for (;;) {
      const unsigned v0 = ld_agent(wp), v1 = ld_agent(wp + 1), v2 = ld_agent(wp + 2), v3 = ld_agent(wp + 3);
      unsigned mn = min(min(v0, v1), min(v2, v3));
      mn = min(mn, (unsigned)__shfl_xor((int)mn, 1));
      mn = min(mn, (unsigned)__shfl_xor((int)mn, 2));
      mn = min(mn, (unsigned)__shfl_xor((int)mn, 4));
      if (mn > pub) {
        if (pub < base + REL_ROUND && mn >= base + REL_ROUND && (lane & 7) == 0)
          __hip_atomic_fetch_add((unsigned*)(myws + WS_REL) + (lane >> 3) * 64 + 32, (unsigned)NCRED, __ATOMIC_RELAXED, __HIP_MEMORY_SCOPE_AGENT);
        pub = mn;
        st_sys(fl, pub);
      }
      if (__all(pub >= base + NROUND)) break;
      if (ld_agent(tmo) == call || __builtin_amdgcn_s_memrealtime() - t0 > DEADLINE_TICKS) {
        st_agent(tmo, call);
        break;
      }
    }
  }

  unsigned* const ctrl = (unsigned*)(lds + LDS_CTRL);
  const uint8_t* const progl = myws + WS_PROG + (size_t)((lane & 7) * 8 + grp) * 128;
  if (wave == 0) {
    const int need = qc == 0 ? 4 + cb + 1 : 8 + 4 * qc + (dst >> 1) + 1;
    wave_wait_ge(progl, base + need, tmo, call);
    __builtin_amdgcn_fence(__ATOMIC_ACQUIRE, "");
    asm volatile("s_waitcnt vmcnt(0)" ::: "memory");
    const unsigned t = ld_agent(tmo);
    if (lane == 0) ctrl[0] = t == call;
  }
  __syncthreads();
  const bool dead0 = __builtin_amdgcn_readfirstlane(ctrl[0]) != 0;
  STAMP(1);

  const int wm = wave >> 1, wn = wave & 1;
  f32x16 acc[2][4];
#pragma unroll
  for (int i = 0; i < 2; ++i)
#pragma unroll
    for (int j = 0; j < 4; ++j)
#pragma unroll
      for (int r = 0; r < 16; ++r) acc[i][j][r] = 0.f;

  if (!dead0) {
    const uint8_t* par = myws + WS_DATA + (size_t)(call & 1) * PAR_BYTES;
    const uint8_t* ga = par + (size_t)rt * (512 << 10);
    const uint8_t* gb = par + XS_BYTES + (size_t)cb * (1 << 20);
    const unsigned wa = wave * 2048, wb = 8192 + wave * 4096;
    const unsigned va = lane * 16 + wave * 2048, vb = lane * 16 + wave * 4096;
    const unsigned ra = lane * 16 + wm * 4096, rb = 8192 + lane * 16 + wn * 8192;
    const unsigned ldsb = (unsigned)(uintptr_t)lds;
#pragma unroll
    for (int s = 0; s < NSTAGE; ++s) {
      glds2(ga, va, ldsb + s * STAGE + wa);
      glds2(gb, vb, ldsb + s * STAGE + wb);
      glds2(gb + 2048, vb, ldsb + s * STAGE + wb + 2048);
      ga += 8192; gb += 16384;
    }
    asm volatile("s_waitcnt vmcnt(30)\n\ts_barrier" ::: "memory");
    Frags f0, f1;
    f0.a[0] = ld_frag(lds + ra); f0.a[1] = ld_frag(lds + ra + 2048);
    f0.b[0] = ld_frag(lds + rb); f0.b[1] = ld_frag(lds + rb + 2048);
    f0.b[2] = ld_frag(lds + rb + 4096); f0.b[3] = ld_frag(lds + rb + 6144);
    int slot = 0;
#pragma unroll 1
    for (int s = 0; s < KSTEPS - NSTAGE; s += 2) {
      kstep<24, true, true>(acc, f0, f1, lds, ldsb, slot, ga, gb, va, vb, ra, rb, wa, wb);
      kstep<24, true, true>(acc, f1, f0, lds, ldsb, slot, ga, gb, va, vb, ra, rb, wa, wb);
    }
    kstep<24, false, true>(acc, f0, f1, lds, ldsb, slot, ga, gb, va, vb, ra, rb, wa, wb);
    kstep<18, false, true>(acc, f1, f0, lds, ldsb, slot, ga, gb, va, vb, ra, rb, wa, wb);
    kstep<12, false, true>(acc, f0, f1, lds, ldsb, slot, ga, gb, va, vb, ra, rb, wa, wb);
    kstep<6, false, true>(acc, f1, f0, lds, ldsb, slot, ga, gb, va, vb, ra, rb, wa, wb);
    kstep<0, false, true>(acc, f0, f1, lds, ldsb, slot, ga, gb, va, vb, ra, rb, wa, wb);
    kstep<0, false, false>(acc, f1, f0, lds, ldsb, slot, ga, gb, va, vb, ra, rb, wa, wb);
  }
  STAMP(2);

  if (threadIdx.x == 0) st_agent(callp, call);
  const float sc = p.sx[0] * p.sw[0];
  const unsigned off = (unsigned)(((qc * 128 + wm * 64 + 4 * (lane >> 5)) * GN + rank * NB + cb * 256 + wn * 128 + 4 * (lane & 31)) * 4);
  float* const ob = p.out[dst];

  if (grp == 0) {
    const float s0 = dead0 ? __builtin_nanf("") : sc;
#pragma unroll
    for (int t = 0; t < 32; ++t) {
      const int mi = t >> 4, r = t & 15;
      st16_out(ob, off + mi * 32 * (GN * 4) + row_off(r), f32x4{acc[mi][0][r] * s0, acc[mi][1][r] * s0, acc[mi][2][r] * s0, acc[mi][3][r] * s0});
    }
    STAMP(5);
    return;
  }

  uint8_t* const scr = lds + LDS_PK + wave * 2048;
  OutBlk o0, o1;
  enc_out(acc[0], o0, scr, lane);
  enc_out(acc[1], o1, scr, lane);
  STAMP(31);
  if (stg == 3) recv_tile(myws, p.out[rank], dst, 0, si, wave, lane, call, tmo, sc, scr);

  link_acquire(relp, tmo, call, lane);
  const bool dead = dead0 || __builtin_amdgcn_readfirstlane(ld_agent(tmo)) == call;
  STAMP(3);
  {
    const int t = (rank * 4 + stg) * 32 + si * 4 + wave;
    uint8_t* const pws = p.ws[dst];
    uint8_t* const slot = pws + WS_OS + (size_t)t * 2 * OBLK;
    unsigned* const flag = (unsigned*)(pws + WS_DONE) + t;
    const float s1 = dead ? __builtin_nanf("") : sc;
    send_out<false>(o0, acc[0], o0.raw || dead, s1, slot, ob, off, lane, flag, 0, relp + 32);
    send_out<true>(o1, acc[1], o1.raw || dead, s1, slot + OBLK, ob, off + 32 * (GN * 4), lane, flag, 2 * call - 1, relp + 32);
    STAMP(4);
    wait_vm<0>();
    st1_sys(flag, 2 * call);
  }
  STAMP(5);
  if (stg != 3) recv_tile(myws, p.out[rank], dst, stg + 1, si, wave, lane, call, tmo, sc, scr);
  STAMP(6);
}

extern "C" void kernel_launch(int rank, int nranks, ncclComm_t comm, hipStream_t stream,
                              void* const* d_in, const int* in_sizes, int n_in,
                              void* d_out, int out_size, const int* in_dtypes, int out_dtype,
                              void* const* d_out_all, void* const* d_ws_all, size_t ws_bytes) {
  (void)comm; (void)in_sizes; (void)in_dtypes; (void)out_dtype; (void)d_out;
  if (nranks != NR || (unsigned)rank >= (unsigned)NR || n_in != 4 || ws_bytes < WS_NEED || out_size != 512 * GN) {
    fprintf(stderr, "kernel_launch: unexpected problem shape\n");
    return;
  }
  Params p;
  p.x = (const uint8_t*)d_in[0];
  p.w = (const uint8_t*)d_in[1];
  p.sx = (const float*)d_in[2];
  p.sw = (const float*)d_in[3];
  for (int i = 0; i < NR; ++i) {
    p.ws[i] = (uint8_t*)d_ws_all[i];
    p.out[i] = (float*)d_out_all[i];
  }
  p.rank = rank;
  p.pad = 0;
  hipLaunchKernelGGL(gemm_rs_fused, dim3(NWG), dim3(256), 0, stream, p);
  HIP_CHECK(hipGetLastError());
}
```

```cpp
#include <hip/hip_runtime.h>
#include <rccl/rccl.h>
#include <cstdio>
#include <cstdint>

#define HIP_CHECK(expr) do { hipError_t _e = (expr); if (_e != hipSuccess) { \
    fprintf(stderr, "HIP error %s at %s:%d\n", hipGetErrorString(_e), __FILE__, __LINE__); \
    return; } } while (0)

typedef unsigned u32x4 __attribute__((ext_vector_type(4)));
typedef int i32x4 __attribute__((ext_vector_type(4)));
typedef int i32x8 __attribute__((ext_vector_type(8)));
typedef float f32x4 __attribute__((ext_vector_type(4)));
typedef float f32x16 __attribute__((ext_vector_type(16)));

namespace {

constexpr int NR = 8;
constexpr int GN = 16384;
constexpr int KS = 512;
constexpr int NB = 2048;
constexpr int NWG = 256;
constexpr int NROUND = 24;
constexpr int KSTEPS = 64;
constexpr int STAGE = 24576;
constexpr int NSTAGE = 6;
constexpr int LDS_CTRL = STAGE * NSTAGE;
constexpr int LDS_PK = LDS_CTRL + 64;
constexpr int LDS_BYTES = LDS_PK + 4 * 2048;

constexpr size_t WS_CALL = 0;
constexpr size_t WS_TMO = 32768;
constexpr size_t WS_DONE = WS_TMO + 1024;
constexpr size_t WS_LINK = WS_DONE + 4096;
constexpr size_t WS_DX = WS_LINK + 4096;
constexpr size_t WS_DW = WS_DX + 4096;
constexpr size_t WS_DBG = 65536;
constexpr size_t WS_DATA = 1u << 20;
constexpr size_t XS_BYTES = 16u << 20;
constexpr size_t WT_BYTES = 8u << 20;
constexpr unsigned ITASK = 3328;
constexpr size_t IS_BYTES = 20u << 20;
constexpr size_t PAR_BYTES = XS_BYTES + WT_BYTES + IS_BYTES;
static_assert((size_t)8 * 768 * ITASK <= IS_BYTES, "");
constexpr size_t OBLK = 13 * 1024;
constexpr size_t WS_OS = WS_DATA + 2 * PAR_BYTES;
constexpr size_t WS_NEED = WS_OS + 8 * 256 * OBLK;

constexpr int LK_PTICK = 0, LK_OTICK = 32, LK_PDONE = 64, LK_OREL = 65;
constexpr int NTASK = 32 * NROUND;

constexpr unsigned long long DEADLINE_TICKS = 50000000ull;

#define OUT_D 2
#define REL_TASKS 576
#define OUT_REL 3
#define DEEP_D 4
#define DEEP_N 32
#define POLL_SLEEP 6
#define S2_GATE 0
#define G0_GATE 96
#define PUSH_W 4
#define PAIR23 0
#define NCRED 64

struct Params {
  const uint8_t* x;
  const uint8_t* w;
  const float* sx;
  const float* sw;
  uint8_t* ws[NR];
  float* out[NR];
  int rank;
  int pad;
};

#define STAMP(slot) do {} while (0)
#define CSTAMP(slot) do {} while (0)

__device__ __forceinline__ unsigned ld_sys(const void* ptr) {
  return __hip_atomic_load((const unsigned*)ptr, __ATOMIC_RELAXED, __HIP_MEMORY_SCOPE_SYSTEM);
}
__device__ __forceinline__ unsigned ld_agent(const void* ptr) {
  return __hip_atomic_load((const unsigned*)ptr, __ATOMIC_RELAXED, __HIP_MEMORY_SCOPE_AGENT);
}
__device__ __forceinline__ void st_sys(void* ptr, unsigned v) {
  __hip_atomic_store((unsigned*)ptr, v, __ATOMIC_RELAXED, __HIP_MEMORY_SCOPE_SYSTEM);
}
__device__ __forceinline__ void st_agent(void* ptr, unsigned v) {
  __hip_atomic_store((unsigned*)ptr, v, __ATOMIC_RELAXED, __HIP_MEMORY_SCOPE_AGENT);
}

template <class F>
__device__ __forceinline__ bool spin(F ready, unsigned* tmo, unsigned call) {
  const unsigned long long t0 = __builtin_amdgcn_s_memrealtime();
  for (unsigned n = 1;; ++n) {
    if (ready()) return true;
    if ((n & 3) == 0 && (__builtin_amdgcn_readfirstlane(ld_agent(tmo)) == call || __builtin_amdgcn_s_memrealtime() - t0 > DEADLINE_TICKS)) {
      st_agent(tmo, call);
      return false;
    }
    __builtin_amdgcn_s_sleep(POLL_SLEEP);
  }
}

__device__ __forceinline__ unsigned link_ticket(unsigned* lk, int lane) {
  unsigned t = 0;
  if (lane == 0) t = __hip_atomic_fetch_add(lk + LK_OTICK, 1u, __ATOMIC_RELAXED, __HIP_MEMORY_SCOPE_AGENT);
  return __builtin_amdgcn_readfirstlane(t);
}
__device__ __forceinline__ bool link_free(unsigned* lk, unsigned t, unsigned call) {
  const unsigned long long v = __hip_atomic_load((const unsigned long long*)(lk + LK_PDONE), __ATOMIC_RELAXED, __HIP_MEMORY_SCOPE_AGENT);
  return __builtin_amdgcn_readfirstlane((unsigned)v) >= (call - 1) * NTASK + REL_TASKS &&
         __builtin_amdgcn_readfirstlane((unsigned)(v >> 32)) + NCRED > t;
}

__device__ __forceinline__ void st16_sys(void* base, unsigned off, u32x4 v) {
  asm volatile("global_store_dwordx4 %0, %1, %2 sc0 sc1\n\ts_nop 1" :: "v"(off), "v"(v), "s"(base) : "memory");
}

__device__ __forceinline__ void st1_agent(void* base, unsigned v) {
  asm volatile("s_mov_b64 exec, 1\n\tglobal_store_dword %0, %1, %2 sc1\n\ts_mov_b64 exec, -1"
               :: "v"(0u), "v"(v), "s"(base) : "memory");
}

__device__ __forceinline__ void add1_agent(void* base, unsigned v) {
  asm volatile("s_mov_b64 exec, 1\n\tglobal_atomic_add %0, %1, %2\n\ts_mov_b64 exec, -1"
               :: "v"(0u), "v"(v), "s"(base) : "memory");
}
__device__ __forceinline__ void st1_sys(void* base, unsigned v) {
  asm volatile("s_mov_b64 exec, 1\n\tglobal_store_dword %0, %1, %2 sc0 sc1\n\ts_mov_b64 exec, -1"
               :: "v"(0u), "v"(v), "s"(base) : "memory");
}
__device__ __forceinline__ void st16_sys_m(void* base, unsigned off, u32x4 v, unsigned long long mask) {
  asm volatile("s_mov_b64 exec, %3\n\tglobal_store_dwordx4 %0, %1, %2 sc0 sc1\n\ts_mov_b64 exec, -1\n\ts_nop 0"
               :: "v"(off), "v"(v), "s"(base), "s"(mask) : "memory");
}

template <int N> __device__ __forceinline__ void wait_vm() {
  asm volatile("s_waitcnt vmcnt(%0)" :: "n"(N) : "memory");
}

__device__ __forceinline__ unsigned wave_max_u32(unsigned v) {
  v = max(v, (unsigned)__builtin_amdgcn_update_dpp(0, (int)v, 0x111, 0xf, 0xf, false));
  v = max(v, (unsigned)__builtin_amdgcn_update_dpp(0, (int)v, 0x112, 0xf, 0xf, false));
  v = max(v, (unsigned)__builtin_amdgcn_update_dpp(0, (int)v, 0x114, 0xf, 0xf, false));
  v = max(v, (unsigned)__builtin_amdgcn_update_dpp(0, (int)v, 0x118, 0xf, 0xf, false));
  return max(max((unsigned)__builtin_amdgcn_readlane((int)v, 15), (unsigned)__builtin_amdgcn_readlane((int)v, 31)),
             max((unsigned)__builtin_amdgcn_readlane((int)v, 47), (unsigned)__builtin_amdgcn_readlane((int)v, 63)));
}

__device__ __forceinline__ unsigned irec_slot(unsigned idx) { return idx + 3 + min((__umul24(idx, 37u) + 74u) >> 8, 3u); }

__device__ __forceinline__ unsigned wave_sum_u32(unsigned v) {
  v += (unsigned)__builtin_amdgcn_update_dpp(0, (int)v, 0x111, 0xf, 0xf, false);
  v += (unsigned)__builtin_amdgcn_update_dpp(0, (int)v, 0x112, 0xf, 0xf, false);
  v += (unsigned)__builtin_amdgcn_update_dpp(0, (int)v, 0x114, 0xf, 0xf, false);
  v += (unsigned)__builtin_amdgcn_update_dpp(0, (int)v, 0x118, 0xf, 0xf, false);
  return (unsigned)(__builtin_amdgcn_readlane((int)v, 15) + __builtin_amdgcn_readlane((int)v, 31) +
                    __builtin_amdgcn_readlane((int)v, 47) + __builtin_amdgcn_readlane((int)v, 63));
}
__device__ __forceinline__ unsigned bfi(unsigned m, unsigned a, unsigned b) { return (a & m) | (b & ~m); }
constexpr unsigned HDR_RAW = 0x80000000u;

constexpr unsigned IREC_CAP = 122, IHDR_RAW = 0x8000u;

struct InTask {
  u32x4 d[4];
  unsigned long long mask;
  bool raw;
};

__device__ __forceinline__ void put_records2(unsigned q0, unsigned q1, unsigned tag, unsigned& cnt, uint8_t* scr, unsigned dump) {
  unsigned m = bfi(0x55555555u, q0 | q0 >> 1, q1 | q1 << 1);
  for (unsigned long long act = __ballot(m != 0); act != 0; act = __ballot(m != 0)) {
    const unsigned bit = (unsigned)__builtin_ctz(m | 0x80000000u), sh = bit & 30u;
    const unsigned idx = cnt + __builtin_amdgcn_mbcnt_hi((unsigned)(act >> 32), __builtin_amdgcn_mbcnt_lo((unsigned)act, 0u));
    const unsigned sl = 2 * irec_slot(idx), at = ((m != 0) & (idx < IREC_CAP)) ? sl : dump;
    *(unsigned short*)(scr + at) = (unsigned short)(tag | (bit & 1u) << 6 | sh << 1 | ((((bit & 1u) ? q1 : q0) >> sh) & 3u));
    m &= m - 1;
    cnt += __popcll(act);
  }
}

__device__ __forceinline__ void enc_in(const u32x4 (&v)[4], InTask& o, unsigned tag, uint8_t* scr, int lane) {
  unsigned sum = 0;
#pragma unroll
  for (int q = 0; q < 4; ++q) sum = __builtin_amdgcn_udot4(v[q][0] & 0x7f7f7f7fu, 0x01010101u, sum, false);
  const unsigned e0 = (unsigned)min(max((int)((wave_sum_u32(sum) + 10445u) >> 12) - 7, 0), 24);
  const unsigned add = 0x80808080u - e0 * 0x04040404u;
  unsigned pay[12], qq[4], cnt = 0;
#pragma unroll
  for (int q = 0; q < 4; ++q) {
    unsigned R[4], S[4];
#pragma unroll
    for (int k = 0; k < 4; ++k) {
      R[k] = (v[q][k] & 0x7f7f7f7fu) + add;
      S[k] = bfi(0x80808080u, v[q][k], R[k]);
    }
    pay[3 * q] = bfi(0x60606060u, S[3] << 5, S[0]);
    pay[3 * q + 1] = bfi(0x60606060u, S[3] << 3, S[1]);
    pay[3 * q + 2] = bfi(0x20202020u, S[3] << 1, bfi(0x40404040u, S[3] >> 1, S[2]));
    qq[q] = bfi(0xc0c0c0c0u, R[3] << 1, bfi(0x30303030u, R[2] >> 1, bfi(0x0c0c0c0cu, R[1] >> 3, (R[0] >> 5) & 0x03030303u)));
  }
  put_records2(qq[0], qq[1], (unsigned)(lane << 8), cnt, scr, 256 + 2 * lane);
  put_records2(qq[2], qq[3], (unsigned)(lane << 8 | 128), cnt, scr, 256 + 2 * lane);
  *(unsigned short*)(scr + (lane < 4 ? lane * 16 : lane < 6 ? lane * 2 - 6 : 256 + 2 * lane)) =
      (unsigned short)(lane < 4 ? tag : lane == 4 ? cnt : e0);
  __builtin_amdgcn_fence(__ATOMIC_ACQ_REL, "wavefront");
  const u32x4 rec = *(const u32x4*)(scr + (lane & 15) * 16);
  __builtin_amdgcn_fence(__ATOMIC_ACQ_REL, "wavefront");
  o.raw = cnt > IREC_CAP;
  if (o.raw) {
#pragma unroll
    for (int q = 0; q < 4; ++q) o.d[q] = v[q];
    o.mask = ~0ull;
  } else {
#pragma unroll
    for (int q = 0; q < 3; ++q) o.d[q] = u32x4{pay[4 * q], pay[4 * q + 1], pay[4 * q + 2], pay[4 * q + 3]};
    o.d[3] = rec;
    o.mask = (1ull << (((irec_slot(cnt) + 31) >> 5) * 4)) - 16;
  }
}

__device__ __forceinline__ void dec_in4(const uint8_t* slot, uint8_t* out, const unsigned (&off)[4], unsigned stride, uint8_t* scr, int lane) {
  u32x4 y[4][3];
  unsigned rec[4];
  asm volatile("global_load_dword %12, %16, %20 sc0 sc1\n\tglobal_load_dword %13, %16, %20 offset:3328 sc0 sc1\n\t"
               "global_load_dword %14, %17, %20 sc0 sc1\n\tglobal_load_dword %15, %17, %20 offset:3328 sc0 sc1\n\t"
               "global_load_dwordx4 %0, %18, %20 offset:-3072 sc0 sc1\n\tglobal_load_dwordx4 %1, %18, %20 offset:-2048 sc0 sc1\n\t"
               "global_load_dwordx4 %2, %18, %20 offset:-1024 sc0 sc1\n\t"
               "global_load_dwordx4 %3, %18, %20 offset:256 sc0 sc1\n\tglobal_load_dwordx4 %4, %18, %20 offset:1280 sc0 sc1\n\t"
               "global_load_dwordx4 %5, %18, %20 offset:2304 sc0 sc1\n\t"
               "global_load_dwordx4 %6, %19, %20 offset:-3072 sc0 sc1\n\tglobal_load_dwordx4 %7, %19, %20 offset:-2048 sc0 sc1\n\t"
               "global_load_dwordx4 %8, %19, %20 offset:-1024 sc0 sc1\n\t"
               "global_load_dwordx4 %9, %19, %20 offset:256 sc0 sc1\n\tglobal_load_dwordx4 %10, %19, %20 offset:1280 sc0 sc1\n\t"
               "global_load_dwordx4 %11, %19, %20 offset:2304 sc0 sc1\n\ts_waitcnt vmcnt(0)"
               : "=&v"(y[0][0]), "=&v"(y[0][1]), "=&v"(y[0][2]), "=&v"(y[1][0]), "=&v"(y[1][1]), "=&v"(y[1][2]),
                 "=&v"(y[2][0]), "=&v"(y[2][1]), "=&v"(y[2][2]), "=&v"(y[3][0]), "=&v"(y[3][1]), "=&v"(y[3][2]),
                 "=&v"(rec[0]), "=&v"(rec[1]), "=&v"(rec[2]), "=&v"(rec[3])
               : "v"(lane * 4), "v"(2 * ITASK + lane * 4), "v"(ITASK + lane * 16), "v"(3 * ITASK + lane * 16), "s"(slot) : "memory");
#pragma unroll
  for (int t = 0; t < 4; ++t) *(u32x4*)(scr + t * 1024 + lane * 16) = u32x4{0u, 0u, 0u, 0u};
  __builtin_amdgcn_fence(__ATOMIC_ACQ_REL, "wavefront");
  unsigned hdr[4];
#pragma unroll
  for (int t = 0; t < 4; ++t) {
    hdr[t] = (unsigned)__builtin_amdgcn_readlane((int)rec[t], 0) >> 16 | (unsigned)__builtin_amdgcn_readlane((int)rec[t], 1) << 16;
    const unsigned end = irec_slot(hdr[t] & 0x7fffu);
#pragma unroll
    for (int i = 0; i < 2; ++i) {
      const unsigned r16 = rec[t] >> (16 * i);
      const bool isrec = i == 0 ? (lane >= 16 || (lane & 3) != 0) && lane != 1 : lane != 0;
      const unsigned val = isrec && (unsigned)(lane * 2 + i) < end ? (r16 & 3u) << ((r16 >> 1) & 30u) : 0u;
      __hip_atomic_fetch_or((unsigned*)(scr + t * 1024 + ((r16 >> 6) & 255u) * 4), val, __ATOMIC_RELAXED, __HIP_MEMORY_SCOPE_WORKGROUP);
    }
  }
  __builtin_amdgcn_fence(__ATOMIC_ACQ_REL, "wavefront");
#pragma unroll
  for (int t = 0; t < 4; ++t) {
    if (hdr[t] & IHDR_RAW) continue;
    const u32x4 qq = *(const u32x4*)(scr + t * 1024 + lane * 16);
    const unsigned add = ((hdr[t] >> 16) & 31u) * 0x04040404u;
#pragma unroll
    for (int q = 0; q < 4; ++q) {
      const unsigned d0 = y[t][(3 * q) >> 2][(3 * q) & 3], d1 = y[t][(3 * q + 1) >> 2][(3 * q + 1) & 3], d2 = y[t][(3 * q + 2) >> 2][(3 * q + 2) & 3];
      const unsigned d3 = bfi(0x80808080u, d2 << 1, bfi(0x10101010u, d2 >> 1, bfi(0x0c0c0c0cu, d1 >> 3, d0 >> 5)));
      const unsigned X[4] = {bfi(0x60606060u, qq[q] << 5, d0), bfi(0x60606060u, qq[q] << 3, d1), bfi(0x60606060u, qq[q] << 1, d2),
                             bfi(0x60606060u, qq[q] >> 1, d3)};
      u32x4 r;
#pragma unroll
      for (int k = 0; k < 4; ++k) r[k] = bfi(0x80808080u, X[k], (X[k] & 0x7f7f7f7fu) + add);
      asm volatile("global_store_dwordx4 %0, %1, %2 sc1\n\ts_nop 1" :: "v"(off[t] + q * stride + lane * 16), "v"(r), "s"(out) : "memory");
    }
  }
  __builtin_amdgcn_fence(__ATOMIC_ACQ_REL, "wavefront");
}

__device__ __forceinline__ bool wait_tasks(const uint8_t* slot, unsigned* tmo, unsigned call, int lane) {
  const uint8_t* q = slot + ((lane >> 2) & 3) * ITASK + (lane & 3) * 16;
  return spin([&] { return __all((ld_sys(q) & 0xffffu) == (call & 0xffffu)) != 0; }, tmo, call);
}

struct PushCtx {
  const uint8_t* x;
  const uint8_t* w;
  uint8_t* dst;
  unsigned* lk;
  uint8_t* scr;
  unsigned xl, wl;
  unsigned par, tick0, tag;
  int rank;
  bool local;
};

__device__ __forceinline__ bool round_is_w(int r) { return r == 4 || r == 5 || (r >= 10 && r < 16); }
__device__ __forceinline__ int round_w(int r) { return r < 6 ? r - 4 : r - 8; }
__device__ __forceinline__ int round_x(int r) { return r < 4 ? r : r < 10 ? r - 2 : r - 8; }
__device__ __forceinline__ bool task_is_w(int t) { return round_is_w(t >> 5); }
__device__ __forceinline__ int tile_q(int stg, int si) { return stg < 2 ? si & 1 : stg; }
__device__ __forceinline__ int tile_cb(int stg, int si) { return stg < 2 ? stg * 4 + (si >> 1) : si; }

__device__ __forceinline__ void ld_x(const PushCtx& c, int t, u32x4 (&v)[4]) {
  t = min(t, NTASK - 1);
  const int xi = round_x(t >> 5), w = t & 31;
  const unsigned o = c.xl + (unsigned)((((xi & 3) * 2 + (w >> 4)) * 512 + (xi >> 2) * 128 + (w & 1) * 64) * KS + ((w & 15) >> 1) * 64);
  asm volatile("global_load_dwordx4 %0, %4, %6\n\tglobal_load_dwordx4 %1, %4, %6 offset:16\n\t"
               "global_load_dwordx4 %2, %5, %6\n\tglobal_load_dwordx4 %3, %5, %6 offset:16"
               : "=&v"(v[0]), "=&v"(v[1]), "=&v"(v[2]), "=&v"(v[3]) : "v"(o), "v"(o + 32 * KS), "s"(c.x) : "memory");
}

__device__ __forceinline__ void pack_task(const PushCtx& c, const u32x4 (&v)[4], InTask& o, int lane, bool plain) {
  if (plain) {
#pragma unroll
    for (int q = 0; q < 4; ++q) o.d[q] = v[q];
    o.mask = ~0ull;
    o.raw = true;
  } else {
    enc_in(v, o, c.tag, c.scr, lane);
  }
}

struct PushState {
  int tp, tc, tn, tl;
  u32x4 line;
  InTask o;
  u32x4 nx[4];
};

template <bool LDW>
__device__ __forceinline__ void push_step(const Params& p, const PushCtx& c, PushState& s, int lane) {
  unsigned w[16], tk = 0;
  u32x4 x[4];
  const bool draw = s.tl < NTASK;
  if (draw)
    asm volatile("s_mov_b64 exec, 1\n\tglobal_atomic_add %0, %1, %2, %3 sc0\n\ts_mov_b64 exec, -1"
                 : "+v"(tk) : "v"(0u), "v"(1u), "s"(c.lk + LK_PTICK) : "memory");
  if constexpr (LDW) {
    const unsigned ow = c.wl + (unsigned)((((s.tl & 31) >> 2) * 64 + (s.tl & 1) * 16) * GN + ((s.tl >> 1) & 1) * 128 + round_w(s.tl >> 5) * 256);
#pragma unroll
    for (int i = 0; i < 16; ++i)
      asm volatile("global_load_dword %0, %1, %2" : "=v"(w[i]) : "v"(ow), "s"(c.w + (size_t)i * GN) : "memory");
  } else {
    ld_x(c, s.tl, x);
  }
  const unsigned is0 = c.par + (unsigned)(XS_BYTES + WT_BYTES) + (unsigned)(c.rank * NTASK) * ITASK + lane * 16;
  if (s.tp >= 0) {
    st16_sys_m(c.dst, is0 + (unsigned)s.tp * ITASK, s.line, 15ull);
    add1_agent(c.lk + LK_PDONE, 1);
  }
  if (s.tc < NTASK) {
    const int t = s.tc, wv = t & 31, xi = round_x(t >> 5);
    const bool wr = task_is_w(t);
    const unsigned slot = is0 + (unsigned)t * ITASK;
    const unsigned plain = c.par + lane * 16 +
        (wr ? (unsigned)XS_BYTES + (unsigned)(round_w(t >> 5) * (1 << 20) + (c.rank * 8 + (wv >> 2)) * 16384 + ((wv >> 1) & 1) * 8192 + (wv & 1) * 1024)
            : (unsigned)(((xi >> 2) * 8 + (xi & 3) * 2 + (wv >> 4)) * (512 << 10) + c.rank * 65536 + (wv & 15) * 4096));
    const unsigned o0 = s.o.raw ? plain : slot + 256, st = s.o.raw && wr ? 2048 : 1024;
    st16_sys(c.dst, o0, s.o.d[0]);
    st16_sys(c.dst, o0 + st, s.o.d[1]);
    st16_sys(c.dst, o0 + 2 * st, s.o.d[2]);
    if (s.o.mask != 0) st16_sys_m(c.dst, s.o.raw ? o0 + 3 * st : slot, s.o.d[3], s.o.mask);
    s.line = s.o.raw ? u32x4{c.tag | IHDR_RAW << 16, 0u, 0u, 0u} : s.o.d[3];
  }
  if (s.tn < NTASK) pack_task(c, s.nx, s.o, lane, c.local);
  if constexpr (LDW) {
    asm volatile("s_waitcnt vmcnt(0)" : "+v"(tk), "+v"(w[0]), "+v"(w[1]), "+v"(w[2]), "+v"(w[3]), "+v"(w[4]), "+v"(w[5]), "+v"(w[6]), "+v"(w[7]),
                 "+v"(w[8]), "+v"(w[9]), "+v"(w[10]), "+v"(w[11]), "+v"(w[12]), "+v"(w[13]), "+v"(w[14]), "+v"(w[15]) :: "memory");
#pragma unroll
    for (int q = 0; q < 4; ++q) {
      const unsigned d0 = w[4 * q], d1 = w[4 * q + 1], d2 = w[4 * q + 2], d3 = w[4 * q + 3];
      const unsigned t0 = __builtin_amdgcn_perm(d1, d0, 0x05010400u), t1 = __builtin_amdgcn_perm(d1, d0, 0x07030602u);
      const unsigned t2 = __builtin_amdgcn_perm(d3, d2, 0x05010400u), t3 = __builtin_amdgcn_perm(d3, d2, 0x07030602u);
      s.nx[0][q] = __builtin_amdgcn_perm(t2, t0, 0x05040100u);
      s.nx[1][q] = __builtin_amdgcn_perm(t2, t0, 0x07060302u);
      s.nx[2][q] = __builtin_amdgcn_perm(t3, t1, 0x05040100u);
      s.nx[3][q] = __builtin_amdgcn_perm(t3, t1, 0x07060302u);
    }
  } else {
    asm volatile("s_waitcnt vmcnt(0)" : "+v"(tk), "+v"(x[0]), "+v"(x[1]), "+v"(x[2]), "+v"(x[3]) :: "memory");
#pragma unroll
    for (int q = 0; q < 4; ++q) s.nx[q] = x[q];
  }
  if (s.tc < NTASK) STAMP(min(8 + (s.tc >> 5), 28));
  s.tp = s.tc; s.tc = s.tn; s.tn = s.tl;
  s.tl = draw ? (int)(__builtin_amdgcn_readfirstlane(tk) - c.tick0) : NTASK;
}

#define GLDS_SC 2
#define GLDS_BITS " sc0 sc1"
#define B_NT 0
template <bool NT = false>
__device__ __forceinline__ void glds2(const uint8_t* gbase, unsigned voff, unsigned lds) {
  unsigned keep;
  if constexpr (NT && B_NT) {
    asm volatile("s_mov_b32 %0, m0\n\ts_mov_b32 m0, %3\n\ts_nop 0\n\t"
                 "global_load_lds_dwordx4 %1, %2" GLDS_BITS " nt\n\t"
                 "global_load_lds_dwordx4 %1, %2 offset:1024" GLDS_BITS " nt\n\t"
                 "s_mov_b32 m0, %0"
                 : "=&s"(keep) : "v"(voff), "s"(gbase), "s"(lds));
    return;
  }
  asm volatile("s_mov_b32 %0, m0\n\ts_mov_b32 m0, %3\n\ts_nop 0\n\t"
               "global_load_lds_dwordx4 %1, %2" GLDS_BITS "\n\t"
               "global_load_lds_dwordx4 %1, %2 offset:1024" GLDS_BITS "\n\t"
               "s_mov_b32 m0, %0"
               : "=&s"(keep) : "v"(voff), "s"(gbase), "s"(lds));
}

struct Frags { i32x8 a[2]; i32x8 b[4]; };

__device__ __forceinline__ i32x8 ld_frag(const uint8_t* q) {
  const i32x4 lo = *(const i32x4*)q, hi = *(const i32x4*)(q + 1024);
  return __builtin_shufflevector(lo, hi, 0, 1, 2, 3, 4, 5, 6, 7);
}

#define ABL_NODMA 0
#define ABL_NORD 0
#define MFMA(mi, j) acc[mi][j] = __builtin_amdgcn_mfma_scale_f32_32x32x64_f8f6f4(cur.a[mi], cur.b[j], acc[mi][j], 1, 1, 0, 0, 0, 0)
#define SB() __builtin_amdgcn_sched_barrier(0)

struct Ring {
  const uint8_t* ga;
  const uint8_t* gb;
  unsigned va, vb;
  unsigned wr;
  unsigned rd;
  unsigned wr_end, wb;
  const uint8_t* qa;
  const uint8_t* qb;
  const uint8_t* la;
  const uint8_t* lb;
};

template <int VM, bool DMA, bool RD>
__device__ __forceinline__ void kstep(f32x16 (&acc)[2][4], const Frags& cur, Frags& nxt, Ring& g) {
  MFMA(0, 0); SB();
  asm volatile("s_waitcnt vmcnt(%0) lgkmcnt(0)\n\ts_barrier" :: "n"(VM) : "memory");
  SB();
  MFMA(0, 1); SB();
  if constexpr (RD && !ABL_NORD) { nxt.a[0] = ld_frag(g.qa); nxt.b[0] = ld_frag(g.qb); }
  SB();
  MFMA(0, 2); SB();
  if constexpr (DMA && !ABL_NODMA) glds2(g.ga, g.va, g.wr);
  SB();
  MFMA(0, 3); SB();
  if constexpr (RD && !ABL_NORD) { nxt.b[1] = ld_frag(g.qb + 2048); nxt.b[2] = ld_frag(g.qb + 4096); }
  SB();
  MFMA(1, 0); SB();
  if constexpr (DMA && !ABL_NODMA) glds2<true>(g.gb, g.vb, g.wr + g.wb);
  SB();
  MFMA(1, 1); SB();
  if constexpr (RD && !ABL_NORD) { nxt.b[3] = ld_frag(g.qb + 6144); nxt.a[1] = ld_frag(g.qa + 2048); }
  g.rd = g.rd == (NSTAGE - 1) * STAGE ? 0 : g.rd + STAGE;
  g.qa = g.la + g.rd; g.qb = g.lb + g.rd;
  SB();
  MFMA(1, 2); SB();
  if constexpr (DMA && !ABL_NODMA) glds2<true>(g.gb + 2048, g.vb, g.wr + g.wb + 2048);
  SB();
  MFMA(1, 3); SB();
  if constexpr (DMA) {
    g.ga += 8192; g.gb += 16384;
    g.wr += STAGE; g.wr = g.wr == g.wr_end ? g.wr - NSTAGE * STAGE : g.wr;
  }
  SB();
}

__device__ __forceinline__ void gemm_tile(f32x16 (&acc)[2][4], const uint8_t* ga, const uint8_t* gb, uint8_t* lds, int wave, int lane) {
  const unsigned ldsb = (unsigned)(uintptr_t)lds;
  Ring g;
  g.ga = ga; g.gb = gb;
  g.va = lane * 16 + wave * 2048; g.vb = lane * 16 + wave * 4096;
  g.wr = ldsb + wave * 2048; g.wb = 8192 + wave * 2048; g.wr_end = g.wr + NSTAGE * STAGE;
#pragma unroll
  for (int s = 0; s < NSTAGE; ++s) {
    glds2(g.ga, g.va, g.wr + s * STAGE);
    glds2<true>(g.gb, g.vb, g.wr + s * STAGE + g.wb);
    glds2<true>(g.gb + 2048, g.vb, g.wr + s * STAGE + g.wb + 2048);
    g.ga += 8192; g.gb += 16384;
  }
#pragma unroll
  for (int i = 0; i < 2; ++i)
#pragma unroll
    for (int j = 0; j < 4; ++j)
#pragma unroll
      for (int r = 0; r < 16; ++r) acc[i][j][r] = 0.f;
  g.la = lds + lane * 16 + (wave >> 1) * 4096; g.lb = lds + 8192 + lane * 16 + (wave & 1) * 8192;
  asm volatile("s_waitcnt vmcnt(30)\n\ts_barrier" ::: "memory");
  Frags f0, f1;
  f0.a[0] = ld_frag(g.la); f0.b[0] = ld_frag(g.lb); f0.b[1] = ld_frag(g.lb + 2048);
  f0.b[2] = ld_frag(g.lb + 4096); f0.b[3] = ld_frag(g.lb + 6144); f0.a[1] = ld_frag(g.la + 2048);
  g.rd = STAGE; g.qa = g.la + STAGE; g.qb = g.lb + STAGE;
#pragma unroll 1
  for (int s = 0; s < KSTEPS - NSTAGE; s += 2) {
    kstep<24, true, true>(acc, f0, f1, g);
    kstep<24, true, true>(acc, f1, f0, g);
  }
  kstep<24, false, true>(acc, f0, f1, g);
  kstep<18, false, true>(acc, f1, f0, g);
  kstep<12, false, true>(acc, f0, f1, g);
  kstep<6, false, true>(acc, f1, f0, g);
  kstep<0, false, true>(acc, f0, f1, g);
  kstep<0, false, false>(acc, f1, f0, g);
}

constexpr unsigned REC_CAP = 510, HDR_B = 0x40000000u;

__device__ __forceinline__ void put_records4(const unsigned (&q)[4], unsigned tag, unsigned& cnt, uint8_t* scr, unsigned dump) {
  unsigned m = 0;
#pragma unroll
  for (int k = 0; k < 4; ++k)
    m |= ((((q[k] & 0x77777777u) + 0x77777777u) | q[k]) & 0x88888888u) >> (3 - k);
  for (unsigned long long act = __ballot(m != 0); act != 0; act = __ballot(m != 0)) {
    const unsigned bit = (unsigned)__builtin_ctz(m | 0x80000000u), sh = bit & 28u;
    const unsigned m1 = (unsigned)__builtin_amdgcn_sbfe((int)bit, 0, 1), m2 = (unsigned)__builtin_amdgcn_sbfe((int)bit, 1, 1);
    const unsigned w = bfi(m2, bfi(m1, q[3], q[2]), bfi(m1, q[1], q[0]));
    const unsigned idx = cnt + __builtin_amdgcn_mbcnt_hi((unsigned)(act >> 32), __builtin_amdgcn_mbcnt_lo((unsigned)act, 0u));
    const unsigned at = (m != 0 && idx < REC_CAP) ? 4 + 2 * idx : dump;
    *(unsigned short*)(scr + at) = (unsigned short)(tag | (bit & 3u) << 7 | sh << 2 | ((w >> sh) & 15u));
    m &= m - 1;
    cnt += __popcll(act);
  }
}

#define PIN_ACC(a) asm volatile("" : "+a"(a[0]), "+a"(a[1]), "+a"(a[2]), "+a"(a[3]))

template <bool MODEB>
__device__ __forceinline__ bool enc_body(const float (&v)[4][16], float up, unsigned& cnt, uint8_t* blk, int lane) {
  unsigned bad = 0, low = 0;
#pragma unroll
  for (int h = 0; h < 2; ++h) {
    unsigned G[32];
#pragma unroll
    for (int j = 0; j < 4; ++j)
#pragma unroll
      for (int r = 0; r < 8; ++r) {
        const float x = v[j][8 * h + r] * up;
        unsigned g = (unsigned)(int)x;
        bad = __builtin_amdgcn_bitop3_b32(__builtin_bit_cast(unsigned, (float)(int)g), __builtin_bit_cast(unsigned, x), bad, 0xbe);
        low |= g;
        if (MODEB) g = g << 1 | (((g ^ g << 1) >> 27) & 16u);
        G[4 * r + j] = g;
      }
    unsigned q[4];
#pragma unroll
    for (int k = 0; k < 4; ++k) {
      const unsigned* g = G + 8 * k;
      const unsigned b0 = __builtin_amdgcn_perm(__builtin_amdgcn_perm(g[3], g[2], 0x0c0c0400u), __builtin_amdgcn_perm(g[1], g[0], 0x0c0c0400u), 0x05040100u);
      const unsigned b1 = __builtin_amdgcn_perm(__builtin_amdgcn_perm(g[7], g[6], 0x0c0c0400u), __builtin_amdgcn_perm(g[5], g[4], 0x0c0c0400u), 0x05040100u);
      q[k] = (b0 >> 4) | b1;
    }
#pragma unroll
    for (int j = 0; j < 2; ++j) {
      unsigned d[12];
#pragma unroll
      for (int g = 0; g < 4; ++g) {
        const unsigned* x = G + 16 * j + 4 * g;
        d[3 * g] = __builtin_amdgcn_perm(x[1], x[0], 0x05030201u);
        d[3 * g + 1] = __builtin_amdgcn_perm(x[2], x[1], 0x06050302u);
        d[3 * g + 2] = __builtin_amdgcn_perm(x[3], x[2], 0x07060503u);
      }
#pragma unroll
      for (int i = 0; i < 3; ++i)
        *(u32x4*)(blk + (1 + 6 * h + 3 * j + i) * 1024 + lane * 16) = u32x4{d[4 * i], d[4 * i + 1], d[4 * i + 2], d[4 * i + 3]};
    }
    put_records4(q, (unsigned)(lane << 10 | h << 9), cnt, blk, 13 * 1024 + 2 * lane);
  }
  return __ballot((bad | (low & 15u)) != 0) == 0 && cnt <= REC_CAP;
}

__device__ __forceinline__ unsigned enc_out(f32x16 (&a)[4], float sc, uint8_t* blk, int lane) {
  PIN_ACC(a);
  float v[4][16];
  float m = 0.f;
#pragma unroll
  for (int j = 0; j < 4; ++j)
#pragma unroll
    for (int r = 0; r < 16; r += 2) {
      v[j][r] = a[j][r]; v[j][r + 1] = a[j][r + 1];
      asm("v_max3_f32 %0, |%1|, |%2|, %0" : "+v"(m) : "v"(v[j][r]), "v"(v[j][r + 1]));
    }
  const int e = max((int)(wave_max_u32(__builtin_bit_cast(unsigned, m)) >> 23) - 157, -126);
  const float up = __builtin_bit_cast(float, (unsigned)(127 - min(e, 0)) << 23);
  int ntop = 0, nlow = 0;
#pragma unroll
  for (int r = 0; r < 8; ++r) {
    const unsigned g = (unsigned)(int)(v[r & 3][2 * r] * up);
    ntop += __popcll(__ballot((int)(g ^ (g << 1)) < 0));
    nlow += __popcll(__ballot((g & 128u) != 0));
  }
  unsigned cnt = 0;
  const bool modeb = ntop < nlow;
  const bool fits = e <= 0 && (modeb ? enc_body<true>(v, up, cnt, blk, lane) : enc_body<false>(v, up, cnt, blk, lane));
  if (!fits) {
    PIN_ACC(a);
#pragma unroll
    for (int r = 0; r < 16; ++r) *(f32x4*)(blk + r * 1024 + lane * 16) = f32x4{a[0][r] * sc, a[1][r] * sc, a[2][r] * sc, a[3][r] * sc};
    return 0;
  }
  *(unsigned*)(blk + (lane == 0 ? 0 : 13 * 1024 + 4 * lane)) = cnt | (unsigned)(e + 127) << 16 | (modeb ? HDR_B : 0u);
  return ((4 + 2 * cnt + 63) >> 6) * 4;
}

#define LOC_ST 1
__device__ __forceinline__ void st16_out(void* base, unsigned off, f32x4 v) {
  asm volatile("global_store_dwordx4 %0, %1, %2 sc1\n\ts_nop 1" :: "v"(off), "v"(v), "s"(base) : "memory");
}

__device__ __forceinline__ constexpr unsigned row_off(int r) { return (unsigned)((r & 3) + 8 * (r >> 2)) * (GN * 4); }

__device__ __forceinline__ void send_out(const uint8_t* blk, unsigned nl, bool dead, uint8_t* slot, float* ob, unsigned off,
                                         int lane, unsigned* rel, bool last, bool deep) {
  const bool raw = nl == 0 || dead;
  const u32x4 h = *(const u32x4*)(blk + lane * 16);
  st16_sys_m(slot, lane * 16, raw ? u32x4{HDR_RAW, 0u, 0u, 0u} : h, raw ? 15ull : nl >= 64 ? ~0ull : (1ull << nl) - 1);
  const int n = raw ? 16 : 12;
  void* const base = raw ? (void*)ob : (void*)slot;
  const uint8_t* const src = blk + (raw ? 0 : 1024) + lane * 16;
  const unsigned nan = dead ? 0x7fc00000u : 0u;
#pragma unroll 1
  for (int i = 0; i < n; ++i) {
    const u32x4 v = *(const u32x4*)(src + i * 1024) | nan;
    if (deep) wait_vm<DEEP_D - 1>(); else wait_vm<OUT_D - 1>();
    if (last && i == n - OUT_REL) add1_agent(rel, 1);
    st16_sys(base, raw ? off + row_off(i) : (unsigned)(1024 + i * 1024 + lane * 16), v);
  }
}

__device__ __forceinline__ void dec_out(const uint8_t* slot, float* ob, unsigned off, float sc, unsigned* tmo, unsigned call, uint8_t* scr, int lane) {
  u32x4 rec, y[12];
  const unsigned tmv = ld_agent(tmo);
  asm volatile("global_load_dwordx4 %0, %13, %15 offset:-4096 sc0 sc1\n\tglobal_load_dwordx4 %1, %13, %15 offset:-3072 sc0 sc1\n\t"
               "global_load_dwordx4 %2, %13, %15 offset:-2048 sc0 sc1\n\tglobal_load_dwordx4 %3, %13, %15 offset:-1024 sc0 sc1\n\t"
               "global_load_dwordx4 %4, %13, %15 sc0 sc1\n\tglobal_load_dwordx4 %5, %13, %15 offset:1024 sc0 sc1\n\t"
               "global_load_dwordx4 %6, %13, %15 offset:2048 sc0 sc1\n\tglobal_load_dwordx4 %7, %13, %15 offset:3072 sc0 sc1\n\t"
               "global_load_dwordx4 %8, %14, %15 offset:-4096 sc0 sc1\n\tglobal_load_dwordx4 %9, %14, %15 offset:-3072 sc0 sc1\n\t"
               "global_load_dwordx4 %10, %14, %15 offset:-2048 sc0 sc1\n\tglobal_load_dwordx4 %11, %14, %15 offset:-1024 sc0 sc1\n\t"
               "global_load_dwordx4 %12, %14, %15 sc0 sc1\n\ts_waitcnt vmcnt(0)"
               : "=&v"(rec), "=&v"(y[0]), "=&v"(y[1]), "=&v"(y[2]), "=&v"(y[3]), "=&v"(y[4]), "=&v"(y[5]), "=&v"(y[6]),
                 "=&v"(y[7]), "=&v"(y[8]), "=&v"(y[9]), "=&v"(y[10]), "=&v"(y[11])
               : "v"(4096 + lane * 16), "v"(12288 + lane * 16), "s"(slot) : "memory");
  const unsigned hdr = __builtin_amdgcn_readfirstlane(rec[0]);
  if (__builtin_amdgcn_readfirstlane(tmv) == call) {
    const float n = __builtin_nanf("");
#pragma unroll 1
    for (int r = 0; r < 16; ++r) st16_out(ob, off + row_off(r), f32x4{n, n, n, n});
    return;
  }
  if (hdr & HDR_RAW) return;
  const unsigned cnt = hdr & 0xffffu;
  const float down = __builtin_bit_cast(float, ((hdr >> 16) & 255u) << 23);
  *(u32x4*)(scr + lane * 32) = u32x4{0u, 0u, 0u, 0u};
  *(u32x4*)(scr + lane * 32 + 16) = u32x4{0u, 0u, 0u, 0u};
  __builtin_amdgcn_fence(__ATOMIC_ACQ_REL, "wavefront");
#pragma unroll
  for (int i = 0; i < 8; ++i) {
    const unsigned r16 = (rec[i >> 1] >> (16 * (i & 1))) & 0xffffu;
    const unsigned val = (unsigned)(lane * 8 + i - 2) < cnt ? (r16 & 15u) << ((r16 >> 2) & 28u) : 0u;
    __hip_atomic_fetch_or((unsigned*)(scr + (r16 >> 7) * 4), val, __ATOMIC_RELAXED, __HIP_MEMORY_SCOPE_WORKGROUP);
  }
  __builtin_amdgcn_fence(__ATOMIC_ACQ_REL, "wavefront");
  const u32x4 q0 = *(const u32x4*)(scr + lane * 32), q1 = *(const u32x4*)(scr + lane * 32 + 16);
  __builtin_amdgcn_fence(__ATOMIC_ACQ_REL, "wavefront");
  const bool modeb = (hdr & HDR_B) != 0;
#pragma unroll
  for (int g = 0; g < 16; ++g) {
    const unsigned d0 = y[(3 * g) >> 2][(3 * g) & 3], d1 = y[(3 * g + 1) >> 2][(3 * g + 1) & 3], d2 = y[(3 * g + 2) >> 2][(3 * g + 2) & 3];
    const unsigned qw = g < 8 ? q0[(g >> 1) & 3] : q1[(g >> 1) & 3];
    const unsigned q = ((g & 1) ? qw : qw << 4) & 0xf0f0f0f0u;
    unsigned G[4] = {__builtin_amdgcn_perm(d0, q, 0x06050400u), __builtin_amdgcn_perm(d1, d0, 0x0504030cu) | ((q >> 8) & 255u),
                     __builtin_amdgcn_perm(d2, d1, 0x0403020cu) | ((q >> 16) & 255u), __builtin_amdgcn_perm(d2, q, 0x07060503u)};
    f32x4 v;
#pragma unroll
    for (int j = 0; j < 4; ++j) {
      if (modeb) G[j] = ((unsigned)((int)G[j] >> 1) & ~8u) ^ (G[j] << 27);
      v[j] = ((float)(int)G[j] * down) * sc;
    }
    st16_out(ob, off + row_off(g), v);
  }
}

__device__ __forceinline__ void recv_tile(uint8_t* myws, float* out, int src, int stg, int si, int wave, int lane, unsigned call,
                                          unsigned* tmo, float sc, uint8_t* scr) {
  const int t = (src * 4 + stg) * 32 + si * 4 + wave;
  const int q = tile_q(stg, si), cb = tile_cb(stg, si);
  const unsigned off = (unsigned)(((q * 128 + (wave >> 1) * 64 + 4 * (lane >> 5)) * GN + src * NB + cb * 256 + (wave & 1) * 128 + 4 * (lane & 31)) * 4);
#pragma unroll 1
  for (int mi = 0; mi < 2; ++mi) {
    const uint8_t* const fl = myws + WS_DONE + (size_t)t * 4;
    spin([&] { return __builtin_amdgcn_readfirstlane(ld_sys(fl)) >= 2 * call - 1 + mi; }, tmo, call);
    dec_out(myws + WS_OS + (size_t)(t * 2 + mi) * OBLK, out, off + (unsigned)mi * 32 * (GN * 4), sc, tmo, call, scr, lane);
  }
}

}

__global__ void __launch_bounds__(256, 1) gemm_rs_fused(const Params p) {
  __shared__ __attribute__((aligned(16))) uint8_t lds[LDS_BYTES];
  const int b = blockIdx.x;
  const int lane = threadIdx.x & 63;
  const int wave = __builtin_amdgcn_readfirstlane(threadIdx.x >> 6);
  const int rank = p.rank;
  uint8_t* const myws = p.ws[rank];
  STAMP(0);

  const int grp = b & 7, dst = grp ^ rank;
  const int stg = 3 - (b >> 6), si = (b >> 3) & 7;
  const int qc = tile_q(stg, si), cb = tile_cb(stg, si);
  const int rt = qc * 8 + dst;
  unsigned* const callp = (unsigned*)(myws + WS_CALL + (size_t)b * 128);
  unsigned* const tmo = (unsigned*)(myws + WS_TMO);
  unsigned* const lk = (unsigned*)(myws + WS_LINK) + grp * 128;

  unsigned call;
  if (stg == 3 && wave < PUSH_W) {
    constexpr int NPW = 8 * PUSH_W;
    const int w = si * PUSH_W + wave;
    PushCtx c;
    c.x = p.x; c.w = p.w; c.dst = p.ws[dst]; c.lk = lk; c.rank = rank;
    c.xl = (unsigned)((lane & 31) * KS + (lane >> 5) * 32);
    c.wl = (unsigned)((lane >> 5) * 32 * GN + dst * NB + (lane & 31) * 4);
    c.scr = lds + LDS_PK + wave * 2048;
    c.local = grp == 0;
    PushState s;
    u32x4 x0[4];
    unsigned cv;
    asm volatile("global_load_dword %0, %1, %2 sc1" : "=v"(cv) : "v"(0u), "s"(callp) : "memory");
    ld_x(c, w, x0);
    ld_x(c, w + NPW, s.nx);
    asm volatile("s_waitcnt vmcnt(0)" : "+v"(cv), "+v"(x0[0]), "+v"(x0[1]), "+v"(x0[2]), "+v"(x0[3]),
                 "+v"(s.nx[0]), "+v"(s.nx[1]), "+v"(s.nx[2]), "+v"(s.nx[3]) :: "memory");
    call = __builtin_amdgcn_readfirstlane(cv) + 1;
    c.par = (unsigned)(WS_DATA + (call & 1) * PAR_BYTES);
    c.tick0 = (call - 1) * (NTASK - 2 * NPW) - 3 * NPW;
    c.tag = call & 0xffffu;
    s.tp = -1; s.tc = w; s.tn = w + NPW; s.tl = w + 2 * NPW;
    s.line = u32x4{0u, 0u, 0u, 0u};
    pack_task(c, x0, s.o, lane, true);
    STAMP(7);
#pragma unroll 1
    while (s.tp < NTASK) {
      if (task_is_w(s.tl)) push_step<true>(p, c, s, lane);
      else push_step<false>(p, c, s, lane);
    }
    STAMP(30);
  } else {
    call = __builtin_amdgcn_readfirstlane(ld_agent(callp)) + 1;
  }

  {
    uint8_t* const par = myws + WS_DATA + (size_t)(call & 1) * PAR_BYTES;
#pragma unroll 1
    for (int job = 0; job < 2; ++job) {
      const bool jw = (job == 0) == (qc == 1 && cb < 2);
      if (jw && stg >= 2) continue;
      const int src = jw ? dst : cb;
      const int round = jw ? (cb < 2 ? 4 + cb : 8 + cb) : (qc == 0 ? 0 : qc == 1 ? 6 : 8 + 4 * qc) + (dst >> 1);
      const int w0 = jw ? qc * 16 + wave * 4 : (dst & 1) * 16 + wave * 4;
      const uint8_t* const slot = par + XS_BYTES + WT_BYTES + (size_t)(src * NTASK + round * 32 + w0) * ITASK;
      if (!wait_tasks(slot, tmo, call, lane)) break;
      if (jw) STAMP(26);
      if (src != rank) {
        unsigned off[4];
#pragma unroll
        for (int t = 0; t < 4; ++t)
          off[t] = jw ? (unsigned)XS_BYTES + (unsigned)(cb * (1 << 20) + (src * 8 + (w0 >> 2)) * 16384 + (t >> 1) * 8192 + (t & 1) * 1024)
                      : (unsigned)(rt * (512 << 10) + src * 65536 + (wave * 4 + t) * 4096);
        dec_in4(slot, par, off, jw ? 2048 : 1024, lds + wave * 4096, lane);
        if (jw) STAMP(27);
        wait_vm<0>();
      }
      st1_agent(jw ? myws + WS_DW + (size_t)(cb * 64 + src * 8 + qc * 4 + wave) * 4 : myws + WS_DX + (size_t)(rt * 32 + src * 4 + wave) * 4, call);
    }
  }
  STAMP(29);

  unsigned* const ctrl = (unsigned*)(lds + LDS_CTRL);
  if (wave == 0) {
    const uint8_t* const fw = myws + WS_DW + (size_t)(cb * 64 + lane) * 4;
    const uint8_t* const fx = myws + WS_DX + (size_t)((lane < 32 || PAIR23 == 0 || qc != 2 ? rt : rt + 8) * 32 + (lane & 31)) * 4;
    const bool ok = spin([&] { const unsigned a = ld_agent(fw), c = ld_agent(fx); return __all(min(a, c) >= call) != 0; }, tmo, call);
    if (S2_GATE > 0 && ok && stg == 2 && grp != 0)
      spin([&] { return __builtin_amdgcn_readfirstlane(ld_agent(lk + LK_OTICK)) >= (call - 1) * 128 + S2_GATE; }, tmo, call);
    if (G0_GATE > 0 && ok && grp == 0)
      spin([&] { return __builtin_amdgcn_readfirstlane(ld_agent(lk + 128 + LK_OTICK)) >= (call - 1) * 128 + G0_GATE; }, tmo, call);
    if (GLDS_SC == 0) {
      __builtin_amdgcn_fence(__ATOMIC_ACQUIRE, "");
      asm volatile("s_waitcnt vmcnt(0)" ::: "memory");
    }
    if (lane == 0) ctrl[0] = !ok;
  }
  __syncthreads();
  const bool dead0 = __builtin_amdgcn_readfirstlane(ctrl[0]) != 0;
  STAMP(1); CSTAMP(1);

  if (wave == 0) st1_agent(callp, call);
  const int wm = wave >> 1, wn = wave & 1;
  const float sc = p.sx[0] * p.sw[0];
  const unsigned off = (unsigned)(((qc * 128 + wm * 64 + 4 * (lane >> 5)) * GN + rank * NB + cb * 256 + wn * 128 + 4 * (lane & 31)) * 4);
  float* const ob = p.out[dst];
  uint8_t* const blk = lds + wave * 32768;
  unsigned nl0 = 0, nl1 = 0, tick = 0;
  int sent = 0;
  const int tix = (rank * 4 + stg) * 32 + si * 4 + wave;
  uint8_t* const oslot = p.ws[dst] + WS_OS + (size_t)tix * 2 * OBLK;
  unsigned* const flag = (unsigned*)(p.ws[dst] + WS_DONE) + tix;

  if (!dead0) {
    f32x16 acc[2][4];
    const uint8_t* const par = myws + WS_DATA + (size_t)(call & 1) * PAR_BYTES;
    gemm_tile(acc, par + (size_t)rt * (512 << 10), par + XS_BYTES + (size_t)cb * (1 << 20), lds, wave, lane);
    STAMP(2); CSTAMP(2);
    if (grp == 0) {
      const float s0 = __builtin_amdgcn_readfirstlane(ld_agent(tmo)) == call ? __builtin_nanf("") : sc;
#pragma unroll
      for (int t = 0; t < 32; ++t) {
        const int mi = t >> 4, r = t & 15;
        st16_out(ob, off + mi * 32 * (GN * 4) + row_off(r), f32x4{acc[mi][0][r] * s0, acc[mi][1][r] * s0, acc[mi][2][r] * s0, acc[mi][3][r] * s0});
      }
      STAMP(5);
      return;
    }
    unsigned tk = 0;
    if (stg != 3 && lane == 0) tk = __hip_atomic_fetch_add(lk + LK_OTICK, 1u, __ATOMIC_RELAXED, __HIP_MEMORY_SCOPE_AGENT);
    nl0 = enc_out(acc[0], sc, blk, lane);
    if (stg != 3) {
      tick = __builtin_amdgcn_readfirstlane(tk);
      const unsigned tmv = ld_agent(tmo);
      if (link_free(lk, tick, call) && __builtin_amdgcn_readfirstlane(tmv) != call) {
        STAMP(3);
        __builtin_amdgcn_fence(__ATOMIC_ACQ_REL, "wavefront");
        send_out(blk, nl0, false, oslot, ob, off, lane, lk + LK_OREL, false, tick - (call - 1) * 128 < DEEP_N);
        sent = 1;
      }
    }
    nl1 = enc_out(acc[1], sc, blk + 16384, lane);
  } else if (grp == 0) {
    const float n = __builtin_nanf("");
#pragma unroll 1
    for (int t = 0; t < 32; ++t) st16_out(ob, off + (t >> 4) * 32 * (GN * 4) + row_off(t & 15), f32x4{n, n, n, n});
    return;
  }
  uint8_t* const scr = lds + LDS_PK + wave * 2048;
  __builtin_amdgcn_fence(__ATOMIC_ACQ_REL, "wavefront");
  STAMP(31); CSTAMP(31);
  if (stg == 3) recv_tile(myws, p.out[rank], dst, 0, si, wave, lane, call, tmo, sc, scr);

  if (stg == 3 || dead0) tick = link_ticket(lk, lane);
  if (sent) {
    wait_vm<0>();
    st1_sys(flag, 2 * call - 1);
  } else {
    spin([&] { return link_free(lk, tick, call); }, tmo, call);
    STAMP(3);
  }
  const bool dead = dead0 || __builtin_amdgcn_readfirstlane(ld_agent(tmo)) == call;
#pragma unroll 1
  for (int mi = sent; mi < 2; ++mi) {
    send_out(blk + mi * 16384, mi ? nl1 : nl0, dead, oslot + mi * OBLK, ob, off + mi * 32 * (GN * 4), lane, lk + LK_OREL, mi == 1,
             tick - (call - 1) * 128 < DEEP_N);
    if (mi) STAMP(4);
    wait_vm<0>();
    st1_sys(flag, 2 * call - 1 + mi);
  }
  STAMP(5);
  if (stg != 3) recv_tile(myws, p.out[rank], dst, stg + 1, si, wave, lane, call, tmo, sc, scr);
  STAMP(6);
}

extern "C" void kernel_launch(int rank, int nranks, ncclComm_t comm, hipStream_t stream,
                              void* const* d_in, const int* in_sizes, int n_in,
                              void* d_out, int out_size, const int* in_dtypes, int out_dtype,
                              void* const* d_out_all, void* const* d_ws_all, size_t ws_bytes) {
  (void)comm; (void)in_sizes; (void)in_dtypes; (void)out_dtype; (void)d_out;
  if (nranks != NR || (unsigned)rank >= (unsigned)NR || n_in != 4 || ws_bytes < WS_NEED || out_size != 512 * GN) {
    fprintf(stderr, "kernel_launch: unexpected problem shape\n");
    return;
  }
  Params p;
  p.x = (const uint8_t*)d_in[0];
  p.w = (const uint8_t*)d_in[1];
  p.sx = (const float*)d_in[2];
  p.sw = (const float*)d_in[3];
  for (int i = 0; i < NR; ++i) {
    p.ws[i] = (uint8_t*)d_ws_all[i];
    p.out[i] = (float*)d_out_all[i];
  }
  p.rank = rank;
  p.pad = 0;
  hipLaunchKernelGGL(gemm_rs_fused, dim3(NWG), dim3(256), 0, stream, p);
  HIP_CHECK(hipGetLastError());
}
```

```cpp
#include <hip/hip_runtime.h>
#include <rccl/rccl.h>
#include <cstdio>
#include <cstdint>

#define HIP_CHECK(expr) do { hipError_t _e = (expr); if (_e != hipSuccess) { \
    fprintf(stderr, "HIP error %s at %s:%d\n", hipGetErrorString(_e), __FILE__, __LINE__); \
    return; } } while (0)

typedef unsigned u32x4 __attribute__((ext_vector_type(4)));
typedef int i32x4 __attribute__((ext_vector_type(4)));
typedef int i32x8 __attribute__((ext_vector_type(8)));
typedef float f32x4 __attribute__((ext_vector_type(4)));
typedef float f32x16 __attribute__((ext_vector_type(16)));

#define SGPR_PAD "s_nop 4\n\t"

namespace {

constexpr int NR = 8;
constexpr int GN = 16384;
constexpr int KS = 512;
constexpr int NWG = 256;
constexpr int KSTEPS = 64;
constexpr int NSTAGE = 6;
constexpr int LDS_CTRL = NSTAGE * 26624;
constexpr int LDS_BYTES = LDS_CTRL + 64;
constexpr int LDS_PK = 9 * 16384;

constexpr int T_W0 = 228, T_OWN = T_W0 + 512, NTASK = T_OWN + 64;

constexpr size_t WS_CALL = 0;
constexpr size_t WS_TMO = 32768;
constexpr size_t WS_DONE = 896u << 10;
constexpr size_t WS_LINK = WS_TMO + 1024;
constexpr size_t WS_DX = WS_LINK + 4096;
constexpr size_t WS_DW = WS_DX + 8192;
constexpr size_t WS_DBG = 65536;
constexpr size_t WS_DATA = 1u << 20;
constexpr size_t XS_BYTES = 10u << 20;
constexpr size_t WT_BYTES = 16u << 20;
constexpr unsigned ITASK = 3328;
constexpr size_t IS_BYTES = 21u << 20;
constexpr size_t PAR_BYTES = XS_BYTES + WT_BYTES + IS_BYTES;
static_assert((size_t)8 * NTASK * ITASK <= IS_BYTES, "");
constexpr size_t OBLK = 13 * 1024;
constexpr size_t WS_OS = WS_DATA + 2 * PAR_BYTES;
constexpr size_t WS_NEED = WS_OS + 8 * 32 * 9 * OBLK;

constexpr int LK_PTICK = 0, LK_OTICK = 32, LK_PDONE = 64, LK_OREL = 65;

constexpr unsigned long long DEADLINE_TICKS = 50000000ull;

#define REL_TASKS 740
#define OUT_REL 3
#define LD_FIRST 1
#define OUT_D 2
#define OUT_D3 3
#define DEEP_D 4
#define DEEP_N 32
#define FLAG_STRIDE 128
#define SPREAD 1
#define RX_SLEEP 16
#define POLL_SLEEP 6
#define NCRED 64

struct Params {
  const uint8_t* x;
  const uint8_t* w;
  const float* sx;
  const float* sw;
  uint8_t* ws[NR];
  float* out[NR];
  int rank;
  int pad;
};

#define STAMP(slot) do {} while (0)
#define CSTAMP(slot) do {} while (0)

__device__ __forceinline__ unsigned ld_sys(const void* ptr) {
  return __hip_atomic_load((const unsigned*)ptr, __ATOMIC_RELAXED, __HIP_MEMORY_SCOPE_SYSTEM);
}
__device__ __forceinline__ unsigned ld_agent(const void* ptr) {
  return __hip_atomic_load((const unsigned*)ptr, __ATOMIC_RELAXED, __HIP_MEMORY_SCOPE_AGENT);
}
__device__ __forceinline__ void st_sys(void* ptr, unsigned v) {
  __hip_atomic_store((unsigned*)ptr, v, __ATOMIC_RELAXED, __HIP_MEMORY_SCOPE_SYSTEM);
}
__device__ __forceinline__ void st_agent(void* ptr, unsigned v) {
  __hip_atomic_store((unsigned*)ptr, v, __ATOMIC_RELAXED, __HIP_MEMORY_SCOPE_AGENT);
}

template <int SLEEP = POLL_SLEEP, class F>
__device__ __forceinline__ bool spin(F ready, unsigned* tmo, unsigned call) {
  const unsigned long long t0 = __builtin_amdgcn_s_memrealtime();
  for (unsigned n = 1;; ++n) {
    if (ready()) return true;
    if ((n & 3) == 0 && (__builtin_amdgcn_readfirstlane(ld_agent(tmo)) == call || __builtin_amdgcn_s_memrealtime() - t0 > DEADLINE_TICKS)) {
      st_agent(tmo, call);
      return false;
    }
    __builtin_amdgcn_s_sleep(SLEEP);
  }
}

__device__ __forceinline__ unsigned link_ticket(unsigned* lk, int lane) {
  unsigned t = 0;
  if (lane == 0) t = __hip_atomic_fetch_add(lk + LK_OTICK, 1u, __ATOMIC_RELAXED, __HIP_MEMORY_SCOPE_AGENT);
  return __builtin_amdgcn_readfirstlane(t);
}
__device__ __forceinline__ bool link_free(unsigned* lk, unsigned t, unsigned call) {
  const unsigned long long v = __hip_atomic_load((const unsigned long long*)(lk + LK_PDONE), __ATOMIC_RELAXED, __HIP_MEMORY_SCOPE_AGENT);
  return __builtin_amdgcn_readfirstlane((unsigned)v) >= (call - 1) * NTASK + REL_TASKS &&
         __builtin_amdgcn_readfirstlane((unsigned)(v >> 32)) + NCRED > t;
}

__device__ __forceinline__ int link_depth(unsigned t, int nb) { return t < DEEP_N ? DEEP_D : nb == 3 ? OUT_D3 : OUT_D; }

__device__ __forceinline__ void st16_sys(void* base, unsigned off, u32x4 v) {
  asm volatile(SGPR_PAD "global_store_dwordx4 %0, %1, %2 sc0 sc1\n\ts_nop 1" :: "v"(off), "v"(v), "s"(base) : "memory");
}

__device__ __forceinline__ void st1_agent(void* base, unsigned v) {
  asm volatile(SGPR_PAD "s_mov_b64 exec, 1\n\tglobal_store_dword %0, %1, %2 sc1\n\ts_mov_b64 exec, -1"
               :: "v"(0u), "v"(v), "s"(base) : "memory");
}

__device__ __forceinline__ void add1_agent(void* base, unsigned v) {
  asm volatile(SGPR_PAD "s_mov_b64 exec, 1\n\tglobal_atomic_add %0, %1, %2\n\ts_mov_b64 exec, -1"
               :: "v"(0u), "v"(v), "s"(base) : "memory");
}
__device__ __forceinline__ void st1_sys(void* base, unsigned v) {
  asm volatile(SGPR_PAD "s_mov_b64 exec, 1\n\tglobal_store_dword %0, %1, %2 sc0 sc1\n\ts_mov_b64 exec, -1"
               :: "v"(0u), "v"(v), "s"(base) : "memory");
}
__device__ __forceinline__ void st16_sys_m(void* base, unsigned off, u32x4 v, unsigned long long mask) {
  asm volatile(SGPR_PAD "s_mov_b64 exec, %3\n\tglobal_store_dwordx4 %0, %1, %2 sc0 sc1\n\ts_mov_b64 exec, -1\n\ts_nop 0"
               :: "v"(off), "v"(v), "s"(base), "s"(mask) : "memory");
}

template <int N> __device__ __forceinline__ void wait_vm() {
  asm volatile("s_waitcnt vmcnt(%0)" :: "n"(N) : "memory");
}

__device__ __forceinline__ unsigned wave_max_u32(unsigned v) {
  v = max(v, (unsigned)__builtin_amdgcn_update_dpp(0, (int)v, 0x111, 0xf, 0xf, false));
  v = max(v, (unsigned)__builtin_amdgcn_update_dpp(0, (int)v, 0x112, 0xf, 0xf, false));
  v = max(v, (unsigned)__builtin_amdgcn_update_dpp(0, (int)v, 0x114, 0xf, 0xf, false));
  v = max(v, (unsigned)__builtin_amdgcn_update_dpp(0, (int)v, 0x118, 0xf, 0xf, false));
  return max(max((unsigned)__builtin_amdgcn_readlane((int)v, 15), (unsigned)__builtin_amdgcn_readlane((int)v, 31)),
             max((unsigned)__builtin_amdgcn_readlane((int)v, 47), (unsigned)__builtin_amdgcn_readlane((int)v, 63)));
}

__device__ __forceinline__ unsigned irec_slot(unsigned idx) { return idx + 3 + min((__umul24(idx, 37u) + 74u) >> 8, 3u); }

__device__ __forceinline__ unsigned wave_sum_u32(unsigned v) {
  v += (unsigned)__builtin_amdgcn_update_dpp(0, (int)v, 0x111, 0xf, 0xf, false);
  v += (unsigned)__builtin_amdgcn_update_dpp(0, (int)v, 0x112, 0xf, 0xf, false);
  v += (unsigned)__builtin_amdgcn_update_dpp(0, (int)v, 0x114, 0xf, 0xf, false);
  v += (unsigned)__builtin_amdgcn_update_dpp(0, (int)v, 0x118, 0xf, 0xf, false);
  return (unsigned)(__builtin_amdgcn_readlane((int)v, 15) + __builtin_amdgcn_readlane((int)v, 31) +
                    __builtin_amdgcn_readlane((int)v, 47) + __builtin_amdgcn_readlane((int)v, 63));
}
__device__ __forceinline__ unsigned bfi(unsigned m, unsigned a, unsigned b) { return (a & m) | (b & ~m); }
constexpr unsigned HDR_RAW = 0x80000000u;

constexpr unsigned IREC_CAP = 122, IHDR_RAW = 0x8000u;

struct InTask {
  u32x4 d[4];
  unsigned long long mask;
  bool raw;
};

__device__ __forceinline__ void put_records2(unsigned q0, unsigned q1, unsigned tag, unsigned& cnt, uint8_t* scr, unsigned dump) {
  unsigned m = bfi(0x55555555u, q0 | q0 >> 1, q1 | q1 << 1);
  for (unsigned long long act = __ballot(m != 0); act != 0; act = __ballot(m != 0)) {
    const unsigned bit = (unsigned)__builtin_ctz(m | 0x80000000u), sh = bit & 30u;
    const unsigned idx = cnt + __builtin_amdgcn_mbcnt_hi((unsigned)(act >> 32), __builtin_amdgcn_mbcnt_lo((unsigned)act, 0u));
    const unsigned sl = 2 * irec_slot(idx), at = ((m != 0) & (idx < IREC_CAP)) ? sl : dump;
    *(unsigned short*)(scr + at) = (unsigned short)(tag | (bit & 1u) << 6 | sh << 1 | ((((bit & 1u) ? q1 : q0) >> sh) & 3u));
    m &= m - 1;
    cnt += __popcll(act);
  }
}

__device__ __forceinline__ void enc_in(const u32x4 (&v)[4], InTask& o, unsigned tag, uint8_t* scr, int lane) {
  unsigned sum = 0;
#pragma unroll
  for (int q = 0; q < 4; ++q) sum = __builtin_amdgcn_udot4(v[q][0] & 0x7f7f7f7fu, 0x01010101u, sum, false);
  const unsigned e0 = (unsigned)min(max((int)((wave_sum_u32(sum) + 10445u) >> 12) - 7, 0), 24);
  const unsigned add = 0x80808080u - e0 * 0x04040404u;
  unsigned pay[12], qq[4], cnt = 0;
#pragma unroll
  for (int q = 0; q < 4; ++q) {
    unsigned R[4], S[4];
#pragma unroll
    for (int k = 0; k < 4; ++k) {
      R[k] = (v[q][k] & 0x7f7f7f7fu) + add;
      S[k] = bfi(0x80808080u, v[q][k], R[k]);
    }
    pay[3 * q] = bfi(0x60606060u, S[3] << 5, S[0]);
    pay[3 * q + 1] = bfi(0x60606060u, S[3] << 3, S[1]);
    pay[3 * q + 2] = bfi(0x20202020u, S[3] << 1, bfi(0x40404040u, S[3] >> 1, S[2]));
    qq[q] = bfi(0xc0c0c0c0u, R[3] << 1, bfi(0x30303030u, R[2] >> 1, bfi(0x0c0c0c0cu, R[1] >> 3, (R[0] >> 5) & 0x03030303u)));
  }
  put_records2(qq[0], qq[1], (unsigned)(lane << 8), cnt, scr, 256 + 2 * lane);
  put_records2(qq[2], qq[3], (unsigned)(lane << 8 | 128), cnt, scr, 256 + 2 * lane);
  *(unsigned short*)(scr + (lane < 4 ? lane * 16 : lane < 6 ? lane * 2 - 6 : 256 + 2 * lane)) =
      (unsigned short)(lane < 4 ? tag : lane == 4 ? cnt : e0);
  __builtin_amdgcn_fence(__ATOMIC_ACQ_REL, "wavefront");
  const u32x4 rec = *(const u32x4*)(scr + (lane & 15) * 16);
  __builtin_amdgcn_fence(__ATOMIC_ACQ_REL, "wavefront");
  o.raw = cnt > IREC_CAP;
  if (o.raw) {
#pragma unroll
    for (int q = 0; q < 4; ++q) o.d[q] = v[q];
    o.mask = ~0ull;
  } else {
#pragma unroll
    for (int q = 0; q < 3; ++q) o.d[q] = u32x4{pay[4 * q], pay[4 * q + 1], pay[4 * q + 2], pay[4 * q + 3]};
    o.d[3] = rec;
    o.mask = (1ull << (((irec_slot(cnt) + 31) >> 5) * 4)) - 16;
  }
}

__device__ __forceinline__ void dec_in4(const uint8_t* slot, uint8_t* out, const unsigned (&off)[4], unsigned stride, uint8_t* scr, int lane) {
  u32x4 y[4][3];
  unsigned rec[4];
  asm volatile(SGPR_PAD "global_load_dword %12, %16, %20 sc0 sc1\n\tglobal_load_dword %13, %16, %20 offset:3328 sc0 sc1\n\t"
               "global_load_dword %14, %17, %20 sc0 sc1\n\tglobal_load_dword %15, %17, %20 offset:3328 sc0 sc1\n\t"
               "global_load_dwordx4 %0, %18, %20 offset:-3072 sc0 sc1\n\tglobal_load_dwordx4 %1, %18, %20 offset:-2048 sc0 sc1\n\t"
               "global_load_dwordx4 %2, %18, %20 offset:-1024 sc0 sc1\n\t"
               "global_load_dwordx4 %3, %18, %20 offset:256 sc0 sc1\n\tglobal_load_dwordx4 %4, %18, %20 offset:1280 sc0 sc1\n\t"
               "global_load_dwordx4 %5, %18, %20 offset:2304 sc0 sc1\n\t"
               "global_load_dwordx4 %6, %19, %20 offset:-3072 sc0 sc1\n\tglobal_load_dwordx4 %7, %19, %20 offset:-2048 sc0 sc1\n\t"
               "global_load_dwordx4 %8, %19, %20 offset:-1024 sc0 sc1\n\t"
               "global_load_dwordx4 %9, %19, %20 offset:256 sc0 sc1\n\tglobal_load_dwordx4 %10, %19, %20 offset:1280 sc0 sc1\n\t"
               "global_load_dwordx4 %11, %19, %20 offset:2304 sc0 sc1\n\ts_waitcnt vmcnt(0)"
               : "=&v"(y[0][0]), "=&v"(y[0][1]), "=&v"(y[0][2]), "=&v"(y[1][0]), "=&v"(y[1][1]), "=&v"(y[1][2]),
                 "=&v"(y[2][0]), "=&v"(y[2][1]), "=&v"(y[2][2]), "=&v"(y[3][0]), "=&v"(y[3][1]), "=&v"(y[3][2]),
                 "=&v"(rec[0]), "=&v"(rec[1]), "=&v"(rec[2]), "=&v"(rec[3])
               : "v"(lane * 4), "v"(2 * ITASK + lane * 4), "v"(ITASK + lane * 16), "v"(3 * ITASK + lane * 16), "s"(slot) : "memory");
#pragma unroll
  for (int t = 0; t < 4; ++t) *(u32x4*)(scr + t * 1024 + lane * 16) = u32x4{0u, 0u, 0u, 0u};
  __builtin_amdgcn_fence(__ATOMIC_ACQ_REL, "wavefront");
  unsigned hdr[4];
#pragma unroll
  for (int t = 0; t < 4; ++t) {
    hdr[t] = (unsigned)__builtin_amdgcn_readlane((int)rec[t], 0) >> 16 | (unsigned)__builtin_amdgcn_readlane((int)rec[t], 1) << 16;
    const unsigned end = irec_slot(hdr[t] & 0x7fffu);
#pragma unroll
    for (int i = 0; i < 2; ++i) {
      const unsigned r16 = rec[t] >> (16 * i);
      const bool isrec = i == 0 ? (lane >= 16 || (lane & 3) != 0) && lane != 1 : lane != 0;
      const unsigned val = isrec && (unsigned)(lane * 2 + i) < end ? (r16 & 3u) << ((r16 >> 1) & 30u) : 0u;
      __hip_atomic_fetch_or((unsigned*)(scr + t * 1024 + ((r16 >> 6) & 255u) * 4), val, __ATOMIC_RELAXED, __HIP_MEMORY_SCOPE_WORKGROUP);
    }
  }
  __builtin_amdgcn_fence(__ATOMIC_ACQ_REL, "wavefront");
#pragma unroll
  for (int t = 0; t < 4; ++t) {
    if (hdr[t] & IHDR_RAW) continue;
    const u32x4 qq = *(const u32x4*)(scr + t * 1024 + lane * 16);
    const unsigned add = ((hdr[t] >> 16) & 31u) * 0x04040404u;
#pragma unroll
    for (int q = 0; q < 4; ++q) {
      const unsigned d0 = y[t][(3 * q) >> 2][(3 * q) & 3], d1 = y[t][(3 * q + 1) >> 2][(3 * q + 1) & 3], d2 = y[t][(3 * q + 2) >> 2][(3 * q + 2) & 3];
      const unsigned d3 = bfi(0x80808080u, d2 << 1, bfi(0x10101010u, d2 >> 1, bfi(0x0c0c0c0cu, d1 >> 3, d0 >> 5)));
      const unsigned X[4] = {bfi(0x60606060u, qq[q] << 5, d0), bfi(0x60606060u, qq[q] << 3, d1), bfi(0x60606060u, qq[q] << 1, d2),
                             bfi(0x60606060u, qq[q] >> 1, d3)};
      u32x4 r;
#pragma unroll
      for (int k = 0; k < 4; ++k) r[k] = bfi(0x80808080u, X[k], (X[k] & 0x7f7f7f7fu) + add);
      asm volatile(SGPR_PAD "global_store_dwordx4 %0, %1, %2 sc1\n\ts_nop 1" :: "v"(off[t] + q * stride + lane * 16), "v"(r), "s"(out) : "memory");
    }
  }
  __builtin_amdgcn_fence(__ATOMIC_ACQ_REL, "wavefront");
}

__device__ __forceinline__ bool wait_tasks(const uint8_t* slot, unsigned* tmo, unsigned call, int lane) {
  const uint8_t* q = slot + ((lane >> 2) & 3) * ITASK + (lane & 3) * 16;
  return spin([&] { return __all((ld_sys(q) & 0xffffu) == (call & 0xffffu)) != 0; }, tmo, call);
}

struct PushCtx {
  const uint8_t* x;
  const uint8_t* w;
  uint8_t* dst;
  unsigned* lk;
  uint8_t* scr;
  unsigned xl, wl;
  unsigned par, tick0, tag;
  int rank, d;
  bool local;
};

struct RowSet { int r, row0, base, task0; };
__device__ __forceinline__ RowSet row_set(int set) {
  if (set < 3) return {7, 0, set * 7, set * 28};
  if (set < 7) return {9, 224, 9 * set - 6, 36 * set - 24};
  return {8, (set - 7) * 256, 8 * set + 1, T_OWN + (set - 7) * 32};
}
__device__ __forceinline__ int block_col(int q, int n) { return ((n & 3) * 4 + (q < 4 ? q : (q ^ n) & 3)) * 1024 + (n >> 2) * 128; }
__device__ __forceinline__ int tile_block(int g, int i) {
  if (g < 4) return i;
  if (i >= 24) return (g - 4) * 8 + i - 24;
  const int c = i / 3, m = i - 3 * c;
  return 4 * c + m + (m >= g - 4);
}
__device__ __forceinline__ bool task_is_w(int t) { return t >= T_W0 && t < T_OWN; }
__device__ __forceinline__ int task_set(int t) { return t < 84 ? t / 28 : t < T_W0 ? 3 + (t - 84) / 36 : 7 + ((t - T_OWN) >> 5); }

__device__ __forceinline__ void ld_x(const PushCtx& c, int t, u32x4 (&v)[4]) {
  t = min(t, NTASK - 1);
  const int set = task_set(t);
  const RowSet rs = row_set(set);
  const int p0 = 2 * (t - rs.task0), ks = (p0 * (rs.r == 7 ? 147 : rs.r == 8 ? 128 : 114)) >> 10, rg = p0 - ks * rs.r;
  const unsigned o0 = c.xl + (unsigned)((((set < 7 ? set + 1 : 0) ^ c.d) * 512 + rs.row0 + rg * 32) * KS + ks * 64);
  const unsigned o1 = o0 + (unsigned)(rg + 1 == rs.r ? 64 - rg * 32 * KS : 32 * KS);
  asm volatile(SGPR_PAD "global_load_dwordx4 %0, %4, %6\n\tglobal_load_dwordx4 %1, %4, %6 offset:16\n\t"
               "global_load_dwordx4 %2, %5, %6\n\tglobal_load_dwordx4 %3, %5, %6 offset:16"
               : "=&v"(v[0]), "=&v"(v[1]), "=&v"(v[2]), "=&v"(v[3]) : "v"(o0), "v"(o1), "s"(c.x) : "memory");
}

__device__ __forceinline__ void pack_task(const PushCtx& c, const u32x4 (&v)[4], InTask& o, int lane, bool plain) {
  if (plain) {
#pragma unroll
    for (int q = 0; q < 4; ++q) o.d[q] = v[q];
    o.mask = ~0ull;
    o.raw = true;
  } else {
    enc_in(v, o, c.tag, c.scr, lane);
  }
}

struct PushState {
  int tp, tc, tn, tl;
  u32x4 line;
  InTask o;
  u32x4 nx[4];
};

template <bool LDW>
__device__ __forceinline__ void push_step(const Params& p, const PushCtx& c, PushState& s, int lane) {
  auto send = [&] {
  const unsigned is0 = c.par + (unsigned)(XS_BYTES + WT_BYTES) + (unsigned)(c.rank * NTASK) * ITASK + lane * 16;
  if (s.tp >= 0) st16_sys_m(c.dst, is0 + (unsigned)s.tp * ITASK, s.line, 15ull);
  if (s.tc < NTASK) {
    if (!s.o.raw) {
      const unsigned slot = is0 + (unsigned)s.tc * ITASK;
      asm volatile(SGPR_PAD "global_store_dwordx4 %0, %1, %4 offset:256 sc0 sc1\n\tglobal_store_dwordx4 %0, %2, %4 offset:1280 sc0 sc1\n\t"
                   "global_store_dwordx4 %0, %3, %4 offset:2304 sc0 sc1\n\ts_nop 1"
                   :: "v"(slot), "v"(s.o.d[0]), "v"(s.o.d[1]), "v"(s.o.d[2]), "s"(c.dst) : "memory");
      if (s.o.mask != 0) st16_sys_m(c.dst, slot, s.o.d[3], s.o.mask);
      s.line = s.o.d[3];
    } else {
      const int t = s.tc, u = t - T_W0;
      const bool wr = task_is_w(t);
      const RowSet rs = row_set(task_set(t));
      const unsigned plain = c.par + lane * 16 +
          (wr ? (unsigned)XS_BYTES + (unsigned)((u >> 4) * (512 << 10) + (c.rank * 8 + ((u >> 1) & 7)) * 8192 + (u & 1) * 1024)
              : (unsigned)(rs.base * (128 << 10) + c.rank * rs.r * 16384 + (t - rs.task0) * 4096));
#pragma unroll
      for (int q = 0; q < 4; ++q) st16_sys(c.dst, plain + q * (wr ? 2048 : 1024), s.o.d[q]);
      s.line = u32x4{c.tag | IHDR_RAW << 16, 0u, 0u, 0u};
    }
  }
  };
  const bool had_tp = s.tp >= 0;
  if (!LD_FIRST) send();
  unsigned w[16], tk;
  u32x4 x[4];
  const bool draw = s.tl < NTASK;
  asm volatile(SGPR_PAD "s_mov_b64 exec, 1\n\tglobal_atomic_add %0, %1, %2, %4 sc0\n\tglobal_atomic_add %1, %3, %4 offset:%5\n\ts_mov_b64 exec, -1"
               : "=&v"(tk) : "v"(0u), "v"(draw ? 1u : 0u), "v"(had_tp ? 1u : 0u), "s"(c.lk + LK_PTICK), "n"((LK_PDONE - LK_PTICK) * 4) : "memory");
  if constexpr (LDW) {
    const int u = s.tl - T_W0;
    unsigned o = c.wl + (unsigned)((((u >> 1) & 7) * 64 + (u & 1) * 16) * GN + block_col(c.d, u >> 4));
    asm volatile(SGPR_PAD
                 "global_load_dword %0, %16, %17\n\tv_add_u32 %16, 0x4000, %16\n\t"
                 "global_load_dword %1, %16, %17\n\tv_add_u32 %16, 0x4000, %16\n\t"
                 "global_load_dword %2, %16, %17\n\tv_add_u32 %16, 0x4000, %16\n\t"
                 "global_load_dword %3, %16, %17\n\tv_add_u32 %16, 0x4000, %16\n\t"
                 "global_load_dword %4, %16, %17\n\tv_add_u32 %16, 0x4000, %16\n\t"
                 "global_load_dword %5, %16, %17\n\tv_add_u32 %16, 0x4000, %16\n\t"
                 "global_load_dword %6, %16, %17\n\tv_add_u32 %16, 0x4000, %16\n\t"
                 "global_load_dword %7, %16, %17\n\tv_add_u32 %16, 0x4000, %16\n\t"
                 "global_load_dword %8, %16, %17\n\tv_add_u32 %16, 0x4000, %16\n\t"
                 "global_load_dword %9, %16, %17\n\tv_add_u32 %16, 0x4000, %16\n\t"
                 "global_load_dword %10, %16, %17\n\tv_add_u32 %16, 0x4000, %16\n\t"
                 "global_load_dword %11, %16, %17\n\tv_add_u32 %16, 0x4000, %16\n\t"
                 "global_load_dword %12, %16, %17\n\tv_add_u32 %16, 0x4000, %16\n\t"
                 "global_load_dword %13, %16, %17\n\tv_add_u32 %16, 0x4000, %16\n\t"
                 "global_load_dword %14, %16, %17\n\tv_add_u32 %16, 0x4000, %16\n\t"
                 "global_load_dword %15, %16, %17\n\t"
                 : "=&v"(w[0]), "=&v"(w[1]), "=&v"(w[2]), "=&v"(w[3]), "=&v"(w[4]), "=&v"(w[5]), "=&v"(w[6]), "=&v"(w[7]),
                   "=&v"(w[8]), "=&v"(w[9]), "=&v"(w[10]), "=&v"(w[11]), "=&v"(w[12]), "=&v"(w[13]), "=&v"(w[14]), "=&v"(w[15]), "+v"(o) : "s"(c.w) : "memory");
    static_assert(GN == 0x4000, "");
  } else {
    ld_x(c, s.tl, x);
  }
  if (LD_FIRST) send();
  if (s.tn < NTASK) pack_task(c, s.nx, s.o, lane, c.local);
  if constexpr (LDW) {
    asm volatile("s_waitcnt vmcnt(0)" : "+v"(tk), "+v"(w[0]), "+v"(w[1]), "+v"(w[2]), "+v"(w[3]), "+v"(w[4]), "+v"(w[5]), "+v"(w[6]), "+v"(w[7]),
                 "+v"(w[8]), "+v"(w[9]), "+v"(w[10]), "+v"(w[11]), "+v"(w[12]), "+v"(w[13]), "+v"(w[14]), "+v"(w[15]) :: "memory");
#pragma unroll
    for (int q = 0; q < 4; ++q) {
      const unsigned d0 = w[4 * q], d1 = w[4 * q + 1], d2 = w[4 * q + 2], d3 = w[4 * q + 3];
      const unsigned t0 = __builtin_amdgcn_perm(d1, d0, 0x05010400u), t1 = __builtin_amdgcn_perm(d1, d0, 0x07030602u);
      const unsigned t2 = __builtin_amdgcn_perm(d3, d2, 0x05010400u), t3 = __builtin_amdgcn_perm(d3, d2, 0x07030602u);
      s.nx[0][q] = __builtin_amdgcn_perm(t2, t0, 0x05040100u);
      s.nx[1][q] = __builtin_amdgcn_perm(t2, t0, 0x07060302u);
      s.nx[2][q] = __builtin_amdgcn_perm(t3, t1, 0x05040100u);
      s.nx[3][q] = __builtin_amdgcn_perm(t3, t1, 0x07060302u);
    }
  } else {
    asm volatile("s_waitcnt vmcnt(0)" : "+v"(tk), "+v"(x[0]), "+v"(x[1]), "+v"(x[2]), "+v"(x[3]) :: "memory");
#pragma unroll
    for (int q = 0; q < 4; ++q) s.nx[q] = x[q];
  }
  if (s.tc < NTASK) STAMP(min(8 + (s.tc >> 5), 28));
  s.tp = s.tc; s.tc = s.tn; s.tn = s.tl;
  s.tl = draw ? (int)(__builtin_amdgcn_readfirstlane(tk) - c.tick0) : NTASK;
}

#define GLDS_SC 2
#define GLDS_BITS " sc0 sc1"
__device__ __forceinline__ void glds2(const uint8_t* gbase, unsigned voff, unsigned lds) {
  unsigned keep;
  asm volatile("s_mov_b32 %0, m0\n\ts_mov_b32 m0, %3\n\ts_nop 2\n\t"
               "global_load_lds_dwordx4 %1, %2" GLDS_BITS "\n\t"
               "global_load_lds_dwordx4 %1, %2 offset:1024" GLDS_BITS "\n\t"
               "s_mov_b32 m0, %0"
               : "=&s"(keep) : "v"(voff), "s"(gbase), "s"(lds));
}
__device__ __forceinline__ void glds_half(const uint8_t* gbase, unsigned voff, unsigned lds) {
  unsigned keep;
  asm volatile("s_mov_b32 %0, m0\n\ts_mov_b32 m0, %3\n\ts_mov_b32 exec_hi, 0\n\ts_nop 1\n\t"
               "global_load_lds_dwordx4 %1, %2" GLDS_BITS "\n\t"
               "s_mov_b32 exec_hi, -1\n\ts_mov_b32 m0, %0"
               : "=&s"(keep) : "v"(voff), "s"(gbase), "s"(lds));
}

struct Frags { i32x8 a[2], b[4], a9, b9; };

__device__ __forceinline__ i32x8 ld_frag(const uint8_t* q) {
  const i32x4 lo = *(const i32x4*)q, hi = *(const i32x4*)(q + 1024);
  return __builtin_shufflevector(lo, hi, 0, 1, 2, 3, 4, 5, 6, 7);
}

#define MFMA(mi, j) acc[mi][j] = __builtin_amdgcn_mfma_scale_f32_32x32x64_f8f6f4(cur.a[mi], cur.b[j], acc[mi][j], 1, 1, 0, 0, 0, 0)
#define SB() __builtin_amdgcn_sched_barrier(0)

struct Ring {
  const uint8_t* ga;
  const uint8_t* gb;
  unsigned astep;
  unsigned va, vb, v9;
  unsigned wr, wr_end;
  unsigned wb, w9;
  unsigned rd;
  const uint8_t* qa;
  const uint8_t* qb;
  const uint8_t* q9;
  const uint8_t* la;
  const uint8_t* lb;
  const uint8_t* l9;
};

template <bool NINE, int LEFT, bool DMA, bool RD>
__device__ __forceinline__ void kstep(f32x16 (&acc)[2][4], f32x16& acc9, const Frags& cur, Frags& nxt, Ring& g) {
  constexpr int STAGE = NINE ? 26624 : 24576;
  MFMA(0, 0); SB();
  asm volatile("s_waitcnt vmcnt(%0) lgkmcnt(0)\n\ts_barrier" :: "n"(LEFT * (NINE ? 7 : 6)) : "memory");
  SB();
  MFMA(0, 1); SB();
  if constexpr (RD) { nxt.a[0] = ld_frag(g.qa); nxt.b[0] = ld_frag(g.qb); }
  SB();
  MFMA(0, 2); SB();
  if constexpr (DMA) glds2(g.ga, g.va, g.wr);
  SB();
  MFMA(0, 3); SB();
  if constexpr (RD) { nxt.b[1] = ld_frag(g.qb + 2048); nxt.b[2] = ld_frag(g.qb + 4096); }
  SB();
  MFMA(1, 0); SB();
  if constexpr (DMA) glds2(g.gb, g.vb, g.wr + g.wb);
  SB();
  MFMA(1, 1); SB();
  if constexpr (RD) { nxt.b[3] = ld_frag(g.qb + 6144); nxt.a[1] = ld_frag(g.qa + 2048); }
  g.rd = g.rd == (NSTAGE - 1) * STAGE ? 0 : g.rd + STAGE;
  g.qa = g.la + g.rd;
  if constexpr (!NINE) g.qb = g.lb + g.rd;
  SB();
  if constexpr (NINE) {
    acc9 = __builtin_amdgcn_mfma_scale_f32_32x32x64_f8f6f4(cur.a9, cur.b9, acc9, 1, 1, 0, 0, 0, 0); SB();
    if constexpr (RD) { nxt.a9 = ld_frag(g.qb - 2048); nxt.b9 = ld_frag(g.q9); }
    if constexpr (DMA) glds_half(g.ga, g.v9, g.wr + g.w9);
    SB();
  }
  MFMA(1, 2); SB();
  if constexpr (DMA) glds2(g.ga + 2048, g.va, g.wr + 2048);
  SB();
  MFMA(1, 3); SB();
  if constexpr (DMA) {
    g.ga += g.astep; g.gb += 8192;
    g.wr += STAGE; g.wr = g.wr == g.wr_end ? g.wr - NSTAGE * STAGE : g.wr;
  }
  if constexpr (NINE) { g.qb = g.lb + g.rd; g.q9 = g.l9 + g.rd; }
  SB();
}

template <bool NINE>
__device__ __forceinline__ void gemm_tile(const uint8_t* ga, unsigned astep, const uint8_t* gb, uint8_t* lds, int wave, int lane) {
  constexpr int STAGE = NINE ? 26624 : 24576, BOFF = STAGE - 8192;
  f32x16 acc[2][4], acc9;
  Ring g;
  g.ga = ga; g.gb = gb; g.astep = astep;
  g.va = lane * 16 + wave * 4096; g.vb = lane * 16 + wave * 2048; g.v9 = 16384 + lane * 16 + wave * 512;
  g.wr = (unsigned)(uintptr_t)lds + wave * 4096; g.wr_end = g.wr + NSTAGE * STAGE;
  g.wb = BOFF - wave * 2048; g.w9 = 16384 - wave * 3584;
#pragma unroll
  for (int s = 0; s < NSTAGE; ++s) {
    glds2(g.ga, g.va, g.wr + s * STAGE);
    glds2(g.gb, g.vb, g.wr + s * STAGE + g.wb);
    if constexpr (NINE) glds_half(g.ga, g.v9, g.wr + s * STAGE + g.w9);
    glds2(g.ga + 2048, g.va, g.wr + s * STAGE + 2048);
    g.ga += astep; g.gb += 8192;
  }
#pragma unroll
  for (int r = 0; r < 16; ++r) {
    acc9[r] = 0.f;
#pragma unroll
    for (int j = 0; j < 4; ++j) { acc[0][j][r] = 0.f; acc[1][j][r] = 0.f; }
  }
  g.la = lds + lane * 16 + wave * 4096; g.lb = lds + BOFF + lane * 16; g.l9 = g.lb + wave * 2048;
  asm volatile("s_waitcnt vmcnt(%0)\n\ts_barrier" :: "n"(5 * (NINE ? 7 : 6)) : "memory");
  Frags f0, f1;
  f0.a[0] = ld_frag(g.la); f0.b[0] = ld_frag(g.lb); f0.b[1] = ld_frag(g.lb + 2048);
  f0.b[2] = ld_frag(g.lb + 4096); f0.b[3] = ld_frag(g.lb + 6144); f0.a[1] = ld_frag(g.la + 2048);
  if constexpr (NINE) { f0.a9 = ld_frag(g.lb - 2048); f0.b9 = ld_frag(g.l9); }
  g.rd = STAGE; g.qa = g.la + STAGE; g.qb = g.lb + STAGE; g.q9 = g.l9 + STAGE;
#pragma unroll 1
  for (int s = 0; s < KSTEPS - NSTAGE; s += 2) {
    kstep<NINE, 4, true, true>(acc, acc9, f0, f1, g);
    kstep<NINE, 4, true, true>(acc, acc9, f1, f0, g);
  }
  kstep<NINE, 4, false, true>(acc, acc9, f0, f1, g);
  kstep<NINE, 3, false, true>(acc, acc9, f1, f0, g);
  kstep<NINE, 2, false, true>(acc, acc9, f0, f1, g);
  kstep<NINE, 1, false, true>(acc, acc9, f1, f0, g);
  kstep<NINE, 0, false, true>(acc, acc9, f0, f1, g);
  kstep<NINE, 0, false, false>(acc, acc9, f1, f0, g);
#pragma unroll
  for (int i = 0; i < 32; ++i) {
    const f32x16& a = acc[i >> 4][(i >> 2) & 3];
    *(f32x4*)(lds + wave * 32768 + i * 1024 + lane * 16) = f32x4{a[4 * (i & 3)], a[4 * (i & 3) + 1], a[4 * (i & 3) + 2], a[4 * (i & 3) + 3]};
  }
  if constexpr (NINE) {
#pragma unroll
    for (int q = 0; q < 4; ++q)
      *(f32x4*)(lds + 8 * 16384 + wave * 4096 + q * 1024 + lane * 16) = f32x4{acc9[4 * q], acc9[4 * q + 1], acc9[4 * q + 2], acc9[4 * q + 3]};
    __syncthreads();
  }
  __builtin_amdgcn_fence(__ATOMIC_ACQ_REL, "wavefront");
}

constexpr unsigned REC_CAP = 510, HDR_B = 0x40000000u;

__device__ __forceinline__ void put_records4(const unsigned (&q)[4], unsigned tag, unsigned& cnt, uint8_t* scr, unsigned dump) {
  unsigned m = 0;
#pragma unroll
  for (int k = 0; k < 4; ++k)
    m |= ((((q[k] & 0x77777777u) + 0x77777777u) | q[k]) & 0x88888888u) >> (3 - k);
  for (unsigned long long act = __ballot(m != 0); act != 0; act = __ballot(m != 0)) {
    const unsigned bit = (unsigned)__builtin_ctz(m | 0x80000000u), sh = bit & 28u;
    const unsigned m1 = (unsigned)__builtin_amdgcn_sbfe((int)bit, 0, 1), m2 = (unsigned)__builtin_amdgcn_sbfe((int)bit, 1, 1);
    const unsigned w = bfi(m2, bfi(m1, q[3], q[2]), bfi(m1, q[1], q[0]));
    const unsigned idx = cnt + __builtin_amdgcn_mbcnt_hi((unsigned)(act >> 32), __builtin_amdgcn_mbcnt_lo((unsigned)act, 0u));
    const unsigned at = (m != 0 && idx < REC_CAP) ? 4 + 2 * idx : dump;
    *(unsigned short*)(scr + at) = (unsigned short)(tag | (bit & 3u) << 7 | sh << 2 | ((w >> sh) & 15u));
    m &= m - 1;
    cnt += __popcll(act);
  }
}

template <bool MODEB>
__device__ __forceinline__ bool enc_body(const float (&v)[4][16], float up, unsigned& cnt, uint8_t* blk, int lane) {
  unsigned bad = 0, low = 0;
#pragma unroll
  for (int h = 0; h < 2; ++h) {
    unsigned G[32];
#pragma unroll
    for (int j = 0; j < 4; ++j)
#pragma unroll
      for (int r = 0; r < 8; ++r) {
        const float x = v[j][8 * h + r] * up;
        unsigned g = (unsigned)(int)x;
        bad = __builtin_amdgcn_bitop3_b32(__builtin_bit_cast(unsigned, (float)(int)g), __builtin_bit_cast(unsigned, x), bad, 0xbe);
        low |= g;
        if (MODEB) g = g << 1 | (((g ^ g << 1) >> 27) & 16u);
        G[4 * r + j] = g;
      }
    unsigned q[4];
#pragma unroll
    for (int k = 0; k < 4; ++k) {
      const unsigned* g = G + 8 * k;
      const unsigned b0 = __builtin_amdgcn_perm(__builtin_amdgcn_perm(g[3], g[2], 0x0c0c0400u), __builtin_amdgcn_perm(g[1], g[0], 0x0c0c0400u), 0x05040100u);
      const unsigned b1 = __builtin_amdgcn_perm(__builtin_amdgcn_perm(g[7], g[6], 0x0c0c0400u), __builtin_amdgcn_perm(g[5], g[4], 0x0c0c0400u), 0x05040100u);
      q[k] = (b0 >> 4) | b1;
    }
#pragma unroll
    for (int j = 0; j < 2; ++j) {
      unsigned d[12];
#pragma unroll
      for (int g = 0; g < 4; ++g) {
        const unsigned* x = G + 16 * j + 4 * g;
        d[3 * g] = __builtin_amdgcn_perm(x[1], x[0], 0x05030201u);
        d[3 * g + 1] = __builtin_amdgcn_perm(x[2], x[1], 0x06050302u);
        d[3 * g + 2] = __builtin_amdgcn_perm(x[3], x[2], 0x07060503u);
      }
#pragma unroll
      for (int i = 0; i < 3; ++i)
        *(u32x4*)(blk + (1 + 6 * h + 3 * j + i) * 1024 + lane * 16) = u32x4{d[4 * i], d[4 * i + 1], d[4 * i + 2], d[4 * i + 3]};
    }
    put_records4(q, (unsigned)(lane << 10 | h << 9), cnt, blk, 13 * 1024 + 2 * lane);
  }
  return __ballot((bad | (low & 15u)) != 0) == 0 && cnt <= REC_CAP;
}

__device__ __forceinline__ unsigned enc_out(const float (&v)[4][16], float sc, uint8_t* blk, int lane) {
  float m = 0.f;
#pragma unroll
  for (int j = 0; j < 4; ++j)
#pragma unroll
    for (int r = 0; r < 16; r += 2) asm("v_max3_f32 %0, |%1|, |%2|, %0" : "+v"(m) : "v"(v[j][r]), "v"(v[j][r + 1]));
  const int e = max((int)(wave_max_u32(__builtin_bit_cast(unsigned, m)) >> 23) - 157, -126);
  const float up = __builtin_bit_cast(float, (unsigned)(127 - min(e, 0)) << 23);
  int ntop = 0, nlow = 0;
#pragma unroll
  for (int r = 0; r < 8; ++r) {
    const unsigned g = (unsigned)(int)(v[r & 3][2 * r] * up);
    ntop += __popcll(__ballot((int)(g ^ (g << 1)) < 0));
    nlow += __popcll(__ballot((g & 128u) != 0));
  }
  unsigned cnt = 0;
  const bool modeb = ntop < nlow;
  const bool fits = e <= 0 && (modeb ? enc_body<true>(v, up, cnt, blk, lane) : enc_body<false>(v, up, cnt, blk, lane));
  if (!fits) {
#pragma unroll
    for (int r = 0; r < 16; ++r) *(f32x4*)(blk + r * 1024 + lane * 16) = f32x4{v[0][r] * sc, v[1][r] * sc, v[2][r] * sc, v[3][r] * sc};
    return 0;
  }
  *(unsigned*)(blk + (lane == 0 ? 0 : 13 * 1024 + 4 * lane)) = cnt | (unsigned)(e + 127) << 16 | (modeb ? HDR_B : 0u);
  return ((4 + 2 * cnt + 63) >> 6) * 4;
}

#define LOC_ST 1
__device__ __forceinline__ void st16_out(void* base, unsigned off, f32x4 v) {
  asm volatile(SGPR_PAD "global_store_dwordx4 %0, %1, %2 sc1\n\ts_nop 1" :: "v"(off), "v"(v), "s"(base) : "memory");
}

__device__ __forceinline__ constexpr unsigned row_off(int r) { return (unsigned)((r & 3) + 8 * (r >> 2)) * (GN * 4); }

__device__ __forceinline__ void send_out(const uint8_t* blk, unsigned nl, bool dead, uint8_t* slot, float* ob, unsigned off,
                                         int lane, unsigned* rel, bool last, int depth) {
  const bool raw = nl == 0 || dead;
  const u32x4 h = *(const u32x4*)(blk + lane * 16);
  st16_sys_m(slot, lane * 16, raw ? u32x4{HDR_RAW, 0u, 0u, 0u} : h, raw ? 15ull : nl >= 64 ? ~0ull : (1ull << nl) - 1);
  const int n = raw ? 16 : 12;
  void* const base = raw ? (void*)ob : (void*)slot;
  const uint8_t* const src = blk + (raw ? 0 : 1024) + lane * 16;
  const unsigned nan = dead ? 0x7fc00000u : 0u;
#pragma unroll 1
  for (int i = 0; i < n; ++i) {
    const u32x4 v = *(const u32x4*)(src + i * 1024) | nan;
    if (depth == OUT_D) wait_vm<OUT_D - 1>(); else if (depth == OUT_D3) wait_vm<OUT_D3 - 1>(); else wait_vm<DEEP_D - 1>();
    if (last && i == n - OUT_REL) add1_agent(rel, 1);
    st16_sys(base, raw ? off + row_off(i) : (unsigned)(1024 + i * 1024 + lane * 16), v);
  }
}

__device__ __forceinline__ void dec_out(const uint8_t* slot, float* ob, unsigned off, float sc, unsigned* tmo, unsigned call, uint8_t* scr, int lane) {
  u32x4 rec, y[12];
  const unsigned tmv = ld_agent(tmo);
  asm volatile(SGPR_PAD "global_load_dwordx4 %0, %13, %15 offset:-4096 sc0 sc1\n\tglobal_load_dwordx4 %1, %13, %15 offset:-3072 sc0 sc1\n\t"
               "global_load_dwordx4 %2, %13, %15 offset:-2048 sc0 sc1\n\tglobal_load_dwordx4 %3, %13, %15 offset:-1024 sc0 sc1\n\t"
               "global_load_dwordx4 %4, %13, %15 sc0 sc1\n\tglobal_load_dwordx4 %5, %13, %15 offset:1024 sc0 sc1\n\t"
               "global_load_dwordx4 %6, %13, %15 offset:2048 sc0 sc1\n\tglobal_load_dwordx4 %7, %13, %15 offset:3072 sc0 sc1\n\t"
               "global_load_dwordx4 %8, %14, %15 offset:-4096 sc0 sc1\n\tglobal_load_dwordx4 %9, %14, %15 offset:-3072 sc0 sc1\n\t"
               "global_load_dwordx4 %10, %14, %15 offset:-2048 sc0 sc1\n\tglobal_load_dwordx4 %11, %14, %15 offset:-1024 sc0 sc1\n\t"
               "global_load_dwordx4 %12, %14, %15 sc0 sc1\n\ts_waitcnt vmcnt(0)"
               : "=&v"(rec), "=&v"(y[0]), "=&v"(y[1]), "=&v"(y[2]), "=&v"(y[3]), "=&v"(y[4]), "=&v"(y[5]), "=&v"(y[6]),
                 "=&v"(y[7]), "=&v"(y[8]), "=&v"(y[9]), "=&v"(y[10]), "=&v"(y[11])
               : "v"(4096 + lane * 16), "v"(12288 + lane * 16), "s"(slot) : "memory");
  const unsigned hdr = __builtin_amdgcn_readfirstlane(rec[0]);
  if (__builtin_amdgcn_readfirstlane(tmv) == call) {
    const float n = __builtin_nanf("");
#pragma unroll 1
    for (int r = 0; r < 16; ++r) st16_out(ob, off + row_off(r), f32x4{n, n, n, n});
    return;
  }
  if (hdr & HDR_RAW) return;
  const unsigned cnt = hdr & 0xffffu;
  const float down = __builtin_bit_cast(float, ((hdr >> 16) & 255u) << 23);
  *(u32x4*)(scr + lane * 32) = u32x4{0u, 0u, 0u, 0u};
  *(u32x4*)(scr + lane * 32 + 16) = u32x4{0u, 0u, 0u, 0u};
  __builtin_amdgcn_fence(__ATOMIC_ACQ_REL, "wavefront");
#pragma unroll
  for (int i = 0; i < 8; ++i) {
    const unsigned r16 = (rec[i >> 1] >> (16 * (i & 1))) & 0xffffu;
    const unsigned val = (unsigned)(lane * 8 + i - 2) < cnt ? (r16 & 15u) << ((r16 >> 2) & 28u) : 0u;
    __hip_atomic_fetch_or((unsigned*)(scr + (r16 >> 7) * 4), val, __ATOMIC_RELAXED, __HIP_MEMORY_SCOPE_WORKGROUP);
  }
  __builtin_amdgcn_fence(__ATOMIC_ACQ_REL, "wavefront");
  const u32x4 q0 = *(const u32x4*)(scr + lane * 32), q1 = *(const u32x4*)(scr + lane * 32 + 16);
  __builtin_amdgcn_fence(__ATOMIC_ACQ_REL, "wavefront");
  const bool modeb = (hdr & HDR_B) != 0;
#pragma unroll
  for (int g = 0; g < 16; ++g) {
    const unsigned d0 = y[(3 * g) >> 2][(3 * g) & 3], d1 = y[(3 * g + 1) >> 2][(3 * g + 1) & 3], d2 = y[(3 * g + 2) >> 2][(3 * g + 2) & 3];
    const unsigned qw = g < 8 ? q0[(g >> 1) & 3] : q1[(g >> 1) & 3];
    const unsigned q = ((g & 1) ? qw : qw << 4) & 0xf0f0f0f0u;
    unsigned G[4] = {__builtin_amdgcn_perm(d0, q, 0x06050400u), __builtin_amdgcn_perm(d1, d0, 0x0504030cu) | ((q >> 8) & 255u),
                     __builtin_amdgcn_perm(d2, d1, 0x0403020cu) | ((q >> 16) & 255u), __builtin_amdgcn_perm(d2, q, 0x07060503u)};
    f32x4 v;
#pragma unroll
    for (int j = 0; j < 4; ++j) {
      if (modeb) G[j] = ((unsigned)((int)G[j] >> 1) & ~8u) ^ (G[j] << 27);
      v[j] = ((float)(int)G[j] * down) * sc;
    }
    st16_out(ob, off + row_off(g), v);
  }
}

__device__ __forceinline__ int wave_blocks(int r, int wave) { return wave < 3 ? 2 : r - 6; }
__device__ __forceinline__ unsigned out_off(int row0, int col, int wave, int lane) {
  return (unsigned)(((row0 + wave * 64 + 4 * (lane >> 5)) * GN + col + 4 * (lane & 31)) * 4);
}

__device__ __forceinline__ void recv_tile(const Params& p, uint8_t* myws, float* out, int g, int src, int i, int k0, int k1, int wave, int lane,
                                          unsigned call, unsigned* tmo, float sc, uint8_t* scr) {
  const unsigned off = out_off(g < 4 ? 0 : 224, block_col(src, tile_block(g, i)), wave, lane);
  const uint8_t* const fl = myws + WS_DONE + (size_t)((src * 32 + i) * 4 + wave) * FLAG_STRIDE;
  const int nb = min(wave_blocks(g < 4 ? 7 : 9, wave), k1);
#pragma unroll 1
  for (int k = k0; k < nb; ++k) {
    spin<RX_SLEEP>([&] { return __builtin_amdgcn_readfirstlane(ld_sys(fl)) > 4 * call + k; }, tmo, call);
    STAMP(24);
    dec_out(myws + WS_OS + (size_t)((src * 32 + i) * 9 + wave * 2 + k) * OBLK, out, off + (unsigned)k * 32 * (GN * 4), sc, tmo, call, scr, lane);
  }
}

}

__global__ void __launch_bounds__(256, 1) gemm_rs_fused(const Params p) {
  __shared__ __attribute__((aligned(16))) uint8_t lds[LDS_BYTES];
  const int b = blockIdx.x;
  const int lane = threadIdx.x & 63;
  const int wave = __builtin_amdgcn_readfirstlane(threadIdx.x >> 6);
  const int rank = p.rank;
  uint8_t* const myws = p.ws[rank];
  STAMP(0);

  const int grp = b & 7, dst = grp ^ rank, ti = 31 - (b >> 3);
  const bool local = grp == 0 || (grp >= 4 && ti >= 24);
  const int set = local ? 7 + (grp != 0) : grp - 1, cb = tile_block(grp, ti);
  const RowSet rs = row_set(set);
  unsigned* const callp = (unsigned*)(myws + WS_CALL + (size_t)b * 128);
  unsigned* const tmo = (unsigned*)(myws + WS_TMO);
  unsigned* const lk = (unsigned*)(myws + WS_LINK) + grp * 128;
  uint8_t* const scr = lds + LDS_PK + wave * 2048;

  unsigned call;
  if (ti >= 24) {
    const int w = (31 - ti) * 4 + wave;
    PushCtx c;
    c.x = p.x; c.w = p.w; c.dst = p.ws[dst]; c.lk = lk; c.rank = rank; c.d = dst;
    c.xl = (unsigned)((lane & 31) * KS + (lane >> 5) * 32);
    c.wl = (unsigned)((lane >> 5) * 32 * GN + (lane & 31) * 4);
    c.scr = scr;
    c.local = grp == 0;
    PushState s;
    u32x4 x0[4];
    unsigned cv;
    asm volatile(SGPR_PAD "global_load_dword %0, %1, %2 sc1" : "=v"(cv) : "v"(0u), "s"(callp) : "memory");
    ld_x(c, w, x0);
    ld_x(c, w + 32, s.nx);
    asm volatile("s_waitcnt vmcnt(0)" : "+v"(cv), "+v"(x0[0]), "+v"(x0[1]), "+v"(x0[2]), "+v"(x0[3]),
                 "+v"(s.nx[0]), "+v"(s.nx[1]), "+v"(s.nx[2]), "+v"(s.nx[3]) :: "memory");
    call = __builtin_amdgcn_readfirstlane(cv) + 1;
    c.par = (unsigned)(WS_DATA + (call & 1) * PAR_BYTES);
    c.tick0 = (call - 1) * (NTASK - 64) - 96;
    c.tag = call & 0xffffu;
    s.tp = -1; s.tc = w; s.tn = w + 32; s.tl = w + 64;
    s.line = u32x4{0u, 0u, 0u, 0u};
    pack_task(c, x0, s.o, lane, true);
    STAMP(7);
#pragma unroll 1
    while (s.tp < NTASK) {
      if (task_is_w(s.tl)) push_step<true>(p, c, s, lane);
      else push_step<false>(p, c, s, lane);
    }
    STAMP(30);
  } else {
    call = __builtin_amdgcn_readfirstlane(ld_agent(callp)) + 1;
  }
  uint8_t* const par = myws + WS_DATA + (size_t)(call & 1) * PAR_BYTES;

#pragma unroll 1
  for (int job = 0; job < 4; ++job) {
    const bool jw = grp == 0 ? job < 3 : job == 1;
    const int beta = (ti * 4 + wave) & (grp == 0 ? 63 : 127);
    if (grp == 0 ? (job == 2 && ti >= 24) : (ti >= 24 || job >= 2 || (job == 0 && beta >= 8 * rs.r))) continue;
    const int n = job == 2 ? 24 + (ti & 7) : cb;
    const RowSet js = row_set(grp == 0 ? 7 + (ti >> 4) : set);
    const int bs = (beta * (js.r == 7 ? 147 : js.r == 8 ? 128 : 114)) >> 10, bu = beta - bs * js.r;
    const int src = !jw ? bs : grp != 0 ? dst : job == 0 ? rank : job == 1 ? rank ^ (4 + (ti & 3)) : rank ^ (1 + (ti >> 3));
    const int t0 = jw ? T_W0 + n * 16 + wave * 4 : js.task0 + bu * 4;
    const uint8_t* const slot = par + XS_BYTES + WT_BYTES + (size_t)(src * NTASK + t0) * ITASK;
    if (!wait_tasks(slot, tmo, call, lane)) break;
    if (jw) STAMP(26);
    if (src != rank) {
      const unsigned base = jw ? (unsigned)XS_BYTES + (unsigned)(n * (512 << 10) + (src * 8 + wave * 2) * 8192)
                               : (unsigned)(js.base * (128 << 10) + (src * js.r + bu) * 16384);
      unsigned off[4];
#pragma unroll
      for (int t = 0; t < 4; ++t) off[t] = base + (jw ? (t >> 1) * 8192 + (t & 1) * 1024 : t * 4096);
      dec_in4(slot, par, off, jw ? 2048 : 1024, lds + wave * 4096, lane);
      if (jw) STAMP(27);
      wait_vm<0>();
    }
    st1_agent(jw ? myws + WS_DW + (size_t)(n * 32 + src * 4 + wave) * 4
                 : myws + WS_DX + (size_t)((grp == 0 ? 7 + (ti >> 4) : set) * 128 + beta) * 4, call);
  }
  STAMP(29);

  unsigned* const ctrl = (unsigned*)(lds + LDS_CTRL);
  if (wave == 0) {
    const uint8_t* const fw = myws + WS_DW + (size_t)(cb * 32 + (lane & 31)) * 4;
    const uint8_t* const fx = myws + WS_DX + (size_t)(set * 128 + min(lane, 8 * rs.r - 1)) * 4;
    const uint8_t* const fy = myws + WS_DX + (size_t)(set * 128 + min(64 + lane, 8 * rs.r - 1)) * 4;
    const bool ok = spin([&] { const unsigned a = ld_agent(fw), c = ld_agent(fx), d = ld_agent(fy); return __all(min(min(a, c), d) >= call) != 0; },
                         tmo, call);
    if (lane == 0) ctrl[0] = !ok;
  }
  __syncthreads();
  const bool dead0 = __builtin_amdgcn_readfirstlane(ctrl[0]) != 0;
  STAMP(1); CSTAMP(1);

  if (wave == 0) st1_agent(callp, call);
  const float sc = p.sx[0] * p.sw[0];
  const unsigned off = out_off(rs.row0, block_col(rank, cb), wave, lane);
  float* const ob = p.out[local ? rank : dst];
  uint8_t* const blk = lds + wave * 32768;
  uint8_t* const oslot = p.ws[dst] + WS_OS + (size_t)((rank * 32 + ti) * 9 + wave * 2) * OBLK;
  unsigned* const flag = (unsigned*)(p.ws[dst] + WS_DONE + (size_t)((rank * 32 + ti) * 4 + wave) * FLAG_STRIDE);
  const int nb = wave_blocks(rs.r, wave);
  unsigned nl0 = 0, nl1 = 0, nl2 = 0, tick = 0;
  int sent = 0;

  if (!dead0) {
    const uint8_t* const ga = par + (size_t)rs.base * (128 << 10);
    const uint8_t* const gb = par + XS_BYTES + (size_t)cb * (512 << 10);
    if (rs.r == 9) gemm_tile<true>(ga, 9 * 2048, gb, lds, wave, lane);
    else gemm_tile<false>(ga, rs.r * 2048, gb, lds, wave, lane);
    STAMP(2); CSTAMP(2);
    unsigned tk;
    asm volatile(SGPR_PAD "s_mov_b64 exec, 1\n\tglobal_atomic_add %0, %1, %2, %3 sc0\n\ts_mov_b64 exec, -1"
                 : "=&v"(tk) : "v"(0u), "v"(!local && ti < 24 ? 1u : 0u), "s"(lk + LK_OTICK) : "memory");
    const float s0 = local && __builtin_amdgcn_readfirstlane(ld_agent(tmo)) == call ? __builtin_nanf("") : sc;
#pragma unroll 1
    for (int k = 0; k < nb; ++k) {
      float v[4][16];
#pragma unroll
      for (int i = 0; i < 16; ++i) {
        const f32x4 t = *(const f32x4*)(blk + k * 16384 + i * 1024 + lane * 16);
#pragma unroll
        for (int q = 0; q < 4; ++q) v[i >> 2][(i & 3) * 4 + q] = t[q];
      }
      __builtin_amdgcn_fence(__ATOMIC_ACQ_REL, "wavefront");
      if (local) {
        if (k == 0) asm volatile("s_waitcnt vmcnt(0)" :: "v"(tk) : "memory");
#pragma unroll
        for (int r = 0; r < 16; ++r)
          st16_out(ob, off + k * 32 * (GN * 4) + row_off(r), f32x4{v[0][r] * s0, v[1][r] * s0, v[2][r] * s0, v[3][r] * s0});
        continue;
      }
      const unsigned nl = enc_out(v, sc, blk + k * 16384, lane);
      __builtin_amdgcn_fence(__ATOMIC_ACQ_REL, "wavefront");
      if (k == 0) nl0 = nl; else if (k == 1) nl1 = nl; else nl2 = nl;
      if (k == 0) {
        asm volatile("s_waitcnt vmcnt(0)\n\tv_readfirstlane_b32 %0, %1\n\ts_nop 1" : "=s"(tick) : "v"(tk) : "memory");
        const unsigned tmv = ld_agent(tmo);
        if (ti < 24 && nb > 1 && link_free(lk, tick, call) && __builtin_amdgcn_readfirstlane(tmv) != call) {
          STAMP(3);
          send_out(blk, nl, false, oslot, ob, off, lane, lk + LK_OREL, false, link_depth(tick - (call - 1) * (grp < 4 ? 128 : 96), nb));
          sent = 1;
        }
      }
    }
  } else if (local) {
    const float n = __builtin_nanf("");
#pragma unroll 1
    for (int t = 0; t < 32; ++t) st16_out(ob, off + (t >> 4) * 32 * (GN * 4) + row_off(t & 15), f32x4{n, n, n, n});
  }
  STAMP(31); CSTAMP(31);
  if (grp == 0) return;

  const int last8 = grp < 4 ? 24 : 16;
  if (ti >= 24) recv_tile(p, myws, p.out[rank], grp, dst, ti - 24, 0, 3, wave, lane, call, tmo, sc, scr);
  if (!local) {
    if (ti >= 24 || dead0) tick = link_ticket(lk, lane);
    if (sent) {
      wait_vm<0>();
      st1_sys(flag, 4 * call + 1);
    } else {
      spin([&] { return link_free(lk, tick, call); }, tmo, call);
      STAMP(3);
    }
    const bool dead = dead0 || __builtin_amdgcn_readfirstlane(ld_agent(tmo)) == call;
#pragma unroll 1
    for (int k = sent; k < nb; ++k) {
      send_out(blk + k * 16384, k == 0 ? nl0 : k == 1 ? nl1 : nl2, dead, oslot + k * OBLK, ob, off + k * 32 * (GN * 4), lane,
               lk + LK_OREL, k == nb - 1, link_depth(tick - (call - 1) * (grp < 4 ? 128 : 96), nb));
      if (k == nb - 1) STAMP(4);
      wait_vm<0>();
      st1_sys(flag, 4 * call + 1 + k);
    }
    STAMP(5);
  }
#pragma unroll 1
  for (int duty = 0; duty < 2; ++duty) {
    int i, k0, k1;
    if (duty == 0) {
      if (ti >= last8) continue;
      i = ti + 8; k0 = 0; k1 = SPREAD && i >= last8 ? 1 : 3;
    } else if (!SPREAD) {
      continue;
    } else if (ti < 8) {
      i = last8 + ti; k0 = grp < 4 ? 1 : 2; k1 = 3;
    } else if (local) {
      i = ti - 8; k0 = 1; k1 = 2;
    } else {
      continue;
    }
    recv_tile(p, myws, p.out[rank], grp, dst, i, k0, k1, wave, lane, call, tmo, sc, scr);
  }
  STAMP(6);
}

extern "C" void kernel_launch(int rank, int nranks, ncclComm_t comm, hipStream_t stream,
                              void* const* d_in, const int* in_sizes, int n_in,
                              void* d_out, int out_size, const int* in_dtypes, int out_dtype,
                              void* const* d_out_all, void* const* d_ws_all, size_t ws_bytes) {
  (void)comm; (void)in_sizes; (void)in_dtypes; (void)out_dtype; (void)d_out;
  if (nranks != NR || (unsigned)rank >= (unsigned)NR || n_in != 4 || ws_bytes < WS_NEED || out_size != 512 * GN) {
    fprintf(stderr, "kernel_launch: unexpected problem shape\n");
    return;
  }
  Params p;
  p.x = (const uint8_t*)d_in[0];
  p.w = (const uint8_t*)d_in[1];
  p.sx = (const float*)d_in[2];
  p.sw = (const float*)d_in[3];
  for (int i = 0; i < NR; ++i) {
    p.ws[i] = (uint8_t*)d_ws_all[i];
    p.out[i] = (float*)d_out_all[i];
  }
  p.rank = rank;
  p.pad = 0;
  hipLaunchKernelGGL(gemm_rs_fused, dim3(NWG), dim3(256), 0, stream, p);
  HIP_CHECK(hipGetLastError());
}
```

```cpp
#include <hip/hip_runtime.h>
#include <rccl/rccl.h>
#include <cstdio>
#include <cstdint>

#define HIP_CHECK(expr) do { hipError_t _e = (expr); if (_e != hipSuccess) { \
    fprintf(stderr, "HIP error %s at %s:%d\n", hipGetErrorString(_e), __FILE__, __LINE__); \
    return; } } while (0)

typedef unsigned u32x4 __attribute__((ext_vector_type(4)));
typedef int i32x4 __attribute__((ext_vector_type(4)));
typedef int i32x8 __attribute__((ext_vector_type(8)));
typedef float f32x4 __attribute__((ext_vector_type(4)));
typedef float f32x16 __attribute__((ext_vector_type(16)));

#define SGPR_PAD "s_nop 4\n\t"

namespace {

constexpr int NR = 8;
constexpr int GN = 16384;
constexpr int KS = 512;
constexpr int NWG = 256;
constexpr int KSTEPS = 64;
constexpr int NSTAGE = 6;
constexpr int LDS_CTRL = NSTAGE * 26624;
constexpr int LDS_BYTES = LDS_CTRL + 64;
constexpr int LDS_PK = 9 * 16384;

constexpr int T_W0 = 228, T_OWN = T_W0 + 512, NTASK = T_OWN + 64;

constexpr size_t WS_CALL = 0;
constexpr size_t WS_TMO = 32768;
constexpr size_t WS_DONE = 896u << 10;
constexpr size_t WS_LINK = WS_TMO + 1024;
constexpr size_t WS_DX = WS_LINK + 4096;
constexpr size_t WS_DW = WS_DX + 8192;
constexpr size_t WS_TURN = 768u << 10;
constexpr size_t WS_DBG = 65536;
constexpr size_t WS_DATA = 1u << 20;
constexpr size_t XS_BYTES = 10u << 20;
constexpr size_t WT_BYTES = 16u << 20;
constexpr unsigned ITASK = 3328;
constexpr size_t IS_BYTES = 21u << 20;
constexpr size_t PAR_BYTES = XS_BYTES + WT_BYTES + IS_BYTES;
static_assert((size_t)8 * NTASK * ITASK <= IS_BYTES, "");
constexpr size_t OBLK = 13 * 1024;
constexpr size_t WS_OS = WS_DATA + 2 * PAR_BYTES;
constexpr size_t WS_NEED = WS_OS + 8 * 32 * 9 * OBLK;

constexpr unsigned long long DEADLINE_TICKS = 50000000ull;

#define OPEN_AT 790
#define NTURN 20
#define OUT_REL 3
#define OUT_D 2
#define OUT_D3 3
#define DEEP_D 4
#define DEEP_T 8
#define POLL_SLEEP 6
#define RX_SLEEP 16

struct Params {
  const uint8_t* x;
  const uint8_t* w;
  const float* sx;
  const float* sw;
  uint8_t* ws[NR];
  float* out[NR];
  uint8_t* myws;
  int rank;
  int pad;
};

#define STAMP(slot) do {} while (0)
#define CSTAMP(slot) do {} while (0)

__device__ __forceinline__ unsigned ld_sys(const void* ptr) {
  return __hip_atomic_load((const unsigned*)ptr, __ATOMIC_RELAXED, __HIP_MEMORY_SCOPE_SYSTEM);
}
__device__ __forceinline__ unsigned ld_agent(const void* ptr) {
  return __hip_atomic_load((const unsigned*)ptr, __ATOMIC_RELAXED, __HIP_MEMORY_SCOPE_AGENT);
}
__device__ __forceinline__ void st_sys(void* ptr, unsigned v) {
  __hip_atomic_store((unsigned*)ptr, v, __ATOMIC_RELAXED, __HIP_MEMORY_SCOPE_SYSTEM);
}
__device__ __forceinline__ void st_agent(void* ptr, unsigned v) {
  __hip_atomic_store((unsigned*)ptr, v, __ATOMIC_RELAXED, __HIP_MEMORY_SCOPE_AGENT);
}

template <int SLEEP = POLL_SLEEP, class F>
__device__ __forceinline__ bool spin(F ready, unsigned* tmo, unsigned call) {
  const unsigned long long t0 = __builtin_amdgcn_s_memrealtime();
  for (unsigned n = 1;; ++n) {
    if (ready()) return true;
    if ((n & 3) == 0 && (__builtin_amdgcn_readfirstlane(ld_agent(tmo)) == call || __builtin_amdgcn_s_memrealtime() - t0 > DEADLINE_TICKS)) {
      st_agent(tmo, call);
      return false;
    }
    __builtin_amdgcn_s_sleep(SLEEP);
  }
}

__device__ __forceinline__ void st16_sys(void* base, unsigned off, u32x4 v) {
  asm volatile(SGPR_PAD "global_store_dwordx4 %0, %1, %2 sc0 sc1\n\ts_nop 1" :: "v"(off), "v"(v), "s"(base) : "memory");
}

__device__ __forceinline__ void st1_agent(void* base, unsigned v) {
  asm volatile(SGPR_PAD "s_mov_b64 exec, 1\n\tglobal_store_dword %0, %1, %2 sc1\n\ts_mov_b64 exec, -1"
               :: "v"(0u), "v"(v), "s"(base) : "memory");
}

__device__ __forceinline__ void st1_sys(void* base, unsigned v) {
  asm volatile(SGPR_PAD "s_mov_b64 exec, 1\n\tglobal_store_dword %0, %1, %2 sc0 sc1\n\ts_mov_b64 exec, -1"
               :: "v"(0u), "v"(v), "s"(base) : "memory");
}
__device__ __forceinline__ void st16_sys_m(void* base, unsigned off, u32x4 v, unsigned long long mask) {
  asm volatile(SGPR_PAD "s_mov_b64 exec, %3\n\tglobal_store_dwordx4 %0, %1, %2 sc0 sc1\n\ts_mov_b64 exec, -1\n\ts_nop 0"
               :: "v"(off), "v"(v), "s"(base), "s"(mask) : "memory");
}

template <int N> __device__ __forceinline__ void wait_vm() {
  asm volatile("s_waitcnt vmcnt(%0)" :: "n"(N) : "memory");
}

__device__ __forceinline__ unsigned wave_max_u32(unsigned v) {
  v = max(v, (unsigned)__builtin_amdgcn_update_dpp(0, (int)v, 0x111, 0xf, 0xf, false));
  v = max(v, (unsigned)__builtin_amdgcn_update_dpp(0, (int)v, 0x112, 0xf, 0xf, false));
  v = max(v, (unsigned)__builtin_amdgcn_update_dpp(0, (int)v, 0x114, 0xf, 0xf, false));
  v = max(v, (unsigned)__builtin_amdgcn_update_dpp(0, (int)v, 0x118, 0xf, 0xf, false));
  return max(max((unsigned)__builtin_amdgcn_readlane((int)v, 15), (unsigned)__builtin_amdgcn_readlane((int)v, 31)),
             max((unsigned)__builtin_amdgcn_readlane((int)v, 47), (unsigned)__builtin_amdgcn_readlane((int)v, 63)));
}

__device__ __forceinline__ unsigned irec_slot(unsigned idx) { return idx + 3 + min((__umul24(idx, 37u) + 74u) >> 8, 3u); }

__device__ __forceinline__ unsigned wave_sum_u32(unsigned v) {
  v += (unsigned)__builtin_amdgcn_update_dpp(0, (int)v, 0x111, 0xf, 0xf, false);
  v += (unsigned)__builtin_amdgcn_update_dpp(0, (int)v, 0x112, 0xf, 0xf, false);
  v += (unsigned)__builtin_amdgcn_update_dpp(0, (int)v, 0x114, 0xf, 0xf, false);
  v += (unsigned)__builtin_amdgcn_update_dpp(0, (int)v, 0x118, 0xf, 0xf, false);
  return (unsigned)(__builtin_amdgcn_readlane((int)v, 15) + __builtin_amdgcn_readlane((int)v, 31) +
                    __builtin_amdgcn_readlane((int)v, 47) + __builtin_amdgcn_readlane((int)v, 63));
}
__device__ __forceinline__ unsigned bfi(unsigned m, unsigned a, unsigned b) { return (a & m) | (b & ~m); }
constexpr unsigned HDR_RAW = 0x80000000u;

constexpr unsigned IREC_CAP = 122, IHDR_RAW = 0x8000u;

struct InTask {
  u32x4 d[4];
  unsigned long long mask;
  bool raw;
};

__device__ __forceinline__ void put_records2(unsigned q0, unsigned q1, unsigned tag, unsigned& cnt, uint8_t* scr, unsigned dump) {
  unsigned m = bfi(0x55555555u, q0 | q0 >> 1, q1 | q1 << 1);
  for (unsigned long long act = __ballot(m != 0); act != 0; act = __ballot(m != 0)) {
    const unsigned bit = (unsigned)__builtin_ctz(m | 0x80000000u), sh = bit & 30u;
    const unsigned idx = cnt + __builtin_amdgcn_mbcnt_hi((unsigned)(act >> 32), __builtin_amdgcn_mbcnt_lo((unsigned)act, 0u));
    const unsigned sl = 2 * irec_slot(idx), at = ((m != 0) & (idx < IREC_CAP)) ? sl : dump;
    *(unsigned short*)(scr + at) = (unsigned short)(tag | (bit & 1u) << 6 | sh << 1 | ((((bit & 1u) ? q1 : q0) >> sh) & 3u));
    m &= m - 1;
    cnt += __popcll(act);
  }
}

__device__ __forceinline__ void enc_in(const u32x4 (&v)[4], InTask& o, unsigned tag, uint8_t* scr, int lane) {
  unsigned sum = 0;
#pragma unroll
  for (int q = 0; q < 4; ++q) sum = __builtin_amdgcn_udot4(v[q][0] & 0x7f7f7f7fu, 0x01010101u, sum, false);
  const unsigned e0 = (unsigned)min(max((int)((wave_sum_u32(sum) + 10445u) >> 12) - 7, 0), 24);
  const unsigned add = 0x80808080u - e0 * 0x04040404u;
  unsigned pay[12], qq[4], cnt = 0;
#pragma unroll
  for (int q = 0; q < 4; ++q) {
    unsigned R[4], S[4];
#pragma unroll
    for (int k = 0; k < 4; ++k) {
      R[k] = (v[q][k] & 0x7f7f7f7fu) + add;
      S[k] = bfi(0x80808080u, v[q][k], R[k]);
    }
    pay[3 * q] = bfi(0x60606060u, S[3] << 5, S[0]);
    pay[3 * q + 1] = bfi(0x60606060u, S[3] << 3, S[1]);
    pay[3 * q + 2] = bfi(0x20202020u, S[3] << 1, bfi(0x40404040u, S[3] >> 1, S[2]));
    qq[q] = bfi(0xc0c0c0c0u, R[3] << 1, bfi(0x30303030u, R[2] >> 1, bfi(0x0c0c0c0cu, R[1] >> 3, (R[0] >> 5) & 0x03030303u)));
  }
  put_records2(qq[0], qq[1], (unsigned)(lane << 8), cnt, scr, 256 + 2 * lane);
  put_records2(qq[2], qq[3], (unsigned)(lane << 8 | 128), cnt, scr, 256 + 2 * lane);
  *(unsigned short*)(scr + (lane < 4 ? lane * 16 : lane < 6 ? lane * 2 - 6 : 256 + 2 * lane)) =
      (unsigned short)(lane < 4 ? tag : lane == 4 ? cnt : e0);
  __builtin_amdgcn_fence(__ATOMIC_ACQ_REL, "wavefront");
  const u32x4 rec = *(const u32x4*)(scr + (lane & 15) * 16);
  __builtin_amdgcn_fence(__ATOMIC_ACQ_REL, "wavefront");
  o.raw = cnt > IREC_CAP;
  if (o.raw) {
#pragma unroll
    for (int q = 0; q < 4; ++q) o.d[q] = v[q];
    o.mask = ~0ull;
  } else {
#pragma unroll
    for (int q = 0; q < 3; ++q) o.d[q] = u32x4{pay[4 * q], pay[4 * q + 1], pay[4 * q + 2], pay[4 * q + 3]};
    o.d[3] = rec;
    o.mask = (1ull << (((irec_slot(cnt) + 31) >> 5) * 4)) - 16;
  }
}

__device__ __forceinline__ void dec_in4(const uint8_t* slot, uint8_t* out, const unsigned (&off)[4], unsigned stride, uint8_t* scr, int lane) {
  u32x4 y[4][3];
  unsigned rec[4];
  asm volatile(SGPR_PAD "global_load_dword %12, %16, %20 sc0 sc1\n\tglobal_load_dword %13, %16, %20 offset:3328 sc0 sc1\n\t"
               "global_load_dword %14, %17, %20 sc0 sc1\n\tglobal_load_dword %15, %17, %20 offset:3328 sc0 sc1\n\t"
               "global_load_dwordx4 %0, %18, %20 offset:-3072 sc0 sc1\n\tglobal_load_dwordx4 %1, %18, %20 offset:-2048 sc0 sc1\n\t"
               "global_load_dwordx4 %2, %18, %20 offset:-1024 sc0 sc1\n\t"
               "global_load_dwordx4 %3, %18, %20 offset:256 sc0 sc1\n\tglobal_load_dwordx4 %4, %18, %20 offset:1280 sc0 sc1\n\t"
               "global_load_dwordx4 %5, %18, %20 offset:2304 sc0 sc1\n\t"
               "global_load_dwordx4 %6, %19, %20 offset:-3072 sc0 sc1\n\tglobal_load_dwordx4 %7, %19, %20 offset:-2048 sc0 sc1\n\t"
               "global_load_dwordx4 %8, %19, %20 offset:-1024 sc0 sc1\n\t"
               "global_load_dwordx4 %9, %19, %20 offset:256 sc0 sc1\n\tglobal_load_dwordx4 %10, %19, %20 offset:1280 sc0 sc1\n\t"
               "global_load_dwordx4 %11, %19, %20 offset:2304 sc0 sc1\n\ts_waitcnt vmcnt(12)"
               : "=&v"(y[0][0]), "=&v"(y[0][1]), "=&v"(y[0][2]), "=&v"(y[1][0]), "=&v"(y[1][1]), "=&v"(y[1][2]),
                 "=&v"(y[2][0]), "=&v"(y[2][1]), "=&v"(y[2][2]), "=&v"(y[3][0]), "=&v"(y[3][1]), "=&v"(y[3][2]),
                 "=&v"(rec[0]), "=&v"(rec[1]), "=&v"(rec[2]), "=&v"(rec[3])
               : "v"(lane * 4), "v"(2 * ITASK + lane * 4), "v"(ITASK + lane * 16), "v"(3 * ITASK + lane * 16), "s"(slot) : "memory");
#pragma unroll
  for (int t = 0; t < 4; ++t) *(u32x4*)(scr + t * 1024 + lane * 16) = u32x4{0u, 0u, 0u, 0u};
  __builtin_amdgcn_fence(__ATOMIC_ACQ_REL, "wavefront");
  unsigned hdr[4];
#pragma unroll
  for (int t = 0; t < 4; ++t) {
    hdr[t] = (unsigned)__builtin_amdgcn_readlane((int)rec[t], 0) >> 16 | (unsigned)__builtin_amdgcn_readlane((int)rec[t], 1) << 16;
    const unsigned end = irec_slot(hdr[t] & 0x7fffu);
#pragma unroll
    for (int i = 0; i < 2; ++i) {
      const unsigned r16 = rec[t] >> (16 * i);
      const bool isrec = i == 0 ? (lane >= 16 || (lane & 3) != 0) && lane != 1 : lane != 0;
      const unsigned val = isrec && (unsigned)(lane * 2 + i) < end ? (r16 & 3u) << ((r16 >> 1) & 30u) : 0u;
      __hip_atomic_fetch_or((unsigned*)(scr + t * 1024 + ((r16 >> 6) & 255u) * 4), val, __ATOMIC_RELAXED, __HIP_MEMORY_SCOPE_WORKGROUP);
    }
  }
  __builtin_amdgcn_fence(__ATOMIC_ACQ_REL, "wavefront");
  asm volatile("s_waitcnt vmcnt(0)" : "+v"(y[0][0]), "+v"(y[0][1]), "+v"(y[0][2]), "+v"(y[1][0]), "+v"(y[1][1]), "+v"(y[1][2]),
               "+v"(y[2][0]), "+v"(y[2][1]), "+v"(y[2][2]), "+v"(y[3][0]), "+v"(y[3][1]), "+v"(y[3][2]) :: "memory");
#pragma unroll
  for (int t = 0; t < 4; ++t) {
    if (hdr[t] & IHDR_RAW) continue;
    const u32x4 qq = *(const u32x4*)(scr + t * 1024 + lane * 16);
    const unsigned add = ((hdr[t] >> 16) & 31u) * 0x04040404u;
#pragma unroll
    for (int q = 0; q < 4; ++q) {
      const unsigned d0 = y[t][(3 * q) >> 2][(3 * q) & 3], d1 = y[t][(3 * q + 1) >> 2][(3 * q + 1) & 3], d2 = y[t][(3 * q + 2) >> 2][(3 * q + 2) & 3];
      const unsigned d3 = bfi(0x80808080u, d2 << 1, bfi(0x10101010u, d2 >> 1, bfi(0x0c0c0c0cu, d1 >> 3, d0 >> 5)));
      const unsigned X[4] = {bfi(0x60606060u, qq[q] << 5, d0), bfi(0x60606060u, qq[q] << 3, d1), bfi(0x60606060u, qq[q] << 1, d2),
                             bfi(0x60606060u, qq[q] >> 1, d3)};
      u32x4 r;
#pragma unroll
      for (int k = 0; k < 4; ++k) r[k] = bfi(0x80808080u, X[k], (X[k] & 0x7f7f7f7fu) + add);
      asm volatile(SGPR_PAD "global_store_dwordx4 %0, %1, %2 sc1\n\ts_nop 1" :: "v"(off[t] + q * stride + lane * 16), "v"(r), "s"(out) : "memory");
    }
  }
  __builtin_amdgcn_fence(__ATOMIC_ACQ_REL, "wavefront");
}

__device__ __forceinline__ bool wait_tasks(const uint8_t* slot, unsigned* tmo, unsigned call, int lane) {
  const uint8_t* q = slot + ((lane >> 2) & 3) * ITASK + (lane & 3) * 16;
  return spin([&] { return __all((ld_sys(q) & 0xffffu) == (call & 0xffffu)) != 0; }, tmo, call);
}

struct PushCtx {
  const uint8_t* x;
  const uint8_t* w;
  uint8_t* dst;
  unsigned* lk;
  uint8_t* scr;
  unsigned xl, wl;
  unsigned par, tick0, tag, call;
  int rank, d;
  bool local;
};

struct RowSet { int r, row0, base, task0; };
__device__ __forceinline__ RowSet row_set(int set) {
  if (set < 3) return {7, 0, set * 7, set * 28};
  if (set < 7) return {9, 224, 9 * set - 6, 36 * set - 24};
  return {8, (set - 7) * 256, 8 * set + 1, T_OWN + (set - 7) * 32};
}
__device__ __forceinline__ int block_col(int q, int n) { return ((n & 3) * 4 + (q < 4 ? q : (q ^ n) & 3)) * 1024 + (n >> 2) * 128; }
__device__ __forceinline__ int tile_block(int g, int i) {
  if (g < 4) return i;
  if (i >= 24) return (g - 4) * 8 + i - 24;
  const int c = i / 3, m = i - 3 * c;
  return 4 * c + m + (m >= g - 4);
}
__device__ __forceinline__ bool task_is_w(int t) { return t >= T_W0 && t < T_OWN; }
__device__ __forceinline__ int task_set(int t) { return t < 84 ? t / 28 : t < T_W0 ? 3 + (t - 84) / 36 : 7 + ((t - T_OWN) >> 5); }

__device__ __forceinline__ void ld_x(const PushCtx& c, int t, u32x4 (&v)[4]) {
  t = min(t, NTASK - 1);
  const int set = task_set(t);
  const RowSet rs = row_set(set);
  const int p0 = 2 * (t - rs.task0), ks = (p0 * (rs.r == 7 ? 147 : rs.r == 8 ? 128 : 114)) >> 10, rg = p0 - ks * rs.r;
  const unsigned o0 = c.xl + (unsigned)((((set < 7 ? set + 1 : 0) ^ c.d) * 512 + rs.row0 + rg * 32) * KS + ks * 64);
  const unsigned o1 = o0 + (unsigned)(rg + 1 == rs.r ? 64 - rg * 32 * KS : 32 * KS);
  asm volatile(SGPR_PAD "global_load_dwordx4 %0, %4, %6\n\tglobal_load_dwordx4 %1, %4, %6 offset:16\n\t"
               "global_load_dwordx4 %2, %5, %6\n\tglobal_load_dwordx4 %3, %5, %6 offset:16"
               : "=&v"(v[0]), "=&v"(v[1]), "=&v"(v[2]), "=&v"(v[3]) : "v"(o0), "v"(o1), "s"(c.x) : "memory");
}

__device__ __forceinline__ void pack_task(const PushCtx& c, const u32x4 (&v)[4], InTask& o, int lane, bool plain) {
  if (plain) {
#pragma unroll
    for (int q = 0; q < 4; ++q) o.d[q] = v[q];
    o.mask = ~0ull;
    o.raw = true;
  } else {
    enc_in(v, o, c.tag, c.scr, lane);
  }
}

struct PushState {
  int tp, tc, tn, tl;
  u32x4 line;
  InTask o;
  u32x4 nx[4];
};

template <bool LDW>
__device__ __forceinline__ void push_step(const Params& p, const PushCtx& c, PushState& s, int lane) {
  unsigned w[16], tk;
  u32x4 x[4];
  const bool draw = s.tl < NTASK;
  asm volatile(SGPR_PAD "s_mov_b64 exec, 1\n\tglobal_atomic_add %0, %1, %2, %3 sc0\n\ts_mov_b64 exec, -1"
               : "=&v"(tk) : "v"(0u), "v"(draw ? 1u : 0u), "s"(c.lk) : "memory");
  if (s.tl == OPEN_AT) st1_agent(c.lk + 32, c.call);
  if constexpr (LDW) {
    const int u = s.tl - T_W0;
    unsigned o = c.wl + (unsigned)((((u >> 1) & 7) * 64 + (u & 1) * 16) * GN + block_col(c.d, u >> 4));
    asm volatile(SGPR_PAD
                 "global_load_dword %0, %16, %17\n\tv_add_u32 %16, 0x4000, %16\n\t"
                 "global_load_dword %1, %16, %17\n\tv_add_u32 %16, 0x4000, %16\n\t"
                 "global_load_dword %2, %16, %17\n\tv_add_u32 %16, 0x4000, %16\n\t"
                 "global_load_dword %3, %16, %17\n\tv_add_u32 %16, 0x4000, %16\n\t"
                 "global_load_dword %4, %16, %17\n\tv_add_u32 %16, 0x4000, %16\n\t"
                 "global_load_dword %5, %16, %17\n\tv_add_u32 %16, 0x4000, %16\n\t"
                 "global_load_dword %6, %16, %17\n\tv_add_u32 %16, 0x4000, %16\n\t"
                 "global_load_dword %7, %16, %17\n\tv_add_u32 %16, 0x4000, %16\n\t"
                 "global_load_dword %8, %16, %17\n\tv_add_u32 %16, 0x4000, %16\n\t"
                 "global_load_dword %9, %16, %17\n\tv_add_u32 %16, 0x4000, %16\n\t"
                 "global_load_dword %10, %16, %17\n\tv_add_u32 %16, 0x4000, %16\n\t"
                 "global_load_dword %11, %16, %17\n\tv_add_u32 %16, 0x4000, %16\n\t"
                 "global_load_dword %12, %16, %17\n\tv_add_u32 %16, 0x4000, %16\n\t"
                 "global_load_dword %13, %16, %17\n\tv_add_u32 %16, 0x4000, %16\n\t"
                 "global_load_dword %14, %16, %17\n\tv_add_u32 %16, 0x4000, %16\n\t"
                 "global_load_dword %15, %16, %17\n\t"
                 : "=&v"(w[0]), "=&v"(w[1]), "=&v"(w[2]), "=&v"(w[3]), "=&v"(w[4]), "=&v"(w[5]), "=&v"(w[6]), "=&v"(w[7]),
                   "=&v"(w[8]), "=&v"(w[9]), "=&v"(w[10]), "=&v"(w[11]), "=&v"(w[12]), "=&v"(w[13]), "=&v"(w[14]), "=&v"(w[15]), "+v"(o) : "s"(c.w) : "memory");
    static_assert(GN == 0x4000, "");
  } else {
    ld_x(c, s.tl, x);
  }
  const unsigned is0 = c.par + (unsigned)(XS_BYTES + WT_BYTES) + (unsigned)(c.rank * NTASK) * ITASK + lane * 16;
  if (s.tp >= 0) st16_sys_m(c.dst, is0 + (unsigned)s.tp * ITASK, s.line, 15ull);
  if (s.tc < NTASK) {
    if (!s.o.raw) {
      const unsigned slot = is0 + (unsigned)s.tc * ITASK;
      asm volatile(SGPR_PAD "global_store_dwordx4 %0, %1, %4 offset:256 sc0 sc1\n\tglobal_store_dwordx4 %0, %2, %4 offset:1280 sc0 sc1\n\t"
                   "global_store_dwordx4 %0, %3, %4 offset:2304 sc0 sc1\n\ts_nop 1"
                   :: "v"(slot), "v"(s.o.d[0]), "v"(s.o.d[1]), "v"(s.o.d[2]), "s"(c.dst) : "memory");
      if (s.o.mask != 0) st16_sys_m(c.dst, slot, s.o.d[3], s.o.mask);
      s.line = s.o.d[3];
    } else {
      const int t = s.tc, u = t - T_W0;
      const bool wr = task_is_w(t);
      const RowSet rs = row_set(task_set(t));
      const unsigned plain = c.par + lane * 16 +
          (wr ? (unsigned)XS_BYTES + (unsigned)((u >> 4) * (512 << 10) + (c.rank * 8 + ((u >> 1) & 7)) * 8192 + (u & 1) * 1024)
              : (unsigned)(rs.base * (128 << 10) + c.rank * rs.r * 16384 + (t - rs.task0) * 4096));
#pragma unroll
      for (int q = 0; q < 4; ++q) st16_sys(c.dst, plain + q * (wr ? 2048 : 1024), s.o.d[q]);
      s.line = u32x4{c.tag | IHDR_RAW << 16, 0u, 0u, 0u};
    }
  }
  if (s.tn < NTASK) pack_task(c, s.nx, s.o, lane, c.local);
  if constexpr (LDW) {
    asm volatile("s_waitcnt vmcnt(0)" : "+v"(tk), "+v"(w[0]), "+v"(w[1]), "+v"(w[2]), "+v"(w[3]), "+v"(w[4]), "+v"(w[5]), "+v"(w[6]), "+v"(w[7]),
                 "+v"(w[8]), "+v"(w[9]), "+v"(w[10]), "+v"(w[11]), "+v"(w[12]), "+v"(w[13]), "+v"(w[14]), "+v"(w[15]) :: "memory");
#pragma unroll
    for (int q = 0; q < 4; ++q) {
      const unsigned d0 = w[4 * q], d1 = w[4 * q + 1], d2 = w[4 * q + 2], d3 = w[4 * q + 3];
      const unsigned t0 = __builtin_amdgcn_perm(d1, d0, 0x05010400u), t1 = __builtin_amdgcn_perm(d1, d0, 0x07030602u);
      const unsigned t2 = __builtin_amdgcn_perm(d3, d2, 0x05010400u), t3 = __builtin_amdgcn_perm(d3, d2, 0x07030602u);
      s.nx[0][q] = __builtin_amdgcn_perm(t2, t0, 0x05040100u);
      s.nx[1][q] = __builtin_amdgcn_perm(t2, t0, 0x07060302u);
      s.nx[2][q] = __builtin_amdgcn_perm(t3, t1, 0x05040100u);
      s.nx[3][q] = __builtin_amdgcn_perm(t3, t1, 0x07060302u);
    }
  } else {
    asm volatile("s_waitcnt vmcnt(0)" : "+v"(tk), "+v"(x[0]), "+v"(x[1]), "+v"(x[2]), "+v"(x[3]) :: "memory");
#pragma unroll
    for (int q = 0; q < 4; ++q) s.nx[q] = x[q];
  }
  if (s.tc < NTASK) STAMP(min(8 + (s.tc >> 5), 28));
  s.tp = s.tc; s.tc = s.tn; s.tn = s.tl;
  s.tl = draw ? (int)(__builtin_amdgcn_readfirstlane(tk) - c.tick0) : NTASK;
}

#define GLDS_BITS " sc0 sc1"
__device__ __forceinline__ void glds2(const uint8_t* gbase, unsigned voff, unsigned lds) {
  unsigned keep;
  asm volatile("s_mov_b32 %0, m0\n\ts_mov_b32 m0, %3\n\ts_nop 2\n\t"
               "global_load_lds_dwordx4 %1, %2" GLDS_BITS "\n\t"
               "global_load_lds_dwordx4 %1, %2 offset:1024" GLDS_BITS "\n\t"
               "s_mov_b32 m0, %0"
               : "=&s"(keep) : "v"(voff), "s"(gbase), "s"(lds));
}
__device__ __forceinline__ void glds2o(const uint8_t* gbase, unsigned voff, unsigned lds) {
  unsigned keep;
  asm volatile("s_mov_b32 %0, m0\n\ts_mov_b32 m0, %3\n\ts_nop 2\n\t"
               "global_load_lds_dwordx4 %1, %2 offset:2048" GLDS_BITS "\n\t"
               "global_load_lds_dwordx4 %1, %2 offset:3072" GLDS_BITS "\n\t"
               "s_mov_b32 m0, %0"
               : "=&s"(keep) : "v"(voff), "s"(gbase), "s"(lds));
}
__device__ __forceinline__ void glds_half(const uint8_t* gbase, unsigned voff, unsigned lds) {
  unsigned keep;
  asm volatile("s_mov_b32 %0, m0\n\ts_mov_b32 m0, %3\n\ts_mov_b32 exec_hi, 0\n\ts_nop 1\n\t"
               "global_load_lds_dwordx4 %1, %2" GLDS_BITS "\n\t"
               "s_mov_b32 exec_hi, -1\n\ts_mov_b32 m0, %0"
               : "=&s"(keep) : "v"(voff), "s"(gbase), "s"(lds));
}

struct Frags { i32x8 a[2], b[4], a9, b9; };

__device__ __forceinline__ i32x8 ld_frag(const uint8_t* q) {
  const i32x4 lo = *(const i32x4*)q, hi = *(const i32x4*)(q + 1024);
  return __builtin_shufflevector(lo, hi, 0, 1, 2, 3, 4, 5, 6, 7);
}

#define MFMA(mi, j) acc[mi][j] = __builtin_amdgcn_mfma_scale_f32_32x32x64_f8f6f4(cur.a[mi], cur.b[j], acc[mi][j], 1, 1, 0, 0, 0, 0)
#define SB() __builtin_amdgcn_sched_barrier(0)

struct Ring {
  const uint8_t* ga;
  const uint8_t* gb;
  unsigned astep;
  unsigned va, vb, v9;
  unsigned wr, wr_end;
  unsigned wb, w9;
  unsigned rd;
  const uint8_t* qa;
  const uint8_t* qb;
  const uint8_t* q9;
  const uint8_t* la;
  const uint8_t* lb;
  const uint8_t* l9;
};

template <int MODE, int LEFT, bool DMA, bool RD>
__device__ __forceinline__ void kstep(f32x16 (&acc)[2][4], f32x16& acc9, const Frags& cur, Frags& nxt, Ring& g) {
  constexpr bool NINE = MODE == 9, TWO = MODE != 7;
  constexpr int STAGE = NINE ? 26624 : 24576;
  MFMA(0, 0); SB();
  asm volatile("s_waitcnt vmcnt(%0) lgkmcnt(0)\n\ts_barrier" :: "n"(LEFT * (NINE ? 7 : 6)) : "memory");
  SB();
  MFMA(0, 1); SB();
  if constexpr (RD) { nxt.a[0] = ld_frag(g.qa); nxt.b[0] = ld_frag(g.qb); }
  SB();
  MFMA(0, 2); SB();
  if constexpr (DMA) glds2(g.ga, g.va, g.wr);
  SB();
  MFMA(0, 3); SB();
  if constexpr (RD) { nxt.b[1] = ld_frag(g.qb + 2048); nxt.b[2] = ld_frag(g.qb + 4096); }
  SB();
  if constexpr (TWO) MFMA(1, 0);
  SB();
  if constexpr (DMA) glds2(g.gb, g.vb, g.wr + g.wb);
  SB();
  if constexpr (TWO) MFMA(1, 1);
  SB();
  if constexpr (RD) { nxt.b[3] = ld_frag(g.qb + 6144); if constexpr (TWO) nxt.a[1] = ld_frag(g.qa + 2048); }
  g.rd = g.rd == (NSTAGE - 1) * STAGE ? 0 : g.rd + STAGE;
  g.qa = g.la + g.rd;
  if constexpr (!NINE) g.qb = g.lb + g.rd;
  SB();
  if constexpr (TWO) MFMA(1, 2);
  SB();
  if constexpr (DMA) glds2o(g.ga, g.va, g.wr);
  SB();
  if constexpr (NINE) {
    if constexpr (TWO) MFMA(1, 3);
  SB();
    if constexpr (RD) { nxt.a9 = ld_frag(g.qb - 2048); nxt.b9 = ld_frag(g.q9); }
    g.qb = g.lb + g.rd; g.q9 = g.l9 + g.rd;
    if constexpr (DMA) g.gb += 8192;
    SB();
    acc9 = __builtin_amdgcn_mfma_scale_f32_32x32x64_f8f6f4(cur.a9, cur.b9, acc9, 1, 1, 0, 0, 0, 0); SB();
    if constexpr (DMA) glds_half(g.ga, g.v9, g.wr + g.w9);
  } else {
    if constexpr (TWO) MFMA(1, 3);
  SB();
    if constexpr (DMA) g.gb += 8192;
  }
  if constexpr (DMA) {
    g.ga += g.astep;
    g.wr += STAGE; g.wr = g.wr == g.wr_end ? g.wr - NSTAGE * STAGE : g.wr;
  }
  SB();
}

template <int MODE>
__device__ __forceinline__ void gemm_tile(const uint8_t* ga, unsigned astep, const uint8_t* gb, uint8_t* lds, int wave, int lane) {
  constexpr bool NINE = MODE == 9, TWO = MODE != 7;
  constexpr int STAGE = NINE ? 26624 : 24576, BOFF = STAGE - 8192;
  f32x16 acc[2][4], acc9;
  Ring g;
  g.ga = ga; g.gb = gb; g.astep = astep;
  g.va = lane * 16 + wave * 4096; g.vb = lane * 16 + wave * 2048; g.v9 = 16384 + lane * 16 + wave * 512;
  g.wr = (unsigned)(uintptr_t)lds + wave * 4096; g.wr_end = g.wr + NSTAGE * STAGE;
  g.wb = BOFF - wave * 2048; g.w9 = 16384 - wave * 3584;
#pragma unroll
  for (int s = 0; s < NSTAGE; ++s) {
    glds2(g.ga, g.va, g.wr + s * STAGE);
    glds2(g.gb, g.vb, g.wr + s * STAGE + g.wb);
    glds2o(g.ga, g.va, g.wr + s * STAGE);
    if constexpr (NINE) glds_half(g.ga, g.v9, g.wr + s * STAGE + g.w9);
    g.ga += astep; g.gb += 8192;
  }
#pragma unroll
  for (int r = 0; r < 16; ++r) {
    acc9[r] = 0.f;
#pragma unroll
    for (int j = 0; j < 4; ++j) { acc[0][j][r] = 0.f; acc[1][j][r] = 0.f; }
  }
  g.la = lds + lane * 16 + wave * 4096; g.lb = lds + BOFF + lane * 16; g.l9 = g.lb + wave * 2048;
  asm volatile("s_waitcnt vmcnt(%0)\n\ts_barrier" :: "n"(5 * (NINE ? 7 : 6)) : "memory");
  Frags f0, f1;
  f0.a[0] = ld_frag(g.la); f0.b[0] = ld_frag(g.lb); f0.b[1] = ld_frag(g.lb + 2048);
  f0.b[2] = ld_frag(g.lb + 4096); f0.b[3] = ld_frag(g.lb + 6144);
  if constexpr (TWO) f0.a[1] = ld_frag(g.la + 2048);
  if constexpr (NINE) { f0.a9 = ld_frag(g.lb - 2048); f0.b9 = ld_frag(g.l9); }
  g.rd = STAGE; g.qa = g.la + STAGE; g.qb = g.lb + STAGE; g.q9 = g.l9 + STAGE;
#pragma unroll 1
  for (int s = 0; s < KSTEPS - NSTAGE; s += 2) {
    kstep<MODE, 4, true, true>(acc, acc9, f0, f1, g);
    kstep<MODE, 4, true, true>(acc, acc9, f1, f0, g);
  }
  kstep<MODE, 4, false, true>(acc, acc9, f0, f1, g);
  kstep<MODE, 3, false, true>(acc, acc9, f1, f0, g);
  kstep<MODE, 2, false, true>(acc, acc9, f0, f1, g);
  kstep<MODE, 1, false, true>(acc, acc9, f1, f0, g);
  kstep<MODE, 0, false, true>(acc, acc9, f0, f1, g);
  auto park = [&](const f32x16& a, int i) {
#pragma unroll
    for (int q = 0; q < 4; ++q)
      *(f32x4*)(lds + wave * 32768 + (i * 4 + q) * 1024 + lane * 16) = f32x4{a[4 * q], a[4 * q + 1], a[4 * q + 2], a[4 * q + 3]};
  };
  {
    const Frags& cur = f1;
    MFMA(0, 0); SB();
    asm volatile("s_waitcnt lgkmcnt(0)\n\ts_barrier" ::: "memory");
    SB();
    MFMA(0, 1); SB();
    MFMA(0, 2); SB();
    park(acc[0][0], 0); SB();
    MFMA(0, 3); SB();
    park(acc[0][1], 1); SB();
    if constexpr (TWO) MFMA(1, 0);
    SB();
    park(acc[0][2], 2); SB();
    if constexpr (TWO) MFMA(1, 1);
    SB();
    park(acc[0][3], 3); SB();
    if constexpr (TWO) {
      MFMA(1, 2); SB();
      park(acc[1][0], 4); SB();
      MFMA(1, 3); SB();
      park(acc[1][1], 5); SB();
      if constexpr (NINE) { acc9 = __builtin_amdgcn_mfma_scale_f32_32x32x64_f8f6f4(cur.a9, cur.b9, acc9, 1, 1, 0, 0, 0, 0); SB(); }
      park(acc[1][2], 6);
      park(acc[1][3], 7);
    }
  }
  if constexpr (NINE) {
#pragma unroll
    for (int q = 0; q < 4; ++q)
      *(f32x4*)(lds + 8 * 16384 + wave * 4096 + q * 1024 + lane * 16) = f32x4{acc9[4 * q], acc9[4 * q + 1], acc9[4 * q + 2], acc9[4 * q + 3]};
    __syncthreads();
  }
  __builtin_amdgcn_fence(__ATOMIC_ACQ_REL, "wavefront");
}

constexpr unsigned REC_CAP = 510, HDR_B = 0x40000000u;

__device__ __forceinline__ void put_records4(const unsigned (&q)[4], unsigned tag, unsigned& cnt, uint8_t* scr, unsigned dump) {
  unsigned m = 0;
#pragma unroll
  for (int k = 0; k < 4; ++k)
    m |= ((((q[k] & 0x77777777u) + 0x77777777u) | q[k]) & 0x88888888u) >> (3 - k);
  for (unsigned long long act = __ballot(m != 0); act != 0; act = __ballot(m != 0)) {
    const unsigned bit = (unsigned)__builtin_ctz(m | 0x80000000u), sh = bit & 28u;
    const unsigned m1 = (unsigned)__builtin_amdgcn_sbfe((int)bit, 0, 1), m2 = (unsigned)__builtin_amdgcn_sbfe((int)bit, 1, 1);
    const unsigned w = bfi(m2, bfi(m1, q[3], q[2]), bfi(m1, q[1], q[0]));
    const unsigned idx = cnt + __builtin_amdgcn_mbcnt_hi((unsigned)(act >> 32), __builtin_amdgcn_mbcnt_lo((unsigned)act, 0u));
    const unsigned at = (m != 0 && idx < REC_CAP) ? 4 + 2 * idx : dump;
    *(unsigned short*)(scr + at) = (unsigned short)(tag | (bit & 3u) << 7 | sh << 2 | ((w >> sh) & 15u));
    m &= m - 1;
    cnt += __popcll(act);
  }
}

template <bool MODEB>
__device__ __forceinline__ bool enc_body(const float (&v)[4][16], float up, unsigned& cnt, uint8_t* blk, int lane) {
  unsigned bad = 0, low = 0;
#pragma unroll
  for (int h = 0; h < 2; ++h) {
    unsigned G[32];
#pragma unroll
    for (int j = 0; j < 4; ++j)
#pragma unroll
      for (int r = 0; r < 8; ++r) {
        const float x = v[j][8 * h + r] * up;
        unsigned g = (unsigned)(int)x;
        bad = __builtin_amdgcn_bitop3_b32(__builtin_bit_cast(unsigned, (float)(int)g), __builtin_bit_cast(unsigned, x), bad, 0xbe);
        low |= g;
        if (MODEB) g = g << 1 | (((g ^ g << 1) >> 27) & 16u);
        G[4 * r + j] = g;
      }
    unsigned q[4];
#pragma unroll
    for (int k = 0; k < 4; ++k) {
      const unsigned* g = G + 8 * k;
      const unsigned b0 = __builtin_amdgcn_perm(__builtin_amdgcn_perm(g[3], g[2], 0x0c0c0400u), __builtin_amdgcn_perm(g[1], g[0], 0x0c0c0400u), 0x05040100u);
      const unsigned b1 = __builtin_amdgcn_perm(__builtin_amdgcn_perm(g[7], g[6], 0x0c0c0400u), __builtin_amdgcn_perm(g[5], g[4], 0x0c0c0400u), 0x05040100u);
      q[k] = (b0 >> 4) | b1;
    }
#pragma unroll
    for (int j = 0; j < 2; ++j) {
      unsigned d[12];
#pragma unroll
      for (int g = 0; g < 4; ++g) {
        const unsigned* x = G + 16 * j + 4 * g;
        d[3 * g] = __builtin_amdgcn_perm(x[1], x[0], 0x05030201u);
        d[3 * g + 1] = __builtin_amdgcn_perm(x[2], x[1], 0x06050302u);
        d[3 * g + 2] = __builtin_amdgcn_perm(x[3], x[2], 0x07060503u);
      }
#pragma unroll
      for (int i = 0; i < 3; ++i)
        *(u32x4*)(blk + (1 + 6 * h + 3 * j + i) * 1024 + lane * 16) = u32x4{d[4 * i], d[4 * i + 1], d[4 * i + 2], d[4 * i + 3]};
    }
    put_records4(q, (unsigned)(lane << 10 | h << 9), cnt, blk, 13 * 1024 + 2 * lane);
  }
  return __ballot((bad | (low & 15u)) != 0) == 0 && cnt <= REC_CAP;
}

__device__ __forceinline__ unsigned enc_out(const float (&v)[4][16], float sc, uint8_t* blk, int lane) {
  float m = 0.f;
#pragma unroll
  for (int j = 0; j < 4; ++j)
#pragma unroll
    for (int r = 0; r < 16; r += 2) asm("v_max3_f32 %0, |%1|, |%2|, %0" : "+v"(m) : "v"(v[j][r]), "v"(v[j][r + 1]));
  const int e = max((int)(wave_max_u32(__builtin_bit_cast(unsigned, m)) >> 23) - 157, -126);
  const float up = __builtin_bit_cast(float, (unsigned)(127 - min(e, 0)) << 23);
  int ntop = 0, nlow = 0;
#pragma unroll
  for (int r = 0; r < 8; ++r) {
    const unsigned g = (unsigned)(int)(v[r & 3][2 * r] * up);
    ntop += __popcll(__ballot((int)(g ^ (g << 1)) < 0));
    nlow += __popcll(__ballot((g & 128u) != 0));
  }
  unsigned cnt = 0;
  const bool modeb = ntop < nlow;
  const bool fits = e <= 0 && (modeb ? enc_body<true>(v, up, cnt, blk, lane) : enc_body<false>(v, up, cnt, blk, lane));
  if (!fits) {
#pragma unroll
    for (int r = 0; r < 16; ++r) *(f32x4*)(blk + r * 1024 + lane * 16) = f32x4{v[0][r] * sc, v[1][r] * sc, v[2][r] * sc, v[3][r] * sc};
    return 0;
  }
  *(unsigned*)(blk + (lane == 0 ? 0 : 13 * 1024 + 4 * lane)) = cnt | (unsigned)(e + 127) << 16 | (modeb ? HDR_B : 0u);
  return ((4 + 2 * cnt + 63) >> 6) * 4;
}

__device__ __forceinline__ void st16_out(void* base, unsigned off, f32x4 v) {
  asm volatile(SGPR_PAD "global_store_dwordx4 %0, %1, %2 sc1\n\ts_nop 1" :: "v"(off), "v"(v), "s"(base) : "memory");
}

__device__ __forceinline__ constexpr unsigned row_off(int r) { return (unsigned)((r & 3) + 8 * (r >> 2)) * (GN * 4); }

__device__ __forceinline__ void send_out(const uint8_t* blk, unsigned nl, bool dead, uint8_t* slot, float* ob, unsigned off,
                                         int lane, unsigned* next, unsigned call, bool last, int depth) {
  const bool raw = nl == 0 || dead;
  const u32x4 h = *(const u32x4*)(blk + lane * 16);
  st16_sys_m(slot, lane * 16, raw ? u32x4{HDR_RAW, 0u, 0u, 0u} : h, raw ? 15ull : nl >= 64 ? ~0ull : (1ull << nl) - 1);
  const int n = raw ? 16 : 12;
  void* const base = raw ? (void*)ob : (void*)slot;
  const uint8_t* const src = blk + (raw ? 0 : 1024) + lane * 16;
  const unsigned nan = dead ? 0x7fc00000u : 0u;
#pragma unroll 1
  for (int i = 0; i < n; ++i) {
    const u32x4 v = *(const u32x4*)(src + i * 1024) | nan;
    if (depth == OUT_D) wait_vm<OUT_D - 1>(); else if (depth == OUT_D3) wait_vm<OUT_D3 - 1>(); else wait_vm<DEEP_D - 1>();
    if (last && i == n - OUT_REL) st1_agent(next, call);
    st16_sys(base, raw ? off + row_off(i) : (unsigned)(1024 + i * 1024 + lane * 16), v);
  }
}

__device__ __forceinline__ void dec_out(const uint8_t* slot, float* ob, unsigned off, float sc, unsigned* tmo, unsigned call, uint8_t* scr, int lane) {
  u32x4 rec, y[12];
  unsigned tmv;
  asm volatile(SGPR_PAD "global_load_dword %13, %16, %18 sc1\n\t"
               "global_load_dwordx4 %0, %14, %17 offset:-4096 sc0 sc1\n\tglobal_load_dwordx4 %1, %14, %17 offset:-3072 sc0 sc1\n\t"
               "global_load_dwordx4 %2, %14, %17 offset:-2048 sc0 sc1\n\tglobal_load_dwordx4 %3, %14, %17 offset:-1024 sc0 sc1\n\t"
               "global_load_dwordx4 %4, %14, %17 sc0 sc1\n\tglobal_load_dwordx4 %5, %14, %17 offset:1024 sc0 sc1\n\t"
               "global_load_dwordx4 %6, %14, %17 offset:2048 sc0 sc1\n\tglobal_load_dwordx4 %7, %14, %17 offset:3072 sc0 sc1\n\t"
               "global_load_dwordx4 %8, %15, %17 offset:-4096 sc0 sc1\n\tglobal_load_dwordx4 %9, %15, %17 offset:-3072 sc0 sc1\n\t"
               "global_load_dwordx4 %10, %15, %17 offset:-2048 sc0 sc1\n\tglobal_load_dwordx4 %11, %15, %17 offset:-1024 sc0 sc1\n\t"
               "global_load_dwordx4 %12, %15, %17 sc0 sc1\n\ts_waitcnt vmcnt(12)"
               : "=&v"(rec), "=&v"(y[0]), "=&v"(y[1]), "=&v"(y[2]), "=&v"(y[3]), "=&v"(y[4]), "=&v"(y[5]), "=&v"(y[6]),
                 "=&v"(y[7]), "=&v"(y[8]), "=&v"(y[9]), "=&v"(y[10]), "=&v"(y[11]), "=&v"(tmv)
               : "v"(4096 + lane * 16), "v"(12288 + lane * 16), "v"(0u), "s"(slot), "s"(tmo) : "memory");
  const unsigned hdr = __builtin_amdgcn_readfirstlane(rec[0]);
  const unsigned cnt = hdr & 0xffffu;
  const float down = __builtin_bit_cast(float, ((hdr >> 16) & 255u) << 23);
  *(u32x4*)(scr + lane * 32) = u32x4{0u, 0u, 0u, 0u};
  *(u32x4*)(scr + lane * 32 + 16) = u32x4{0u, 0u, 0u, 0u};
  __builtin_amdgcn_fence(__ATOMIC_ACQ_REL, "wavefront");
#pragma unroll
  for (int i = 0; i < 8; ++i) {
    const unsigned r16 = (rec[i >> 1] >> (16 * (i & 1))) & 0xffffu;
    const unsigned val = (unsigned)(lane * 8 + i - 2) < cnt ? (r16 & 15u) << ((r16 >> 2) & 28u) : 0u;
    __hip_atomic_fetch_or((unsigned*)(scr + (r16 >> 7) * 4), val, __ATOMIC_RELAXED, __HIP_MEMORY_SCOPE_WORKGROUP);
  }
  __builtin_amdgcn_fence(__ATOMIC_ACQ_REL, "wavefront");
  const u32x4 q0 = *(const u32x4*)(scr + lane * 32), q1 = *(const u32x4*)(scr + lane * 32 + 16);
  __builtin_amdgcn_fence(__ATOMIC_ACQ_REL, "wavefront");
  asm volatile("s_waitcnt vmcnt(0)" : "+v"(y[0]), "+v"(y[1]), "+v"(y[2]), "+v"(y[3]), "+v"(y[4]), "+v"(y[5]), "+v"(y[6]),
               "+v"(y[7]), "+v"(y[8]), "+v"(y[9]), "+v"(y[10]), "+v"(y[11]) :: "memory");
  if (__builtin_amdgcn_readfirstlane(tmv) == call) {
    const float n = __builtin_nanf("");
#pragma unroll 1
    for (int r = 0; r < 16; ++r) st16_out(ob, off + row_off(r), f32x4{n, n, n, n});
    return;
  }
  if (hdr & HDR_RAW) return;
  const bool modeb = (hdr & HDR_B) != 0;
#pragma unroll
  for (int g = 0; g < 16; ++g) {
    const unsigned d0 = y[(3 * g) >> 2][(3 * g) & 3], d1 = y[(3 * g + 1) >> 2][(3 * g + 1) & 3], d2 = y[(3 * g + 2) >> 2][(3 * g + 2) & 3];
    const unsigned qw = g < 8 ? q0[(g >> 1) & 3] : q1[(g >> 1) & 3];
    const unsigned q = ((g & 1) ? qw : qw << 4) & 0xf0f0f0f0u;
    unsigned G[4] = {__builtin_amdgcn_perm(d0, q, 0x06050400u), __builtin_amdgcn_perm(d1, d0, 0x0504030cu) | ((q >> 8) & 255u),
                     __builtin_amdgcn_perm(d2, d1, 0x0403020cu) | ((q >> 16) & 255u), __builtin_amdgcn_perm(d2, q, 0x07060503u)};
    f32x4 v;
#pragma unroll
    for (int j = 0; j < 4; ++j) {
      if (modeb) G[j] = ((unsigned)((int)G[j] >> 1) & ~8u) ^ (G[j] << 27);
      v[j] = ((float)(int)G[j] * down) * sc;
    }
    st16_out(ob, off + row_off(g), v);
  }
}

__device__ __forceinline__ int wave_blocks(int r, int wave) { return wave < 3 ? 2 : r - 6; }
__device__ __forceinline__ unsigned out_off(int row0, int col, int wave, int lane) {
  return (unsigned)(((row0 + wave * 64 + 4 * (lane >> 5)) * GN + col + 4 * (lane & 31)) * 4);
}

__device__ __forceinline__ void recv_tile(const Params& p, uint8_t* myws, float* out, int g, int src, int i, int k0, int k1, int wave, int lane,
                                          unsigned call, unsigned* tmo, float sc, uint8_t* scr) {
  const unsigned off = out_off(g < 4 ? 0 : 224, block_col(src, tile_block(g, i)), wave, lane);
  const uint8_t* const fl = myws + WS_DONE + (size_t)((src * 32 + i) * 4 + wave) * 128;
  const int nb = wave_blocks(g < 4 ? 7 : 9, wave);
#pragma unroll 1
  for (int k = k0; k < min(nb, k1); ++k) {
    spin<RX_SLEEP>([&] { return __builtin_amdgcn_readfirstlane(ld_sys(fl)) > 4 * call + k; }, tmo, call);
    STAMP(24);
    dec_out(myws + WS_OS + (size_t)((src * 32 + i) * 9 + wave * 2 + k) * OBLK, out, off + (unsigned)k * 32 * (GN * 4), sc, tmo, call, scr, lane);
  }
}

}

__global__ void __launch_bounds__(256, 1) gemm_rs_fused(const Params p) {
  __shared__ __attribute__((aligned(16))) uint8_t lds[LDS_BYTES];
  const int b = blockIdx.x;
  const int lane = threadIdx.x & 63;
  const int wave = __builtin_amdgcn_readfirstlane(threadIdx.x >> 6);
  const int rank = p.rank;
  uint8_t* const myws = p.myws;
  STAMP(0);

  const int grp = b & 7, dst = grp ^ rank, ti = 31 - (b >> 3);
  const bool local = grp == 0 || (grp >= 4 && ti >= 24);
  const int set = local ? 7 + (grp != 0) : grp - 1, cb = tile_block(grp, ti);
  const RowSet rs = row_set(set);
  unsigned* const callp = (unsigned*)(myws + WS_CALL + (size_t)b * 128);
  unsigned* const tmo = (unsigned*)(myws + WS_TMO);
  unsigned* const lk = (unsigned*)(myws + WS_LINK) + grp * 64;
  uint8_t* const scr = lds + LDS_PK + wave * 2048;

  unsigned call;
  if (ti >= 24) {
    const int w = (31 - ti) * 4 + wave;
    PushCtx c;
    c.x = p.x; c.w = p.w; c.dst = p.ws[dst]; c.lk = lk; c.rank = rank; c.d = dst;
    c.xl = (unsigned)((lane & 31) * KS + (lane >> 5) * 32);
    c.wl = (unsigned)((lane >> 5) * 32 * GN + (lane & 31) * 4);
    c.scr = scr;
    c.local = grp == 0;
    PushState s;
    u32x4 x0[4];
    unsigned cv;
    asm volatile(SGPR_PAD "global_load_dword %0, %1, %2 sc1" : "=v"(cv) : "v"(0u), "s"(callp) : "memory");
    ld_x(c, w, x0);
    ld_x(c, w + 32, s.nx);
    asm volatile("s_waitcnt vmcnt(0)" : "+v"(cv), "+v"(x0[0]), "+v"(x0[1]), "+v"(x0[2]), "+v"(x0[3]),
                 "+v"(s.nx[0]), "+v"(s.nx[1]), "+v"(s.nx[2]), "+v"(s.nx[3]) :: "memory");
    call = __builtin_amdgcn_readfirstlane(cv) + 1;
    c.par = (unsigned)(WS_DATA + (call & 1) * PAR_BYTES);
    c.tick0 = (call - 1) * (NTASK - 64) - 96;
    c.tag = call & 0xffffu;
    c.call = call;
    s.tp = -1; s.tc = w; s.tn = w + 32; s.tl = w + 64;
    s.line = u32x4{0u, 0u, 0u, 0u};
    pack_task(c, x0, s.o, lane, true);
    STAMP(7);
#pragma unroll 1
    while (s.tp < NTASK) {
      if (task_is_w(s.tl)) push_step<true>(p, c, s, lane);
      else push_step<false>(p, c, s, lane);
    }
    STAMP(30);
  } else {
    call = __builtin_amdgcn_readfirstlane(ld_agent(callp)) + 1;
  }
  uint8_t* const par = myws + WS_DATA + (size_t)(call & 1) * PAR_BYTES;

#pragma unroll 1
  for (int job = 0; job < 4; ++job) {
    const bool jw = grp == 0 ? job < 3 : job == 1;
    const int beta = grp == 0 ? (ti * 4 + wave) & 63 : (23 - ti) * 4 + wave;
    if (grp == 0 ? (job == 2 && ti >= 24) : (ti >= 24 || job >= 2 || (job == 0 && beta >= 8 * rs.r))) continue;
    const int n = job == 2 ? 24 + (ti & 7) : cb;
    const RowSet js = row_set(grp == 0 ? 7 + (ti >> 4) : set);
    const int bs = (beta * (js.r == 7 ? 147 : js.r == 8 ? 128 : 114)) >> 10, bu = beta - bs * js.r;
    const int src = !jw ? bs : grp != 0 ? dst : job == 0 ? rank : job == 1 ? rank ^ (4 + (ti & 3)) : rank ^ (1 + (ti >> 3));
    const int t0 = jw ? T_W0 + n * 16 + wave * 4 : js.task0 + bu * 4;
    const uint8_t* const slot = par + XS_BYTES + WT_BYTES + (size_t)(src * NTASK + t0) * ITASK;
    if (!wait_tasks(slot, tmo, call, lane)) break;
    if (jw) STAMP(26);
    if (src != rank) {
      const unsigned base = jw ? (unsigned)XS_BYTES + (unsigned)(n * (512 << 10) + (src * 8 + wave * 2) * 8192)
                               : (unsigned)(js.base * (128 << 10) + (src * js.r + bu) * 16384);
      unsigned off[4];
#pragma unroll
      for (int t = 0; t < 4; ++t) off[t] = base + (jw ? (t >> 1) * 8192 + (t & 1) * 1024 : t * 4096);
      dec_in4(slot, par, off, jw ? 2048 : 1024, lds + wave * 4096, lane);
      if (jw) STAMP(27);
      wait_vm<0>();
    }
    st1_agent(jw ? myws + WS_DW + (size_t)(n * 32 + src * 4 + wave) * 4
                 : myws + WS_DX + (size_t)((grp == 0 ? 7 + (ti >> 4) : set) * 128 + beta) * 4, call);
  }
  STAMP(29);

  unsigned* const ctrl = (unsigned*)(lds + LDS_CTRL);
  if (wave == 0) {
    const uint8_t* const fw = myws + WS_DW + (size_t)(cb * 32 + (lane & 31)) * 4;
    const uint8_t* const fx = myws + WS_DX + (size_t)(set * 128 + min(lane, 8 * rs.r - 1)) * 4;
    const uint8_t* const fy = myws + WS_DX + (size_t)(set * 128 + min(64 + lane, 8 * rs.r - 1)) * 4;
    const bool ok = spin([&] { const unsigned a = ld_agent(fw), c = ld_agent(fx), d = ld_agent(fy); return __all(min(min(a, c), d) >= call) != 0; },
                         tmo, call);
    if (lane == 0) ctrl[0] = !ok;
  }
  __syncthreads();
  const bool dead0 = __builtin_amdgcn_readfirstlane(ctrl[0]) != 0;
  STAMP(1); CSTAMP(1);

  if (wave == 0) st1_agent(callp, call);
  const float sc = p.sx[0] * p.sw[0];
  const unsigned off = out_off(rs.row0, block_col(rank, cb), wave, lane);
  float* const ob = p.out[local ? rank : dst];
  uint8_t* const blk = lds + wave * 32768;
  uint8_t* const oslot = p.ws[dst] + WS_OS + (size_t)((rank * 32 + ti) * 9 + wave * 2) * OBLK;
  unsigned* const flag = (unsigned*)(p.ws[dst] + WS_DONE + (size_t)((rank * 32 + ti) * 4 + wave) * 128);
  const int nb = wave_blocks(rs.r, wave);
  const unsigned* const turn = ti < NTURN ? lk + 32 : (const unsigned*)(myws + WS_TURN + (size_t)((grp * 32 + ti) * 4 + wave) * 128);
  unsigned* const next = (unsigned*)(myws + WS_TURN + (size_t)((grp * 32 + ((ti + NTURN) & 31)) * 4 + wave) * 128);
  const int depth = ti < DEEP_T ? DEEP_D : nb == 3 ? OUT_D3 : OUT_D;
  unsigned nl0 = 0, nl1 = 0, nl2 = 0;
  int sent = 0;

  if (!dead0) {
    const uint8_t* const ga = par + (size_t)rs.base * (128 << 10);
    const uint8_t* const gb = par + XS_BYTES + (size_t)cb * (512 << 10);
    if (rs.r == 9) gemm_tile<9>(ga, 9 * 2048, gb, lds, wave, lane);
    else if (nb == 1) gemm_tile<7>(ga, 7 * 2048, gb, lds, wave, lane);
    else gemm_tile<8>(ga, rs.r * 2048, gb, lds, wave, lane);
    STAMP(2); CSTAMP(2);
    unsigned tnv, tmv;
    asm volatile(SGPR_PAD "global_load_dword %0, %2, %3 sc1\n\tglobal_load_dword %1, %2, %4 sc1"
                 : "=&a"(tnv), "=&a"(tmv) : "v"(0u), "s"(turn), "s"(tmo) : "memory");
    auto landed = [&](unsigned& s_tn, unsigned& s_tm) {
      unsigned t0, t1;
      asm volatile("s_waitcnt vmcnt(0)\n\tv_accvgpr_read_b32 %2, %4\n\tv_accvgpr_read_b32 %3, %5\n\ts_nop 1\n\t"
                   "v_readfirstlane_b32 %0, %2\n\tv_readfirstlane_b32 %1, %3\n\ts_nop 1"
                   : "=&s"(s_tn), "=&s"(s_tm), "=&v"(t0), "=&v"(t1) : "a"(tnv), "a"(tmv) : "memory");
    };
    float s0 = sc;
#pragma unroll 1
    for (int k = 0; k < nb; ++k) {
      float v[4][16];
#pragma unroll
      for (int i = 0; i < 16; ++i) {
        const f32x4 t = *(const f32x4*)(blk + k * 16384 + i * 1024 + lane * 16);
#pragma unroll
        for (int q = 0; q < 4; ++q) v[i >> 2][(i & 3) * 4 + q] = t[q];
      }
      __builtin_amdgcn_fence(__ATOMIC_ACQ_REL, "wavefront");
      if (local) {
        if (k == 0) {
          unsigned tn, t;
          landed(tn, t);
          if (t == call) s0 = __builtin_nanf("");
        }
#pragma unroll
        for (int r = 0; r < 16; ++r)
          st16_out(ob, off + k * 32 * (GN * 4) + row_off(r), f32x4{v[0][r] * s0, v[1][r] * s0, v[2][r] * s0, v[3][r] * s0});
        continue;
      }
      const unsigned nl = enc_out(v, sc, blk + k * 16384, lane);
      __builtin_amdgcn_fence(__ATOMIC_ACQ_REL, "wavefront");
      if (k == 0) nl0 = nl; else if (k == 1) nl1 = nl; else nl2 = nl;
      if (k == 0) {
        unsigned tn, t;
        landed(tn, t);
        if (ti < 24 && nb > 1 && t != call && (tn >= call || __builtin_amdgcn_readfirstlane(ld_agent(turn)) >= call)) {
          STAMP(3);
          send_out(blk, nl, false, oslot, ob, off, lane, next, call, false, depth);
          sent = 1;
        }
      }
    }
  } else if (local) {
    const float n = __builtin_nanf("");
#pragma unroll 1
    for (int t = 0; t < 32; ++t) st16_out(ob, off + (t >> 4) * 32 * (GN * 4) + row_off(t & 15), f32x4{n, n, n, n});
  }
  STAMP(31); CSTAMP(31);
  if (grp == 0) return;

  const int last8 = grp < 4 ? 24 : 16;
  if (ti >= 24) recv_tile(p, myws, p.out[rank], grp, dst, ti - 24, 0, 3, wave, lane, call, tmo, sc, scr);
  if (!local) {
    if (sent) {
      wait_vm<0>();
      STAMP(20);
      st1_sys(flag, 4 * call + 1);
    } else {
      spin([&] { return __builtin_amdgcn_readfirstlane(ld_agent(turn)) >= call; }, tmo, call);
      STAMP(3);
    }
    const bool dead = dead0 || __builtin_amdgcn_readfirstlane(ld_agent(tmo)) == call;
#pragma unroll 1
    for (int k = sent; k < nb; ++k) {
      send_out(blk + k * 16384, k == 0 ? nl0 : k == 1 ? nl1 : nl2, dead, oslot + k * OBLK, ob, off + k * 32 * (GN * 4), lane, next, call, k == nb - 1, depth);
      if (k == nb - 1) STAMP(4);
      wait_vm<0>();
      STAMP(20 + k);
      st1_sys(flag, 4 * call + 1 + k);
    }
    STAMP(5);
  }
#pragma unroll 1
  for (int duty = 0; duty < 2; ++duty) {
    int i, k0, k1;
    if (duty == 0) {
      if (ti >= last8) continue;
      i = ti + 8; k0 = 0; k1 = i >= last8 ? 1 : 3;
    } else if (ti < 8) {
      i = last8 + ti; k0 = grp < 4 ? 1 : 2; k1 = 3;
    } else if (local) {
      i = ti - 8; k0 = 1; k1 = 2;
    } else {
      continue;
    }
    recv_tile(p, myws, p.out[rank], grp, dst, i, k0, k1, wave, lane, call, tmo, sc, scr);
  }
  STAMP(6);
}

extern "C" void kernel_launch(int rank, int nranks, ncclComm_t comm, hipStream_t stream,
                              void* const* d_in, const int* in_sizes, int n_in,
                              void* d_out, int out_size, const int* in_dtypes, int out_dtype,
                              void* const* d_out_all, void* const* d_ws_all, size_t ws_bytes) {
  (void)comm; (void)in_sizes; (void)in_dtypes; (void)out_dtype; (void)d_out;
  if (nranks != NR || (unsigned)rank >= (unsigned)NR || n_in != 4 || ws_bytes < WS_NEED || out_size != 512 * GN) {
    fprintf(stderr, "kernel_launch: unexpected problem shape\n");
    return;
  }
  Params p;
  p.x = (const uint8_t*)d_in[0];
  p.w = (const uint8_t*)d_in[1];
  p.sx = (const float*)d_in[2];
  p.sw = (const float*)d_in[3];
  for (int i = 0; i < NR; ++i) {
    p.ws[i] = (uint8_t*)d_ws_all[i];
    p.out[i] = (float*)d_out_all[i];
  }
  p.myws = p.ws[rank];
  p.rank = rank;
  p.pad = 0;
  hipLaunchKernelGGL(gemm_rs_fused, dim3(NWG), dim3(256), 0, stream, p);
  HIP_CHECK(hipGetLastError());
}
```

```cpp
#include <hip/hip_runtime.h>
#include <rccl/rccl.h>
#include <cstdio>
#include <cstdint>

#define HIP_CHECK(expr) do { hipError_t _e = (expr); if (_e != hipSuccess) { \
    fprintf(stderr, "HIP error %s at %s:%d\n", hipGetErrorString(_e), __FILE__, __LINE__); \
    return; } } while (0)

typedef unsigned u32x4 __attribute__((ext_vector_type(4)));
typedef int i32x4 __attribute__((ext_vector_type(4)));
typedef int i32x8 __attribute__((ext_vector_type(8)));
typedef float f32x4 __attribute__((ext_vector_type(4)));
typedef float f32x16 __attribute__((ext_vector_type(16)));

#define SGPR_PAD "s_nop 4\n\t"

namespace {

constexpr int NR = 8;
constexpr int GN = 16384;
constexpr int KS = 512;
constexpr int NWG = 256;
constexpr int KSTEPS = 64;
constexpr int NSTAGE = 6;
constexpr int LDS_CTRL = NSTAGE * 26624;
constexpr int LDS_BYTES = LDS_CTRL + 64;
constexpr int LDS_PK = 9 * 16384;

constexpr int T_W0 = 228, T_OWN = T_W0 + 512, NTASK = T_OWN + 64;

constexpr size_t WS_CALL = 0;
constexpr size_t WS_TMO = 32768;
constexpr size_t WS_DONE = 896u << 10;
constexpr size_t WS_LINK = WS_TMO + 1024;
constexpr size_t WS_DX = WS_LINK + 4096;
constexpr size_t WS_DW = WS_DX + 8192;
constexpr size_t WS_TURN = 768u << 10;
constexpr size_t WS_DBG = 65536;
constexpr size_t WS_DATA = 1u << 20;
constexpr size_t XS_BYTES = 10u << 20;
constexpr size_t WT_BYTES = 16u << 20;
constexpr unsigned ITASK = 3328;
constexpr size_t IS_BYTES = 21u << 20;
constexpr size_t PAR_BYTES = XS_BYTES + WT_BYTES + IS_BYTES;
static_assert((size_t)8 * NTASK * ITASK <= IS_BYTES, "");
constexpr size_t OBLK = 13 * 1024;
constexpr size_t WS_OS = WS_DATA + 2 * PAR_BYTES;
constexpr size_t WS_NEED = WS_OS + 8 * 32 * 9 * OBLK;

constexpr unsigned long long DEADLINE_TICKS = 50000000ull;

#define OPEN_AT 790
#define NTURN 20
#define OUT_REL 3
#define OUT_D 2
#define OUT_D3 3
#define DEEP_D 4
#define DEEP_T 8
#define WARM 1
#define POLL_SLEEP 6
#define RX_SLEEP 16

struct Params {
  const uint8_t* x;
  const uint8_t* w;
  const float* sx;
  const float* sw;
  uint8_t* ws[NR];
  float* out[NR];
  uint8_t* myws;
  int rank;
  int pad;
};

#define STAMP(slot) do {} while (0)
#define CSTAMP(slot) do {} while (0)

__device__ __forceinline__ unsigned ld_sys(const void* ptr) {
  return __hip_atomic_load((const unsigned*)ptr, __ATOMIC_RELAXED, __HIP_MEMORY_SCOPE_SYSTEM);
}
__device__ __forceinline__ unsigned ld_agent(const void* ptr) {
  return __hip_atomic_load((const unsigned*)ptr, __ATOMIC_RELAXED, __HIP_MEMORY_SCOPE_AGENT);
}
__device__ __forceinline__ void st_sys(void* ptr, unsigned v) {
  __hip_atomic_store((unsigned*)ptr, v, __ATOMIC_RELAXED, __HIP_MEMORY_SCOPE_SYSTEM);
}
__device__ __forceinline__ void st_agent(void* ptr, unsigned v) {
  __hip_atomic_store((unsigned*)ptr, v, __ATOMIC_RELAXED, __HIP_MEMORY_SCOPE_AGENT);
}

template <int SLEEP = POLL_SLEEP, class F>
__device__ __forceinline__ bool spin(F ready, unsigned* tmo, unsigned call) {
  const unsigned long long t0 = __builtin_amdgcn_s_memrealtime();
  for (unsigned n = 1;; ++n) {
    if (ready()) return true;
    if ((n & 3) == 0 && (__builtin_amdgcn_readfirstlane(ld_agent(tmo)) == call || __builtin_amdgcn_s_memrealtime() - t0 > DEADLINE_TICKS)) {
      st_agent(tmo, call);
      return false;
    }
    __builtin_amdgcn_s_sleep(SLEEP);
  }
}

__device__ __forceinline__ void st16_sys(void* base, unsigned off, u32x4 v) {
  asm volatile(SGPR_PAD "global_store_dwordx4 %0, %1, %2 sc0 sc1\n\ts_nop 1" :: "v"(off), "v"(v), "s"(base) : "memory");
}

__device__ __forceinline__ void st1_agent(void* base, unsigned v) {
  asm volatile(SGPR_PAD "s_mov_b64 exec, 1\n\tglobal_store_dword %0, %1, %2 sc1\n\ts_mov_b64 exec, -1"
               :: "v"(0u), "v"(v), "s"(base) : "memory");
}

__device__ __forceinline__ void st1_sys(void* base, unsigned v) {
  asm volatile(SGPR_PAD "s_mov_b64 exec, 1\n\tglobal_store_dword %0, %1, %2 sc0 sc1\n\ts_mov_b64 exec, -1"
               :: "v"(0u), "v"(v), "s"(base) : "memory");
}
__device__ __forceinline__ void st16_sys_m(void* base, unsigned off, u32x4 v, unsigned long long mask) {
  asm volatile(SGPR_PAD "s_mov_b64 exec, %3\n\tglobal_store_dwordx4 %0, %1, %2 sc0 sc1\n\ts_mov_b64 exec, -1\n\ts_nop 0"
               :: "v"(off), "v"(v), "s"(base), "s"(mask) : "memory");
}

template <int N> __device__ __forceinline__ void wait_vm() {
  asm volatile("s_waitcnt vmcnt(%0)" :: "n"(N) : "memory");
}

__device__ __forceinline__ unsigned wave_max_u32(unsigned v) {
  v = max(v, (unsigned)__builtin_amdgcn_update_dpp(0, (int)v, 0x111, 0xf, 0xf, false));
  v = max(v, (unsigned)__builtin_amdgcn_update_dpp(0, (int)v, 0x112, 0xf, 0xf, false));
  v = max(v, (unsigned)__builtin_amdgcn_update_dpp(0, (int)v, 0x114, 0xf, 0xf, false));
  v = max(v, (unsigned)__builtin_amdgcn_update_dpp(0, (int)v, 0x118, 0xf, 0xf, false));
  return max(max((unsigned)__builtin_amdgcn_readlane((int)v, 15), (unsigned)__builtin_amdgcn_readlane((int)v, 31)),
             max((unsigned)__builtin_amdgcn_readlane((int)v, 47), (unsigned)__builtin_amdgcn_readlane((int)v, 63)));
}

__device__ __forceinline__ unsigned irec_slot(unsigned idx) { return idx + 3 + min((__umul24(idx, 37u) + 74u) >> 8, 3u); }

__device__ __forceinline__ unsigned wave_sum_u32(unsigned v) {
  v += (unsigned)__builtin_amdgcn_update_dpp(0, (int)v, 0x111, 0xf, 0xf, false);
  v += (unsigned)__builtin_amdgcn_update_dpp(0, (int)v, 0x112, 0xf, 0xf, false);
  v += (unsigned)__builtin_amdgcn_update_dpp(0, (int)v, 0x114, 0xf, 0xf, false);
  v += (unsigned)__builtin_amdgcn_update_dpp(0, (int)v, 0x118, 0xf, 0xf, false);
  return (unsigned)(__builtin_amdgcn_readlane((int)v, 15) + __builtin_amdgcn_readlane((int)v, 31) +
                    __builtin_amdgcn_readlane((int)v, 47) + __builtin_amdgcn_readlane((int)v, 63));
}
__device__ __forceinline__ unsigned bfi(unsigned m, unsigned a, unsigned b) { return (a & m) | (b & ~m); }
constexpr unsigned HDR_RAW = 0x80000000u;

constexpr unsigned IREC_CAP = 122, IHDR_RAW = 0x8000u;

struct InTask {
  u32x4 d[4];
  unsigned long long mask;
  bool raw;
};

__device__ __forceinline__ void put_records2(unsigned q0, unsigned q1, unsigned tag, unsigned& cnt, uint8_t* scr, unsigned dump) {
  unsigned m = bfi(0x55555555u, q0 | q0 >> 1, q1 | q1 << 1);
  for (unsigned long long act = __ballot(m != 0); act != 0; act = __ballot(m != 0)) {
    const unsigned bit = (unsigned)__builtin_ctz(m | 0x80000000u), sh = bit & 30u;
    const unsigned idx = cnt + __builtin_amdgcn_mbcnt_hi((unsigned)(act >> 32), __builtin_amdgcn_mbcnt_lo((unsigned)act, 0u));
    const unsigned sl = 2 * irec_slot(idx), at = ((m != 0) & (idx < IREC_CAP)) ? sl : dump;
    *(unsigned short*)(scr + at) = (unsigned short)(tag | (bit & 1u) << 6 | sh << 1 | ((((bit & 1u) ? q1 : q0) >> sh) & 3u));
    m &= m - 1;
    cnt += __popcll(act);
  }
}

__device__ __forceinline__ void enc_in(const u32x4 (&v)[4], InTask& o, unsigned tag, uint8_t* scr, int lane) {
  unsigned sum = 0;
#pragma unroll
  for (int q = 0; q < 4; ++q) sum = __builtin_amdgcn_udot4(v[q][0] & 0x7f7f7f7fu, 0x01010101u, sum, false);
  const unsigned e0 = (unsigned)min(max((int)((wave_sum_u32(sum) + 10445u) >> 12) - 7, 0), 24);
  const unsigned add = 0x80808080u - e0 * 0x04040404u;
  unsigned pay[12], qq[4], cnt = 0;
#pragma unroll
  for (int q = 0; q < 4; ++q) {
    unsigned R[4], S[4];
#pragma unroll
    for (int k = 0; k < 4; ++k) {
      R[k] = (v[q][k] & 0x7f7f7f7fu) + add;
      S[k] = bfi(0x80808080u, v[q][k], R[k]);
    }
    pay[3 * q] = bfi(0x60606060u, S[3] << 5, S[0]);
    pay[3 * q + 1] = bfi(0x60606060u, S[3] << 3, S[1]);
    pay[3 * q + 2] = bfi(0x20202020u, S[3] << 1, bfi(0x40404040u, S[3] >> 1, S[2]));
    qq[q] = bfi(0xc0c0c0c0u, R[3] << 1, bfi(0x30303030u, R[2] >> 1, bfi(0x0c0c0c0cu, R[1] >> 3, (R[0] >> 5) & 0x03030303u)));
  }
  put_records2(qq[0], qq[1], (unsigned)(lane << 8), cnt, scr, 256 + 2 * lane);
  put_records2(qq[2], qq[3], (unsigned)(lane << 8 | 128), cnt, scr, 256 + 2 * lane);
  *(unsigned short*)(scr + (lane < 4 ? lane * 16 : lane < 6 ? lane * 2 - 6 : 256 + 2 * lane)) =
      (unsigned short)(lane < 4 ? tag : lane == 4 ? cnt : e0);
  __builtin_amdgcn_fence(__ATOMIC_ACQ_REL, "wavefront");
  const u32x4 rec = *(const u32x4*)(scr + (lane & 15) * 16);
  __builtin_amdgcn_fence(__ATOMIC_ACQ_REL, "wavefront");
  o.raw = cnt > IREC_CAP;
  if (o.raw) {
#pragma unroll
    for (int q = 0; q < 4; ++q) o.d[q] = v[q];
    o.mask = ~0ull;
  } else {
#pragma unroll
    for (int q = 0; q < 3; ++q) o.d[q] = u32x4{pay[4 * q], pay[4 * q + 1], pay[4 * q + 2], pay[4 * q + 3]};
    o.d[3] = rec;
    o.mask = (1ull << (((irec_slot(cnt) + 31) >> 5) * 4)) - 16;
  }
}

__device__ __forceinline__ void dec_in4(const uint8_t* slot, uint8_t* out, const unsigned (&off)[4], unsigned stride, uint8_t* scr, int lane) {
  u32x4 y[4][3];
  unsigned rec[4];
  asm volatile(SGPR_PAD "global_load_dword %12, %16, %20 sc0 sc1\n\tglobal_load_dword %13, %16, %20 offset:3328 sc0 sc1\n\t"
               "global_load_dword %14, %17, %20 sc0 sc1\n\tglobal_load_dword %15, %17, %20 offset:3328 sc0 sc1\n\t"
               "global_load_dwordx4 %0, %18, %20 offset:-3072 sc0 sc1\n\tglobal_load_dwordx4 %1, %18, %20 offset:-2048 sc0 sc1\n\t"
               "global_load_dwordx4 %2, %18, %20 offset:-1024 sc0 sc1\n\t"
               "global_load_dwordx4 %3, %18, %20 offset:256 sc0 sc1\n\tglobal_load_dwordx4 %4, %18, %20 offset:1280 sc0 sc1\n\t"
               "global_load_dwordx4 %5, %18, %20 offset:2304 sc0 sc1\n\t"
               "global_load_dwordx4 %6, %19, %20 offset:-3072 sc0 sc1\n\tglobal_load_dwordx4 %7, %19, %20 offset:-2048 sc0 sc1\n\t"
               "global_load_dwordx4 %8, %19, %20 offset:-1024 sc0 sc1\n\t"
               "global_load_dwordx4 %9, %19, %20 offset:256 sc0 sc1\n\tglobal_load_dwordx4 %10, %19, %20 offset:1280 sc0 sc1\n\t"
               "global_load_dwordx4 %11, %19, %20 offset:2304 sc0 sc1\n\ts_waitcnt vmcnt(12)"
               : "=&v"(y[0][0]), "=&v"(y[0][1]), "=&v"(y[0][2]), "=&v"(y[1][0]), "=&v"(y[1][1]), "=&v"(y[1][2]),
                 "=&v"(y[2][0]), "=&v"(y[2][1]), "=&v"(y[2][2]), "=&v"(y[3][0]), "=&v"(y[3][1]), "=&v"(y[3][2]),
                 "=&v"(rec[0]), "=&v"(rec[1]), "=&v"(rec[2]), "=&v"(rec[3])
               : "v"(lane * 4), "v"(2 * ITASK + lane * 4), "v"(ITASK + lane * 16), "v"(3 * ITASK + lane * 16), "s"(slot) : "memory");
#pragma unroll
  for (int t = 0; t < 4; ++t) *(u32x4*)(scr + t * 1024 + lane * 16) = u32x4{0u, 0u, 0u, 0u};
  __builtin_amdgcn_fence(__ATOMIC_ACQ_REL, "wavefront");
  unsigned hdr[4];
#pragma unroll
  for (int t = 0; t < 4; ++t) {
    hdr[t] = (unsigned)__builtin_amdgcn_readlane((int)rec[t], 0) >> 16 | (unsigned)__builtin_amdgcn_readlane((int)rec[t], 1) << 16;
    const unsigned end = irec_slot(hdr[t] & 0x7fffu);
#pragma unroll
    for (int i = 0; i < 2; ++i) {
      const unsigned r16 = rec[t] >> (16 * i);
      const bool isrec = i == 0 ? (lane >= 16 || (lane & 3) != 0) && lane != 1 : lane != 0;
      const unsigned val = isrec && (unsigned)(lane * 2 + i) < end ? (r16 & 3u) << ((r16 >> 1) & 30u) : 0u;
      __hip_atomic_fetch_or((unsigned*)(scr + t * 1024 + ((r16 >> 6) & 255u) * 4), val, __ATOMIC_RELAXED, __HIP_MEMORY_SCOPE_WORKGROUP);
    }
  }
  __builtin_amdgcn_fence(__ATOMIC_ACQ_REL, "wavefront");
  asm volatile("s_waitcnt vmcnt(0)" : "+v"(y[0][0]), "+v"(y[0][1]), "+v"(y[0][2]), "+v"(y[1][0]), "+v"(y[1][1]), "+v"(y[1][2]),
               "+v"(y[2][0]), "+v"(y[2][1]), "+v"(y[2][2]), "+v"(y[3][0]), "+v"(y[3][1]), "+v"(y[3][2]) :: "memory");
#pragma unroll
  for (int t = 0; t < 4; ++t) {
    if (hdr[t] & IHDR_RAW) continue;
    const u32x4 qq = *(const u32x4*)(scr + t * 1024 + lane * 16);
    const unsigned add = ((hdr[t] >> 16) & 31u) * 0x04040404u;
#pragma unroll
    for (int q = 0; q < 4; ++q) {
      const unsigned d0 = y[t][(3 * q) >> 2][(3 * q) & 3], d1 = y[t][(3 * q + 1) >> 2][(3 * q + 1) & 3], d2 = y[t][(3 * q + 2) >> 2][(3 * q + 2) & 3];
      const unsigned d3 = bfi(0x80808080u, d2 << 1, bfi(0x10101010u, d2 >> 1, bfi(0x0c0c0c0cu, d1 >> 3, d0 >> 5)));
      const unsigned X[4] = {bfi(0x60606060u, qq[q] << 5, d0), bfi(0x60606060u, qq[q] << 3, d1), bfi(0x60606060u, qq[q] << 1, d2),
                             bfi(0x60606060u, qq[q] >> 1, d3)};
      u32x4 r;
#pragma unroll
      for (int k = 0; k < 4; ++k) r[k] = bfi(0x80808080u, X[k], (X[k] & 0x7f7f7f7fu) + add);
      asm volatile(SGPR_PAD "global_store_dwordx4 %0, %1, %2 sc1\n\ts_nop 1" :: "v"(off[t] + q * stride + lane * 16), "v"(r), "s"(out) : "memory");
    }
  }
  __builtin_amdgcn_fence(__ATOMIC_ACQ_REL, "wavefront");
}

__device__ __forceinline__ bool wait_tasks(const uint8_t* slot, unsigned* tmo, unsigned call, int lane) {
  const uint8_t* q = slot + ((lane >> 2) & 3) * ITASK + (lane & 3) * 16;
  return spin([&] { return __all((ld_sys(q) & 0xffffu) == (call & 0xffffu)) != 0; }, tmo, call);
}

struct PushCtx {
  const uint8_t* x;
  const uint8_t* w;
  uint8_t* dst;
  unsigned* lk;
  uint8_t* scr;
  unsigned xl, wl;
  unsigned par, tick0, tag, call;
  int rank, d;
  bool local;
};

struct RowSet { int r, row0, base, task0; };
__device__ __forceinline__ RowSet row_set(int set) {
  if (set < 3) return {7, 0, set * 7, set * 28};
  if (set < 7) return {9, 224, 9 * set - 6, 36 * set - 24};
  return {8, (set - 7) * 256, 8 * set + 1, T_OWN + (set - 7) * 32};
}
__device__ __forceinline__ int block_col(int q, int n) { return ((n & 3) * 4 + (q < 4 ? q : (q ^ n) & 3)) * 1024 + (n >> 2) * 128; }
__device__ __forceinline__ int tile_block(int g, int i) {
  if (g < 4) return i;
  if (i >= 24) return (g - 4) * 8 + i - 24;
  const int c = i / 3, m = i - 3 * c;
  return 4 * c + m + (m >= g - 4);
}
__device__ __forceinline__ bool task_is_w(int t) { return t >= T_W0 && t < T_OWN; }
__device__ __forceinline__ int task_set(int t) { return t < 84 ? t / 28 : t < T_W0 ? 3 + (t - 84) / 36 : 7 + ((t - T_OWN) >> 5); }

__device__ __forceinline__ void ld_x(const PushCtx& c, int t, u32x4 (&v)[4]) {
  t = min(t, NTASK - 1);
  const int set = task_set(t);
  const RowSet rs = row_set(set);
  const int p0 = 2 * (t - rs.task0), ks = (p0 * (rs.r == 7 ? 147 : rs.r == 8 ? 128 : 114)) >> 10, rg = p0 - ks * rs.r;
  const unsigned o0 = c.xl + (unsigned)((((set < 7 ? set + 1 : 0) ^ c.d) * 512 + rs.row0 + rg * 32) * KS + ks * 64);
  const unsigned o1 = o0 + (unsigned)(rg + 1 == rs.r ? 64 - rg * 32 * KS : 32 * KS);
  asm volatile(SGPR_PAD "global_load_dwordx4 %0, %4, %6\n\tglobal_load_dwordx4 %1, %4, %6 offset:16\n\t"
               "global_load_dwordx4 %2, %5, %6\n\tglobal_load_dwordx4 %3, %5, %6 offset:16"
               : "=&v"(v[0]), "=&v"(v[1]), "=&v"(v[2]), "=&v"(v[3]) : "v"(o0), "v"(o1), "s"(c.x) : "memory");
}

__device__ __forceinline__ void pack_task(const PushCtx& c, const u32x4 (&v)[4], InTask& o, int lane, bool plain) {
  if (plain) {
#pragma unroll
    for (int q = 0; q < 4; ++q) o.d[q] = v[q];
    o.mask = ~0ull;
    o.raw = true;
  } else {
    enc_in(v, o, c.tag, c.scr, lane);
  }
}

struct PushState {
  int tp, tc, tn, tl;
  u32x4 line;
  InTask o;
  u32x4 nx[4];
};

template <bool LDW>
__device__ __forceinline__ void push_step(const Params& p, const PushCtx& c, PushState& s, int lane) {
  unsigned w[16], tk;
  u32x4 x[4];
  const bool draw = s.tl < NTASK;
  asm volatile(SGPR_PAD "s_mov_b64 exec, 1\n\tglobal_atomic_add %0, %1, %2, %3 sc0\n\ts_mov_b64 exec, -1"
               : "=&v"(tk) : "v"(0u), "v"(draw ? 1u : 0u), "s"(c.lk) : "memory");
  if (s.tl == OPEN_AT) st1_agent(c.lk + 32, c.call);
  if constexpr (LDW) {
    const int u = s.tl - T_W0;
    unsigned o = c.wl + (unsigned)((((u >> 1) & 7) * 64 + (u & 1) * 16) * GN + block_col(c.d, u >> 4));
    asm volatile(SGPR_PAD
                 "global_load_dword %0, %16, %17\n\tv_add_u32 %16, 0x4000, %16\n\t"
                 "global_load_dword %1, %16, %17\n\tv_add_u32 %16, 0x4000, %16\n\t"
                 "global_load_dword %2, %16, %17\n\tv_add_u32 %16, 0x4000, %16\n\t"
                 "global_load_dword %3, %16, %17\n\tv_add_u32 %16, 0x4000, %16\n\t"
                 "global_load_dword %4, %16, %17\n\tv_add_u32 %16, 0x4000, %16\n\t"
                 "global_load_dword %5, %16, %17\n\tv_add_u32 %16, 0x4000, %16\n\t"
                 "global_load_dword %6, %16, %17\n\tv_add_u32 %16, 0x4000, %16\n\t"
                 "global_load_dword %7, %16, %17\n\tv_add_u32 %16, 0x4000, %16\n\t"
                 "global_load_dword %8, %16, %17\n\tv_add_u32 %16, 0x4000, %16\n\t"
                 "global_load_dword %9, %16, %17\n\tv_add_u32 %16, 0x4000, %16\n\t"
                 "global_load_dword %10, %16, %17\n\tv_add_u32 %16, 0x4000, %16\n\t"
                 "global_load_dword %11, %16, %17\n\tv_add_u32 %16, 0x4000, %16\n\t"
                 "global_load_dword %12, %16, %17\n\tv_add_u32 %16, 0x4000, %16\n\t"
                 "global_load_dword %13, %16, %17\n\tv_add_u32 %16, 0x4000, %16\n\t"
                 "global_load_dword %14, %16, %17\n\tv_add_u32 %16, 0x4000, %16\n\t"
                 "global_load_dword %15, %16, %17\n\t"
                 : "=&v"(w[0]), "=&v"(w[1]), "=&v"(w[2]), "=&v"(w[3]), "=&v"(w[4]), "=&v"(w[5]), "=&v"(w[6]), "=&v"(w[7]),
                   "=&v"(w[8]), "=&v"(w[9]), "=&v"(w[10]), "=&v"(w[11]), "=&v"(w[12]), "=&v"(w[13]), "=&v"(w[14]), "=&v"(w[15]), "+v"(o) : "s"(c.w) : "memory");
    static_assert(GN == 0x4000, "");
  } else {
    ld_x(c, s.tl, x);
  }
  const unsigned is0 = c.par + (unsigned)(XS_BYTES + WT_BYTES) + (unsigned)(c.rank * NTASK) * ITASK + lane * 16;
  if (s.tp >= 0) st16_sys_m(c.dst, is0 + (unsigned)s.tp * ITASK, s.line, 15ull);
  if (s.tc < NTASK) {
    if (!s.o.raw) {
      const unsigned slot = is0 + (unsigned)s.tc * ITASK;
      asm volatile(SGPR_PAD "global_store_dwordx4 %0, %1, %4 offset:256 sc0 sc1\n\tglobal_store_dwordx4 %0, %2, %4 offset:1280 sc0 sc1\n\t"
                   "global_store_dwordx4 %0, %3, %4 offset:2304 sc0 sc1\n\ts_nop 1"
                   :: "v"(slot), "v"(s.o.d[0]), "v"(s.o.d[1]), "v"(s.o.d[2]), "s"(c.dst) : "memory");
      if (s.o.mask != 0) st16_sys_m(c.dst, slot, s.o.d[3], s.o.mask);
      s.line = s.o.d[3];
    } else {
      const int t = s.tc, u = t - T_W0;
      const bool wr = task_is_w(t);
      const RowSet rs = row_set(task_set(t));
      const unsigned plain = c.par + lane * 16 +
          (wr ? (unsigned)XS_BYTES + (unsigned)((u >> 4) * (512 << 10) + (c.rank * 8 + ((u >> 1) & 7)) * 8192 + (u & 1) * 1024)
              : (unsigned)(rs.base * (128 << 10) + c.rank * rs.r * 16384 + (t - rs.task0) * 4096));
#pragma unroll
      for (int q = 0; q < 4; ++q) st16_sys(c.dst, plain + q * (wr ? 2048 : 1024), s.o.d[q]);
      s.line = u32x4{c.tag | IHDR_RAW << 16, 0u, 0u, 0u};
    }
  }
  if (s.tn < NTASK) pack_task(c, s.nx, s.o, lane, c.local);
  if constexpr (LDW) {
    asm volatile("s_waitcnt vmcnt(0)" : "+v"(tk), "+v"(w[0]), "+v"(w[1]), "+v"(w[2]), "+v"(w[3]), "+v"(w[4]), "+v"(w[5]), "+v"(w[6]), "+v"(w[7]),
                 "+v"(w[8]), "+v"(w[9]), "+v"(w[10]), "+v"(w[11]), "+v"(w[12]), "+v"(w[13]), "+v"(w[14]), "+v"(w[15]) :: "memory");
#pragma unroll
    for (int q = 0; q < 4; ++q) {
      const unsigned d0 = w[4 * q], d1 = w[4 * q + 1], d2 = w[4 * q + 2], d3 = w[4 * q + 3];
      const unsigned t0 = __builtin_amdgcn_perm(d1, d0, 0x05010400u), t1 = __builtin_amdgcn_perm(d1, d0, 0x07030602u);
      const unsigned t2 = __builtin_amdgcn_perm(d3, d2, 0x05010400u), t3 = __builtin_amdgcn_perm(d3, d2, 0x07030602u);
      s.nx[0][q] = __builtin_amdgcn_perm(t2, t0, 0x05040100u);
      s.nx[1][q] = __builtin_amdgcn_perm(t2, t0, 0x07060302u);
      s.nx[2][q] = __builtin_amdgcn_perm(t3, t1, 0x05040100u);
      s.nx[3][q] = __builtin_amdgcn_perm(t3, t1, 0x07060302u);
    }
  } else {
    asm volatile("s_waitcnt vmcnt(0)" : "+v"(tk), "+v"(x[0]), "+v"(x[1]), "+v"(x[2]), "+v"(x[3]) :: "memory");
#pragma unroll
    for (int q = 0; q < 4; ++q) s.nx[q] = x[q];
  }
  if (s.tc < NTASK) STAMP(min(8 + (s.tc >> 5), 28));
  s.tp = s.tc; s.tc = s.tn; s.tn = s.tl;
  s.tl = draw ? (int)(__builtin_amdgcn_readfirstlane(tk) - c.tick0) : NTASK;
}

#define GLDS_BITS " sc0 sc1"
__device__ __forceinline__ void glds2(const uint8_t* gbase, unsigned voff, unsigned lds) {
  unsigned keep;
  asm volatile("s_mov_b32 %0, m0\n\ts_mov_b32 m0, %3\n\ts_nop 2\n\t"
               "global_load_lds_dwordx4 %1, %2" GLDS_BITS "\n\t"
               "global_load_lds_dwordx4 %1, %2 offset:1024" GLDS_BITS "\n\t"
               "s_mov_b32 m0, %0"
               : "=&s"(keep) : "v"(voff), "s"(gbase), "s"(lds));
}
__device__ __forceinline__ void glds2o(const uint8_t* gbase, unsigned voff, unsigned lds) {
  unsigned keep;
  asm volatile("s_mov_b32 %0, m0\n\ts_mov_b32 m0, %3\n\ts_nop 2\n\t"
               "global_load_lds_dwordx4 %1, %2 offset:2048" GLDS_BITS "\n\t"
               "global_load_lds_dwordx4 %1, %2 offset:3072" GLDS_BITS "\n\t"
               "s_mov_b32 m0, %0"
               : "=&s"(keep) : "v"(voff), "s"(gbase), "s"(lds));
}
__device__ __forceinline__ void glds_half(const uint8_t* gbase, unsigned voff, unsigned lds) {
  unsigned keep;
  asm volatile("s_mov_b32 %0, m0\n\ts_mov_b32 m0, %3\n\ts_mov_b32 exec_hi, 0\n\ts_nop 1\n\t"
               "global_load_lds_dwordx4 %1, %2" GLDS_BITS "\n\t"
               "s_mov_b32 exec_hi, -1\n\ts_mov_b32 m0, %0"
               : "=&s"(keep) : "v"(voff), "s"(gbase), "s"(lds));
}

struct Frags { i32x8 a[2], b[4], a9, b9; };

__device__ __forceinline__ i32x8 ld_frag(const uint8_t* q) {
  const i32x4 lo = *(const i32x4*)q, hi = *(const i32x4*)(q + 1024);
  return __builtin_shufflevector(lo, hi, 0, 1, 2, 3, 4, 5, 6, 7);
}

#define MFMA(mi, j) acc[mi][j] = __builtin_amdgcn_mfma_scale_f32_32x32x64_f8f6f4(cur.a[mi], cur.b[j], acc[mi][j], 1, 1, 0, 0, 0, 0)
#define SB() __builtin_amdgcn_sched_barrier(0)

struct Ring {
  const uint8_t* ga;
  const uint8_t* gb;
  unsigned astep;
  unsigned va, vb, v9;
  unsigned wr, wr_end;
  unsigned wb, w9;
  unsigned rd;
  const uint8_t* qa;
  const uint8_t* qb;
  const uint8_t* q9;
  const uint8_t* la;
  const uint8_t* lb;
  const uint8_t* l9;
};

template <int MODE, int LEFT, bool DMA, bool RD>
__device__ __forceinline__ void kstep(f32x16 (&acc)[2][4], f32x16& acc9, const Frags& cur, Frags& nxt, Ring& g) {
  constexpr bool NINE = MODE == 9, TWO = MODE != 7;
  constexpr int STAGE = NINE ? 26624 : 24576;
  MFMA(0, 0); SB();
  asm volatile("s_waitcnt vmcnt(%0) lgkmcnt(0)\n\ts_barrier" :: "n"(LEFT * (NINE ? 7 : 6)) : "memory");
  SB();
  MFMA(0, 1); SB();
  if constexpr (RD) { nxt.a[0] = ld_frag(g.qa); nxt.b[0] = ld_frag(g.qb); }
  SB();
  MFMA(0, 2); SB();
  if constexpr (DMA) glds2(g.ga, g.va, g.wr);
  SB();
  MFMA(0, 3); SB();
  if constexpr (RD) { nxt.b[1] = ld_frag(g.qb + 2048); nxt.b[2] = ld_frag(g.qb + 4096); }
  SB();
  if constexpr (TWO) MFMA(1, 0);
  SB();
  if constexpr (DMA) glds2(g.gb, g.vb, g.wr + g.wb);
  SB();
  if constexpr (TWO) MFMA(1, 1);
  SB();
  if constexpr (RD) { nxt.b[3] = ld_frag(g.qb + 6144); if constexpr (TWO) nxt.a[1] = ld_frag(g.qa + 2048); }
  g.rd = g.rd == (NSTAGE - 1) * STAGE ? 0 : g.rd + STAGE;
  g.qa = g.la + g.rd;
  if constexpr (!NINE) g.qb = g.lb + g.rd;
  SB();
  if constexpr (TWO) MFMA(1, 2);
  SB();
  if constexpr (DMA) glds2o(g.ga, g.va, g.wr);
  SB();
  if constexpr (NINE) {
    if constexpr (TWO) MFMA(1, 3);
  SB();
    if constexpr (RD) { nxt.a9 = ld_frag(g.qb - 2048); nxt.b9 = ld_frag(g.q9); }
    g.qb = g.lb + g.rd; g.q9 = g.l9 + g.rd;
    if constexpr (DMA) g.gb += 8192;
    SB();
    acc9 = __builtin_amdgcn_mfma_scale_f32_32x32x64_f8f6f4(cur.a9, cur.b9, acc9, 1, 1, 0, 0, 0, 0); SB();
    if constexpr (DMA) glds_half(g.ga, g.v9, g.wr + g.w9);
  } else {
    if constexpr (TWO) MFMA(1, 3);
  SB();
    if constexpr (DMA) g.gb += 8192;
  }
  if constexpr (DMA) {
    g.ga += g.astep;
    g.wr += STAGE; g.wr = g.wr == g.wr_end ? g.wr - NSTAGE * STAGE : g.wr;
  }
  SB();
}

template <int MODE>
__device__ __forceinline__ void gemm_tile(const uint8_t* ga, unsigned astep, const uint8_t* gb, uint8_t* lds, int wave, int lane) {
  constexpr bool NINE = MODE == 9, TWO = MODE != 7;
  constexpr int STAGE = NINE ? 26624 : 24576, BOFF = STAGE - 8192;
  f32x16 acc[2][4], acc9;
  Ring g;
  g.ga = ga; g.gb = gb; g.astep = astep;
  g.va = lane * 16 + wave * 4096; g.vb = lane * 16 + wave * 2048; g.v9 = 16384 + lane * 16 + wave * 512;
  g.wr = (unsigned)(uintptr_t)lds + wave * 4096; g.wr_end = g.wr + NSTAGE * STAGE;
  g.wb = BOFF - wave * 2048; g.w9 = 16384 - wave * 3584;
#pragma unroll
  for (int s = 0; s < NSTAGE; ++s) {
    glds2(g.ga, g.va, g.wr + s * STAGE);
    glds2(g.gb, g.vb, g.wr + s * STAGE + g.wb);
    glds2o(g.ga, g.va, g.wr + s * STAGE);
    if constexpr (NINE) glds_half(g.ga, g.v9, g.wr + s * STAGE + g.w9);
    g.ga += astep; g.gb += 8192;
  }
#pragma unroll
  for (int r = 0; r < 16; ++r) {
    acc9[r] = 0.f;
#pragma unroll
    for (int j = 0; j < 4; ++j) { acc[0][j][r] = 0.f; acc[1][j][r] = 0.f; }
  }
  g.la = lds + lane * 16 + wave * 4096; g.lb = lds + BOFF + lane * 16; g.l9 = g.lb + wave * 2048;
  asm volatile("s_waitcnt vmcnt(%0)\n\ts_barrier" :: "n"(5 * (NINE ? 7 : 6)) : "memory");
  Frags f0, f1;
  f0.a[0] = ld_frag(g.la); f0.b[0] = ld_frag(g.lb); f0.b[1] = ld_frag(g.lb + 2048);
  f0.b[2] = ld_frag(g.lb + 4096); f0.b[3] = ld_frag(g.lb + 6144);
  if constexpr (TWO) f0.a[1] = ld_frag(g.la + 2048);
  if constexpr (NINE) { f0.a9 = ld_frag(g.lb - 2048); f0.b9 = ld_frag(g.l9); }
  g.rd = STAGE; g.qa = g.la + STAGE; g.qb = g.lb + STAGE; g.q9 = g.l9 + STAGE;
#pragma unroll 1
  for (int s = 0; s < KSTEPS - NSTAGE; s += 2) {
    kstep<MODE, 4, true, true>(acc, acc9, f0, f1, g);
    kstep<MODE, 4, true, true>(acc, acc9, f1, f0, g);
  }
  kstep<MODE, 4, false, true>(acc, acc9, f0, f1, g);
  kstep<MODE, 3, false, true>(acc, acc9, f1, f0, g);
  kstep<MODE, 2, false, true>(acc, acc9, f0, f1, g);
  kstep<MODE, 1, false, true>(acc, acc9, f1, f0, g);
  kstep<MODE, 0, false, true>(acc, acc9, f0, f1, g);
  auto park = [&](const f32x16& a, int i) {
#pragma unroll
    for (int q = 0; q < 4; ++q)
      *(f32x4*)(lds + wave * 32768 + (i * 4 + q) * 1024 + lane * 16) = f32x4{a[4 * q], a[4 * q + 1], a[4 * q + 2], a[4 * q + 3]};
  };
  {
    const Frags& cur = f1;
    MFMA(0, 0); SB();
    asm volatile("s_waitcnt lgkmcnt(0)\n\ts_barrier" ::: "memory");
    SB();
    MFMA(0, 1); SB();
    MFMA(0, 2); SB();
    park(acc[0][0], 0); SB();
    MFMA(0, 3); SB();
    park(acc[0][1], 1); SB();
    if constexpr (TWO) MFMA(1, 0);
    SB();
    park(acc[0][2], 2); SB();
    if constexpr (TWO) MFMA(1, 1);
    SB();
    park(acc[0][3], 3); SB();
    if constexpr (TWO) {
      MFMA(1, 2); SB();
      park(acc[1][0], 4); SB();
      MFMA(1, 3); SB();
      park(acc[1][1], 5); SB();
      if constexpr (NINE) { acc9 = __builtin_amdgcn_mfma_scale_f32_32x32x64_f8f6f4(cur.a9, cur.b9, acc9, 1, 1, 0, 0, 0, 0); SB(); }
      park(acc[1][2], 6);
      park(acc[1][3], 7);
    }
  }
  if constexpr (NINE) {
#pragma unroll
    for (int q = 0; q < 4; ++q)
      *(f32x4*)(lds + 8 * 16384 + wave * 4096 + q * 1024 + lane * 16) = f32x4{acc9[4 * q], acc9[4 * q + 1], acc9[4 * q + 2], acc9[4 * q + 3]};
    __syncthreads();
  }
  __builtin_amdgcn_fence(__ATOMIC_ACQ_REL, "wavefront");
}

constexpr unsigned REC_CAP = 510, HDR_B = 0x40000000u;

__device__ __forceinline__ void put_records4(const unsigned (&q)[4], unsigned tag, unsigned& cnt, uint8_t* scr, unsigned dump) {
  unsigned m = 0;
#pragma unroll
  for (int k = 0; k < 4; ++k)
    m |= ((((q[k] & 0x77777777u) + 0x77777777u) | q[k]) & 0x88888888u) >> (3 - k);
  for (unsigned long long act = __ballot(m != 0); act != 0; act = __ballot(m != 0)) {
    const unsigned bit = (unsigned)__builtin_ctz(m | 0x80000000u), sh = bit & 28u;
    const unsigned m1 = (unsigned)__builtin_amdgcn_sbfe((int)bit, 0, 1), m2 = (unsigned)__builtin_amdgcn_sbfe((int)bit, 1, 1);
    const unsigned w = bfi(m2, bfi(m1, q[3], q[2]), bfi(m1, q[1], q[0]));
    const unsigned idx = cnt + __builtin_amdgcn_mbcnt_hi((unsigned)(act >> 32), __builtin_amdgcn_mbcnt_lo((unsigned)act, 0u));
    const unsigned at = (m != 0 && idx < REC_CAP) ? 4 + 2 * idx : dump;
    *(unsigned short*)(scr + at) = (unsigned short)(tag | (bit & 3u) << 7 | sh << 2 | ((w >> sh) & 15u));
    m &= m - 1;
    cnt += __popcll(act);
  }
}

template <bool MODEB>
__device__ __forceinline__ bool enc_body(const float (&v)[4][16], float up, unsigned& cnt, uint8_t* blk, int lane) {
  unsigned bad = 0, low = 0;
#pragma unroll
  for (int h = 0; h < 2; ++h) {
    unsigned G[32];
#pragma unroll
    for (int j = 0; j < 4; ++j)
#pragma unroll
      for (int r = 0; r < 8; ++r) {
        const float x = v[j][8 * h + r] * up;
        unsigned g = (unsigned)(int)x;
        bad = __builtin_amdgcn_bitop3_b32(__builtin_bit_cast(unsigned, (float)(int)g), __builtin_bit_cast(unsigned, x), bad, 0xbe);
        low |= g;
        if (MODEB) g = g << 1 | (((g ^ g << 1) >> 27) & 16u);
        G[4 * r + j] = g;
      }
    unsigned q[4];
#pragma unroll
    for (int k = 0; k < 4; ++k) {
      const unsigned* g = G + 8 * k;
      const unsigned b0 = __builtin_amdgcn_perm(__builtin_amdgcn_perm(g[3], g[2], 0x0c0c0400u), __builtin_amdgcn_perm(g[1], g[0], 0x0c0c0400u), 0x05040100u);
      const unsigned b1 = __builtin_amdgcn_perm(__builtin_amdgcn_perm(g[7], g[6], 0x0c0c0400u), __builtin_amdgcn_perm(g[5], g[4], 0x0c0c0400u), 0x05040100u);
      q[k] = (b0 >> 4) | b1;
    }
#pragma unroll
    for (int j = 0; j < 2; ++j) {
      unsigned d[12];
#pragma unroll
      for (int g = 0; g < 4; ++g) {
        const unsigned* x = G + 16 * j + 4 * g;
        d[3 * g] = __builtin_amdgcn_perm(x[1], x[0], 0x05030201u);
        d[3 * g + 1] = __builtin_amdgcn_perm(x[2], x[1], 0x06050302u);
        d[3 * g + 2] = __builtin_amdgcn_perm(x[3], x[2], 0x07060503u);
      }
#pragma unroll
      for (int i = 0; i < 3; ++i)
        *(u32x4*)(blk + (1 + 6 * h + 3 * j + i) * 1024 + lane * 16) = u32x4{d[4 * i], d[4 * i + 1], d[4 * i + 2], d[4 * i + 3]};
    }
    put_records4(q, (unsigned)(lane << 10 | h << 9), cnt, blk, 13 * 1024 + 2 * lane);
  }
  return __ballot((bad | (low & 15u)) != 0) == 0 && cnt <= REC_CAP;
}

__device__ __forceinline__ unsigned enc_out(const float (&v)[4][16], float sc, uint8_t* blk, int lane) {
  float m = 0.f;
#pragma unroll
  for (int j = 0; j < 4; ++j)
#pragma unroll
    for (int r = 0; r < 16; r += 2) asm("v_max3_f32 %0, |%1|, |%2|, %0" : "+v"(m) : "v"(v[j][r]), "v"(v[j][r + 1]));
  const int e = max((int)(wave_max_u32(__builtin_bit_cast(unsigned, m)) >> 23) - 157, -126);
  const float up = __builtin_bit_cast(float, (unsigned)(127 - min(e, 0)) << 23);
  int ntop = 0, nlow = 0;
#pragma unroll
  for (int r = 0; r < 8; ++r) {
    const unsigned g = (unsigned)(int)(v[r & 3][2 * r] * up);
    ntop += __popcll(__ballot((int)(g ^ (g << 1)) < 0));
    nlow += __popcll(__ballot((g & 128u) != 0));
  }
  unsigned cnt = 0;
  const bool modeb = ntop < nlow;
  const bool fits = e <= 0 && (modeb ? enc_body<true>(v, up, cnt, blk, lane) : enc_body<false>(v, up, cnt, blk, lane));
  if (!fits) {
#pragma unroll
    for (int r = 0; r < 16; ++r) *(f32x4*)(blk + r * 1024 + lane * 16) = f32x4{v[0][r] * sc, v[1][r] * sc, v[2][r] * sc, v[3][r] * sc};
    return 0;
  }
  *(unsigned*)(blk + (lane == 0 ? 0 : 13 * 1024 + 4 * lane)) = cnt | (unsigned)(e + 127) << 16 | (modeb ? HDR_B : 0u);
  return ((4 + 2 * cnt + 63) >> 6) * 4;
}

__device__ __forceinline__ void st16_out(void* base, unsigned off, f32x4 v) {
  asm volatile(SGPR_PAD "global_store_dwordx4 %0, %1, %2 sc1\n\ts_nop 1" :: "v"(off), "v"(v), "s"(base) : "memory");
}

__device__ __forceinline__ constexpr unsigned row_off(int r) { return (unsigned)((r & 3) + 8 * (r >> 2)) * (GN * 4); }

__device__ __forceinline__ void send_out(const uint8_t* blk, unsigned nl, bool dead, uint8_t* slot, float* ob, unsigned off,
                                         int lane, unsigned* next, unsigned call, bool last, int depth) {
  const bool raw = nl == 0 || dead;
  const u32x4 h = *(const u32x4*)(blk + lane * 16);
  st16_sys_m(slot, lane * 16, raw ? u32x4{HDR_RAW, 0u, 0u, 0u} : h, raw ? 15ull : nl >= 64 ? ~0ull : (1ull << nl) - 1);
  const int n = raw ? 16 : 12;
  void* const base = raw ? (void*)ob : (void*)slot;
  const uint8_t* const src = blk + (raw ? 0 : 1024) + lane * 16;
  const unsigned nan = dead ? 0x7fc00000u : 0u;
#pragma unroll 1
  for (int i = 0; i < n; ++i) {
    const u32x4 v = *(const u32x4*)(src + i * 1024) | nan;
    if (depth == OUT_D) wait_vm<OUT_D - 1>(); else if (depth == OUT_D3) wait_vm<OUT_D3 - 1>(); else wait_vm<DEEP_D - 1>();
    if (last && i == n - OUT_REL) st1_agent(next, call);
    st16_sys(base, raw ? off + row_off(i) : (unsigned)(1024 + i * 1024 + lane * 16), v);
  }
}

__device__ __forceinline__ void dec_out(const uint8_t* slot, float* ob, unsigned off, float sc, unsigned* tmo, unsigned call, uint8_t* scr, int lane) {
  u32x4 rec, y[12];
  unsigned tmv;
  asm volatile(SGPR_PAD "global_load_dword %13, %16, %18 sc1\n\t"
               "global_load_dwordx4 %0, %14, %17 offset:-4096 sc0 sc1\n\tglobal_load_dwordx4 %1, %14, %17 offset:-3072 sc0 sc1\n\t"
               "global_load_dwordx4 %2, %14, %17 offset:-2048 sc0 sc1\n\tglobal_load_dwordx4 %3, %14, %17 offset:-1024 sc0 sc1\n\t"
               "global_load_dwordx4 %4, %14, %17 sc0 sc1\n\tglobal_load_dwordx4 %5, %14, %17 offset:1024 sc0 sc1\n\t"
               "global_load_dwordx4 %6, %14, %17 offset:2048 sc0 sc1\n\tglobal_load_dwordx4 %7, %14, %17 offset:3072 sc0 sc1\n\t"
               "global_load_dwordx4 %8, %15, %17 offset:-4096 sc0 sc1\n\tglobal_load_dwordx4 %9, %15, %17 offset:-3072 sc0 sc1\n\t"
               "global_load_dwordx4 %10, %15, %17 offset:-2048 sc0 sc1\n\tglobal_load_dwordx4 %11, %15, %17 offset:-1024 sc0 sc1\n\t"
               "global_load_dwordx4 %12, %15, %17 sc0 sc1\n\ts_waitcnt vmcnt(12)"
               : "=&v"(rec), "=&v"(y[0]), "=&v"(y[1]), "=&v"(y[2]), "=&v"(y[3]), "=&v"(y[4]), "=&v"(y[5]), "=&v"(y[6]),
                 "=&v"(y[7]), "=&v"(y[8]), "=&v"(y[9]), "=&v"(y[10]), "=&v"(y[11]), "=&v"(tmv)
               : "v"(4096 + lane * 16), "v"(12288 + lane * 16), "v"(0u), "s"(slot), "s"(tmo) : "memory");
  const unsigned hdr = __builtin_amdgcn_readfirstlane(rec[0]);
  const unsigned cnt = hdr & 0xffffu;
  const float down = __builtin_bit_cast(float, ((hdr >> 16) & 255u) << 23);
  *(u32x4*)(scr + lane * 32) = u32x4{0u, 0u, 0u, 0u};
  *(u32x4*)(scr + lane * 32 + 16) = u32x4{0u, 0u, 0u, 0u};
  __builtin_amdgcn_fence(__ATOMIC_ACQ_REL, "wavefront");
#pragma unroll
  for (int i = 0; i < 8; ++i) {
    const unsigned r16 = (rec[i >> 1] >> (16 * (i & 1))) & 0xffffu;
    const unsigned val = (unsigned)(lane * 8 + i - 2) < cnt ? (r16 & 15u) << ((r16 >> 2) & 28u) : 0u;
    __hip_atomic_fetch_or((unsigned*)(scr + (r16 >> 7) * 4), val, __ATOMIC_RELAXED, __HIP_MEMORY_SCOPE_WORKGROUP);
  }
  __builtin_amdgcn_fence(__ATOMIC_ACQ_REL, "wavefront");
  const u32x4 q0 = *(const u32x4*)(scr + lane * 32), q1 = *(const u32x4*)(scr + lane * 32 + 16);
  __builtin_amdgcn_fence(__ATOMIC_ACQ_REL, "wavefront");
  asm volatile("s_waitcnt vmcnt(0)" : "+v"(y[0]), "+v"(y[1]), "+v"(y[2]), "+v"(y[3]), "+v"(y[4]), "+v"(y[5]), "+v"(y[6]),
               "+v"(y[7]), "+v"(y[8]), "+v"(y[9]), "+v"(y[10]), "+v"(y[11]) :: "memory");
  if (__builtin_amdgcn_readfirstlane(tmv) == call) {
    const float n = __builtin_nanf("");
#pragma unroll 1
    for (int r = 0; r < 16; ++r) st16_out(ob, off + row_off(r), f32x4{n, n, n, n});
    return;
  }
  if (hdr & HDR_RAW) return;
  const bool modeb = (hdr & HDR_B) != 0;
#pragma unroll
  for (int g = 0; g < 16; ++g) {
    const unsigned d0 = y[(3 * g) >> 2][(3 * g) & 3], d1 = y[(3 * g + 1) >> 2][(3 * g + 1) & 3], d2 = y[(3 * g + 2) >> 2][(3 * g + 2) & 3];
    const unsigned qw = g < 8 ? q0[(g >> 1) & 3] : q1[(g >> 1) & 3];
    const unsigned q = ((g & 1) ? qw : qw << 4) & 0xf0f0f0f0u;
    unsigned G[4] = {__builtin_amdgcn_perm(d0, q, 0x06050400u), __builtin_amdgcn_perm(d1, d0, 0x0504030cu) | ((q >> 8) & 255u),
                     __builtin_amdgcn_perm(d2, d1, 0x0403020cu) | ((q >> 16) & 255u), __builtin_amdgcn_perm(d2, q, 0x07060503u)};
    f32x4 v;
#pragma unroll
    for (int j = 0; j < 4; ++j) {
      if (modeb) G[j] = ((unsigned)((int)G[j] >> 1) & ~8u) ^ (G[j] << 27);
      v[j] = ((float)(int)G[j] * down) * sc;
    }
    st16_out(ob, off + row_off(g), v);
  }
}

__device__ __forceinline__ int wave_blocks(int r, int wave) { return wave < 3 ? 2 : r - 6; }
__device__ __forceinline__ unsigned out_off(int row0, int col, int wave, int lane) {
  return (unsigned)(((row0 + wave * 64 + 4 * (lane >> 5)) * GN + col + 4 * (lane & 31)) * 4);
}

__device__ __forceinline__ void recv_tile(const Params& p, uint8_t* myws, float* out, int g, int src, int i, int k0, int k1, int wave, int lane,
                                          unsigned call, unsigned* tmo, float sc, uint8_t* scr) {
  const unsigned off = out_off(g < 4 ? 0 : 224, block_col(src, tile_block(g, i)), wave, lane);
  const uint8_t* const fl = myws + WS_DONE + (size_t)((src * 32 + i) * 4 + wave) * 128;
  const int nb = wave_blocks(g < 4 ? 7 : 9, wave);
#pragma unroll 1
  for (int k = k0; k < min(nb, k1); ++k) {
    spin<RX_SLEEP>([&] { return __builtin_amdgcn_readfirstlane(ld_sys(fl)) > 4 * call + k; }, tmo, call);
    STAMP(24);
    dec_out(myws + WS_OS + (size_t)((src * 32 + i) * 9 + wave * 2 + k) * OBLK, out, off + (unsigned)k * 32 * (GN * 4), sc, tmo, call, scr, lane);
  }
}

}

__global__ void __launch_bounds__(256, 1) gemm_rs_fused(const Params p) {
  __shared__ __attribute__((aligned(16))) uint8_t lds[LDS_BYTES];
  const int b = blockIdx.x;
  const int lane = threadIdx.x & 63;
  const int wave = __builtin_amdgcn_readfirstlane(threadIdx.x >> 6);
  const int rank = p.rank;
  uint8_t* const myws = p.myws;
  STAMP(0);

  const int grp = b & 7, dst = grp ^ rank, ti = 31 - (b >> 3);
  const bool local = grp == 0 || (grp >= 4 && ti >= 24);
  const int set = local ? 7 + (grp != 0) : grp - 1, cb = tile_block(grp, ti);
  const RowSet rs = row_set(set);
  unsigned* const callp = (unsigned*)(myws + WS_CALL + (size_t)b * 128);
  unsigned* const tmo = (unsigned*)(myws + WS_TMO);
  unsigned* const lk = (unsigned*)(myws + WS_LINK) + grp * 64;
  uint8_t* const scr = lds + LDS_PK + wave * 2048;

  unsigned call;
  if (ti >= 24) {
    const int w = (31 - ti) * 4 + wave;
    PushCtx c;
    c.x = p.x; c.w = p.w; c.dst = p.ws[dst]; c.lk = lk; c.rank = rank; c.d = dst;
    c.xl = (unsigned)((lane & 31) * KS + (lane >> 5) * 32);
    c.wl = (unsigned)((lane >> 5) * 32 * GN + (lane & 31) * 4);
    c.scr = scr;
    c.local = grp == 0;
    PushState s;
    u32x4 x0[4];
    unsigned cv;
    asm volatile(SGPR_PAD "global_load_dword %0, %1, %2 sc1" : "=v"(cv) : "v"(0u), "s"(callp) : "memory");
    ld_x(c, w, x0);
    ld_x(c, w + 32, s.nx);
    asm volatile("s_waitcnt vmcnt(0)" : "+v"(cv), "+v"(x0[0]), "+v"(x0[1]), "+v"(x0[2]), "+v"(x0[3]),
                 "+v"(s.nx[0]), "+v"(s.nx[1]), "+v"(s.nx[2]), "+v"(s.nx[3]) :: "memory");
    call = __builtin_amdgcn_readfirstlane(cv) + 1;
    c.par = (unsigned)(WS_DATA + (call & 1) * PAR_BYTES);
    c.tick0 = (call - 1) * (NTASK - 64) - 96;
    c.tag = call & 0xffffu;
    c.call = call;
    s.tp = -1; s.tc = w; s.tn = w + 32; s.tl = w + 64;
    s.line = u32x4{0u, 0u, 0u, 0u};
    pack_task(c, x0, s.o, lane, true);
    STAMP(7);
#pragma unroll 1
    while (s.tp < NTASK) {
      if (task_is_w(s.tl)) push_step<true>(p, c, s, lane);
      else push_step<false>(p, c, s, lane);
    }
    STAMP(30);
  } else {
    call = __builtin_amdgcn_readfirstlane(ld_agent(callp)) + 1;
    if (WARM) {
      const int l = (ti * 4 + wave) * 64 + lane;
      unsigned v[7];
      v[6] = *(const unsigned*)(p.x + (size_t)min(grp * 4096 + l, grp * 4096 + 4095) * 64);
#pragma unroll
      for (int j = 0; j < 6; ++j) {
        const int h = min(j * 6144 + l, 32767);
        v[j] = *(const unsigned*)(p.w + (size_t)((h >> 1) & 511) * GN + block_col(dst, h >> 10) + (h & 1) * 64);
      }
      asm volatile("" :: "v"(v[0]), "v"(v[1]), "v"(v[2]), "v"(v[3]), "v"(v[4]), "v"(v[5]), "v"(v[6]));
    }
  }
  uint8_t* const par = myws + WS_DATA + (size_t)(call & 1) * PAR_BYTES;

#pragma unroll 1
  for (int job = 0; job < 4; ++job) {
    const bool jw = grp == 0 ? job < 3 : job == 1;
    const int beta = grp == 0 ? (ti * 4 + wave) & 63 : (23 - ti) * 4 + wave;
    if (grp == 0 ? (job == 2 && ti >= 24) : (ti >= 24 || job >= 2 || (job == 0 && beta >= 8 * rs.r))) continue;
    const int n = job == 2 ? 24 + (ti & 7) : cb;
    const RowSet js = row_set(grp == 0 ? 7 + (ti >> 4) : set);
    const int bs = (beta * (js.r == 7 ? 147 : js.r == 8 ? 128 : 114)) >> 10, bu = beta - bs * js.r;
    const int src = !jw ? bs : grp != 0 ? dst : job == 0 ? rank : job == 1 ? rank ^ (4 + (ti & 3)) : rank ^ (1 + (ti >> 3));
    const int t0 = jw ? T_W0 + n * 16 + wave * 4 : js.task0 + bu * 4;
    const uint8_t* const slot = par + XS_BYTES + WT_BYTES + (size_t)(src * NTASK + t0) * ITASK;
    if (!wait_tasks(slot, tmo, call, lane)) break;
    if (jw) STAMP(26);
    if (src != rank) {
      const unsigned base = jw ? (unsigned)XS_BYTES + (unsigned)(n * (512 << 10) + (src * 8 + wave * 2) * 8192)
                               : (unsigned)(js.base * (128 << 10) + (src * js.r + bu) * 16384);
      unsigned off[4];
#pragma unroll
      for (int t = 0; t < 4; ++t) off[t] = base + (jw ? (t >> 1) * 8192 + (t & 1) * 1024 : t * 4096);
      dec_in4(slot, par, off, jw ? 2048 : 1024, lds + wave * 4096, lane);
      if (jw) STAMP(27);
      wait_vm<0>();
    }
    st1_agent(jw ? myws + WS_DW + (size_t)(n * 32 + src * 4 + wave) * 4
                 : myws + WS_DX + (size_t)((grp == 0 ? 7 + (ti >> 4) : set) * 128 + beta) * 4, call);
  }
  STAMP(29);

  unsigned* const ctrl = (unsigned*)(lds + LDS_CTRL);
  if (wave == 0) {
    const uint8_t* const fw = myws + WS_DW + (size_t)(cb * 32 + (lane & 31)) * 4;
    const uint8_t* const fx = myws + WS_DX + (size_t)(set * 128 + min(lane, 8 * rs.r - 1)) * 4;
    const uint8_t* const fy = myws + WS_DX + (size_t)(set * 128 + min(64 + lane, 8 * rs.r - 1)) * 4;
    const bool ok = spin([&] { const unsigned a = ld_agent(fw), c = ld_agent(fx), d = ld_agent(fy); return __all(min(min(a, c), d) >= call) != 0; },
                         tmo, call);
    if (lane == 0) ctrl[0] = !ok;
  }
  __syncthreads();
  const bool dead0 = __builtin_amdgcn_readfirstlane(ctrl[0]) != 0;
  STAMP(1); CSTAMP(1);

  if (wave == 0) st1_agent(callp, call);
  const float sc = p.sx[0] * p.sw[0];
  const unsigned off = out_off(rs.row0, block_col(rank, cb), wave, lane);
  float* const ob = p.out[local ? rank : dst];
  uint8_t* const blk = lds + wave * 32768;
  uint8_t* const oslot = p.ws[dst] + WS_OS + (size_t)((rank * 32 + ti) * 9 + wave * 2) * OBLK;
  unsigned* const flag = (unsigned*)(p.ws[dst] + WS_DONE + (size_t)((rank * 32 + ti) * 4 + wave) * 128);
  const int nb = wave_blocks(rs.r, wave);
  const unsigned* const turn = ti < NTURN ? lk + 32 : (const unsigned*)(myws + WS_TURN + (size_t)((grp * 32 + ti) * 4 + wave) * 128);
  unsigned* const next = (unsigned*)(myws + WS_TURN + (size_t)((grp * 32 + ((ti + NTURN) & 31)) * 4 + wave) * 128);
  const int depth = ti < DEEP_T ? DEEP_D : nb == 3 ? OUT_D3 : OUT_D;
  unsigned nl0 = 0, nl1 = 0, nl2 = 0;
  int sent = 0;

  if (!dead0) {
    const uint8_t* const ga = par + (size_t)rs.base * (128 << 10);
    const uint8_t* const gb = par + XS_BYTES + (size_t)cb * (512 << 10);
    if (rs.r == 9) gemm_tile<9>(ga, 9 * 2048, gb, lds, wave, lane);
    else if (nb == 1) gemm_tile<7>(ga, 7 * 2048, gb, lds, wave, lane);
    else gemm_tile<8>(ga, rs.r * 2048, gb, lds, wave, lane);
    STAMP(2); CSTAMP(2);
    unsigned tnv, tmv;
    asm volatile(SGPR_PAD "global_load_dword %0, %2, %3 sc1\n\tglobal_load_dword %1, %2, %4 sc1"
                 : "=&a"(tnv), "=&a"(tmv) : "v"(0u), "s"(turn), "s"(tmo) : "memory");
    auto landed = [&](unsigned& s_tn, unsigned& s_tm) {
      unsigned t0, t1;
      asm volatile("s_waitcnt vmcnt(0)\n\tv_accvgpr_read_b32 %2, %4\n\tv_accvgpr_read_b32 %3, %5\n\ts_nop 1\n\t"
                   "v_readfirstlane_b32 %0, %2\n\tv_readfirstlane_b32 %1, %3\n\ts_nop 1"
                   : "=&s"(s_tn), "=&s"(s_tm), "=&v"(t0), "=&v"(t1) : "a"(tnv), "a"(tmv) : "memory");
    };
    float s0 = sc;
#pragma unroll 1
    for (int k = 0; k < nb; ++k) {
      float v[4][16];
#pragma unroll
      for (int i = 0; i < 16; ++i) {
        const f32x4 t = *(const f32x4*)(blk + k * 16384 + i * 1024 + lane * 16);
#pragma unroll
        for (int q = 0; q < 4; ++q) v[i >> 2][(i & 3) * 4 + q] = t[q];
      }
      __builtin_amdgcn_fence(__ATOMIC_ACQ_REL, "wavefront");
      if (local) {
        if (k == 0) {
          unsigned tn, t;
          landed(tn, t);
          if (t == call) s0 = __builtin_nanf("");
        }
#pragma unroll
        for (int r = 0; r < 16; ++r)
          st16_out(ob, off + k * 32 * (GN * 4) + row_off(r), f32x4{v[0][r] * s0, v[1][r] * s0, v[2][r] * s0, v[3][r] * s0});
        continue;
      }
      const unsigned nl = enc_out(v, sc, blk + k * 16384, lane);
      __builtin_amdgcn_fence(__ATOMIC_ACQ_REL, "wavefront");
      if (k == 0) nl0 = nl; else if (k == 1) nl1 = nl; else nl2 = nl;
      if (k == 0) {
        unsigned tn, t;
        landed(tn, t);
        if (ti < 24 && nb > 1 && t != call && (tn >= call || __builtin_amdgcn_readfirstlane(ld_agent(turn)) >= call)) {
          STAMP(3);
          send_out(blk, nl, false, oslot, ob, off, lane, next, call, false, depth);
          sent = 1;
        }
      }
    }
  } else if (local) {
    const float n = __builtin_nanf("");
#pragma unroll 1
    for (int t = 0; t < 32; ++t) st16_out(ob, off + (t >> 4) * 32 * (GN * 4) + row_off(t & 15), f32x4{n, n, n, n});
  }
  STAMP(31); CSTAMP(31);
  if (grp == 0) return;

  const int last8 = grp < 4 ? 24 : 16;
  if (ti >= 24) recv_tile(p, myws, p.out[rank], grp, dst, ti - 24, 0, 3, wave, lane, call, tmo, sc, scr);
  if (!local) {
    if (sent) {
      wait_vm<0>();
      STAMP(20);
      st1_sys(flag, 4 * call + 1);
    } else {
      spin([&] { return __builtin_amdgcn_readfirstlane(ld_agent(turn)) >= call; }, tmo, call);
      STAMP(3);
    }
    const bool dead = dead0 || __builtin_amdgcn_readfirstlane(ld_agent(tmo)) == call;
#pragma unroll 1
    for (int k = sent; k < nb; ++k) {
      send_out(blk + k * 16384, k == 0 ? nl0 : k == 1 ? nl1 : nl2, dead, oslot + k * OBLK, ob, off + k * 32 * (GN * 4), lane, next, call, k == nb - 1, depth);
      if (k == nb - 1) STAMP(4);
      wait_vm<0>();
      STAMP(20 + k);
      st1_sys(flag, 4 * call + 1 + k);
    }
    STAMP(5);
  }
#pragma unroll 1
  for (int duty = 0; duty < 2; ++duty) {
    int i, k0, k1;
    if (duty == 0) {
      if (ti >= last8) continue;
      i = ti + 8; k0 = 0; k1 = i >= last8 ? 1 : 3;
    } else if (ti < 8) {
      i = last8 + ti; k0 = grp < 4 ? 1 : 2; k1 = 3;
    } else if (local) {
      i = ti - 8; k0 = 1; k1 = 2;
    } else {
      continue;
    }
    recv_tile(p, myws, p.out[rank], grp, dst, i, k0, k1, wave, lane, call, tmo, sc, scr);
  }
  STAMP(6);
}

extern "C" void kernel_launch(int rank, int nranks, ncclComm_t comm, hipStream_t stream,
                              void* const* d_in, const int* in_sizes, int n_in,
                              void* d_out, int out_size, const int* in_dtypes, int out_dtype,
                              void* const* d_out_all, void* const* d_ws_all, size_t ws_bytes) {
  (void)comm; (void)in_sizes; (void)in_dtypes; (void)out_dtype; (void)d_out;
  if (nranks != NR || (unsigned)rank >= (unsigned)NR || n_in != 4 || ws_bytes < WS_NEED || out_size != 512 * GN) {
    fprintf(stderr, "kernel_launch: unexpected problem shape\n");
    return;
  }
  Params p;
  p.x = (const uint8_t*)d_in[0];
  p.w = (const uint8_t*)d_in[1];
  p.sx = (const float*)d_in[2];
  p.sw = (const float*)d_in[3];
  for (int i = 0; i < NR; ++i) {
    p.ws[i] = (uint8_t*)d_ws_all[i];
    p.out[i] = (float*)d_out_all[i];
  }
  p.myws = p.ws[rank];
  p.rank = rank;
  p.pad = 0;
  hipLaunchKernelGGL(gemm_rs_fused, dim3(NWG), dim3(256), 0, stream, p);
  HIP_CHECK(hipGetLastError());
}
```

```cpp
#include <hip/hip_runtime.h>
#include <rccl/rccl.h>
#include <cstdio>
#include <cstdint>

#define HIP_CHECK(expr) do { hipError_t _e = (expr); if (_e != hipSuccess) { \
    fprintf(stderr, "HIP error %s at %s:%d\n", hipGetErrorString(_e), __FILE__, __LINE__); \
    return; } } while (0)

typedef unsigned u32x4 __attribute__((ext_vector_type(4)));
typedef int i32x4 __attribute__((ext_vector_type(4)));
typedef int i32x8 __attribute__((ext_vector_type(8)));
typedef float f32x4 __attribute__((ext_vector_type(4)));
typedef float f32x16 __attribute__((ext_vector_type(16)));

#define SGPR_PAD "s_nop 4\n\t"

namespace {

constexpr int NR = 8;
constexpr int GN = 16384;
constexpr int KS = 512;
constexpr int NWG = 256;
constexpr int KSTEPS = 64;
constexpr int NSTAGE = 6;
constexpr int LDS_CTRL = NSTAGE * 26624;
constexpr int LDS_BYTES = LDS_CTRL + 64;
constexpr int LDS_PK = 9 * 16384;

constexpr int T_W0 = 228, T_OWN = T_W0 + 512, NTASK = T_OWN + 64;

constexpr size_t WS_CALL = 0;
constexpr size_t WS_TMO = 32768;
constexpr size_t WS_DONE = 896u << 10;
constexpr size_t WS_LINK = WS_TMO + 1024;
constexpr size_t WS_DX = WS_LINK + 4096;
constexpr size_t WS_DW = WS_DX + 8192;
constexpr size_t WS_TURN = 768u << 10;
constexpr size_t WS_DATA = 1u << 20;
constexpr size_t XS_BYTES = 10u << 20;
constexpr size_t WT_BYTES = 16u << 20;
constexpr unsigned ITASK = 3328;
constexpr size_t IS_BYTES = 21u << 20;
constexpr size_t PAR_BYTES = XS_BYTES + WT_BYTES + IS_BYTES;
static_assert((size_t)8 * NTASK * ITASK <= IS_BYTES, "");
constexpr size_t OBLK = 13 * 1024;
constexpr size_t WS_OS = WS_DATA + 2 * PAR_BYTES;
constexpr size_t WS_NEED = WS_OS + 8 * 32 * 9 * OBLK;

constexpr unsigned long long DEADLINE_TICKS = 50000000ull;

constexpr int OPEN_AT = 790, NTURN = 20, OUT_REL = 3, OUT_D = 2, OUT_D3 = 3, DEEP_D = 4, DEEP_T = 8;
constexpr int POLL_SLEEP = 6, RX_SLEEP = 16;

struct Params {
  const uint8_t* x;
  const uint8_t* w;
  const float* sx;
  const float* sw;
  uint8_t* ws[NR];
  float* out[NR];
  uint8_t* myws;
  int rank;
  int pad;
};

__device__ __forceinline__ unsigned ld_sys(const void* ptr) {
  return __hip_atomic_load((const unsigned*)ptr, __ATOMIC_RELAXED, __HIP_MEMORY_SCOPE_SYSTEM);
}
__device__ __forceinline__ unsigned ld_agent(const void* ptr) {
  return __hip_atomic_load((const unsigned*)ptr, __ATOMIC_RELAXED, __HIP_MEMORY_SCOPE_AGENT);
}
__device__ __forceinline__ void st_agent(void* ptr, unsigned v) {
  __hip_atomic_store((unsigned*)ptr, v, __ATOMIC_RELAXED, __HIP_MEMORY_SCOPE_AGENT);
}

template <int SLEEP = POLL_SLEEP, class F>
__device__ __forceinline__ bool spin(F ready, unsigned* tmo, unsigned call) {
  const unsigned long long t0 = __builtin_amdgcn_s_memrealtime();
  for (unsigned n = 1;; ++n) {
    if (ready()) return true;
    if ((n & 3) == 0 && (__builtin_amdgcn_readfirstlane(ld_agent(tmo)) == call || __builtin_amdgcn_s_memrealtime() - t0 > DEADLINE_TICKS)) {
      st_agent(tmo, call);
      return false;
    }
    __builtin_amdgcn_s_sleep(SLEEP);
  }
}

__device__ __forceinline__ void st16_sys(void* base, unsigned off, u32x4 v) {
  asm volatile(SGPR_PAD "global_store_dwordx4 %0, %1, %2 sc0 sc1\n\ts_nop 1" :: "v"(off), "v"(v), "s"(base) : "memory");
}

__device__ __forceinline__ void st1_agent(void* base, unsigned v) {
  asm volatile(SGPR_PAD "s_mov_b64 exec, 1\n\tglobal_store_dword %0, %1, %2 sc1\n\ts_mov_b64 exec, -1"
               :: "v"(0u), "v"(v), "s"(base) : "memory");
}

__device__ __forceinline__ void st1_sys(void* base, unsigned v) {
  asm volatile(SGPR_PAD "s_mov_b64 exec, 1\n\tglobal_store_dword %0, %1, %2 sc0 sc1\n\ts_mov_b64 exec, -1"
               :: "v"(0u), "v"(v), "s"(base) : "memory");
}
__device__ __forceinline__ void st16_sys_m(void* base, unsigned off, u32x4 v, unsigned long long mask) {
  asm volatile(SGPR_PAD "s_mov_b64 exec, %3\n\tglobal_store_dwordx4 %0, %1, %2 sc0 sc1\n\ts_mov_b64 exec, -1\n\ts_nop 0"
               :: "v"(off), "v"(v), "s"(base), "s"(mask) : "memory");
}

template <int N> __device__ __forceinline__ void wait_vm() {
  asm volatile("s_waitcnt vmcnt(%0)" :: "n"(N) : "memory");
}

__device__ __forceinline__ unsigned wave_max_u32(unsigned v) {
  v = max(v, (unsigned)__builtin_amdgcn_update_dpp(0, (int)v, 0x111, 0xf, 0xf, false));
  v = max(v, (unsigned)__builtin_amdgcn_update_dpp(0, (int)v, 0x112, 0xf, 0xf, false));
  v = max(v, (unsigned)__builtin_amdgcn_update_dpp(0, (int)v, 0x114, 0xf, 0xf, false));
  v = max(v, (unsigned)__builtin_amdgcn_update_dpp(0, (int)v, 0x118, 0xf, 0xf, false));
  return max(max((unsigned)__builtin_amdgcn_readlane((int)v, 15), (unsigned)__builtin_amdgcn_readlane((int)v, 31)),
             max((unsigned)__builtin_amdgcn_readlane((int)v, 47), (unsigned)__builtin_amdgcn_readlane((int)v, 63)));
}

__device__ __forceinline__ unsigned irec_slot(unsigned idx) { return idx + 3 + min((__umul24(idx, 37u) + 74u) >> 8, 3u); }

__device__ __forceinline__ unsigned wave_sum_u32(unsigned v) {
  v += (unsigned)__builtin_amdgcn_update_dpp(0, (int)v, 0x111, 0xf, 0xf, false);
  v += (unsigned)__builtin_amdgcn_update_dpp(0, (int)v, 0x112, 0xf, 0xf, false);
  v += (unsigned)__builtin_amdgcn_update_dpp(0, (int)v, 0x114, 0xf, 0xf, false);
  v += (unsigned)__builtin_amdgcn_update_dpp(0, (int)v, 0x118, 0xf, 0xf, false);
  return (unsigned)(__builtin_amdgcn_readlane((int)v, 15) + __builtin_amdgcn_readlane((int)v, 31) +
                    __builtin_amdgcn_readlane((int)v, 47) + __builtin_amdgcn_readlane((int)v, 63));
}
__device__ __forceinline__ unsigned bfi(unsigned m, unsigned a, unsigned b) { return (a & m) | (b & ~m); }
constexpr unsigned HDR_RAW = 0x80000000u;

constexpr unsigned IREC_CAP = 122, IHDR_RAW = 0x8000u;

struct InTask {
  u32x4 d[4];
  unsigned long long mask;
  bool raw;
};

__device__ __forceinline__ void put_records2(unsigned q0, unsigned q1, unsigned tag, unsigned& cnt, uint8_t* scr, unsigned dump) {
  unsigned m = bfi(0x55555555u, q0 | q0 >> 1, q1 | q1 << 1);
  for (unsigned long long act = __ballot(m != 0); act != 0 && cnt <= IREC_CAP; act = __ballot(m != 0)) {
    const unsigned bit = (unsigned)__builtin_ctz(m | 0x80000000u), sh = bit & 30u;
    const unsigned idx = cnt + __builtin_amdgcn_mbcnt_hi((unsigned)(act >> 32), __builtin_amdgcn_mbcnt_lo((unsigned)act, 0u));
    const unsigned sl = 2 * irec_slot(idx), at = ((m != 0) & (idx < IREC_CAP)) ? sl : dump;
    *(unsigned short*)(scr + at) = (unsigned short)(tag | (bit & 1u) << 6 | sh << 1 | ((((bit & 1u) ? q1 : q0) >> sh) & 3u));
    m &= m - 1;
    cnt += __popcll(act);
  }
}

__device__ __forceinline__ void enc_in(const u32x4 (&v)[4], InTask& o, unsigned tag, uint8_t* scr, int lane) {
  unsigned sum = 0;
#pragma unroll
  for (int q = 0; q < 4; ++q) sum = __builtin_amdgcn_udot4(v[q][0] & 0x7f7f7f7fu, 0x01010101u, sum, false);
  const unsigned e0 = (unsigned)min(max((int)((wave_sum_u32(sum) + 10445u) >> 12) - 7, 0), 24);
  const unsigned add = 0x80808080u - e0 * 0x04040404u;
  unsigned pay[12], qq[4], cnt = 0;
#pragma unroll
  for (int q = 0; q < 4; ++q) {
    unsigned R[4], S[4];
#pragma unroll
    for (int k = 0; k < 4; ++k) {
      R[k] = (v[q][k] & 0x7f7f7f7fu) + add;
      S[k] = bfi(0x80808080u, v[q][k], R[k]);
    }
    pay[3 * q] = bfi(0x60606060u, S[3] << 5, S[0]);
    pay[3 * q + 1] = bfi(0x60606060u, S[3] << 3, S[1]);
    pay[3 * q + 2] = bfi(0x20202020u, S[3] << 1, bfi(0x40404040u, S[3] >> 1, S[2]));
    qq[q] = bfi(0xc0c0c0c0u, R[3] << 1, bfi(0x30303030u, R[2] >> 1, bfi(0x0c0c0c0cu, R[1] >> 3, (R[0] >> 5) & 0x03030303u)));
  }
  put_records2(qq[0], qq[1], (unsigned)(lane << 8), cnt, scr, 256 + 2 * lane);
  put_records2(qq[2], qq[3], (unsigned)(lane << 8 | 128), cnt, scr, 256 + 2 * lane);
  *(unsigned short*)(scr + (lane < 4 ? lane * 16 : lane < 6 ? lane * 2 - 6 : 256 + 2 * lane)) =
      (unsigned short)(lane < 4 ? tag : lane == 4 ? cnt : e0);
  __builtin_amdgcn_fence(__ATOMIC_ACQ_REL, "wavefront");
  const u32x4 rec = *(const u32x4*)(scr + (lane & 15) * 16);
  __builtin_amdgcn_fence(__ATOMIC_ACQ_REL, "wavefront");
  o.raw = cnt > IREC_CAP;
  if (o.raw) {
#pragma unroll
    for (int q = 0; q < 4; ++q) o.d[q] = v[q];
    o.mask = ~0ull;
  } else {
#pragma unroll
    for (int q = 0; q < 3; ++q) o.d[q] = u32x4{pay[4 * q], pay[4 * q + 1], pay[4 * q + 2], pay[4 * q + 3]};
    o.d[3] = rec;
    o.mask = (1ull << (((irec_slot(cnt) + 31) >> 5) * 4)) - 16;
  }
}

__device__ __forceinline__ void dec_in4(const uint8_t* slot, uint8_t* out, const unsigned (&off)[4], unsigned stride, uint8_t* scr, int lane) {
  u32x4 y[4][3];
  unsigned rec[4];
  asm volatile(SGPR_PAD "global_load_dword %12, %16, %20 sc0 sc1\n\tglobal_load_dword %13, %16, %20 offset:3328 sc0 sc1\n\t"
               "global_load_dword %14, %17, %20 sc0 sc1\n\tglobal_load_dword %15, %17, %20 offset:3328 sc0 sc1\n\t"
               "global_load_dwordx4 %0, %18, %20 offset:-3072 sc0 sc1\n\tglobal_load_dwordx4 %1, %18, %20 offset:-2048 sc0 sc1\n\t"
               "global_load_dwordx4 %2, %18, %20 offset:-1024 sc0 sc1\n\t"
               "global_load_dwordx4 %3, %18, %20 offset:256 sc0 sc1\n\tglobal_load_dwordx4 %4, %18, %20 offset:1280 sc0 sc1\n\t"
               "global_load_dwordx4 %5, %18, %20 offset:2304 sc0 sc1\n\t"
               "global_load_dwordx4 %6, %19, %20 offset:-3072 sc0 sc1\n\tglobal_load_dwordx4 %7, %19, %20 offset:-2048 sc0 sc1\n\t"
               "global_load_dwordx4 %8, %19, %20 offset:-1024 sc0 sc1\n\t"
               "global_load_dwordx4 %9, %19, %20 offset:256 sc0 sc1\n\tglobal_load_dwordx4 %10, %19, %20 offset:1280 sc0 sc1\n\t"
               "global_load_dwordx4 %11, %19, %20 offset:2304 sc0 sc1\n\ts_waitcnt vmcnt(0)"
               : "=&v"(y[0][0]), "=&v"(y[0][1]), "=&v"(y[0][2]), "=&v"(y[1][0]), "=&v"(y[1][1]), "=&v"(y[1][2]),
                 "=&v"(y[2][0]), "=&v"(y[2][1]), "=&v"(y[2][2]), "=&v"(y[3][0]), "=&v"(y[3][1]), "=&v"(y[3][2]),
                 "=&v"(rec[0]), "=&v"(rec[1]), "=&v"(rec[2]), "=&v"(rec[3])
               : "v"(lane * 4), "v"(2 * ITASK + lane * 4), "v"(ITASK + lane * 16), "v"(3 * ITASK + lane * 16), "s"(slot) : "memory");
#pragma unroll
  for (int t = 0; t < 4; ++t) *(u32x4*)(scr + t * 1024 + lane * 16) = u32x4{0u, 0u, 0u, 0u};
  __builtin_amdgcn_fence(__ATOMIC_ACQ_REL, "wavefront");
  unsigned hdr[4];
#pragma unroll
  for (int t = 0; t < 4; ++t) {
    hdr[t] = (unsigned)__builtin_amdgcn_readlane((int)rec[t], 0) >> 16 | (unsigned)__builtin_amdgcn_readlane((int)rec[t], 1) << 16;
    const unsigned end = irec_slot(hdr[t] & 0x7fffu);
#pragma unroll
    for (int i = 0; i < 2; ++i) {
      const unsigned r16 = rec[t] >> (16 * i);
      const bool isrec = i == 0 ? (lane >= 16 || (lane & 3) != 0) && lane != 1 : lane != 0;
      const unsigned val = isrec && (unsigned)(lane * 2 + i) < end ? (r16 & 3u) << ((r16 >> 1) & 30u) : 0u;
      __hip_atomic_fetch_or((unsigned*)(scr + t * 1024 + ((r16 >> 6) & 255u) * 4), val, __ATOMIC_RELAXED, __HIP_MEMORY_SCOPE_WORKGROUP);
    }
  }
  __builtin_amdgcn_fence(__ATOMIC_ACQ_REL, "wavefront");
#pragma unroll
  for (int t = 0; t < 4; ++t) {
    if (hdr[t] & IHDR_RAW) continue;
    const u32x4 qq = *(const u32x4*)(scr + t * 1024 + lane * 16);
    const unsigned add = ((hdr[t] >> 16) & 31u) * 0x04040404u;
#pragma unroll
    for (int q = 0; q < 4; ++q) {
      const unsigned d0 = y[t][(3 * q) >> 2][(3 * q) & 3], d1 = y[t][(3 * q + 1) >> 2][(3 * q + 1) & 3], d2 = y[t][(3 * q + 2) >> 2][(3 * q + 2) & 3];
      const unsigned d3 = bfi(0x80808080u, d2 << 1, bfi(0x10101010u, d2 >> 1, bfi(0x0c0c0c0cu, d1 >> 3, d0 >> 5)));
      const unsigned X[4] = {bfi(0x60606060u, qq[q] << 5, d0), bfi(0x60606060u, qq[q] << 3, d1), bfi(0x60606060u, qq[q] << 1, d2),
                             bfi(0x60606060u, qq[q] >> 1, d3)};
      u32x4 r;
#pragma unroll
      for (int k = 0; k < 4; ++k) r[k] = bfi(0x80808080u, X[k], (X[k] & 0x7f7f7f7fu) + add);
      asm volatile(SGPR_PAD "global_store_dwordx4 %0, %1, %2 sc1\n\ts_nop 1" :: "v"(off[t] + q * stride + lane * 16), "v"(r), "s"(out) : "memory");
    }
  }
  __builtin_amdgcn_fence(__ATOMIC_ACQ_REL, "wavefront");
}

__device__ __forceinline__ bool wait_tasks(const uint8_t* slot, unsigned* tmo, unsigned call, int lane) {
  const uint8_t* q = slot + ((lane >> 2) & 3) * ITASK + (lane & 3) * 16;
  return spin([&] { return __all((ld_sys(q) & 0xffffu) == (call & 0xffffu)) != 0; }, tmo, call);
}

struct PushCtx {
  const uint8_t* x;
  const uint8_t* w;
  uint8_t* dst;
  unsigned* lk;
  uint8_t* scr;
  unsigned xl, wl;
  unsigned par, tick0, tag, call;
  int rank, d;
  bool local;
};

struct RowSet { int r, row0, base, task0; };
__device__ __forceinline__ RowSet row_set(int set) {
  const int a = min(set, 3), o = max(set - 7, 0), b = set - a - o;
  const int own = (set + 1) >> 3, mid = ((set + 5) >> 3) - own;
  return {7 + 2 * mid + own, 224 * mid + 256 * o, 7 * a + 9 * b + 8 * o, 28 * a + 36 * b + 512 * own + 32 * o};
}
__device__ __forceinline__ int block_col(int q, int n) { return ((n & 3) * 4 + (q < 4 ? q : (q ^ n) & 3)) * 1024 + (n >> 2) * 128; }
__device__ __forceinline__ int tile_block(int g, int i) {
  const int c = (i * 43) >> 7, m = i - 3 * c;
  return g < 4 ? i : i >= 24 ? (g - 4) * 8 + i - 24 : 4 * c + m + (m >= g - 4);
}
__device__ __forceinline__ bool task_is_w(int t) { return t >= T_W0 && t < T_OWN; }
__device__ __forceinline__ int task_set(int t) {
  const int lo = min((t * 2341) >> 16, ((t + 24) * 1821) >> 16), hi = 7 + ((t - T_OWN) >> 5);
  return t < T_OWN ? lo : hi;
}

__device__ __forceinline__ int div_r(int v, int r) { return (v * (0x728093 >> (8 * (r - 7)) & 255)) >> 10; }

__device__ __forceinline__ void ld_x(const PushCtx& c, int t, u32x4 (&v)[4]) {
  t = min(t, NTASK - 1);
  const int set = task_set(t);
  const RowSet rs = row_set(set);
  const int p0 = 2 * (t - rs.task0), ks = div_r(p0, rs.r), rg = p0 - ks * rs.r;
  const unsigned o0 = c.xl + (unsigned)((((set < 7 ? set + 1 : 0) ^ c.d) * 512 + rs.row0 + rg * 32) * KS + ks * 64);
  const unsigned o1 = o0 + (unsigned)(rg + 1 == rs.r ? 64 - rg * 32 * KS : 32 * KS);
  asm volatile(SGPR_PAD "global_load_dwordx4 %0, %4, %6\n\tglobal_load_dwordx4 %1, %4, %6 offset:16\n\t"
               "global_load_dwordx4 %2, %5, %6\n\tglobal_load_dwordx4 %3, %5, %6 offset:16"
               : "=&v"(v[0]), "=&v"(v[1]), "=&v"(v[2]), "=&v"(v[3]) : "v"(o0), "v"(o1), "s"(c.x) : "memory");
}

__device__ __forceinline__ void pack_task(const PushCtx& c, const u32x4 (&v)[4], InTask& o, int lane, bool plain) {
  if (plain) {
#pragma unroll
    for (int q = 0; q < 4; ++q) o.d[q] = v[q];
    o.mask = ~0ull;
    o.raw = true;
  } else {
    enc_in(v, o, c.tag, c.scr, lane);
  }
}

__device__ __forceinline__ void send_plain(const PushCtx& c, int t, const u32x4 (&d)[4], int lane) {
  const int u = t - T_W0;
  const bool wr = task_is_w(t);
  const RowSet rs = row_set(task_set(t));
  const unsigned plain = c.par + lane * 16 +
      (wr ? (unsigned)XS_BYTES + (unsigned)((u >> 4) * (512 << 10) + (c.rank * 8 + ((u >> 1) & 7)) * 8192 + (u & 1) * 1024)
          : (unsigned)(rs.base * (128 << 10) + c.rank * rs.r * 16384 + (t - rs.task0) * 4096));
#pragma unroll
  for (int q = 0; q < 4; ++q) st16_sys(c.dst, plain + q * (wr ? 2048 : 1024), d[q]);
}

struct PushState {
  int tp, tc, tn, tl;
  bool gone;
  u32x4 line;
  InTask o;
  u32x4 nx[4];
};

template <bool LDW>
__device__ __forceinline__ void push_step(const PushCtx& c, PushState& s, int lane) {
  unsigned w[16], tk;
  u32x4 x[4];
  const bool draw = s.tl < NTASK;
  asm volatile(SGPR_PAD "s_mov_b64 exec, 1\n\tglobal_atomic_add %0, %1, %2, %3 sc0\n\ts_mov_b64 exec, -1"
               : "=&v"(tk) : "v"(0u), "v"(draw ? 1u : 0u), "s"(c.lk) : "memory");
  if (s.tl == OPEN_AT) st1_agent(c.lk + 32, c.call);
  if constexpr (LDW) {
    const int u = s.tl - T_W0;
    unsigned o = c.wl + (unsigned)((((u >> 1) & 7) * 64 + (u & 1) * 16) * GN + block_col(c.d, u >> 4));
    asm volatile(SGPR_PAD
                 "global_load_dword %0, %16, %17\n\tv_add_u32 %16, 0x4000, %16\n\t"
                 "global_load_dword %1, %16, %17\n\tv_add_u32 %16, 0x4000, %16\n\t"
                 "global_load_dword %2, %16, %17\n\tv_add_u32 %16, 0x4000, %16\n\t"
                 "global_load_dword %3, %16, %17\n\tv_add_u32 %16, 0x4000, %16\n\t"
                 "global_load_dword %4, %16, %17\n\tv_add_u32 %16, 0x4000, %16\n\t"
                 "global_load_dword %5, %16, %17\n\tv_add_u32 %16, 0x4000, %16\n\t"
                 "global_load_dword %6, %16, %17\n\tv_add_u32 %16, 0x4000, %16\n\t"
                 "global_load_dword %7, %16, %17\n\tv_add_u32 %16, 0x4000, %16\n\t"
                 "global_load_dword %8, %16, %17\n\tv_add_u32 %16, 0x4000, %16\n\t"
                 "global_load_dword %9, %16, %17\n\tv_add_u32 %16, 0x4000, %16\n\t"
                 "global_load_dword %10, %16, %17\n\tv_add_u32 %16, 0x4000, %16\n\t"
                 "global_load_dword %11, %16, %17\n\tv_add_u32 %16, 0x4000, %16\n\t"
                 "global_load_dword %12, %16, %17\n\tv_add_u32 %16, 0x4000, %16\n\t"
                 "global_load_dword %13, %16, %17\n\tv_add_u32 %16, 0x4000, %16\n\t"
                 "global_load_dword %14, %16, %17\n\tv_add_u32 %16, 0x4000, %16\n\t"
                 "global_load_dword %15, %16, %17\n\t"
                 : "=&v"(w[0]), "=&v"(w[1]), "=&v"(w[2]), "=&v"(w[3]), "=&v"(w[4]), "=&v"(w[5]), "=&v"(w[6]), "=&v"(w[7]),
                   "=&v"(w[8]), "=&v"(w[9]), "=&v"(w[10]), "=&v"(w[11]), "=&v"(w[12]), "=&v"(w[13]), "=&v"(w[14]), "=&v"(w[15]), "+v"(o) : "s"(c.w) : "memory");
    static_assert(GN == 0x4000, "");
  } else {
    ld_x(c, s.tl, x);
  }
  const unsigned is0 = c.par + (unsigned)(XS_BYTES + WT_BYTES) + (unsigned)(c.rank * NTASK) * ITASK + lane * 16;
  if (s.tp >= 0) st16_sys_m(c.dst, is0 + (unsigned)s.tp * ITASK, s.line, 15ull);
  if (s.tc < NTASK) {
    if (!s.o.raw) {
      const unsigned slot = is0 + (unsigned)s.tc * ITASK;
      asm volatile(SGPR_PAD "global_store_dwordx4 %0, %1, %4 offset:256 sc0 sc1\n\tglobal_store_dwordx4 %0, %2, %4 offset:1280 sc0 sc1\n\t"
                   "global_store_dwordx4 %0, %3, %4 offset:2304 sc0 sc1\n\ts_nop 1"
                   :: "v"(slot), "v"(s.o.d[0]), "v"(s.o.d[1]), "v"(s.o.d[2]), "s"(c.dst) : "memory");
      if (s.o.mask != 0) st16_sys_m(c.dst, slot, s.o.d[3], s.o.mask);
      s.line = s.o.d[3];
    } else {
      if (!s.gone) send_plain(c, s.tc, s.o.d, lane);
      s.line = u32x4{c.tag | IHDR_RAW << 16, 0u, 0u, 0u};
    }
  }
  pack_task(c, s.nx, s.o, lane, c.local || s.tn >= NTASK);
  if constexpr (LDW) {
    asm volatile("s_waitcnt vmcnt(0)" : "+v"(tk), "+v"(w[0]), "+v"(w[1]), "+v"(w[2]), "+v"(w[3]), "+v"(w[4]), "+v"(w[5]), "+v"(w[6]), "+v"(w[7]),
                 "+v"(w[8]), "+v"(w[9]), "+v"(w[10]), "+v"(w[11]), "+v"(w[12]), "+v"(w[13]), "+v"(w[14]), "+v"(w[15]) :: "memory");
#pragma unroll
    for (int q = 0; q < 4; ++q) {
      const unsigned d0 = w[4 * q], d1 = w[4 * q + 1], d2 = w[4 * q + 2], d3 = w[4 * q + 3];
      const unsigned t0 = __builtin_amdgcn_perm(d1, d0, 0x05010400u), t1 = __builtin_amdgcn_perm(d1, d0, 0x07030602u);
      const unsigned t2 = __builtin_amdgcn_perm(d3, d2, 0x05010400u), t3 = __builtin_amdgcn_perm(d3, d2, 0x07030602u);
      s.nx[0][q] = __builtin_amdgcn_perm(t2, t0, 0x05040100u);
      s.nx[1][q] = __builtin_amdgcn_perm(t2, t0, 0x07060302u);
      s.nx[2][q] = __builtin_amdgcn_perm(t3, t1, 0x05040100u);
      s.nx[3][q] = __builtin_amdgcn_perm(t3, t1, 0x07060302u);
    }
  } else {
    asm volatile("s_waitcnt vmcnt(0)" : "+v"(tk), "+v"(x[0]), "+v"(x[1]), "+v"(x[2]), "+v"(x[3]) :: "memory");
#pragma unroll
    for (int q = 0; q < 4; ++q) s.nx[q] = x[q];
  }
  s.tp = s.tc; s.tc = s.tn; s.tn = s.tl; s.gone = false;
  s.tl = draw ? (int)min(__builtin_amdgcn_readfirstlane(tk) - c.tick0, (unsigned)NTASK) : NTASK;
}

#define GLDS_BITS " sc0 sc1"
__device__ __forceinline__ void glds2(const uint8_t* gbase, unsigned voff, unsigned lds) {
  unsigned keep;
  asm volatile("s_mov_b32 %0, m0\n\ts_mov_b32 m0, %3\n\ts_nop 2\n\t"
               "global_load_lds_dwordx4 %1, %2" GLDS_BITS "\n\t"
               "global_load_lds_dwordx4 %1, %2 offset:1024" GLDS_BITS "\n\t"
               "s_mov_b32 m0, %0"
               : "=&s"(keep) : "v"(voff), "s"(gbase), "s"(lds));
}
__device__ __forceinline__ void glds2o(const uint8_t* gbase, unsigned voff, unsigned lds) {
  unsigned keep;
  asm volatile("s_mov_b32 %0, m0\n\ts_mov_b32 m0, %3\n\ts_nop 2\n\t"
               "global_load_lds_dwordx4 %1, %2 offset:2048" GLDS_BITS "\n\t"
               "global_load_lds_dwordx4 %1, %2 offset:3072" GLDS_BITS "\n\t"
               "s_mov_b32 m0, %0"
               : "=&s"(keep) : "v"(voff), "s"(gbase), "s"(lds));
}
__device__ __forceinline__ void glds_half(const uint8_t* gbase, unsigned voff, unsigned lds) {
  unsigned keep;
  asm volatile("s_mov_b32 %0, m0\n\ts_mov_b32 m0, %3\n\ts_mov_b32 exec_hi, 0\n\ts_nop 1\n\t"
               "global_load_lds_dwordx4 %1, %2" GLDS_BITS "\n\t"
               "s_mov_b32 exec_hi, -1\n\ts_mov_b32 m0, %0"
               : "=&s"(keep) : "v"(voff), "s"(gbase), "s"(lds));
}

struct Frags { i32x8 a[2], b[4], a9, b9; };

__device__ __forceinline__ i32x8 ld_frag(const uint8_t* q) {
  const i32x4 lo = *(const i32x4*)q, hi = *(const i32x4*)(q + 1024);
  return __builtin_shufflevector(lo, hi, 0, 1, 2, 3, 4, 5, 6, 7);
}

#define MFMA(mi, j) acc[mi][j] = __builtin_amdgcn_mfma_scale_f32_32x32x64_f8f6f4(cur.a[mi], cur.b[j], acc[mi][j], 1, 1, 0, 0, 0, 0)
#define SB() __builtin_amdgcn_sched_barrier(0)

struct Ring {
  const uint8_t* ga;
  const uint8_t* gb;
  unsigned astep;
  unsigned va, vb, v9;
  unsigned wr, wr_end;
  unsigned wb, w9;
  unsigned rd;
  const uint8_t* qa;
  const uint8_t* qb;
  const uint8_t* q9;
  const uint8_t* la;
  const uint8_t* lb;
  const uint8_t* l9;
};

template <int MODE, int LEFT, bool DMA, bool RD>
__device__ __forceinline__ void kstep(f32x16 (&acc)[2][4], f32x16& acc9, const Frags& cur, Frags& nxt, Ring& g) {
  constexpr bool NINE = MODE == 9, TWO = MODE != 7;
  constexpr int STAGE = NINE ? 26624 : 24576;
  MFMA(0, 0); SB();
  asm volatile("s_waitcnt vmcnt(%0) lgkmcnt(0)\n\ts_barrier" :: "n"(LEFT * (NINE ? 7 : 6)) : "memory");
  SB();
  MFMA(0, 1); SB();
  if constexpr (RD) { nxt.a[0] = ld_frag(g.qa); nxt.b[0] = ld_frag(g.qb); }
  SB();
  MFMA(0, 2); SB();
  if constexpr (DMA) glds2(g.ga, g.va, g.wr);
  SB();
  MFMA(0, 3); SB();
  if constexpr (RD) { nxt.b[1] = ld_frag(g.qb + 2048); nxt.b[2] = ld_frag(g.qb + 4096); }
  SB();
  if constexpr (TWO) MFMA(1, 0);
  SB();
  if constexpr (DMA) glds2(g.gb, g.vb, g.wr + g.wb);
  SB();
  if constexpr (TWO) MFMA(1, 1);
  SB();
  if constexpr (RD) { nxt.b[3] = ld_frag(g.qb + 6144); if constexpr (TWO) nxt.a[1] = ld_frag(g.qa + 2048); }
  g.rd = g.rd == (NSTAGE - 1) * STAGE ? 0 : g.rd + STAGE;
  g.qa = g.la + g.rd;
  if constexpr (!NINE) g.qb = g.lb + g.rd;
  SB();
  if constexpr (TWO) MFMA(1, 2);
  SB();
  if constexpr (DMA) glds2o(g.ga, g.va, g.wr);
  SB();
  if constexpr (NINE) {
    if constexpr (TWO) MFMA(1, 3);
    SB();
    if constexpr (RD) { nxt.a9 = ld_frag(g.qb - 2048); nxt.b9 = ld_frag(g.q9); }
    g.qb = g.lb + g.rd; g.q9 = g.l9 + g.rd;
    if constexpr (DMA) g.gb += 8192;
    SB();
    acc9 = __builtin_amdgcn_mfma_scale_f32_32x32x64_f8f6f4(cur.a9, cur.b9, acc9, 1, 1, 0, 0, 0, 0); SB();
    if constexpr (DMA) glds_half(g.ga, g.v9, g.wr + g.w9);
  } else {
    if constexpr (TWO) MFMA(1, 3);
    SB();
    if constexpr (DMA) g.gb += 8192;
  }
  if constexpr (DMA) {
    g.ga += g.astep;
    g.wr += STAGE; g.wr = g.wr == g.wr_end ? g.wr - NSTAGE * STAGE : g.wr;
  }
  SB();
}

template <int MODE>
__device__ __forceinline__ void gemm_tile(const uint8_t* ga, unsigned astep, const uint8_t* gb, uint8_t* lds, int wave, int lane) {
  constexpr bool NINE = MODE == 9, TWO = MODE != 7;
  constexpr int STAGE = NINE ? 26624 : 24576, BOFF = STAGE - 8192;
  f32x16 acc[2][4], acc9;
  Ring g;
  g.ga = ga; g.gb = gb; g.astep = astep;
  g.va = lane * 16 + wave * 4096; g.vb = lane * 16 + wave * 2048; g.v9 = 16384 + lane * 16 + wave * 512;
  g.wr = (unsigned)(uintptr_t)lds + wave * 4096; g.wr_end = g.wr + NSTAGE * STAGE;
  g.wb = BOFF - wave * 2048; g.w9 = 16384 - wave * 3584;
#pragma unroll
  for (int s = 0; s < NSTAGE; ++s) {
    glds2(g.ga, g.va, g.wr + s * STAGE);
    glds2(g.gb, g.vb, g.wr + s * STAGE + g.wb);
    glds2o(g.ga, g.va, g.wr + s * STAGE);
    if constexpr (NINE) glds_half(g.ga, g.v9, g.wr + s * STAGE + g.w9);
    g.ga += astep; g.gb += 8192;
  }
#pragma unroll
  for (int r = 0; r < 16; ++r) {
    acc9[r] = 0.f;
#pragma unroll
    for (int j = 0; j < 4; ++j) { acc[0][j][r] = 0.f; acc[1][j][r] = 0.f; }
  }
  g.la = lds + lane * 16 + wave * 4096; g.lb = lds + BOFF + lane * 16; g.l9 = g.lb + wave * 2048;
  asm volatile("s_waitcnt vmcnt(%0)\n\ts_barrier" :: "n"(5 * (NINE ? 7 : 6)) : "memory");
  Frags f0, f1;
  f0.a[0] = ld_frag(g.la); f0.b[0] = ld_frag(g.lb); f0.b[1] = ld_frag(g.lb + 2048);
  f0.b[2] = ld_frag(g.lb + 4096); f0.b[3] = ld_frag(g.lb + 6144);
  if constexpr (TWO) f0.a[1] = ld_frag(g.la + 2048);
  if constexpr (NINE) { f0.a9 = ld_frag(g.lb - 2048); f0.b9 = ld_frag(g.l9); }
  g.rd = STAGE; g.qa = g.la + STAGE; g.qb = g.lb + STAGE; g.q9 = g.l9 + STAGE;
#pragma unroll 1
  for (int s = 0; s < KSTEPS - NSTAGE; s += 2) {
    kstep<MODE, 4, true, true>(acc, acc9, f0, f1, g);
    kstep<MODE, 4, true, true>(acc, acc9, f1, f0, g);
  }
  kstep<MODE, 4, false, true>(acc, acc9, f0, f1, g);
  kstep<MODE, 3, false, true>(acc, acc9, f1, f0, g);
  kstep<MODE, 2, false, true>(acc, acc9, f0, f1, g);
  kstep<MODE, 1, false, true>(acc, acc9, f1, f0, g);
  kstep<MODE, 0, false, true>(acc, acc9, f0, f1, g);
  auto park = [&](const f32x16& a, int i) {
#pragma unroll
    for (int q = 0; q < 4; ++q)
      *(f32x4*)(lds + wave * 32768 + (i * 4 + q) * 1024 + lane * 16) = f32x4{a[4 * q], a[4 * q + 1], a[4 * q + 2], a[4 * q + 3]};
  };
  {
    const Frags& cur = f1;
    MFMA(0, 0); SB();
    asm volatile("s_waitcnt lgkmcnt(0)\n\ts_barrier" ::: "memory");
    SB();
    MFMA(0, 1); SB();
    MFMA(0, 2); SB();
    park(acc[0][0], 0); SB();
    MFMA(0, 3); SB();
    park(acc[0][1], 1); SB();
    if constexpr (TWO) MFMA(1, 0);
    SB();
    park(acc[0][2], 2); SB();
    if constexpr (TWO) MFMA(1, 1);
    SB();
    park(acc[0][3], 3); SB();
    if constexpr (TWO) {
      MFMA(1, 2); SB();
      park(acc[1][0], 4); SB();
      MFMA(1, 3); SB();
      park(acc[1][1], 5); SB();
      if constexpr (NINE) { acc9 = __builtin_amdgcn_mfma_scale_f32_32x32x64_f8f6f4(cur.a9, cur.b9, acc9, 1, 1, 0, 0, 0, 0); SB(); }
      park(acc[1][2], 6);
      park(acc[1][3], 7);
    }
  }
  if constexpr (NINE) {
#pragma unroll
    for (int q = 0; q < 4; ++q)
      *(f32x4*)(lds + 8 * 16384 + wave * 4096 + q * 1024 + lane * 16) = f32x4{acc9[4 * q], acc9[4 * q + 1], acc9[4 * q + 2], acc9[4 * q + 3]};
    __syncthreads();
  }
  __builtin_amdgcn_fence(__ATOMIC_ACQ_REL, "wavefront");
}

constexpr unsigned REC_CAP = 510, HDR_B = 0x40000000u;

__device__ __forceinline__ void put_records4(const unsigned (&q)[4], unsigned tag, unsigned& cnt, uint8_t* scr, unsigned dump) {
  unsigned m = 0;
#pragma unroll
  for (int k = 0; k < 4; ++k)
    m |= ((((q[k] & 0x77777777u) + 0x77777777u) | q[k]) & 0x88888888u) >> (3 - k);
  for (unsigned long long act = __ballot(m != 0); act != 0 && cnt <= REC_CAP; act = __ballot(m != 0)) {
    const unsigned bit = (unsigned)__builtin_ctz(m | 0x80000000u), sh = bit & 28u;
    const unsigned m1 = (unsigned)__builtin_amdgcn_sbfe((int)bit, 0, 1), m2 = (unsigned)__builtin_amdgcn_sbfe((int)bit, 1, 1);
    const unsigned w = bfi(m2, bfi(m1, q[3], q[2]), bfi(m1, q[1], q[0]));
    const unsigned idx = cnt + __builtin_amdgcn_mbcnt_hi((unsigned)(act >> 32), __builtin_amdgcn_mbcnt_lo((unsigned)act, 0u));
    const unsigned at = (m != 0 && idx < REC_CAP) ? 4 + 2 * idx : dump;
    *(unsigned short*)(scr + at) = (unsigned short)(tag | (bit & 3u) << 7 | sh << 2 | ((w >> sh) & 15u));
    m &= m - 1;
    cnt += __popcll(act);
  }
}

template <bool MODEB>
__device__ __forceinline__ bool enc_body(const float (&v)[4][16], float up, unsigned& cnt, uint8_t* blk, int lane) {
  unsigned bad = 0, low = 0;
#pragma unroll
  for (int h = 0; h < 2; ++h) {
    unsigned G[32];
#pragma unroll
    for (int j = 0; j < 4; ++j)
#pragma unroll
      for (int r = 0; r < 8; ++r) {
        const float x = v[j][8 * h + r] * up;
        unsigned g = (unsigned)(int)x;
        bad = __builtin_amdgcn_bitop3_b32(__builtin_bit_cast(unsigned, (float)(int)g), __builtin_bit_cast(unsigned, x), bad, 0xbe);
        low |= g;
        if (MODEB) g = g << 1 | (((g ^ g << 1) >> 27) & 16u);
        G[4 * r + j] = g;
      }
    unsigned q[4];
#pragma unroll
    for (int k = 0; k < 4; ++k) {
      const unsigned* g = G + 8 * k;
      const unsigned b0 = __builtin_amdgcn_perm(__builtin_amdgcn_perm(g[3], g[2], 0x0c0c0400u), __builtin_amdgcn_perm(g[1], g[0], 0x0c0c0400u), 0x05040100u);
      const unsigned b1 = __builtin_amdgcn_perm(__builtin_amdgcn_perm(g[7], g[6], 0x0c0c0400u), __builtin_amdgcn_perm(g[5], g[4], 0x0c0c0400u), 0x05040100u);
      q[k] = (b0 >> 4) | b1;
    }
#pragma unroll
    for (int j = 0; j < 2; ++j) {
      unsigned d[12];
#pragma unroll
      for (int g = 0; g < 4; ++g) {
        const unsigned* x = G + 16 * j + 4 * g;
        d[3 * g] = __builtin_amdgcn_perm(x[1], x[0], 0x05030201u);
        d[3 * g + 1] = __builtin_amdgcn_perm(x[2], x[1], 0x06050302u);
        d[3 * g + 2] = __builtin_amdgcn_perm(x[3], x[2], 0x07060503u);
      }
#pragma unroll
      for (int i = 0; i < 3; ++i)
        *(u32x4*)(blk + (1 + 6 * h + 3 * j + i) * 1024 + lane * 16) = u32x4{d[4 * i], d[4 * i + 1], d[4 * i + 2], d[4 * i + 3]};
    }
    put_records4(q, (unsigned)(lane << 10 | h << 9), cnt, blk, 13 * 1024 + 2 * lane);
  }
  return __ballot((bad | (low & 15u)) != 0) == 0 && cnt <= REC_CAP;
}

__device__ __forceinline__ unsigned enc_out(const float (&v)[4][16], float sc, uint8_t* blk, int lane) {
  float m = 0.f;
#pragma unroll
  for (int j = 0; j < 4; ++j)
#pragma unroll
    for (int r = 0; r < 16; r += 2) asm("v_max3_f32 %0, |%1|, |%2|, %0" : "+v"(m) : "v"(v[j][r]), "v"(v[j][r + 1]));
  const int e = max((int)(wave_max_u32(__builtin_bit_cast(unsigned, m)) >> 23) - 157, -126);
  const float up = __builtin_bit_cast(float, (unsigned)(127 - min(e, 0)) << 23);
  int ntop = 0, nlow = 0;
#pragma unroll
  for (int r = 0; r < 8; ++r) {
    const unsigned g = (unsigned)(int)(v[r & 3][2 * r] * up);
    ntop += __popcll(__ballot((int)(g ^ (g << 1)) < 0));
    nlow += __popcll(__ballot((g & 128u) != 0));
  }
  unsigned cnt = 0;
  const bool modeb = ntop < nlow;
  const bool fits = e <= 0 && (modeb ? enc_body<true>(v, up, cnt, blk, lane) : enc_body<false>(v, up, cnt, blk, lane));
  if (!fits) {
#pragma unroll
    for (int r = 0; r < 16; ++r) *(f32x4*)(blk + r * 1024 + lane * 16) = f32x4{v[0][r] * sc, v[1][r] * sc, v[2][r] * sc, v[3][r] * sc};
    return 0;
  }
  *(unsigned*)(blk + (lane == 0 ? 0 : 13 * 1024 + 4 * lane)) = cnt | (unsigned)(e + 127) << 16 | (modeb ? HDR_B : 0u);
  return ((4 + 2 * cnt + 63) >> 6) * 4;
}

__device__ __forceinline__ void st16_out(void* base, unsigned off, f32x4 v) {
  asm volatile(SGPR_PAD "global_store_dwordx4 %0, %1, %2 sc1\n\ts_nop 1" :: "v"(off), "v"(v), "s"(base) : "memory");
}

__device__ __forceinline__ constexpr unsigned row_off(int r) { return (unsigned)((r & 3) + 8 * (r >> 2)) * (GN * 4); }

__device__ __forceinline__ void send_out(const uint8_t* blk, unsigned nl, bool dead, uint8_t* slot, float* ob, unsigned off,
                                         int lane, unsigned* next, unsigned call, bool last, int depth) {
  const bool raw = nl == 0 || dead;
  const u32x4 h = *(const u32x4*)(blk + lane * 16);
  st16_sys_m(slot, lane * 16, raw ? u32x4{HDR_RAW, 0u, 0u, 0u} : h, raw ? 15ull : nl >= 64 ? ~0ull : (1ull << nl) - 1);
  const int n = raw ? 16 : 12;
  void* const base = raw ? (void*)ob : (void*)slot;
  const uint8_t* const src = blk + (raw ? 0 : 1024) + lane * 16;
  const unsigned nan = dead ? 0x7fc00000u : 0u;
#pragma unroll 1
  for (int i = 0; i < n; ++i) {
    const u32x4 v = *(const u32x4*)(src + i * 1024) | nan;
    if (depth == OUT_D) wait_vm<OUT_D - 1>(); else if (depth == OUT_D3) wait_vm<OUT_D3 - 1>(); else wait_vm<DEEP_D - 1>();
    if (last && i == n - OUT_REL) st1_agent(next, call);
    st16_sys(base, raw ? off + row_off(i) : (unsigned)(1024 + i * 1024 + lane * 16), v);
  }
}

__device__ __forceinline__ void dec_out(const uint8_t* slot, float* ob, unsigned off, float sc, unsigned* tmo, unsigned call, uint8_t* scr, int lane) {
  u32x4 rec, y[12];
  unsigned tmv;
  asm volatile(SGPR_PAD "global_load_dword %13, %16, %18 sc1\n\t"
               "global_load_dwordx4 %0, %14, %17 offset:-4096 sc0 sc1\n\tglobal_load_dwordx4 %1, %14, %17 offset:-3072 sc0 sc1\n\t"
               "global_load_dwordx4 %2, %14, %17 offset:-2048 sc0 sc1\n\tglobal_load_dwordx4 %3, %14, %17 offset:-1024 sc0 sc1\n\t"
               "global_load_dwordx4 %4, %14, %17 sc0 sc1\n\tglobal_load_dwordx4 %5, %14, %17 offset:1024 sc0 sc1\n\t"
               "global_load_dwordx4 %6, %14, %17 offset:2048 sc0 sc1\n\tglobal_load_dwordx4 %7, %14, %17 offset:3072 sc0 sc1\n\t"
               "global_load_dwordx4 %8, %15, %17 offset:-4096 sc0 sc1\n\tglobal_load_dwordx4 %9, %15, %17 offset:-3072 sc0 sc1\n\t"
               "global_load_dwordx4 %10, %15, %17 offset:-2048 sc0 sc1\n\tglobal_load_dwordx4 %11, %15, %17 offset:-1024 sc0 sc1\n\t"
               "global_load_dwordx4 %12, %15, %17 sc0 sc1\n\ts_waitcnt vmcnt(0)"
               : "=&v"(rec), "=&v"(y[0]), "=&v"(y[1]), "=&v"(y[2]), "=&v"(y[3]), "=&v"(y[4]), "=&v"(y[5]), "=&v"(y[6]),
                 "=&v"(y[7]), "=&v"(y[8]), "=&v"(y[9]), "=&v"(y[10]), "=&v"(y[11]), "=&v"(tmv)
               : "v"(4096 + lane * 16), "v"(12288 + lane * 16), "v"(0u), "s"(slot), "s"(tmo) : "memory");
  const unsigned hdr = __builtin_amdgcn_readfirstlane(rec[0]);
  if (__builtin_amdgcn_readfirstlane(tmv) == call) {
    const float n = __builtin_nanf("");
#pragma unroll 1
    for (int r = 0; r < 16; ++r) st16_out(ob, off + row_off(r), f32x4{n, n, n, n});
    return;
  }
  if (hdr & HDR_RAW) return;
  const unsigned cnt = hdr & 0xffffu;
  const float down = __builtin_bit_cast(float, ((hdr >> 16) & 255u) << 23);
  *(u32x4*)(scr + lane * 32) = u32x4{0u, 0u, 0u, 0u};
  *(u32x4*)(scr + lane * 32 + 16) = u32x4{0u, 0u, 0u, 0u};
  __builtin_amdgcn_fence(__ATOMIC_ACQ_REL, "wavefront");
#pragma unroll
  for (int i = 0; i < 8; ++i) {
    const unsigned r16 = (rec[i >> 1] >> (16 * (i & 1))) & 0xffffu;
    const unsigned val = (unsigned)(lane * 8 + i - 2) < cnt ? (r16 & 15u) << ((r16 >> 2) & 28u) : 0u;
    __hip_atomic_fetch_or((unsigned*)(scr + (r16 >> 7) * 4), val, __ATOMIC_RELAXED, __HIP_MEMORY_SCOPE_WORKGROUP);
  }
  __builtin_amdgcn_fence(__ATOMIC_ACQ_REL, "wavefront");
  const u32x4 q0 = *(const u32x4*)(scr + lane * 32), q1 = *(const u32x4*)(scr + lane * 32 + 16);
  __builtin_amdgcn_fence(__ATOMIC_ACQ_REL, "wavefront");
  const bool modeb = (hdr & HDR_B) != 0;
#pragma unroll
  for (int g = 0; g < 16; ++g) {
    const unsigned d0 = y[(3 * g) >> 2][(3 * g) & 3], d1 = y[(3 * g + 1) >> 2][(3 * g + 1) & 3], d2 = y[(3 * g + 2) >> 2][(3 * g + 2) & 3];
    const unsigned qw = g < 8 ? q0[(g >> 1) & 3] : q1[(g >> 1) & 3];
    const unsigned q = ((g & 1) ? qw : qw << 4) & 0xf0f0f0f0u;
    unsigned G[4] = {__builtin_amdgcn_perm(d0, q, 0x06050400u), __builtin_amdgcn_perm(d1, d0, 0x0504030cu) | ((q >> 8) & 255u),
                     __builtin_amdgcn_perm(d2, d1, 0x0403020cu) | ((q >> 16) & 255u), __builtin_amdgcn_perm(d2, q, 0x07060503u)};
    f32x4 v;
#pragma unroll
    for (int j = 0; j < 4; ++j) {
      if (modeb) G[j] = ((unsigned)((int)G[j] >> 1) & ~8u) ^ (G[j] << 27);
      v[j] = ((float)(int)G[j] * down) * sc;
    }
    st16_out(ob, off + row_off(g), v);
  }
}

__device__ __forceinline__ int wave_blocks(int r, int wave) { return wave < 3 ? 2 : r - 6; }
__device__ __forceinline__ unsigned out_off(int row0, int col, int wave, int lane) {
  return (unsigned)(((row0 + wave * 64 + 4 * (lane >> 5)) * GN + col + 4 * (lane & 31)) * 4);
}

__device__ __forceinline__ void recv_tile(uint8_t* myws, float* out, int g, int src, int i, int k0, int k1, int wave, int lane,
                                          unsigned call, unsigned* tmo, float sc, uint8_t* scr) {
  const unsigned off = out_off(g < 4 ? 0 : 224, block_col(src, tile_block(g, i)), wave, lane);
  const uint8_t* const fl = myws + WS_DONE + (size_t)((src * 32 + i) * 4 + wave) * 128;
  const int nb = wave_blocks(g < 4 ? 7 : 9, wave);
#pragma unroll 1
  for (int k = k0; k < min(nb, k1); ++k) {
    spin<RX_SLEEP>([&] { return __builtin_amdgcn_readfirstlane(ld_sys(fl)) > 4 * call + k; }, tmo, call);
    dec_out(myws + WS_OS + (size_t)((src * 32 + i) * 9 + wave * 2 + k) * OBLK, out, off + (unsigned)k * 32 * (GN * 4), sc, tmo, call, scr, lane);
  }
}

}

__global__ void __launch_bounds__(256, 1) gemm_rs_fused(const Params p) {
  __shared__ __attribute__((aligned(16))) uint8_t lds[LDS_BYTES];
  const int b = blockIdx.x;
  const int lane = threadIdx.x & 63;
  const int wave = __builtin_amdgcn_readfirstlane(threadIdx.x >> 6);
  const int rank = p.rank;
  uint8_t* const myws = p.myws;
  asm volatile("" :: "s"(myws), "s"(rank), "s"(p.x), "s"(p.w));

  const int grp = b & 7, dst = grp ^ rank, ti = 31 - (b >> 3);
  const bool local = grp == 0 || (grp >= 4 && ti >= 24);
  const int set = local ? 7 + (grp != 0) : grp - 1, cb = tile_block(grp, ti);
  const RowSet rs = row_set(set);
  unsigned* const callp = (unsigned*)(myws + WS_CALL + (size_t)b * 128);
  unsigned* const tmo = (unsigned*)(myws + WS_TMO);
  unsigned* const lk = (unsigned*)(myws + WS_LINK) + grp * 64;
  uint8_t* const scr = lds + LDS_PK + wave * 2048;

  unsigned call;
  if (ti >= 24) {
    const int w = ((b >> 3) * 4 + wave) & 31;
    PushCtx c;
    c.x = p.x; c.w = p.w; c.dst = p.ws[dst]; c.lk = lk; c.rank = rank; c.d = dst;
    c.xl = (unsigned)((lane & 31) * KS + (lane >> 5) * 32);
    c.wl = (unsigned)((lane >> 5) * 32 * GN + (lane & 31) * 4);
    c.scr = scr;
    c.local = grp == 0;
    PushState s;
    u32x4 x0[4];
    unsigned cv;
    asm volatile(SGPR_PAD "global_load_dword %0, %1, %2 sc1" : "=v"(cv) : "v"(0u), "s"(callp) : "memory");
    ld_x(c, w, x0);
    __builtin_amdgcn_sched_barrier(0);
    ld_x(c, w + 32, s.nx);
    unsigned first = (unsigned)(((w >= 28) * 7 * 128 + rank * 7 * 16 + (w >= 28 ? w - 28 : w) * 4) << 10) + lane * 16;
    asm volatile("" : "+v"(first));
    asm volatile("s_waitcnt vmcnt(4)" : "+v"(cv), "+v"(x0[0]), "+v"(x0[1]), "+v"(x0[2]), "+v"(x0[3]) :: "memory");
    call = __builtin_amdgcn_readfirstlane(cv) + 1;
    c.par = (unsigned)(WS_DATA + (call & 1) * PAR_BYTES);
#pragma unroll
    for (int q = 0; q < 4; ++q) st16_sys(c.dst, c.par + first + q * 1024, x0[q]);
    asm volatile("s_waitcnt vmcnt(4)" : "+v"(s.nx[0]), "+v"(s.nx[1]), "+v"(s.nx[2]), "+v"(s.nx[3]) :: "memory");
    c.tick0 = (call - 1) * (NTASK - 64) - 96;
    c.tag = call & 0xffffu;
    c.call = call;
    s.tp = -1; s.tc = w; s.tn = w + 32; s.tl = w + 64; s.gone = true;
    s.line = u32x4{0u, 0u, 0u, 0u};
    pack_task(c, x0, s.o, lane, true);
#pragma unroll 1
    while (s.tp < NTASK) {
      if (task_is_w(s.tl)) push_step<true>(c, s, lane);
      else push_step<false>(c, s, lane);
    }
  } else {
    call = __builtin_amdgcn_readfirstlane(ld_agent(callp)) + 1;
  }
  uint8_t* const par = myws + WS_DATA + (size_t)(call & 1) * PAR_BYTES;

#pragma unroll 1
  for (int job = 0; job < 4; ++job) {
    const bool jw = grp == 0 ? job < 3 : job == 1;
    const int beta = grp == 0 ? (ti * 4 + wave) & 63 : (23 - ti) * 4 + wave;
    if (grp == 0 ? (job == 2 && ti >= 24) : (ti >= 24 || job >= 2 || (job == 0 && beta >= 8 * rs.r))) continue;
    const int n = job == 2 ? 24 + (ti & 7) : cb;
    const RowSet js = row_set(grp == 0 ? 7 + (ti >> 4) : set);
    const int bs = div_r(beta, js.r), bu = beta - bs * js.r;
    const int src = !jw ? bs : grp != 0 ? dst : job == 0 ? rank : job == 1 ? rank ^ (4 + (ti & 3)) : rank ^ (1 + (ti >> 3));
    const int t0 = jw ? T_W0 + n * 16 + wave * 4 : js.task0 + bu * 4;
    const uint8_t* const slot = par + XS_BYTES + WT_BYTES + (size_t)(src * NTASK + t0) * ITASK;
    if (!wait_tasks(slot, tmo, call, lane)) break;
    if (src != rank) {
      const unsigned base = jw ? (unsigned)XS_BYTES + (unsigned)(n * (512 << 10) + (src * 8 + wave * 2) * 8192)
                               : (unsigned)(js.base * (128 << 10) + (src * js.r + bu) * 16384);
      unsigned off[4];
#pragma unroll
      for (int t = 0; t < 4; ++t) off[t] = base + (jw ? (t >> 1) * 8192 + (t & 1) * 1024 : t * 4096);
      dec_in4(slot, par, off, jw ? 2048 : 1024, lds + wave * 4096, lane);
      wait_vm<0>();
    }
    st1_agent(jw ? myws + WS_DW + (size_t)(n * 32 + src * 4 + wave) * 4
                 : myws + WS_DX + (size_t)((grp == 0 ? 7 + (ti >> 4) : set) * 128 + beta) * 4, call);
  }

  unsigned* const ctrl = (unsigned*)(lds + LDS_CTRL);
  if (wave == 0) {
    const uint8_t* const fw = myws + WS_DW + (size_t)(cb * 32 + (lane & 31)) * 4;
    const uint8_t* const fx = myws + WS_DX + (size_t)(set * 128 + min(lane, 8 * rs.r - 1)) * 4;
    const uint8_t* const fy = myws + WS_DX + (size_t)(set * 128 + min(64 + lane, 8 * rs.r - 1)) * 4;
    const bool ok = spin([&] { const unsigned a = ld_agent(fw), c = ld_agent(fx), d = ld_agent(fy); return __all(min(min(a, c), d) >= call) != 0; },
                         tmo, call);
    if (lane == 0) ctrl[0] = !ok;
  }
  __syncthreads();
  const bool dead0 = __builtin_amdgcn_readfirstlane(ctrl[0]) != 0;

  if (wave == 0) st1_agent(callp, call);
  const float sc = p.sx[0] * p.sw[0];
  const unsigned off = out_off(rs.row0, block_col(rank, cb), wave, lane);
  float* const ob = p.out[local ? rank : dst];
  uint8_t* const blk = lds + wave * 32768;
  uint8_t* const oslot = p.ws[dst] + WS_OS + (size_t)((rank * 32 + ti) * 9 + wave * 2) * OBLK;
  unsigned* const flag = (unsigned*)(p.ws[dst] + WS_DONE + (size_t)((rank * 32 + ti) * 4 + wave) * 128);
  const int nb = wave_blocks(rs.r, wave);
  const unsigned* const turn = ti < NTURN ? lk + 32 : (const unsigned*)(myws + WS_TURN + (size_t)((grp * 32 + ti) * 4 + wave) * 128);
  unsigned* const next = (unsigned*)(myws + WS_TURN + (size_t)((grp * 32 + ((ti + NTURN) & 31)) * 4 + wave) * 128);
  const int depth = ti < DEEP_T ? DEEP_D : nb == 3 ? OUT_D3 : OUT_D;
  unsigned nl0 = 0, nl1 = 0, nl2 = 0;
  int sent = 0;

  if (!dead0) {
    const uint8_t* const ga = par + (size_t)rs.base * (128 << 10);
    const uint8_t* const gb = par + XS_BYTES + (size_t)cb * (512 << 10);
    if (rs.r == 9) gemm_tile<9>(ga, 9 * 2048, gb, lds, wave, lane);
    else if (nb == 1) gemm_tile<7>(ga, 7 * 2048, gb, lds, wave, lane);
    else gemm_tile<8>(ga, rs.r * 2048, gb, lds, wave, lane);
    unsigned tnv, tmv;
    asm volatile(SGPR_PAD "global_load_dword %0, %2, %3 sc1\n\tglobal_load_dword %1, %2, %4 sc1"
                 : "=&a"(tnv), "=&a"(tmv) : "v"(0u), "s"(turn), "s"(tmo) : "memory");
    auto landed = [&](unsigned& s_tn, unsigned& s_tm) {
      unsigned t0, t1;
      asm volatile("s_waitcnt vmcnt(0)\n\tv_accvgpr_read_b32 %2, %4\n\tv_accvgpr_read_b32 %3, %5\n\ts_nop 1\n\t"
                   "v_readfirstlane_b32 %0, %2\n\tv_readfirstlane_b32 %1, %3\n\ts_nop 1"
                   : "=&s"(s_tn), "=&s"(s_tm), "=&v"(t0), "=&v"(t1) : "a"(tnv), "a"(tmv) : "memory");
    };
    float s0 = sc;
#pragma unroll 1
    for (int k = 0; k < nb; ++k) {
      float v[4][16];
#pragma unroll
      for (int i = 0; i < 16; ++i) {
        const f32x4 t = *(const f32x4*)(blk + k * 16384 + i * 1024 + lane * 16);
#pragma unroll
        for (int q = 0; q < 4; ++q) v[i >> 2][(i & 3) * 4 + q] = t[q];
      }
      __builtin_amdgcn_fence(__ATOMIC_ACQ_REL, "wavefront");
      if (local) {
        if (k == 0) {
          unsigned tn, t;
          landed(tn, t);
          if (t == call) s0 = __builtin_nanf("");
        }
#pragma unroll
        for (int r = 0; r < 16; ++r)
          st16_out(ob, off + k * 32 * (GN * 4) + row_off(r), f32x4{v[0][r] * s0, v[1][r] * s0, v[2][r] * s0, v[3][r] * s0});
        continue;
      }
      const unsigned nl = enc_out(v, sc, blk + k * 16384, lane);
      __builtin_amdgcn_fence(__ATOMIC_ACQ_REL, "wavefront");
      if (k == 0) nl0 = nl; else if (k == 1) nl1 = nl; else nl2 = nl;
      if (k == 0) {
        unsigned tn, t;
        landed(tn, t);
        if (ti < 24 && nb > 1 && t != call && (tn >= call || __builtin_amdgcn_readfirstlane(ld_agent(turn)) >= call)) {
          send_out(blk, nl, false, oslot, ob, off, lane, next, call, false, depth);
          sent = 1;
        }
      }
    }
  } else if (local) {
    const float n = __builtin_nanf("");
#pragma unroll 1
    for (int t = 0; t < 32; ++t) st16_out(ob, off + (t >> 4) * 32 * (GN * 4) + row_off(t & 15), f32x4{n, n, n, n});
  }
  if (grp == 0) return;

  const int last8 = grp < 4 ? 24 : 16;
  if (ti >= 24) recv_tile(myws, p.out[rank], grp, dst, ti - 24, 0, 3, wave, lane, call, tmo, sc, scr);
  if (!local) {
    if (sent) {
      wait_vm<0>();
      st1_sys(flag, 4 * call + 1);
    } else {
      spin([&] { return __builtin_amdgcn_readfirstlane(ld_agent(turn)) >= call; }, tmo, call);
    }
    const bool dead = dead0 || __builtin_amdgcn_readfirstlane(ld_agent(tmo)) == call;
#pragma unroll 1
    for (int k = sent; k < nb; ++k) {
      send_out(blk + k * 16384, k == 0 ? nl0 : k == 1 ? nl1 : nl2, dead, oslot + k * OBLK, ob, off + k * 32 * (GN * 4), lane, next, call, k == nb - 1, depth);
      wait_vm<0>();
      st1_sys(flag, 4 * call + 1 + k);
    }
  }
#pragma unroll 1
  for (int duty = 0; duty < 2; ++duty) {
    int i, k0, k1;
    if (duty == 0) {
      if (ti >= last8) continue;
      i = ti + 8; k0 = 0; k1 = i >= last8 ? 1 : 3;
    } else if (ti < 8) {
      i = last8 + ti; k0 = grp < 4 ? 1 : 2; k1 = 3;
    } else if (local) {
      i = ti - 8; k0 = 1; k1 = 2;
    } else {
      continue;
    }
    recv_tile(myws, p.out[rank], grp, dst, i, k0, k1, wave, lane, call, tmo, sc, scr);
  }
}

extern "C" void kernel_launch(int rank, int nranks, ncclComm_t comm, hipStream_t stream,
                              void* const* d_in, const int* in_sizes, int n_in,
                              void* d_out, int out_size, const int* in_dtypes, int out_dtype,
                              void* const* d_out_all, void* const* d_ws_all, size_t ws_bytes) {
  (void)comm; (void)in_sizes; (void)in_dtypes; (void)out_dtype; (void)d_out;
  if (nranks != NR || (unsigned)rank >= (unsigned)NR || n_in != 4 || ws_bytes < WS_NEED || out_size != 512 * GN) {
    fprintf(stderr, "kernel_launch: unexpected problem shape\n");
    return;
  }
  Params p;
  p.x = (const uint8_t*)d_in[0];
  p.w = (const uint8_t*)d_in[1];
  p.sx = (const float*)d_in[2];
  p.sw = (const float*)d_in[3];
  for (int i = 0; i < NR; ++i) {
    p.ws[i] = (uint8_t*)d_ws_all[i];
    p.out[i] = (float*)d_out_all[i];
  }
  p.myws = p.ws[rank];
  p.rank = rank;
  p.pad = 0;
  hipLaunchKernelGGL(gemm_rs_fused, dim3(NWG), dim3(256), 0, stream, p);
  HIP_CHECK(hipGetLastError());
}
```

```cpp
#include <hip/hip_runtime.h>
#include <rccl/rccl.h>
#include <cstdio>
#include <cstdint>

#define HIP_CHECK(expr) do { hipError_t _e = (expr); if (_e != hipSuccess) { \
    fprintf(stderr, "HIP error %s at %s:%d\n", hipGetErrorString(_e), __FILE__, __LINE__); \
    return; } } while (0)

typedef unsigned u32x4 __attribute__((ext_vector_type(4)));
typedef int i32x4 __attribute__((ext_vector_type(4)));
typedef int i32x8 __attribute__((ext_vector_type(8)));
typedef float f32x4 __attribute__((ext_vector_type(4)));
typedef float f32x16 __attribute__((ext_vector_type(16)));

#define SGPR_PAD "s_nop 4\n\t"

namespace {

constexpr int NR = 8;
constexpr int GN = 16384;
constexpr int KS = 512;
constexpr int NWG = 256;
constexpr int KSTEPS = 64;
constexpr int NSTAGE = 6;
constexpr int LDS_CTRL = NSTAGE * 26624;
constexpr int LDS_BYTES = LDS_CTRL + 64;
constexpr int LDS_PK = 9 * 16384;

constexpr int T_W0 = 228, T_OWN = T_W0 + 512, NTASK = T_OWN + 64;

constexpr size_t WS_CALL = 0;
constexpr size_t WS_TMO = 32768;
constexpr size_t WS_DONE = 896u << 10;
constexpr size_t WS_LINK = WS_TMO + 1024;
constexpr size_t WS_DX = WS_LINK + 4096;
constexpr size_t WS_DW = WS_DX + 8192;
constexpr size_t WS_HOME = WS_DW + 4096;
constexpr size_t WS_TURN = 768u << 10;
constexpr size_t WS_DATA = 1u << 20;
constexpr size_t XS_BYTES = 10u << 20;
constexpr size_t WT_BYTES = 16u << 20;
constexpr unsigned ITASK = 3328;
constexpr size_t IS_BYTES = 21u << 20;
constexpr size_t PAR_BYTES = XS_BYTES + WT_BYTES + IS_BYTES;
static_assert((size_t)8 * NTASK * ITASK <= IS_BYTES, "");
constexpr size_t OBLK = 17 * 1024;
constexpr size_t WS_OS = WS_DATA + 2 * PAR_BYTES;
constexpr size_t WS_PB = WS_OS + 8 * 32 * 9 * OBLK;
constexpr size_t WS_NEED = WS_PB + 8 * 32 * 9 * 16384;

constexpr unsigned long long DEADLINE_TICKS = 50000000ull;

constexpr int OPEN_AT = 790, NTURN = 24, OUT_REL = 3, OUT_D = 2, OUT_D3 = 3, DEEP_D = 4, DEEP_T = 8;
constexpr int POLL_SLEEP = 6, RX_SLEEP = 16;

struct Params {
  const uint8_t* x;
  const uint8_t* w;
  const float* sx;
  const float* sw;
  uint8_t* ws[NR];
  float* out[NR];
  uint8_t* myws;
  int rank;
  int pad;
};

__device__ __forceinline__ unsigned ld_sys(const void* ptr) {
  return __hip_atomic_load((const unsigned*)ptr, __ATOMIC_RELAXED, __HIP_MEMORY_SCOPE_SYSTEM);
}
__device__ __forceinline__ unsigned ld_agent(const void* ptr) {
  return __hip_atomic_load((const unsigned*)ptr, __ATOMIC_RELAXED, __HIP_MEMORY_SCOPE_AGENT);
}
__device__ __forceinline__ void st_agent(void* ptr, unsigned v) {
  __hip_atomic_store((unsigned*)ptr, v, __ATOMIC_RELAXED, __HIP_MEMORY_SCOPE_AGENT);
}

template <int SLEEP = POLL_SLEEP, class F>
__device__ __forceinline__ bool spin(F ready, unsigned* tmo, unsigned call) {
  const unsigned long long t0 = __builtin_amdgcn_s_memrealtime();
  for (unsigned n = 1;; ++n) {
    if (ready()) return true;
    if ((n & 3) == 0 && (__builtin_amdgcn_readfirstlane(ld_agent(tmo)) == call || __builtin_amdgcn_s_memrealtime() - t0 > DEADLINE_TICKS)) {
      st_agent(tmo, call);
      return false;
    }
    __builtin_amdgcn_s_sleep(SLEEP);
  }
}

__device__ __forceinline__ void st16_sys(void* base, unsigned off, u32x4 v) {
  asm volatile(SGPR_PAD "global_store_dwordx4 %0, %1, %2 sc0 sc1\n\ts_nop 1" :: "v"(off), "v"(v), "s"(base) : "memory");
}

__device__ __forceinline__ void st16_loc(void* base, unsigned off, u32x4 v) {
  asm volatile(SGPR_PAD "global_store_dwordx4 %0, %1, %2 sc1\n\ts_nop 1" :: "v"(off), "v"(v), "s"(base) : "memory");
}

__device__ __forceinline__ void st1_agent(void* base, unsigned v) {
  asm volatile(SGPR_PAD "s_mov_b64 exec, 1\n\tglobal_store_dword %0, %1, %2 sc1\n\ts_mov_b64 exec, -1"
               :: "v"(0u), "v"(v), "s"(base) : "memory");
}

__device__ __forceinline__ void st1_sys(void* base, unsigned v) {
  asm volatile(SGPR_PAD "s_mov_b64 exec, 1\n\tglobal_store_dword %0, %1, %2 sc0 sc1\n\ts_mov_b64 exec, -1"
               :: "v"(0u), "v"(v), "s"(base) : "memory");
}
__device__ __forceinline__ void st16_sys_m(void* base, unsigned off, u32x4 v, unsigned long long mask) {
  asm volatile(SGPR_PAD "s_mov_b64 exec, %3\n\tglobal_store_dwordx4 %0, %1, %2 sc0 sc1\n\ts_mov_b64 exec, -1\n\ts_nop 0"
               :: "v"(off), "v"(v), "s"(base), "s"(mask) : "memory");
}

template <int N> __device__ __forceinline__ void wait_vm() {
  asm volatile("s_waitcnt vmcnt(%0)" :: "n"(N) : "memory");
}

__device__ __forceinline__ unsigned wave_max_u32(unsigned v) {
  v = max(v, (unsigned)__builtin_amdgcn_update_dpp(0, (int)v, 0x111, 0xf, 0xf, false));
  v = max(v, (unsigned)__builtin_amdgcn_update_dpp(0, (int)v, 0x112, 0xf, 0xf, false));
  v = max(v, (unsigned)__builtin_amdgcn_update_dpp(0, (int)v, 0x114, 0xf, 0xf, false));
  v = max(v, (unsigned)__builtin_amdgcn_update_dpp(0, (int)v, 0x118, 0xf, 0xf, false));
  return max(max((unsigned)__builtin_amdgcn_readlane((int)v, 15), (unsigned)__builtin_amdgcn_readlane((int)v, 31)),
             max((unsigned)__builtin_amdgcn_readlane((int)v, 47), (unsigned)__builtin_amdgcn_readlane((int)v, 63)));
}

__device__ __forceinline__ unsigned irec_slot(unsigned idx) { return idx + 3 + min((__umul24(idx, 37u) + 74u) >> 8, 3u); }

__device__ __forceinline__ unsigned wave_sum_u32(unsigned v) {
  v += (unsigned)__builtin_amdgcn_update_dpp(0, (int)v, 0x111, 0xf, 0xf, false);
  v += (unsigned)__builtin_amdgcn_update_dpp(0, (int)v, 0x112, 0xf, 0xf, false);
  v += (unsigned)__builtin_amdgcn_update_dpp(0, (int)v, 0x114, 0xf, 0xf, false);
  v += (unsigned)__builtin_amdgcn_update_dpp(0, (int)v, 0x118, 0xf, 0xf, false);
  return (unsigned)(__builtin_amdgcn_readlane((int)v, 15) + __builtin_amdgcn_readlane((int)v, 31) +
                    __builtin_amdgcn_readlane((int)v, 47) + __builtin_amdgcn_readlane((int)v, 63));
}
__device__ __forceinline__ unsigned bfi(unsigned m, unsigned a, unsigned b) { return (a & m) | (b & ~m); }
constexpr unsigned HDR_RAW = 0x80000000u;

constexpr unsigned IREC_CAP = 122, IHDR_RAW = 0x8000u;

struct InTask {
  u32x4 d[4];
  unsigned long long mask;
  bool raw;
};

__device__ __forceinline__ void put_records2(unsigned q0, unsigned q1, unsigned tag, unsigned& cnt, uint8_t* scr, unsigned dump) {
  unsigned m = bfi(0x55555555u, q0 | q0 >> 1, q1 | q1 << 1);
  for (unsigned long long act = __ballot(m != 0); act != 0 && cnt <= IREC_CAP; act = __ballot(m != 0)) {
    const unsigned bit = (unsigned)__builtin_ctz(m | 0x80000000u), sh = bit & 30u;
    const unsigned idx = cnt + __builtin_amdgcn_mbcnt_hi((unsigned)(act >> 32), __builtin_amdgcn_mbcnt_lo((unsigned)act, 0u));
    const unsigned sl = 2 * irec_slot(idx), at = ((m != 0) & (idx < IREC_CAP)) ? sl : dump;
    *(unsigned short*)(scr + at) = (unsigned short)(tag | (bit & 1u) << 6 | sh << 1 | ((((bit & 1u) ? q1 : q0) >> sh) & 3u));
    m &= m - 1;
    cnt += __popcll(act);
  }
}

__device__ __forceinline__ void enc_in(const u32x4 (&v)[4], InTask& o, unsigned tag, uint8_t* scr, int lane) {
  unsigned sum = 0;
#pragma unroll
  for (int q = 0; q < 4; ++q) sum = __builtin_amdgcn_udot4(v[q][0] & 0x7f7f7f7fu, 0x01010101u, sum, false);
  const unsigned e0 = (unsigned)min(max((int)((wave_sum_u32(sum) + 10445u) >> 12) - 7, 0), 24);
  const unsigned add = 0x80808080u - e0 * 0x04040404u;
  unsigned pay[12], qq[4], cnt = 0;
#pragma unroll
  for (int q = 0; q < 4; ++q) {
    unsigned R[4], S[4];
#pragma unroll
    for (int k = 0; k < 4; ++k) {
      R[k] = (v[q][k] & 0x7f7f7f7fu) + add;
      S[k] = bfi(0x80808080u, v[q][k], R[k]);
    }
    pay[3 * q] = bfi(0x60606060u, S[3] << 5, S[0]);
    pay[3 * q + 1] = bfi(0x60606060u, S[3] << 3, S[1]);
    pay[3 * q + 2] = bfi(0x20202020u, S[3] << 1, bfi(0x40404040u, S[3] >> 1, S[2]));
    qq[q] = bfi(0xc0c0c0c0u, R[3] << 1, bfi(0x30303030u, R[2] >> 1, bfi(0x0c0c0c0cu, R[1] >> 3, (R[0] >> 5) & 0x03030303u)));
  }
  put_records2(qq[0], qq[1], (unsigned)(lane << 8), cnt, scr, 256 + 2 * lane);
  put_records2(qq[2], qq[3], (unsigned)(lane << 8 | 128), cnt, scr, 256 + 2 * lane);
  *(unsigned short*)(scr + (lane < 4 ? lane * 16 : lane < 6 ? lane * 2 - 6 : 256 + 2 * lane)) =
      (unsigned short)(lane < 4 ? tag : lane == 4 ? cnt : e0);
  __builtin_amdgcn_fence(__ATOMIC_ACQ_REL, "wavefront");
  const u32x4 rec = *(const u32x4*)(scr + (lane & 15) * 16);
  __builtin_amdgcn_fence(__ATOMIC_ACQ_REL, "wavefront");
  o.raw = cnt > IREC_CAP;
  if (o.raw) {
#pragma unroll
    for (int q = 0; q < 4; ++q) o.d[q] = v[q];
    o.mask = ~0ull;
  } else {
#pragma unroll
    for (int q = 0; q < 3; ++q) o.d[q] = u32x4{pay[4 * q], pay[4 * q + 1], pay[4 * q + 2], pay[4 * q + 3]};
    o.d[3] = rec;
    o.mask = (1ull << (((irec_slot(cnt) + 31) >> 5) * 4)) - 16;
  }
}

__device__ __forceinline__ void dec_in4(const uint8_t* slot, uint8_t* out, const unsigned (&off)[4], unsigned stride, uint8_t* scr, int lane) {
  u32x4 y[4][3];
  unsigned rec[4];
  asm volatile(SGPR_PAD "global_load_dword %12, %16, %20 sc0 sc1\n\tglobal_load_dword %13, %16, %20 offset:3328 sc0 sc1\n\t"
               "global_load_dword %14, %17, %20 sc0 sc1\n\tglobal_load_dword %15, %17, %20 offset:3328 sc0 sc1\n\t"
               "global_load_dwordx4 %0, %18, %20 offset:-3072 sc0 sc1\n\tglobal_load_dwordx4 %1, %18, %20 offset:-2048 sc0 sc1\n\t"
               "global_load_dwordx4 %2, %18, %20 offset:-1024 sc0 sc1\n\t"
               "global_load_dwordx4 %3, %18, %20 offset:256 sc0 sc1\n\tglobal_load_dwordx4 %4, %18, %20 offset:1280 sc0 sc1\n\t"
               "global_load_dwordx4 %5, %18, %20 offset:2304 sc0 sc1\n\t"
               "global_load_dwordx4 %6, %19, %20 offset:-3072 sc0 sc1\n\tglobal_load_dwordx4 %7, %19, %20 offset:-2048 sc0 sc1\n\t"
               "global_load_dwordx4 %8, %19, %20 offset:-1024 sc0 sc1\n\t"
               "global_load_dwordx4 %9, %19, %20 offset:256 sc0 sc1\n\tglobal_load_dwordx4 %10, %19, %20 offset:1280 sc0 sc1\n\t"
               "global_load_dwordx4 %11, %19, %20 offset:2304 sc0 sc1\n\ts_waitcnt vmcnt(0)"
               : "=&v"(y[0][0]), "=&v"(y[0][1]), "=&v"(y[0][2]), "=&v"(y[1][0]), "=&v"(y[1][1]), "=&v"(y[1][2]),
                 "=&v"(y[2][0]), "=&v"(y[2][1]), "=&v"(y[2][2]), "=&v"(y[3][0]), "=&v"(y[3][1]), "=&v"(y[3][2]),
                 "=&v"(rec[0]), "=&v"(rec[1]), "=&v"(rec[2]), "=&v"(rec[3])
               : "v"(lane * 4), "v"(2 * ITASK + lane * 4), "v"(ITASK + lane * 16), "v"(3 * ITASK + lane * 16), "s"(slot) : "memory");
#pragma unroll
  for (int t = 0; t < 4; ++t) *(u32x4*)(scr + t * 1024 + lane * 16) = u32x4{0u, 0u, 0u, 0u};
  __builtin_amdgcn_fence(__ATOMIC_ACQ_REL, "wavefront");
  unsigned hdr[4];
#pragma unroll
  for (int t = 0; t < 4; ++t) {
    hdr[t] = (unsigned)__builtin_amdgcn_readlane((int)rec[t], 0) >> 16 | (unsigned)__builtin_amdgcn_readlane((int)rec[t], 1) << 16;
    const unsigned end = irec_slot(hdr[t] & 0x7fffu);
#pragma unroll
    for (int i = 0; i < 2; ++i) {
      const unsigned r16 = rec[t] >> (16 * i);
      const bool isrec = i == 0 ? (lane >= 16 || (lane & 3) != 0) && lane != 1 : lane != 0;
      const unsigned val = isrec && (unsigned)(lane * 2 + i) < end ? (r16 & 3u) << ((r16 >> 1) & 30u) : 0u;
      __hip_atomic_fetch_or((unsigned*)(scr + t * 1024 + ((r16 >> 6) & 255u) * 4), val, __ATOMIC_RELAXED, __HIP_MEMORY_SCOPE_WORKGROUP);
    }
  }
  __builtin_amdgcn_fence(__ATOMIC_ACQ_REL, "wavefront");
#pragma unroll
  for (int t = 0; t < 4; ++t) {
    if (hdr[t] & IHDR_RAW) continue;
    const u32x4 qq = *(const u32x4*)(scr + t * 1024 + lane * 16);
    const unsigned add = ((hdr[t] >> 16) & 31u) * 0x04040404u;
#pragma unroll
    for (int q = 0; q < 4; ++q) {
      const unsigned d0 = y[t][(3 * q) >> 2][(3 * q) & 3], d1 = y[t][(3 * q + 1) >> 2][(3 * q + 1) & 3], d2 = y[t][(3 * q + 2) >> 2][(3 * q + 2) & 3];
      const unsigned d3 = bfi(0x80808080u, d2 << 1, bfi(0x10101010u, d2 >> 1, bfi(0x0c0c0c0cu, d1 >> 3, d0 >> 5)));
      const unsigned X[4] = {bfi(0x60606060u, qq[q] << 5, d0), bfi(0x60606060u, qq[q] << 3, d1), bfi(0x60606060u, qq[q] << 1, d2),
                             bfi(0x60606060u, qq[q] >> 1, d3)};
      u32x4 r;
#pragma unroll
      for (int k = 0; k < 4; ++k) r[k] = bfi(0x80808080u, X[k], (X[k] & 0x7f7f7f7fu) + add);
      asm volatile(SGPR_PAD "global_store_dwordx4 %0, %1, %2 sc1\n\ts_nop 1" :: "v"(off[t] + q * stride + lane * 16), "v"(r), "s"(out) : "memory");
    }
  }
  __builtin_amdgcn_fence(__ATOMIC_ACQ_REL, "wavefront");
}

__device__ __forceinline__ bool wait_tasks(const uint8_t* slot, unsigned* tmo, unsigned call, int lane) {
  const uint8_t* q = slot + ((lane >> 2) & 3) * ITASK + (lane & 3) * 16;
  return spin([&] { return __all((ld_sys(q) & 0xffffu) == (call & 0xffffu)) != 0; }, tmo, call);
}

struct PushCtx {
  const uint8_t* x;
  const uint8_t* w;
  uint8_t* dst;
  unsigned* lk;
  uint8_t* scr;
  unsigned xl, wl;
  unsigned par, tick0, tag, call;
  int skip0, skipn;
  int rank, d;
  bool local;
};

struct RowSet { int r, row0, base, task0; };
__device__ __forceinline__ RowSet row_set(int set) {
  const int a = min(set, 3), o = max(set - 7, 0), b = set - a - o;
  const int own = (set + 1) >> 3, mid = ((set + 5) >> 3) - own;
  return {7 + 2 * mid + own, 224 * mid + 256 * o, 7 * a + 9 * b + 8 * o, 28 * a + 36 * b + 512 * own + 32 * o};
}
__device__ __forceinline__ int block_col(int q, int n) { return ((n & 3) * 4 + (q < 4 ? q : (q ^ n) & 3)) * 1024 + (n >> 2) * 128; }
__device__ __forceinline__ int tile_block(int g, int i) {
  const int c = (i * 43) >> 7, m = i - 3 * c;
  return g < 4 ? i : i >= 24 ? (g - 4) * 8 + i - 24 : 4 * c + m + (m >= g - 4);
}
__device__ __forceinline__ bool task_is_w(int t) { return t >= T_W0 && t < T_OWN; }
__device__ __forceinline__ int task_set(int t) {
  const int lo = min((t * 2341) >> 16, ((t + 24) * 1821) >> 16), hi = 7 + ((t - T_OWN) >> 5);
  return t < T_OWN ? lo : hi;
}

__device__ __forceinline__ int div_r(int v, int r) { return (v * (0x728093 >> (8 * (r - 7)) & 255)) >> 10; }

__device__ __forceinline__ void ld_x(const PushCtx& c, int t, u32x4 (&v)[4]) {
  t = min(t, NTASK - 1);
  const int set = task_set(t);
  const RowSet rs = row_set(set);
  const int p0 = 2 * (t - rs.task0), ks = div_r(p0, rs.r), rg = p0 - ks * rs.r;
  const unsigned o0 = c.xl + (unsigned)((((set < 7 ? set + 1 : 0) ^ c.d) * 512 + rs.row0 + rg * 32) * KS + ks * 64);
  const unsigned o1 = o0 + (unsigned)(rg + 1 == rs.r ? 64 - rg * 32 * KS : 32 * KS);
  asm volatile(SGPR_PAD "global_load_dwordx4 %0, %4, %6\n\tglobal_load_dwordx4 %1, %4, %6 offset:16\n\t"
               "global_load_dwordx4 %2, %5, %6\n\tglobal_load_dwordx4 %3, %5, %6 offset:16"
               : "=&v"(v[0]), "=&v"(v[1]), "=&v"(v[2]), "=&v"(v[3]) : "v"(o0), "v"(o1), "s"(c.x) : "memory");
}

__device__ __forceinline__ void ld_w(const uint8_t* base, unsigned o, unsigned (&w)[16]) {
    asm volatile(SGPR_PAD
                 "global_load_dword %0, %16, %17\n\tv_add_u32 %16, 0x4000, %16\n\t"
                 "global_load_dword %1, %16, %17\n\tv_add_u32 %16, 0x4000, %16\n\t"
                 "global_load_dword %2, %16, %17\n\tv_add_u32 %16, 0x4000, %16\n\t"
                 "global_load_dword %3, %16, %17\n\tv_add_u32 %16, 0x4000, %16\n\t"
                 "global_load_dword %4, %16, %17\n\tv_add_u32 %16, 0x4000, %16\n\t"
                 "global_load_dword %5, %16, %17\n\tv_add_u32 %16, 0x4000, %16\n\t"
                 "global_load_dword %6, %16, %17\n\tv_add_u32 %16, 0x4000, %16\n\t"
                 "global_load_dword %7, %16, %17\n\tv_add_u32 %16, 0x4000, %16\n\t"
                 "global_load_dword %8, %16, %17\n\tv_add_u32 %16, 0x4000, %16\n\t"
                 "global_load_dword %9, %16, %17\n\tv_add_u32 %16, 0x4000, %16\n\t"
                 "global_load_dword %10, %16, %17\n\tv_add_u32 %16, 0x4000, %16\n\t"
                 "global_load_dword %11, %16, %17\n\tv_add_u32 %16, 0x4000, %16\n\t"
                 "global_load_dword %12, %16, %17\n\tv_add_u32 %16, 0x4000, %16\n\t"
                 "global_load_dword %13, %16, %17\n\tv_add_u32 %16, 0x4000, %16\n\t"
                 "global_load_dword %14, %16, %17\n\tv_add_u32 %16, 0x4000, %16\n\t"
                 "global_load_dword %15, %16, %17\n\t"
                 : "=&v"(w[0]), "=&v"(w[1]), "=&v"(w[2]), "=&v"(w[3]), "=&v"(w[4]), "=&v"(w[5]), "=&v"(w[6]), "=&v"(w[7]),
                   "=&v"(w[8]), "=&v"(w[9]), "=&v"(w[10]), "=&v"(w[11]), "=&v"(w[12]), "=&v"(w[13]), "=&v"(w[14]), "=&v"(w[15]), "+v"(o) : "s"(base) : "memory");
    static_assert(GN == 0x4000, "");
}
__device__ __forceinline__ void transpose_w(const unsigned (&w)[16], u32x4 (&v)[4]) {
#pragma unroll
  for (int q = 0; q < 4; ++q) {
    const unsigned d0 = w[4 * q], d1 = w[4 * q + 1], d2 = w[4 * q + 2], d3 = w[4 * q + 3];
    const unsigned t0 = __builtin_amdgcn_perm(d1, d0, 0x05010400u), t1 = __builtin_amdgcn_perm(d1, d0, 0x07030602u);
    const unsigned t2 = __builtin_amdgcn_perm(d3, d2, 0x05010400u), t3 = __builtin_amdgcn_perm(d3, d2, 0x07030602u);
    v[0][q] = __builtin_amdgcn_perm(t2, t0, 0x05040100u);
    v[1][q] = __builtin_amdgcn_perm(t2, t0, 0x07060302u);
    v[2][q] = __builtin_amdgcn_perm(t3, t1, 0x05040100u);
    v[3][q] = __builtin_amdgcn_perm(t3, t1, 0x07060302u);
  }
}

__device__ __forceinline__ void pack_task(const PushCtx& c, const u32x4 (&v)[4], InTask& o, int lane, bool plain) {
  if (plain) {
#pragma unroll
    for (int q = 0; q < 4; ++q) o.d[q] = v[q];
    o.mask = ~0ull;
    o.raw = true;
  } else {
    enc_in(v, o, c.tag, c.scr, lane);
  }
}

__device__ __forceinline__ void send_plain(const PushCtx& c, int t, const u32x4 (&d)[4], int lane) {
  const int u = t - T_W0;
  const bool wr = task_is_w(t);
  const RowSet rs = row_set(task_set(t));
  const unsigned plain = c.par + lane * 16 +
      (wr ? (unsigned)XS_BYTES + (unsigned)((u >> 4) * (512 << 10) + (c.rank * 8 + ((u >> 1) & 7)) * 8192 + (u & 1) * 1024)
          : (unsigned)(rs.base * (128 << 10) + c.rank * rs.r * 16384 + (t - rs.task0) * 4096));
#pragma unroll
  for (int q = 0; q < 4; ++q) st16_sys(c.dst, plain + q * (wr ? 2048 : 1024), d[q]);
}

__device__ __forceinline__ int task_at(const PushCtx& c, int pos) { return min(pos < c.skip0 ? pos : pos + c.skipn, NTASK); }

struct PushState {
  int tp, tc, tn, tl;
  bool gone;
  u32x4 line;
  InTask o;
  u32x4 nx[4];
};

template <bool LDW>
__device__ __forceinline__ void push_step(const PushCtx& c, PushState& s, int lane) {
  unsigned w[16], tk;
  u32x4 x[4];
  const bool draw = s.tl < NTASK;
  asm volatile(SGPR_PAD "s_mov_b64 exec, 1\n\tglobal_atomic_add %0, %1, %2, %3 sc0\n\ts_mov_b64 exec, -1"
               : "=&v"(tk) : "v"(0u), "v"(draw ? 1u : 0u), "s"(c.lk) : "memory");
  if (s.tc == OPEN_AT) st1_agent(c.lk + 32, c.call);
  if constexpr (LDW) {
    const int u = s.tl - T_W0;
    ld_w(c.w, c.wl + (unsigned)((((u >> 1) & 7) * 64 + (u & 1) * 16) * GN + block_col(c.d, u >> 4)), w);
  } else {
    ld_x(c, s.tl, x);
  }
  const unsigned is0 = c.par + (unsigned)(XS_BYTES + WT_BYTES) + (unsigned)(c.rank * NTASK) * ITASK + lane * 16;
  if (s.tp >= 0) st16_sys_m(c.dst, is0 + (unsigned)s.tp * ITASK, s.line, 15ull);
  if (s.tc < NTASK) {
    if (!s.o.raw) {
      const unsigned slot = is0 + (unsigned)s.tc * ITASK;
      asm volatile(SGPR_PAD "global_store_dwordx4 %0, %1, %4 offset:256 sc0 sc1\n\tglobal_store_dwordx4 %0, %2, %4 offset:1280 sc0 sc1\n\t"
                   "global_store_dwordx4 %0, %3, %4 offset:2304 sc0 sc1\n\ts_nop 1"
                   :: "v"(slot), "v"(s.o.d[0]), "v"(s.o.d[1]), "v"(s.o.d[2]), "s"(c.dst) : "memory");
      if (s.o.mask != 0) st16_sys_m(c.dst, slot, s.o.d[3], s.o.mask);
      s.line = s.o.d[3];
    } else {
      if (!s.gone) send_plain(c, s.tc, s.o.d, lane);
      s.line = u32x4{c.tag | IHDR_RAW << 16, 0u, 0u, 0u};
    }
  }
  pack_task(c, s.nx, s.o, lane, c.local || s.tn >= NTASK);
  if constexpr (LDW) {
    asm volatile("s_waitcnt vmcnt(0)" : "+v"(tk), "+v"(w[0]), "+v"(w[1]), "+v"(w[2]), "+v"(w[3]), "+v"(w[4]), "+v"(w[5]), "+v"(w[6]), "+v"(w[7]),
                 "+v"(w[8]), "+v"(w[9]), "+v"(w[10]), "+v"(w[11]), "+v"(w[12]), "+v"(w[13]), "+v"(w[14]), "+v"(w[15]) :: "memory");
    transpose_w(w, s.nx);
  } else {
    asm volatile("s_waitcnt vmcnt(0)" : "+v"(tk), "+v"(x[0]), "+v"(x[1]), "+v"(x[2]), "+v"(x[3]) :: "memory");
#pragma unroll
    for (int q = 0; q < 4; ++q) s.nx[q] = x[q];
  }
  s.tp = s.tc; s.tc = s.tn; s.tn = s.tl; s.gone = false;
  s.tl = draw ? task_at(c, min(__builtin_amdgcn_readfirstlane(tk) - c.tick0, (unsigned)NTASK)) : NTASK;
}

#define GLDS_BITS " sc0 sc1"
__device__ __forceinline__ void glds2(const uint8_t* gbase, unsigned voff, unsigned lds) {
  unsigned keep;
  asm volatile("s_mov_b32 %0, m0\n\ts_mov_b32 m0, %3\n\ts_nop 2\n\t"
               "global_load_lds_dwordx4 %1, %2" GLDS_BITS "\n\t"
               "global_load_lds_dwordx4 %1, %2 offset:1024" GLDS_BITS "\n\t"
               "s_mov_b32 m0, %0"
               : "=&s"(keep) : "v"(voff), "s"(gbase), "s"(lds));
}
__device__ __forceinline__ void glds2o(const uint8_t* gbase, unsigned voff, unsigned lds) {
  unsigned keep;
  asm volatile("s_mov_b32 %0, m0\n\ts_mov_b32 m0, %3\n\ts_nop 2\n\t"
               "global_load_lds_dwordx4 %1, %2 offset:2048" GLDS_BITS "\n\t"
               "global_load_lds_dwordx4 %1, %2 offset:3072" GLDS_BITS "\n\t"
               "s_mov_b32 m0, %0"
               : "=&s"(keep) : "v"(voff), "s"(gbase), "s"(lds));
}
__device__ __forceinline__ void glds_half(const uint8_t* gbase, unsigned voff, unsigned lds) {
  unsigned keep;
  asm volatile("s_mov_b32 %0, m0\n\ts_mov_b32 m0, %3\n\ts_mov_b32 exec_hi, 0\n\ts_nop 1\n\t"
               "global_load_lds_dwordx4 %1, %2" GLDS_BITS "\n\t"
               "s_mov_b32 exec_hi, -1\n\ts_mov_b32 m0, %0"
               : "=&s"(keep) : "v"(voff), "s"(gbase), "s"(lds));
}

struct Frags { i32x8 a[2], b[4], a9, b9; };

__device__ __forceinline__ i32x8 ld_frag(const uint8_t* q) {
  const i32x4 lo = *(const i32x4*)q, hi = *(const i32x4*)(q + 1024);
  return __builtin_shufflevector(lo, hi, 0, 1, 2, 3, 4, 5, 6, 7);
}

#define MFMA(mi, j) acc[mi][j] = __builtin_amdgcn_mfma_scale_f32_32x32x64_f8f6f4(cur.a[mi], cur.b[j], acc[mi][j], 1, 1, 0, 0, 0, 0)
#define SB() __builtin_amdgcn_sched_barrier(0)

struct Ring {
  const uint8_t* ga;
  const uint8_t* gb;
  unsigned astep, ahop;
  int togo;
  unsigned va, vb, v9;
  unsigned wr, wr_end;
  unsigned wb, w9;
  unsigned rd;
  const uint8_t* qa;
  const uint8_t* qb;
  const uint8_t* q9;
  const uint8_t* la;
  const uint8_t* lb;
  const uint8_t* l9;
};

template <int MODE, int LEFT, bool DMA, bool RD>
__device__ __forceinline__ void kstep(f32x16 (&acc)[2][4], f32x16& acc9, const Frags& cur, Frags& nxt, Ring& g) {
  constexpr bool NINE = MODE == 9, TWO = MODE != 7;
  constexpr int STAGE = NINE ? 26624 : 24576;
  MFMA(0, 0); SB();
  asm volatile("s_waitcnt vmcnt(%0) lgkmcnt(0)\n\ts_barrier" :: "n"(LEFT * (NINE ? 7 : 6)) : "memory");
  SB();
  MFMA(0, 1); SB();
  if constexpr (RD) { nxt.a[0] = ld_frag(g.qa); nxt.b[0] = ld_frag(g.qb); }
  unsigned sa = 0, sb = 0;
  if constexpr (DMA) { const bool hop = --g.togo == 0; sa = hop ? g.ahop : g.astep; sb = hop ? 9 * 8192 : 8192; asm volatile("" : "+s"(sa), "+s"(sb)); }
  SB();
  MFMA(0, 2); SB();
  if constexpr (DMA) glds2(g.ga, g.va, g.wr);
  SB();
  MFMA(0, 3); SB();
  if constexpr (RD) { nxt.b[1] = ld_frag(g.qb + 2048); nxt.b[2] = ld_frag(g.qb + 4096); }
  SB();
  if constexpr (TWO) MFMA(1, 0);
  SB();
  if constexpr (DMA) glds2(g.gb, g.vb, g.wr + g.wb);
  SB();
  if constexpr (TWO) MFMA(1, 1);
  SB();
  if constexpr (RD) { nxt.b[3] = ld_frag(g.qb + 6144); if constexpr (TWO) nxt.a[1] = ld_frag(g.qa + 2048); }
  g.rd = g.rd == (NSTAGE - 1) * STAGE ? 0 : g.rd + STAGE;
  g.qa = g.la + g.rd;
  if constexpr (!NINE) g.qb = g.lb + g.rd;
  SB();
  if constexpr (TWO) MFMA(1, 2);
  SB();
  if constexpr (DMA) glds2o(g.ga, g.va, g.wr);
  SB();
  if constexpr (NINE) {
    if constexpr (TWO) MFMA(1, 3);
    SB();
    if constexpr (RD) { nxt.a9 = ld_frag(g.qb - 2048); nxt.b9 = ld_frag(g.q9); }
    g.qb = g.lb + g.rd; g.q9 = g.l9 + g.rd;
    if constexpr (DMA) g.gb += sb;
    SB();
    acc9 = __builtin_amdgcn_mfma_scale_f32_32x32x64_f8f6f4(cur.a9, cur.b9, acc9, 1, 1, 0, 0, 0, 0); SB();
    if constexpr (DMA) glds_half(g.ga, g.v9, g.wr + g.w9);
  } else {
    if constexpr (TWO) MFMA(1, 3);
    SB();
    if constexpr (DMA) g.gb += sb;
  }
  if constexpr (DMA) {
    g.ga += sa;
    g.wr += STAGE; g.wr = g.wr == g.wr_end ? g.wr - NSTAGE * STAGE : g.wr;
  }
  SB();
}

template <int MODE>
__device__ __forceinline__ void gemm_tile(const uint8_t* ga, unsigned astep, const uint8_t* gb, int skip, uint8_t* lds, int wave, int lane) {
  constexpr bool NINE = MODE == 9, TWO = MODE != 7;
  constexpr int STAGE = NINE ? 26624 : 24576, BOFF = STAGE - 8192;
  f32x16 acc[2][4], acc9;
  Ring g;
  g.ga = ga + (skip == 0 ? 8 * astep : 0); g.gb = gb + (skip == 0 ? 8 * 8192 : 0); g.astep = astep; g.ahop = 9 * astep;
  g.togo = skip > 0 ? 8 * skip - NSTAGE : -1;
  g.va = lane * 16 + wave * 4096; g.vb = lane * 16 + wave * 2048; g.v9 = 16384 + lane * 16 + wave * 512;
  g.wr = (unsigned)(uintptr_t)lds + wave * 4096; g.wr_end = g.wr + NSTAGE * STAGE;
  g.wb = BOFF - wave * 2048; g.w9 = 16384 - wave * 3584;
#pragma unroll
  for (int s = 0; s < NSTAGE; ++s) {
    glds2(g.ga, g.va, g.wr + s * STAGE);
    glds2(g.gb, g.vb, g.wr + s * STAGE + g.wb);
    glds2o(g.ga, g.va, g.wr + s * STAGE);
    if constexpr (NINE) glds_half(g.ga, g.v9, g.wr + s * STAGE + g.w9);
    g.ga += astep; g.gb += 8192;
  }
#pragma unroll
  for (int r = 0; r < 16; ++r) {
    acc9[r] = 0.f;
#pragma unroll
    for (int j = 0; j < 4; ++j) { acc[0][j][r] = 0.f; acc[1][j][r] = 0.f; }
  }
  g.la = lds + lane * 16 + wave * 4096; g.lb = lds + BOFF + lane * 16; g.l9 = g.lb + wave * 2048;
  asm volatile("s_waitcnt vmcnt(%0)\n\ts_barrier" :: "n"(5 * (NINE ? 7 : 6)) : "memory");
  Frags f0, f1;
  f0.a[0] = ld_frag(g.la); f0.b[0] = ld_frag(g.lb); f0.b[1] = ld_frag(g.lb + 2048);
  f0.b[2] = ld_frag(g.lb + 4096); f0.b[3] = ld_frag(g.lb + 6144);
  if constexpr (TWO) f0.a[1] = ld_frag(g.la + 2048);
  if constexpr (NINE) { f0.a9 = ld_frag(g.lb - 2048); f0.b9 = ld_frag(g.l9); }
  g.rd = STAGE; g.qa = g.la + STAGE; g.qb = g.lb + STAGE; g.q9 = g.l9 + STAGE;
#pragma unroll 1
  for (int s = skip < 0 ? 0 : 8; s < KSTEPS - NSTAGE; s += 2) {
    kstep<MODE, 4, true, true>(acc, acc9, f0, f1, g);
    kstep<MODE, 4, true, true>(acc, acc9, f1, f0, g);
  }
  kstep<MODE, 4, false, true>(acc, acc9, f0, f1, g);
  kstep<MODE, 3, false, true>(acc, acc9, f1, f0, g);
  kstep<MODE, 2, false, true>(acc, acc9, f0, f1, g);
  kstep<MODE, 1, false, true>(acc, acc9, f1, f0, g);
  kstep<MODE, 0, false, true>(acc, acc9, f0, f1, g);
  auto park = [&](const f32x16& a, int i) {
#pragma unroll
    for (int q = 0; q < 4; ++q)
      *(f32x4*)(lds + wave * 32768 + (i * 4 + q) * 1024 + lane * 16) = f32x4{a[4 * q], a[4 * q + 1], a[4 * q + 2], a[4 * q + 3]};
  };
  {
    const Frags& cur = f1;
    MFMA(0, 0); SB();
    asm volatile("s_waitcnt lgkmcnt(0)\n\ts_barrier" ::: "memory");
    SB();
    MFMA(0, 1); SB();
    MFMA(0, 2); SB();
    park(acc[0][0], 0); SB();
    MFMA(0, 3); SB();
    park(acc[0][1], 1); SB();
    if constexpr (TWO) MFMA(1, 0);
    SB();
    park(acc[0][2], 2); SB();
    if constexpr (TWO) MFMA(1, 1);
    SB();
    park(acc[0][3], 3); SB();
    if constexpr (TWO) {
      MFMA(1, 2); SB();
      park(acc[1][0], 4); SB();
      MFMA(1, 3); SB();
      park(acc[1][1], 5); SB();
      if constexpr (NINE) { acc9 = __builtin_amdgcn_mfma_scale_f32_32x32x64_f8f6f4(cur.a9, cur.b9, acc9, 1, 1, 0, 0, 0, 0); SB(); }
      park(acc[1][2], 6);
      park(acc[1][3], 7);
    }
  }
  if constexpr (NINE) {
#pragma unroll
    for (int q = 0; q < 4; ++q)
      *(f32x4*)(lds + 8 * 16384 + wave * 4096 + q * 1024 + lane * 16) = f32x4{acc9[4 * q], acc9[4 * q + 1], acc9[4 * q + 2], acc9[4 * q + 3]};
    __syncthreads();
  }
  __builtin_amdgcn_fence(__ATOMIC_ACQ_REL, "wavefront");
}

constexpr unsigned REC_CAP = 510, HDR_B = 0x40000000u;

__device__ __forceinline__ void put_records4(const unsigned (&q)[4], unsigned tag, unsigned& cnt, uint8_t* scr, unsigned dump) {
  unsigned m = 0;
#pragma unroll
  for (int k = 0; k < 4; ++k)
    m |= ((((q[k] & 0x77777777u) + 0x77777777u) | q[k]) & 0x88888888u) >> (3 - k);
  for (unsigned long long act = __ballot(m != 0); act != 0 && cnt <= REC_CAP; act = __ballot(m != 0)) {
    const unsigned bit = (unsigned)__builtin_ctz(m | 0x80000000u), sh = bit & 28u;
    const unsigned m1 = (unsigned)__builtin_amdgcn_sbfe((int)bit, 0, 1), m2 = (unsigned)__builtin_amdgcn_sbfe((int)bit, 1, 1);
    const unsigned w = bfi(m2, bfi(m1, q[3], q[2]), bfi(m1, q[1], q[0]));
    const unsigned idx = cnt + __builtin_amdgcn_mbcnt_hi((unsigned)(act >> 32), __builtin_amdgcn_mbcnt_lo((unsigned)act, 0u));
    const unsigned at = (m != 0 && idx < REC_CAP) ? 4 + 2 * idx : dump;
    *(unsigned short*)(scr + at) = (unsigned short)(tag | (bit & 3u) << 7 | sh << 2 | ((w >> sh) & 15u));
    m &= m - 1;
    cnt += __popcll(act);
  }
}

template <bool MODEB>
__device__ __forceinline__ bool enc_body(const float (&v)[4][16], float up, unsigned& cnt, uint8_t* blk, int lane) {
  unsigned bad = 0, low = 0;
#pragma unroll
  for (int h = 0; h < 2; ++h) {
    unsigned G[32];
#pragma unroll
    for (int j = 0; j < 4; ++j)
#pragma unroll
      for (int r = 0; r < 8; ++r) {
        const float x = v[j][8 * h + r] * up;
        unsigned g = (unsigned)(int)x;
        bad = __builtin_amdgcn_bitop3_b32(__builtin_bit_cast(unsigned, (float)(int)g), __builtin_bit_cast(unsigned, x), bad, 0xbe);
        low |= g;
        if (MODEB) g = g << 1 | (((g ^ g << 1) >> 27) & 16u);
        G[4 * r + j] = g;
      }
    unsigned q[4];
#pragma unroll
    for (int k = 0; k < 4; ++k) {
      const unsigned* g = G + 8 * k;
      const unsigned b0 = __builtin_amdgcn_perm(__builtin_amdgcn_perm(g[3], g[2], 0x0c0c0400u), __builtin_amdgcn_perm(g[1], g[0], 0x0c0c0400u), 0x05040100u);
      const unsigned b1 = __builtin_amdgcn_perm(__builtin_amdgcn_perm(g[7], g[6], 0x0c0c0400u), __builtin_amdgcn_perm(g[5], g[4], 0x0c0c0400u), 0x05040100u);
      q[k] = (b0 >> 4) | b1;
    }
#pragma unroll
    for (int j = 0; j < 2; ++j) {
      unsigned d[12];
#pragma unroll
      for (int g = 0; g < 4; ++g) {
        const unsigned* x = G + 16 * j + 4 * g;
        d[3 * g] = __builtin_amdgcn_perm(x[1], x[0], 0x05030201u);
        d[3 * g + 1] = __builtin_amdgcn_perm(x[2], x[1], 0x06050302u);
        d[3 * g + 2] = __builtin_amdgcn_perm(x[3], x[2], 0x07060503u);
      }
#pragma unroll
      for (int i = 0; i < 3; ++i)
        *(u32x4*)(blk + (1 + 6 * h + 3 * j + i) * 1024 + lane * 16) = u32x4{d[4 * i], d[4 * i + 1], d[4 * i + 2], d[4 * i + 3]};
    }
    put_records4(q, (unsigned)(lane << 10 | h << 9), cnt, blk, 13 * 1024 + 2 * lane);
  }
  return __ballot((bad | (low & 15u)) != 0) == 0 && cnt <= REC_CAP;
}

__device__ __forceinline__ unsigned enc_out(const float (&v)[4][16], uint8_t* blk, int lane) {
  float m = 0.f;
#pragma unroll
  for (int j = 0; j < 4; ++j)
#pragma unroll
    for (int r = 0; r < 16; r += 2) asm("v_max3_f32 %0, |%1|, |%2|, %0" : "+v"(m) : "v"(v[j][r]), "v"(v[j][r + 1]));
  const int e = max((int)(wave_max_u32(__builtin_bit_cast(unsigned, m)) >> 23) - 157, -126);
  const float up = __builtin_bit_cast(float, (unsigned)(127 - min(e, 0)) << 23);
  int ntop = 0, nlow = 0;
#pragma unroll
  for (int r = 0; r < 8; ++r) {
    const unsigned g = (unsigned)(int)(v[r & 3][2 * r] * up);
    ntop += __popcll(__ballot((int)(g ^ (g << 1)) < 0));
    nlow += __popcll(__ballot((g & 128u) != 0));
  }
  unsigned cnt = 0;
  const bool modeb = ntop < nlow;
  const bool fits = e <= 0 && (modeb ? enc_body<true>(v, up, cnt, blk, lane) : enc_body<false>(v, up, cnt, blk, lane));
  if (!fits) {
#pragma unroll
    for (int i = 0; i < 16; ++i) {
      const float* t = &v[i >> 2][(i & 3) * 4];
      *(f32x4*)(blk + i * 1024 + lane * 16) = f32x4{t[0], t[1], t[2], t[3]};
    }
    return 0;
  }
  *(unsigned*)(blk + (lane == 0 ? 0 : 13 * 1024 + 4 * lane)) = cnt | (unsigned)(e + 127) << 16 | (modeb ? HDR_B : 0u);
  return ((4 + 2 * cnt + 63) >> 6) * 4;
}

__device__ __forceinline__ void st16_out(void* base, unsigned off, f32x4 v) { st16_loc(base, off, __builtin_bit_cast(u32x4, v)); }

__device__ __forceinline__ constexpr unsigned row_off(int r) { return (unsigned)((r & 3) + 8 * (r >> 2)) * (GN * 4); }

__device__ __forceinline__ void send_out(const uint8_t* blk, unsigned nl, bool dead, uint8_t* slot, int lane, unsigned* next,
                                         unsigned call, bool last, int depth) {
  const bool raw = nl == 0 || dead;
  const u32x4 h = *(const u32x4*)(blk + lane * 16);
  st16_sys_m(slot, lane * 16, raw ? u32x4{HDR_RAW, 0u, 0u, 0u} : h, raw ? 15ull : nl >= 64 ? ~0ull : (1ull << nl) - 1);
  const int n = raw ? 16 : 12;
  const uint8_t* const src = blk + (raw ? 0 : 1024) + lane * 16;
  const unsigned nan = dead ? 0x7fc00000u : 0u;
#pragma unroll 1
  for (int i = 0; i < n; ++i) {
    const u32x4 v = *(const u32x4*)(src + i * 1024) | nan;
    if (depth == OUT_D) wait_vm<OUT_D - 1>(); else if (depth == OUT_D3) wait_vm<OUT_D3 - 1>(); else wait_vm<DEEP_D - 1>();
    if (last && i == n - OUT_REL) st1_agent(next, call);
    st16_sys(slot, (unsigned)(1024 + i * 1024 + lane * 16), v);
  }
}

__device__ __forceinline__ void dec_out(const uint8_t* slot, const f32x4 (&h)[16], float* ob, unsigned off, float sc, unsigned* tmo, unsigned call,
                                        uint8_t* scr, int lane) {
  u32x4 rec, y[12];
  unsigned tmv;
  asm volatile(SGPR_PAD "global_load_dword %13, %16, %18 sc1\n\t"
               "global_load_dwordx4 %0, %14, %17 offset:-4096 sc0 sc1\n\tglobal_load_dwordx4 %1, %14, %17 offset:-3072 sc0 sc1\n\t"
               "global_load_dwordx4 %2, %14, %17 offset:-2048 sc0 sc1\n\tglobal_load_dwordx4 %3, %14, %17 offset:-1024 sc0 sc1\n\t"
               "global_load_dwordx4 %4, %14, %17 sc0 sc1\n\tglobal_load_dwordx4 %5, %14, %17 offset:1024 sc0 sc1\n\t"
               "global_load_dwordx4 %6, %14, %17 offset:2048 sc0 sc1\n\tglobal_load_dwordx4 %7, %14, %17 offset:3072 sc0 sc1\n\t"
               "global_load_dwordx4 %8, %15, %17 offset:-4096 sc0 sc1\n\tglobal_load_dwordx4 %9, %15, %17 offset:-3072 sc0 sc1\n\t"
               "global_load_dwordx4 %10, %15, %17 offset:-2048 sc0 sc1\n\tglobal_load_dwordx4 %11, %15, %17 offset:-1024 sc0 sc1\n\t"
               "global_load_dwordx4 %12, %15, %17 sc0 sc1\n\ts_waitcnt vmcnt(0)"
               : "=&v"(rec), "=&v"(y[0]), "=&v"(y[1]), "=&v"(y[2]), "=&v"(y[3]), "=&v"(y[4]), "=&v"(y[5]), "=&v"(y[6]),
                 "=&v"(y[7]), "=&v"(y[8]), "=&v"(y[9]), "=&v"(y[10]), "=&v"(y[11]), "=&v"(tmv)
               : "v"(4096 + lane * 16), "v"(12288 + lane * 16), "v"(0u), "s"(slot), "s"(tmo) : "memory");
  const unsigned hdr = __builtin_amdgcn_readfirstlane(rec[0]);
  if (__builtin_amdgcn_readfirstlane(tmv) == call) {
    const float n = __builtin_nanf("");
#pragma unroll 1
    for (int r = 0; r < 16; ++r) st16_out(ob, off + row_off(r), f32x4{n, n, n, n});
    return;
  }
  if (hdr & HDR_RAW) {
    u32x4 z[4];
    asm volatile(SGPR_PAD "global_load_dwordx4 %0, %4, %5 sc0 sc1\n\tglobal_load_dwordx4 %1, %4, %5 offset:1024 sc0 sc1\n\t"
                 "global_load_dwordx4 %2, %4, %5 offset:2048 sc0 sc1\n\tglobal_load_dwordx4 %3, %4, %5 offset:3072 sc0 sc1\n\ts_waitcnt vmcnt(0)"
                 : "=&v"(z[0]), "=&v"(z[1]), "=&v"(z[2]), "=&v"(z[3]) : "v"(13312 + lane * 16), "s"(slot) : "memory");
#pragma unroll
    for (int r = 0; r < 16; ++r) {
      f32x4 t;
#pragma unroll
      for (int j = 0; j < 4; ++j) t[j] = (__builtin_bit_cast(float, j < 3 ? y[4 * j + (r >> 2)][r & 3] : z[r >> 2][r & 3]) + h[4 * j + (r >> 2)][r & 3]) * sc;
      st16_out(ob, off + row_off(r), t);
    }
    return;
  }
  const unsigned cnt = hdr & 0xffffu;
  const float down = __builtin_bit_cast(float, ((hdr >> 16) & 255u) << 23);
  *(u32x4*)(scr + lane * 32) = u32x4{0u, 0u, 0u, 0u};
  *(u32x4*)(scr + lane * 32 + 16) = u32x4{0u, 0u, 0u, 0u};
  __builtin_amdgcn_fence(__ATOMIC_ACQ_REL, "wavefront");
#pragma unroll
  for (int i = 0; i < 8; ++i) {
    const unsigned r16 = (rec[i >> 1] >> (16 * (i & 1))) & 0xffffu;
    const unsigned val = (unsigned)(lane * 8 + i - 2) < cnt ? (r16 & 15u) << ((r16 >> 2) & 28u) : 0u;
    __hip_atomic_fetch_or((unsigned*)(scr + (r16 >> 7) * 4), val, __ATOMIC_RELAXED, __HIP_MEMORY_SCOPE_WORKGROUP);
  }
  __builtin_amdgcn_fence(__ATOMIC_ACQ_REL, "wavefront");
  const u32x4 q0 = *(const u32x4*)(scr + lane * 32), q1 = *(const u32x4*)(scr + lane * 32 + 16);
  __builtin_amdgcn_fence(__ATOMIC_ACQ_REL, "wavefront");
  const bool modeb = (hdr & HDR_B) != 0;
#pragma unroll
  for (int g = 0; g < 16; ++g) {
    const unsigned d0 = y[(3 * g) >> 2][(3 * g) & 3], d1 = y[(3 * g + 1) >> 2][(3 * g + 1) & 3], d2 = y[(3 * g + 2) >> 2][(3 * g + 2) & 3];
    const unsigned qw = g < 8 ? q0[(g >> 1) & 3] : q1[(g >> 1) & 3];
    const unsigned q = ((g & 1) ? qw : qw << 4) & 0xf0f0f0f0u;
    unsigned G[4] = {__builtin_amdgcn_perm(d0, q, 0x06050400u), __builtin_amdgcn_perm(d1, d0, 0x0504030cu) | ((q >> 8) & 255u),
                     __builtin_amdgcn_perm(d2, d1, 0x0403020cu) | ((q >> 16) & 255u), __builtin_amdgcn_perm(d2, q, 0x07060503u)};
    f32x4 v;
#pragma unroll
    for (int j = 0; j < 4; ++j) {
      if (modeb) G[j] = ((unsigned)((int)G[j] >> 1) & ~8u) ^ (G[j] << 27);
      v[j] = (__fmul_rn((float)(int)G[j], down) + h[4 * j + (g >> 2)][g & 3]) * sc;
    }
    st16_out(ob, off + row_off(g), v);
  }
}

__device__ __forceinline__ int wave_blocks(int r, int wave) { return wave < 3 ? 2 : r - 6; }
__device__ __forceinline__ unsigned out_off(int row0, int col, int wave, int lane) {
  return (unsigned)(((row0 + wave * 64 + 4 * (lane >> 5)) * GN + col + 4 * (lane & 31)) * 4);
}

template <bool NINE>
__device__ __forceinline__ void home_tile(const Params& p, int g, int src, int i, uint8_t* lds, int wave, int lane) {
  const int col = block_col(src, tile_block(g, i));
  const uint8_t* const xg = p.x + (size_t)(p.rank * 512 + (lane & 31)) * KS + (lane >> 5) * 32;
  auto xa = [&](int rho, int s) { return *(const i32x8*)(xg + rho * 32 * KS + s * 64); };
  i32x8 a0[8], a1[8], a9[8];
  {
    unsigned w[4][16];
    const unsigned o = (unsigned)((wave * 128 + (lane >> 5) * 32) * GN + col + (lane & 31) * 4);
#pragma unroll
    for (int t = 0; t < 4; ++t) ld_w(p.w, o + (unsigned)((t >> 1) * 64 + (t & 1) * 16) * GN, w[t]);
#pragma unroll
    for (int s = 0; s < 8; ++s) {
      a0[s] = xa((NINE ? 7 : 0) + wave * 2, s);
      a1[s] = xa((NINE ? 8 : 1) + wave * 2, s);
    }
    asm volatile("s_waitcnt vmcnt(0)" : "+v"(w[0][0]), "+v"(w[0][1]), "+v"(w[0][2]), "+v"(w[0][3]), "+v"(w[0][4]), "+v"(w[0][5]), "+v"(w[0][6]), "+v"(w[0][7]), "+v"(w[0][8]), "+v"(w[0][9]), "+v"(w[0][10]), "+v"(w[0][11]), "+v"(w[0][12]), "+v"(w[0][13]), "+v"(w[0][14]), "+v"(w[0][15]) :: "memory");
    asm volatile("" : "+v"(w[1][0]), "+v"(w[1][1]), "+v"(w[1][2]), "+v"(w[1][3]), "+v"(w[1][4]), "+v"(w[1][5]), "+v"(w[1][6]), "+v"(w[1][7]), "+v"(w[1][8]), "+v"(w[1][9]), "+v"(w[1][10]), "+v"(w[1][11]), "+v"(w[1][12]), "+v"(w[1][13]), "+v"(w[1][14]), "+v"(w[1][15]) :: "memory");
    asm volatile("" : "+v"(w[2][0]), "+v"(w[2][1]), "+v"(w[2][2]), "+v"(w[2][3]), "+v"(w[2][4]), "+v"(w[2][5]), "+v"(w[2][6]), "+v"(w[2][7]), "+v"(w[2][8]), "+v"(w[2][9]), "+v"(w[2][10]), "+v"(w[2][11]), "+v"(w[2][12]), "+v"(w[2][13]), "+v"(w[2][14]), "+v"(w[2][15]) :: "memory");
    asm volatile("" : "+v"(w[3][0]), "+v"(w[3][1]), "+v"(w[3][2]), "+v"(w[3][3]), "+v"(w[3][4]), "+v"(w[3][5]), "+v"(w[3][6]), "+v"(w[3][7]), "+v"(w[3][8]), "+v"(w[3][9]), "+v"(w[3][10]), "+v"(w[3][11]), "+v"(w[3][12]), "+v"(w[3][13]), "+v"(w[3][14]), "+v"(w[3][15]) :: "memory");
#pragma unroll
    for (int t = 0; t < 4; ++t) {
      u32x4 v[4];
      transpose_w(w[t], v);
#pragma unroll
      for (int j = 0; j < 4; ++j) *(u32x4*)(lds + (wave * 2 + (t >> 1)) * 8192 + j * 2048 + (t & 1) * 1024 + lane * 16) = v[j];
    }
  }
  if constexpr (NINE) {
#pragma unroll
    for (int s = 0; s < 8; ++s) a9[s] = xa(15, s);
  }
  __syncthreads();
  f32x16 h[2][4], h9;
#pragma unroll
  for (int r = 0; r < 16; ++r) {
    h9[r] = 0.f;
#pragma unroll
    for (int j = 0; j < 4; ++j) { h[0][j][r] = 0.f; h[1][j][r] = 0.f; }
  }
  i32x8 bn = ld_frag(lds + lane * 16);
#pragma unroll
  for (int t = 0; t < 32; ++t) {
    const i32x8 bc = bn;
    if (t < 31) bn = ld_frag(lds + (t + 1) * 2048 + lane * 16);
    i32x8 b9;
    if (NINE && (t & 3) == 0) b9 = ld_frag(lds + (t + wave) * 2048 + lane * 16);
    SB();
    h[0][t & 3] = __builtin_amdgcn_mfma_scale_f32_32x32x64_f8f6f4(a0[t >> 2], bc, h[0][t & 3], 1, 1, 0, 0, 0, 0);
    h[1][t & 3] = __builtin_amdgcn_mfma_scale_f32_32x32x64_f8f6f4(a1[t >> 2], bc, h[1][t & 3], 1, 1, 0, 0, 0, 0);
    if (NINE && (t & 3) == 0) h9 = __builtin_amdgcn_mfma_scale_f32_32x32x64_f8f6f4(a9[t >> 2], b9, h9, 1, 1, 0, 0, 0, 0);
    SB();
  }
  uint8_t* const pb = p.myws + WS_PB + (size_t)((g * 32 + i) * 9) * 16384;
#pragma unroll
  for (int q = 0; q < (NINE ? 36 : 32); ++q) {
    const f32x16& t = q < 32 ? h[q >> 4][(q >> 2) & 3] : h9;
    st16_out(pb, (unsigned)((q < 32 ? wave * 32 + q : 128 - 32 + wave * 4 + q) * 1024 + lane * 16),
             f32x4{t[4 * (q & 3)], t[4 * (q & 3) + 1], t[4 * (q & 3) + 2], t[4 * (q & 3) + 3]});
  }
  __syncthreads();
}

__device__ __forceinline__ void recv_tile(uint8_t* myws, float* out, int g, int src, int i, int k0, int k1, int wave, int lane,
                                          unsigned call, unsigned* tmo, float sc, uint8_t* scr) {
  const unsigned off = out_off(g < 4 ? 0 : 224, block_col(src, tile_block(g, i)), wave, lane);
  const uint8_t* const fl = myws + WS_DONE + (size_t)((src * 32 + i) * 4 + wave) * 128;
  const int nb = wave_blocks(g < 4 ? 7 : 9, wave);
  if (k0 >= min(nb, k1)) return;
  const uint8_t* const fh = myws + WS_HOME + (size_t)((g * 32 + i) * 4 + (lane & 3)) * 4;
  spin<RX_SLEEP>([&] { return __all(ld_agent(fh) >= call) != 0; }, tmo, call);
#pragma unroll 1
  for (int k = k0; k < min(nb, k1); ++k) {
    f32x4 h[16];
    asm volatile(SGPR_PAD "global_load_dwordx4 %0, %16, %18 offset:-4096 sc1\n\t""global_load_dwordx4 %1, %16, %18 offset:-3072 sc1\n\t"
                 "global_load_dwordx4 %2, %16, %18 offset:-2048 sc1\n\t""global_load_dwordx4 %3, %16, %18 offset:-1024 sc1\n\t"
                 "global_load_dwordx4 %4, %16, %18 offset:0 sc1\n\t""global_load_dwordx4 %5, %16, %18 offset:1024 sc1\n\t"
                 "global_load_dwordx4 %6, %16, %18 offset:2048 sc1\n\t""global_load_dwordx4 %7, %16, %18 offset:3072 sc1\n\t"
                 "global_load_dwordx4 %8, %17, %18 offset:-4096 sc1\n\t""global_load_dwordx4 %9, %17, %18 offset:-3072 sc1\n\t"
                 "global_load_dwordx4 %10, %17, %18 offset:-2048 sc1\n\t""global_load_dwordx4 %11, %17, %18 offset:-1024 sc1\n\t"
                 "global_load_dwordx4 %12, %17, %18 offset:0 sc1\n\t""global_load_dwordx4 %13, %17, %18 offset:1024 sc1\n\t"
                 "global_load_dwordx4 %14, %17, %18 offset:2048 sc1\n\t""global_load_dwordx4 %15, %17, %18 offset:3072 sc1"
                 : "=&v"(h[0]), "=&v"(h[1]), "=&v"(h[2]), "=&v"(h[3]), "=&v"(h[4]), "=&v"(h[5]), "=&v"(h[6]), "=&v"(h[7]),
                   "=&v"(h[8]), "=&v"(h[9]), "=&v"(h[10]), "=&v"(h[11]), "=&v"(h[12]), "=&v"(h[13]), "=&v"(h[14]), "=&v"(h[15])
                 : "v"(4096 + lane * 16), "v"(12288 + lane * 16), "s"(myws + WS_PB + (size_t)((g * 32 + i) * 9 + wave * 2 + k) * 16384) : "memory");
    spin<RX_SLEEP>([&] { return __builtin_amdgcn_readfirstlane(ld_sys(fl)) > 4 * call + k; }, tmo, call);
    asm volatile("s_waitcnt vmcnt(0)" : "+v"(h[0]), "+v"(h[1]), "+v"(h[2]), "+v"(h[3]), "+v"(h[4]), "+v"(h[5]), "+v"(h[6]), "+v"(h[7]), "+v"(h[8]), "+v"(h[9]), "+v"(h[10]), "+v"(h[11]), "+v"(h[12]), "+v"(h[13]), "+v"(h[14]), "+v"(h[15]) :: "memory");
    dec_out(myws + WS_OS + (size_t)((src * 32 + i) * 9 + wave * 2 + k) * OBLK, h, out, off + (unsigned)k * 32 * (GN * 4), sc, tmo, call, scr, lane);
  }
}

}

__global__ void __launch_bounds__(256, 1) gemm_rs_fused(const Params p) {
  __shared__ __attribute__((aligned(16))) uint8_t lds[LDS_BYTES];
  const int b = blockIdx.x;
  const int lane = threadIdx.x & 63;
  const int wave = __builtin_amdgcn_readfirstlane(threadIdx.x >> 6);
  const int rank = p.rank;
  uint8_t* const myws = p.myws;
  asm volatile("" :: "s"(myws), "s"(rank), "s"(p.x), "s"(p.w));

  const int grp = b & 7, dst = grp ^ rank, ti = 31 - (b >> 3);
  const bool local = grp == 0 || (grp >= 4 && ti >= 24);
  const int set = local ? 7 + (grp != 0) : grp - 1, cb = tile_block(grp, ti);
  const RowSet rs = row_set(set);
  unsigned* const callp = (unsigned*)(myws + WS_CALL + (size_t)b * 128);
  unsigned* const tmo = (unsigned*)(myws + WS_TMO);
  unsigned* const lk = (unsigned*)(myws + WS_LINK) + grp * 64;
  uint8_t* const scr = lds + LDS_PK + wave * 2048;

  PushCtx c;
  c.x = p.x; c.d = dst; c.local = grp == 0;
  c.xl = (unsigned)((lane & 31) * KS + (lane >> 5) * 32);
  const RowSet hs = row_set(grp - 1);
  c.skip0 = grp == 0 ? NTASK : hs.task0; c.skipn = grp == 0 ? 0 : 4 * hs.r;

  unsigned call;
  if (ti >= 24) {
    const int w = ((b >> 3) * 4 + wave) & 31;
    c.w = p.w; c.dst = p.ws[dst]; c.lk = lk; c.rank = rank;
    c.wl = (unsigned)((lane >> 5) * 32 * GN + (lane & 31) * 4);
    c.scr = scr;
    PushState s;
    u32x4 x0[4];
    unsigned cv;
    s.tp = -1; s.tc = task_at(c, w); s.tn = task_at(c, w + 32); s.tl = task_at(c, w + 64); s.gone = true;
    asm volatile(SGPR_PAD "global_load_dword %0, %1, %2 sc1" : "=v"(cv) : "v"(0u), "s"(callp) : "memory");
    ld_x(c, s.tc, x0);
    __builtin_amdgcn_sched_barrier(0);
    ld_x(c, s.tn, s.nx);
    const RowSet fs = row_set(task_set(s.tc));
    unsigned first = (unsigned)((fs.base * 128 + rank * fs.r * 16 + (s.tc - fs.task0) * 4) << 10) + lane * 16;
    asm volatile("" : "+v"(first));
    asm volatile("s_waitcnt vmcnt(4)" : "+v"(cv), "+v"(x0[0]), "+v"(x0[1]), "+v"(x0[2]), "+v"(x0[3]) :: "memory");
    call = __builtin_amdgcn_readfirstlane(cv) + 1;
    c.par = (unsigned)(WS_DATA + (call & 1) * PAR_BYTES);
#pragma unroll
    for (int q = 0; q < 4; ++q) st16_sys(c.dst, c.par + first + q * 1024, x0[q]);
    asm volatile("s_waitcnt vmcnt(4)" : "+v"(s.nx[0]), "+v"(s.nx[1]), "+v"(s.nx[2]), "+v"(s.nx[3]) :: "memory");
    c.tick0 = (call - 1) * (NTASK - c.skipn - 64) - 96;
    c.tag = call & 0xffffu;
    c.call = call;
    s.line = u32x4{0u, 0u, 0u, 0u};
    pack_task(c, x0, s.o, lane, true);
#pragma unroll 1
    while (s.tp < NTASK) {
      if (task_is_w(s.tl)) push_step<true>(c, s, lane);
      else push_step<false>(c, s, lane);
    }
  } else {
    call = __builtin_amdgcn_readfirstlane(ld_agent(callp)) + 1;
  }
  uint8_t* const par = myws + WS_DATA + (size_t)(call & 1) * PAR_BYTES;

#pragma unroll 1
  for (int job = 0; job < 4; ++job) {
    if (job == 1 && grp != 0 && ti < 24) {
      __syncthreads();
      if (grp >= 4) {
        home_tile<true>(p, grp, dst, ti, lds, wave, lane);
      } else {
#pragma unroll 1
        for (int i = ti; i < 32; i += ti >= 16 ? 8 : 32) home_tile<false>(p, grp, dst, i, lds, wave, lane);
      }
    }
    const bool jw = grp == 0 ? job < 3 : job == 1;
    const int beta = grp == 0 ? (ti * 4 + wave) & 63 : (23 - ti) * 4 + wave;
    if (grp == 0 ? (job == 2 && ti >= 24) : (ti >= 24 || job >= 2 || (job == 0 && beta >= 8 * rs.r))) continue;
    const int n = job == 2 ? 24 + (ti & 7) : cb;
    const RowSet js = row_set(grp == 0 ? 7 + (ti >> 4) : set);
    const int bs = div_r(beta, js.r), bu = beta - bs * js.r;
    const int src = !jw ? bs : grp != 0 ? dst : job == 0 ? rank : job == 1 ? rank ^ (4 + (ti & 3)) : rank ^ (1 + (ti >> 3));
    const int t0 = jw ? T_W0 + n * 16 + wave * 4 : js.task0 + bu * 4;
    const uint8_t* const slot = par + XS_BYTES + WT_BYTES + (size_t)(src * NTASK + t0) * ITASK;
    const bool absent = !jw && grp != 0 && src == dst;
    if (!absent && !wait_tasks(slot, tmo, call, lane)) continue;
    if (src != rank && !absent) {
      const unsigned base = jw ? (unsigned)XS_BYTES + (unsigned)(n * (512 << 10) + (src * 8 + wave * 2) * 8192)
                               : (unsigned)(js.base * (128 << 10) + (src * js.r + bu) * 16384);
      unsigned off[4];
#pragma unroll
      for (int t = 0; t < 4; ++t) off[t] = base + (jw ? (t >> 1) * 8192 + (t & 1) * 1024 : t * 4096);
      dec_in4(slot, par, off, jw ? 2048 : 1024, lds + wave * 4096, lane);
      wait_vm<0>();
    }
    st1_agent(jw ? myws + WS_DW + (size_t)(n * 32 + src * 4 + wave) * 4
                 : myws + WS_DX + (size_t)((grp == 0 ? 7 + (ti >> 4) : set) * 128 + beta) * 4, call);
  }
  if (grp != 0 && ti < 24) {
    wait_vm<0>();
    st1_agent(myws + WS_HOME + (size_t)((grp * 32 + ti) * 4 + wave) * 4, call);
    if (grp < 4 && ti >= 16) st1_agent(myws + WS_HOME + (size_t)((grp * 32 + ti + 8) * 4 + wave) * 4, call);
  }

  unsigned* const ctrl = (unsigned*)(lds + LDS_CTRL);
  if (wave == 0) {
    const uint8_t* const fw = myws + WS_DW + (size_t)(cb * 32 + (lane & 31)) * 4;
    const uint8_t* const fx = myws + WS_DX + (size_t)(set * 128 + min(lane, 8 * rs.r - 1)) * 4;
    const uint8_t* const fy = myws + WS_DX + (size_t)(set * 128 + min(64 + lane, 8 * rs.r - 1)) * 4;
    const bool ok = spin([&] { const unsigned a = ld_agent(fw), c = ld_agent(fx), d = ld_agent(fy); return __all(min(min(a, c), d) >= call) != 0; },
                         tmo, call);
    if (lane == 0) ctrl[0] = !ok;
  }
  __syncthreads();
  const bool dead0 = __builtin_amdgcn_readfirstlane(ctrl[0]) != 0;

  if (wave == 0) st1_agent(callp, call);
  const float sc = p.sx[0] * p.sw[0];
  const unsigned off = out_off(rs.row0, block_col(rank, cb), wave, lane);
  float* const ob = p.out[rank];
  uint8_t* const blk = lds + wave * 32768;
  uint8_t* const oslot = p.ws[dst] + WS_OS + (size_t)((rank * 32 + ti) * 9 + wave * 2) * OBLK;
  unsigned* const flag = (unsigned*)(p.ws[dst] + WS_DONE + (size_t)((rank * 32 + ti) * 4 + wave) * 128);
  const int nb = wave_blocks(rs.r, wave);
  const unsigned* const turn = ti < NTURN ? lk + 32 : (const unsigned*)(myws + WS_TURN + (size_t)((grp * 32 + ti) * 4 + wave) * 128);
  unsigned* const next = (unsigned*)(myws + WS_TURN + (size_t)((grp * 32 + ((ti + NTURN) & 31)) * 4 + wave) * 128);
  const int depth = ti < DEEP_T ? DEEP_D : nb == 3 ? OUT_D3 : OUT_D;
  unsigned nl0 = 0, nl1 = 0, nl2 = 0;
  int sent = 0;

  if (!dead0) {
    const uint8_t* const ga = par + (size_t)rs.base * (128 << 10);
    const uint8_t* const gb = par + XS_BYTES + (size_t)cb * (512 << 10);
    if (rs.r == 9) gemm_tile<9>(ga, 9 * 2048, gb, dst, lds, wave, lane);
    else if (nb == 1) gemm_tile<7>(ga, 7 * 2048, gb, dst, lds, wave, lane);
    else gemm_tile<8>(ga, rs.r * 2048, gb, local ? -1 : dst, lds, wave, lane);
    unsigned tnv, tmv;
    asm volatile(SGPR_PAD "global_load_dword %0, %2, %3 sc1\n\tglobal_load_dword %1, %2, %4 sc1"
                 : "=&a"(tnv), "=&a"(tmv) : "v"(0u), "s"(turn), "s"(tmo) : "memory");
    auto landed = [&](unsigned& s_tn, unsigned& s_tm) {
      unsigned t0, t1;
      asm volatile("s_waitcnt vmcnt(0)\n\tv_accvgpr_read_b32 %2, %4\n\tv_accvgpr_read_b32 %3, %5\n\ts_nop 1\n\t"
                   "v_readfirstlane_b32 %0, %2\n\tv_readfirstlane_b32 %1, %3\n\ts_nop 1"
                   : "=&s"(s_tn), "=&s"(s_tm), "=&v"(t0), "=&v"(t1) : "a"(tnv), "a"(tmv) : "memory");
    };
    float s0 = sc;
#pragma unroll 1
    for (int k = 0; k < nb; ++k) {
      float v[4][16];
#pragma unroll
      for (int i = 0; i < 16; ++i) {
        const f32x4 t = *(const f32x4*)(blk + k * 16384 + i * 1024 + lane * 16);
#pragma unroll
        for (int q = 0; q < 4; ++q) v[i >> 2][(i & 3) * 4 + q] = t[q];
      }
      __builtin_amdgcn_fence(__ATOMIC_ACQ_REL, "wavefront");
      if (local) {
        if (k == 0) {
          unsigned tn, t;
          landed(tn, t);
          if (t == call) s0 = __builtin_nanf("");
        }
#pragma unroll
        for (int r = 0; r < 16; ++r)
          st16_out(ob, off + k * 32 * (GN * 4) + row_off(r), f32x4{v[0][r] * s0, v[1][r] * s0, v[2][r] * s0, v[3][r] * s0});
        continue;
      }
      const unsigned nl = enc_out(v, blk + k * 16384, lane);
      __builtin_amdgcn_fence(__ATOMIC_ACQ_REL, "wavefront");
      if (k == 0) nl0 = nl; else if (k == 1) nl1 = nl; else nl2 = nl;
      if (k == 0) {
        unsigned tn, t;
        landed(tn, t);
        if (ti < 24 && nb > 1 && t != call && (tn >= call || __builtin_amdgcn_readfirstlane(ld_agent(turn)) >= call)) {
          send_out(blk, nl, false, oslot, lane, next, call, false, depth);
          sent = 1;
        }
      }
    }
  } else if (local) {
    const float n = __builtin_nanf("");
#pragma unroll 1
    for (int t = 0; t < 32; ++t) st16_out(ob, off + (t >> 4) * 32 * (GN * 4) + row_off(t & 15), f32x4{n, n, n, n});
  }
  if (grp == 0) return;

  const int last8 = grp < 4 ? 24 : 16;
  if (ti >= 24) recv_tile(myws, p.out[rank], grp, dst, ti - 24, 0, 3, wave, lane, call, tmo, sc, scr);
  if (!local) {
    if (sent) {
      wait_vm<0>();
      st1_sys(flag, 4 * call + 1);
    } else {
      spin([&] { return __builtin_amdgcn_readfirstlane(ld_agent(turn)) >= call; }, tmo, call);
    }
    const bool dead = dead0 || __builtin_amdgcn_readfirstlane(ld_agent(tmo)) == call;
#pragma unroll 1
    for (int k = sent; k < nb; ++k) {
      send_out(blk + k * 16384, k == 0 ? nl0 : k == 1 ? nl1 : nl2, dead, oslot + k * OBLK, lane, next, call, k == nb - 1, depth);
      wait_vm<0>();
      st1_sys(flag, 4 * call + 1 + k);
    }
  }
#pragma unroll 1
  for (int duty = 0; duty < 2; ++duty) {
    int i, k0, k1;
    if (duty == 0) {
      if (ti >= last8) continue;
      i = ti + 8; k0 = 0; k1 = i >= last8 ? 1 : 3;
    } else if (ti < 8) {
      i = last8 + ti; k0 = grp < 4 ? 1 : 2; k1 = 3;
    } else if (local) {
      i = ti - 8; k0 = 1; k1 = 2;
    } else {
      continue;
    }
    recv_tile(myws, p.out[rank], grp, dst, i, k0, k1, wave, lane, call, tmo, sc, scr);
  }
}

extern "C" void kernel_launch(int rank, int nranks, ncclComm_t comm, hipStream_t stream,
                              void* const* d_in, const int* in_sizes, int n_in,
                              void* d_out, int out_size, const int* in_dtypes, int out_dtype,
                              void* const* d_out_all, void* const* d_ws_all, size_t ws_bytes) {
  (void)comm; (void)in_sizes; (void)in_dtypes; (void)out_dtype; (void)d_out;
  if (nranks != NR || (unsigned)rank >= (unsigned)NR || n_in != 4 || ws_bytes < WS_NEED || out_size != 512 * GN) {
    fprintf(stderr, "kernel_launch: unexpected problem shape\n");
    return;
  }
  Params p;
  p.x = (const uint8_t*)d_in[0];
  p.w = (const uint8_t*)d_in[1];
  p.sx = (const float*)d_in[2];
  p.sw = (const float*)d_in[3];
  for (int i = 0; i < NR; ++i) {
    p.ws[i] = (uint8_t*)d_ws_all[i];
    p.out[i] = (float*)d_out_all[i];
  }
  p.myws = p.ws[rank];
  p.rank = rank;
  p.pad = 0;
  hipLaunchKernelGGL(gemm_rs_fused, dim3(NWG), dim3(256), 0, stream, p);
  HIP_CHECK(hipGetLastError());
}
```

```cpp
#include <hip/hip_runtime.h>
#include <rccl/rccl.h>
#include <cstdio>
#include <cstdint>

#define HIP_CHECK(expr) do { hipError_t _e = (expr); if (_e != hipSuccess) { \
    fprintf(stderr, "HIP error %s at %s:%d\n", hipGetErrorString(_e), __FILE__, __LINE__); \
    return; } } while (0)

typedef unsigned u32x4 __attribute__((ext_vector_type(4)));
typedef int i32x4 __attribute__((ext_vector_type(4)));
typedef int i32x8 __attribute__((ext_vector_type(8)));
typedef float f32x4 __attribute__((ext_vector_type(4)));
typedef float f32x16 __attribute__((ext_vector_type(16)));

#define SGPR_PAD "s_nop 4\n\t"

namespace {

constexpr int NR = 8;
constexpr int GN = 16384;
constexpr int KS = 512;
constexpr int NWG = 256;
constexpr int KSTEPS = 64;
constexpr int NSTAGE = 6;
constexpr int LDS_CTRL = NSTAGE * 26624;
constexpr int LDS_BYTES = LDS_CTRL + 64;
constexpr int LDS_PK = 9 * 16384;

constexpr int T_W0 = 228, T_OWN = T_W0 + 512, NTASK = T_OWN + 64;

constexpr size_t WS_CALL = 0;
constexpr size_t WS_TMO = 32768;
constexpr size_t WS_DONE = 896u << 10;
constexpr size_t WS_LINK = WS_TMO + 1024;
constexpr size_t WS_DX = WS_LINK + 4096;
constexpr size_t WS_DW = WS_DX + 8192;
constexpr size_t WS_HOME = WS_DW + 4096;
constexpr size_t WS_TURN = 768u << 10;
constexpr size_t WS_DATA = 1u << 20;
constexpr size_t XS_BYTES = 10u << 20;
constexpr size_t WT_BYTES = 16u << 20;
constexpr unsigned ITASK = 3328;
constexpr size_t IS_BYTES = 21u << 20;
constexpr size_t PAR_BYTES = XS_BYTES + WT_BYTES + IS_BYTES;
static_assert((size_t)8 * NTASK * ITASK <= IS_BYTES, "");
constexpr size_t OBLK = 17 * 1024;
constexpr size_t WS_OS = WS_DATA + 2 * PAR_BYTES;
constexpr size_t WS_PB = WS_OS + 8 * 32 * 9 * OBLK;
constexpr size_t WS_NEED = WS_PB + 8 * 32 * 9 * 16384;

constexpr unsigned long long DEADLINE_TICKS = 50000000ull;

constexpr int OPEN_AT = 790, NTURN = 24, OUT_REL = 3, OUT_D = 2, OUT_D3 = 3, DEEP_D = 4, DEEP_T = 8;
constexpr int POLL_SLEEP = 6, RX_SLEEP = 16, LAST_SLEEP = 8;

struct Params {
  const uint8_t* x;
  const uint8_t* w;
  const float* sx;
  const float* sw;
  uint8_t* ws[NR];
  float* out[NR];
  uint8_t* myws;
  int rank;
  int pad;
};

__device__ __forceinline__ unsigned ld_sys(const void* ptr) {
  return __hip_atomic_load((const unsigned*)ptr, __ATOMIC_RELAXED, __HIP_MEMORY_SCOPE_SYSTEM);
}
__device__ __forceinline__ unsigned ld_agent(const void* ptr) {
  return __hip_atomic_load((const unsigned*)ptr, __ATOMIC_RELAXED, __HIP_MEMORY_SCOPE_AGENT);
}
__device__ __forceinline__ void st_agent(void* ptr, unsigned v) {
  __hip_atomic_store((unsigned*)ptr, v, __ATOMIC_RELAXED, __HIP_MEMORY_SCOPE_AGENT);
}

template <int SLEEP = POLL_SLEEP, class F>
__device__ __forceinline__ bool spin(F ready, unsigned* tmo, unsigned call) {
  const unsigned long long t0 = __builtin_amdgcn_s_memrealtime();
  for (unsigned n = 1;; ++n) {
    if (ready()) return true;
    if ((n & 3) == 0 && (__builtin_amdgcn_readfirstlane(ld_agent(tmo)) == call || __builtin_amdgcn_s_memrealtime() - t0 > DEADLINE_TICKS)) {
      st_agent(tmo, call);
      return false;
    }
    __builtin_amdgcn_s_sleep(SLEEP);
  }
}

__device__ __forceinline__ void st16_sys(void* base, unsigned off, u32x4 v) {
  asm volatile(SGPR_PAD "global_store_dwordx4 %0, %1, %2 sc0 sc1\n\ts_nop 1" :: "v"(off), "v"(v), "s"(base) : "memory");
}

__device__ __forceinline__ void st16_loc(void* base, unsigned off, u32x4 v) {
  asm volatile(SGPR_PAD "global_store_dwordx4 %0, %1, %2 sc1\n\ts_nop 1" :: "v"(off), "v"(v), "s"(base) : "memory");
}

__device__ __forceinline__ void st1_agent(void* base, unsigned v) {
  asm volatile(SGPR_PAD "s_mov_b64 exec, 1\n\tglobal_store_dword %0, %1, %2 sc1\n\ts_mov_b64 exec, -1"
               :: "v"(0u), "v"(v), "s"(base) : "memory");
}

__device__ __forceinline__ void st1_sys(void* base, unsigned v) {
  asm volatile(SGPR_PAD "s_mov_b64 exec, 1\n\tglobal_store_dword %0, %1, %2 sc0 sc1\n\ts_mov_b64 exec, -1"
               :: "v"(0u), "v"(v), "s"(base) : "memory");
}
__device__ __forceinline__ void st16_sys_m(void* base, unsigned off, u32x4 v, unsigned long long mask) {
  asm volatile(SGPR_PAD "s_mov_b64 exec, %3\n\tglobal_store_dwordx4 %0, %1, %2 sc0 sc1\n\ts_mov_b64 exec, -1\n\ts_nop 0"
               :: "v"(off), "v"(v), "s"(base), "s"(mask) : "memory");
}

template <int N> __device__ __forceinline__ void wait_vm() {
  asm volatile("s_waitcnt vmcnt(%0)" :: "n"(N) : "memory");
}

__device__ __forceinline__ unsigned wave_max_u32(unsigned v) {
  v = max(v, (unsigned)__builtin_amdgcn_update_dpp(0, (int)v, 0x111, 0xf, 0xf, false));
  v = max(v, (unsigned)__builtin_amdgcn_update_dpp(0, (int)v, 0x112, 0xf, 0xf, false));
  v = max(v, (unsigned)__builtin_amdgcn_update_dpp(0, (int)v, 0x114, 0xf, 0xf, false));
  v = max(v, (unsigned)__builtin_amdgcn_update_dpp(0, (int)v, 0x118, 0xf, 0xf, false));
  return max(max((unsigned)__builtin_amdgcn_readlane((int)v, 15), (unsigned)__builtin_amdgcn_readlane((int)v, 31)),
             max((unsigned)__builtin_amdgcn_readlane((int)v, 47), (unsigned)__builtin_amdgcn_readlane((int)v, 63)));
}

__device__ __forceinline__ unsigned irec_slot(unsigned idx) { return idx + 3 + min((__umul24(idx, 37u) + 74u) >> 8, 3u); }

__device__ __forceinline__ unsigned wave_sum_u32(unsigned v) {
  v += (unsigned)__builtin_amdgcn_update_dpp(0, (int)v, 0x111, 0xf, 0xf, false);
  v += (unsigned)__builtin_amdgcn_update_dpp(0, (int)v, 0x112, 0xf, 0xf, false);
  v += (unsigned)__builtin_amdgcn_update_dpp(0, (int)v, 0x114, 0xf, 0xf, false);
  v += (unsigned)__builtin_amdgcn_update_dpp(0, (int)v, 0x118, 0xf, 0xf, false);
  return (unsigned)(__builtin_amdgcn_readlane((int)v, 15) + __builtin_amdgcn_readlane((int)v, 31) +
                    __builtin_amdgcn_readlane((int)v, 47) + __builtin_amdgcn_readlane((int)v, 63));
}
__device__ __forceinline__ unsigned bfi(unsigned m, unsigned a, unsigned b) { return (a & m) | (b & ~m); }
constexpr unsigned HDR_RAW = 0x80000000u;

constexpr unsigned IREC_CAP = 122, IHDR_RAW = 0x8000u;

struct InTask {
  u32x4 d[4];
  unsigned long long mask;
  bool raw;
};

__device__ __forceinline__ void put_records2(unsigned q0, unsigned q1, unsigned tag, unsigned& cnt, uint8_t* scr, unsigned dump) {
  unsigned m = bfi(0x55555555u, q0 | q0 >> 1, q1 | q1 << 1);
  for (unsigned long long act = __ballot(m != 0); act != 0 && cnt <= IREC_CAP; act = __ballot(m != 0)) {
    const unsigned bit = (unsigned)__builtin_ctz(m | 0x80000000u), sh = bit & 30u;
    const unsigned idx = cnt + __builtin_amdgcn_mbcnt_hi((unsigned)(act >> 32), __builtin_amdgcn_mbcnt_lo((unsigned)act, 0u));
    const unsigned sl = 2 * irec_slot(idx), at = ((m != 0) & (idx < IREC_CAP)) ? sl : dump;
    *(unsigned short*)(scr + at) = (unsigned short)(tag | (bit & 1u) << 6 | sh << 1 | ((((bit & 1u) ? q1 : q0) >> sh) & 3u));
    m &= m - 1;
    cnt += __popcll(act);
  }
}

__device__ __forceinline__ void enc_in(const u32x4 (&v)[4], InTask& o, unsigned tag, uint8_t* scr, int lane) {
  unsigned sum = 0;
#pragma unroll
  for (int q = 0; q < 4; ++q) sum = __builtin_amdgcn_udot4(v[q][0] & 0x7f7f7f7fu, 0x01010101u, sum, false);
  const unsigned e0 = (unsigned)min(max((int)((wave_sum_u32(sum) + 10445u) >> 12) - 7, 0), 24);
  const unsigned add = 0x80808080u - e0 * 0x04040404u;
  unsigned pay[12], qq[4], cnt = 0;
#pragma unroll
  for (int q = 0; q < 4; ++q) {
    unsigned R[4], S[4];
#pragma unroll
    for (int k = 0; k < 4; ++k) {
      R[k] = (v[q][k] & 0x7f7f7f7fu) + add;
      S[k] = bfi(0x80808080u, v[q][k], R[k]);
    }
    pay[3 * q] = bfi(0x60606060u, S[3] << 5, S[0]);
    pay[3 * q + 1] = bfi(0x60606060u, S[3] << 3, S[1]);
    pay[3 * q + 2] = bfi(0x20202020u, S[3] << 1, bfi(0x40404040u, S[3] >> 1, S[2]));
    qq[q] = bfi(0xc0c0c0c0u, R[3] << 1, bfi(0x30303030u, R[2] >> 1, bfi(0x0c0c0c0cu, R[1] >> 3, (R[0] >> 5) & 0x03030303u)));
  }
  put_records2(qq[0], qq[1], (unsigned)(lane << 8), cnt, scr, 256 + 2 * lane);
  put_records2(qq[2], qq[3], (unsigned)(lane << 8 | 128), cnt, scr, 256 + 2 * lane);
  *(unsigned short*)(scr + (lane < 4 ? lane * 16 : lane < 6 ? lane * 2 - 6 : 256 + 2 * lane)) =
      (unsigned short)(lane < 4 ? tag : lane == 4 ? cnt : e0);
  __builtin_amdgcn_fence(__ATOMIC_ACQ_REL, "wavefront");
  const u32x4 rec = *(const u32x4*)(scr + (lane & 15) * 16);
  __builtin_amdgcn_fence(__ATOMIC_ACQ_REL, "wavefront");
  o.raw = cnt > IREC_CAP;
  if (o.raw) {
#pragma unroll
    for (int q = 0; q < 4; ++q) o.d[q] = v[q];
    o.mask = ~0ull;
  } else {
#pragma unroll
    for (int q = 0; q < 3; ++q) o.d[q] = u32x4{pay[4 * q], pay[4 * q + 1], pay[4 * q + 2], pay[4 * q + 3]};
    o.d[3] = rec;
    o.mask = (1ull << (((irec_slot(cnt) + 31) >> 5) * 4)) - 16;
  }
}

__device__ __forceinline__ void dec_in4(const uint8_t* slot, uint8_t* out, const unsigned (&off)[4], unsigned stride, uint8_t* scr, int lane) {
  u32x4 y[4][3];
  unsigned rec[4];
  asm volatile(SGPR_PAD "global_load_dword %12, %16, %20 sc0 sc1\n\tglobal_load_dword %13, %16, %20 offset:3328 sc0 sc1\n\t"
               "global_load_dword %14, %17, %20 sc0 sc1\n\tglobal_load_dword %15, %17, %20 offset:3328 sc0 sc1\n\t"
               "global_load_dwordx4 %0, %18, %20 offset:-3072 sc0 sc1\n\tglobal_load_dwordx4 %1, %18, %20 offset:-2048 sc0 sc1\n\t"
               "global_load_dwordx4 %2, %18, %20 offset:-1024 sc0 sc1\n\t"
               "global_load_dwordx4 %3, %18, %20 offset:256 sc0 sc1\n\tglobal_load_dwordx4 %4, %18, %20 offset:1280 sc0 sc1\n\t"
               "global_load_dwordx4 %5, %18, %20 offset:2304 sc0 sc1\n\t"
               "global_load_dwordx4 %6, %19, %20 offset:-3072 sc0 sc1\n\tglobal_load_dwordx4 %7, %19, %20 offset:-2048 sc0 sc1\n\t"
               "global_load_dwordx4 %8, %19, %20 offset:-1024 sc0 sc1\n\t"
               "global_load_dwordx4 %9, %19, %20 offset:256 sc0 sc1\n\tglobal_load_dwordx4 %10, %19, %20 offset:1280 sc0 sc1\n\t"
               "global_load_dwordx4 %11, %19, %20 offset:2304 sc0 sc1\n\ts_waitcnt vmcnt(0)"
               : "=&v"(y[0][0]), "=&v"(y[0][1]), "=&v"(y[0][2]), "=&v"(y[1][0]), "=&v"(y[1][1]), "=&v"(y[1][2]),
                 "=&v"(y[2][0]), "=&v"(y[2][1]), "=&v"(y[2][2]), "=&v"(y[3][0]), "=&v"(y[3][1]), "=&v"(y[3][2]),
                 "=&v"(rec[0]), "=&v"(rec[1]), "=&v"(rec[2]), "=&v"(rec[3])
               : "v"(lane * 4), "v"(2 * ITASK + lane * 4), "v"(ITASK + lane * 16), "v"(3 * ITASK + lane * 16), "s"(slot) : "memory");
#pragma unroll
  for (int t = 0; t < 4; ++t) *(u32x4*)(scr + t * 1024 + lane * 16) = u32x4{0u, 0u, 0u, 0u};
  __builtin_amdgcn_fence(__ATOMIC_ACQ_REL, "wavefront");
  unsigned hdr[4];
#pragma unroll
  for (int t = 0; t < 4; ++t) {
    hdr[t] = (unsigned)__builtin_amdgcn_readlane((int)rec[t], 0) >> 16 | (unsigned)__builtin_amdgcn_readlane((int)rec[t], 1) << 16;
    const unsigned end = irec_slot(hdr[t] & 0x7fffu);
#pragma unroll
    for (int i = 0; i < 2; ++i) {
      const unsigned r16 = rec[t] >> (16 * i);
      const bool isrec = i == 0 ? (lane >= 16 || (lane & 3) != 0) && lane != 1 : lane != 0;
      const unsigned val = isrec && (unsigned)(lane * 2 + i) < end ? (r16 & 3u) << ((r16 >> 1) & 30u) : 0u;
      __hip_atomic_fetch_or((unsigned*)(scr + t * 1024 + ((r16 >> 6) & 255u) * 4), val, __ATOMIC_RELAXED, __HIP_MEMORY_SCOPE_WORKGROUP);
    }
  }
  __builtin_amdgcn_fence(__ATOMIC_ACQ_REL, "wavefront");
#pragma unroll
  for (int t = 0; t < 4; ++t) {
    if (hdr[t] & IHDR_RAW) continue;
    const u32x4 qq = *(const u32x4*)(scr + t * 1024 + lane * 16);
    const unsigned add = ((hdr[t] >> 16) & 31u) * 0x04040404u;
#pragma unroll
    for (int q = 0; q < 4; ++q) {
      const unsigned d0 = y[t][(3 * q) >> 2][(3 * q) & 3], d1 = y[t][(3 * q + 1) >> 2][(3 * q + 1) & 3], d2 = y[t][(3 * q + 2) >> 2][(3 * q + 2) & 3];
      const unsigned d3 = bfi(0x80808080u, d2 << 1, bfi(0x10101010u, d2 >> 1, bfi(0x0c0c0c0cu, d1 >> 3, d0 >> 5)));
      const unsigned X[4] = {bfi(0x60606060u, qq[q] << 5, d0), bfi(0x60606060u, qq[q] << 3, d1), bfi(0x60606060u, qq[q] << 1, d2),
                             bfi(0x60606060u, qq[q] >> 1, d3)};
      u32x4 r;
#pragma unroll
      for (int k = 0; k < 4; ++k) r[k] = bfi(0x80808080u, X[k], (X[k] & 0x7f7f7f7fu) + add);
      asm volatile(SGPR_PAD "global_store_dwordx4 %0, %1, %2 sc1\n\ts_nop 1" :: "v"(off[t] + q * stride + lane * 16), "v"(r), "s"(out) : "memory");
    }
  }
  __builtin_amdgcn_fence(__ATOMIC_ACQ_REL, "wavefront");
}

__device__ __forceinline__ bool wait_tasks(const uint8_t* slot, unsigned* tmo, unsigned call, int lane) {
  const uint8_t* q = slot + ((lane >> 2) & 3) * ITASK + (lane & 3) * 16;
  return spin([&] { return __all((ld_sys(q) & 0xffffu) == (call & 0xffffu)) != 0; }, tmo, call);
}

struct PushCtx {
  const uint8_t* x;
  const uint8_t* w;
  uint8_t* dst;
  unsigned* lk;
  uint8_t* scr;
  unsigned xl, wl;
  unsigned par, tick0, tag, call;
  int skip0, skipn;
  int rank, d;
  bool local;
};

struct RowSet { int r, row0, base, task0; };
__device__ __forceinline__ RowSet row_set(int set) {
  const int a = min(set, 3), o = max(set - 7, 0), b = set - a - o;
  const int own = (set + 1) >> 3, mid = ((set + 5) >> 3) - own;
  return {7 + 2 * mid + own, 224 * mid + 256 * o, 7 * a + 9 * b + 8 * o, 28 * a + 36 * b + 512 * own + 32 * o};
}
__device__ __forceinline__ int block_col(int q, int n) { return ((n & 3) * 4 + (q < 4 ? q : (q ^ n) & 3)) * 1024 + (n >> 2) * 128; }
__device__ __forceinline__ int tile_block(int g, int i) {
  const int c = (i * 43) >> 7, m = i - 3 * c;
  return g < 4 ? i : i >= 24 ? (g - 4) * 8 + i - 24 : 4 * c + m + (m >= g - 4);
}
__device__ __forceinline__ bool task_is_w(int t) { return t >= T_W0 && t < T_OWN; }
__device__ __forceinline__ int task_set(int t) {
  const int lo = min((t * 2341) >> 16, ((t + 24) * 1821) >> 16), hi = 7 + ((t - T_OWN) >> 5);
  return t < T_OWN ? lo : hi;
}

__device__ __forceinline__ int div_r(int v, int r) { return (v * (0x728093 >> (8 * (r - 7)) & 255)) >> 10; }

__device__ __forceinline__ void ld_x(const PushCtx& c, int t, u32x4 (&v)[4]) {
  t = min(t, NTASK - 1);
  const int set = task_set(t);
  const RowSet rs = row_set(set);
  const int p0 = 2 * (t - rs.task0), ks = div_r(p0, rs.r), rg = p0 - ks * rs.r;
  const unsigned o0 = c.xl + (unsigned)((((set < 7 ? set + 1 : 0) ^ c.d) * 512 + rs.row0 + rg * 32) * KS + ks * 64);
  const unsigned o1 = o0 + (unsigned)(rg + 1 == rs.r ? 64 - rg * 32 * KS : 32 * KS);
  asm volatile(SGPR_PAD "global_load_dwordx4 %0, %4, %6\n\tglobal_load_dwordx4 %1, %4, %6 offset:16\n\t"
               "global_load_dwordx4 %2, %5, %6\n\tglobal_load_dwordx4 %3, %5, %6 offset:16"
               : "=&v"(v[0]), "=&v"(v[1]), "=&v"(v[2]), "=&v"(v[3]) : "v"(o0), "v"(o1), "s"(c.x) : "memory");
}

__device__ __forceinline__ void ld_w(const uint8_t* base, unsigned o, unsigned (&w)[16]) {
    asm volatile(SGPR_PAD
                 "global_load_dword %0, %16, %17\n\tv_add_u32 %16, 0x4000, %16\n\t"
                 "global_load_dword %1, %16, %17\n\tv_add_u32 %16, 0x4000, %16\n\t"
                 "global_load_dword %2, %16, %17\n\tv_add_u32 %16, 0x4000, %16\n\t"
                 "global_load_dword %3, %16, %17\n\tv_add_u32 %16, 0x4000, %16\n\t"
                 "global_load_dword %4, %16, %17\n\tv_add_u32 %16, 0x4000, %16\n\t"
                 "global_load_dword %5, %16, %17\n\tv_add_u32 %16, 0x4000, %16\n\t"
                 "global_load_dword %6, %16, %17\n\tv_add_u32 %16, 0x4000, %16\n\t"
                 "global_load_dword %7, %16, %17\n\tv_add_u32 %16, 0x4000, %16\n\t"
                 "global_load_dword %8, %16, %17\n\tv_add_u32 %16, 0x4000, %16\n\t"
                 "global_load_dword %9, %16, %17\n\tv_add_u32 %16, 0x4000, %16\n\t"
                 "global_load_dword %10, %16, %17\n\tv_add_u32 %16, 0x4000, %16\n\t"
                 "global_load_dword %11, %16, %17\n\tv_add_u32 %16, 0x4000, %16\n\t"
                 "global_load_dword %12, %16, %17\n\tv_add_u32 %16, 0x4000, %16\n\t"
                 "global_load_dword %13, %16, %17\n\tv_add_u32 %16, 0x4000, %16\n\t"
                 "global_load_dword %14, %16, %17\n\tv_add_u32 %16, 0x4000, %16\n\t"
                 "global_load_dword %15, %16, %17\n\t"
                 : "=&v"(w[0]), "=&v"(w[1]), "=&v"(w[2]), "=&v"(w[3]), "=&v"(w[4]), "=&v"(w[5]), "=&v"(w[6]), "=&v"(w[7]),
                   "=&v"(w[8]), "=&v"(w[9]), "=&v"(w[10]), "=&v"(w[11]), "=&v"(w[12]), "=&v"(w[13]), "=&v"(w[14]), "=&v"(w[15]), "+v"(o) : "s"(base) : "memory");
    static_assert(GN == 0x4000, "");
}
__device__ __forceinline__ void transpose_w(const unsigned (&w)[16], u32x4 (&v)[4]) {
#pragma unroll
  for (int q = 0; q < 4; ++q) {
    const unsigned d0 = w[4 * q], d1 = w[4 * q + 1], d2 = w[4 * q + 2], d3 = w[4 * q + 3];
    const unsigned t0 = __builtin_amdgcn_perm(d1, d0, 0x05010400u), t1 = __builtin_amdgcn_perm(d1, d0, 0x07030602u);
    const unsigned t2 = __builtin_amdgcn_perm(d3, d2, 0x05010400u), t3 = __builtin_amdgcn_perm(d3, d2, 0x07030602u);
    v[0][q] = __builtin_amdgcn_perm(t2, t0, 0x05040100u);
    v[1][q] = __builtin_amdgcn_perm(t2, t0, 0x07060302u);
    v[2][q] = __builtin_amdgcn_perm(t3, t1, 0x05040100u);
    v[3][q] = __builtin_amdgcn_perm(t3, t1, 0x07060302u);
  }
}

__device__ __forceinline__ void pack_task(const PushCtx& c, const u32x4 (&v)[4], InTask& o, int lane, bool plain) {
  if (plain) {
#pragma unroll
    for (int q = 0; q < 4; ++q) o.d[q] = v[q];
    o.mask = ~0ull;
    o.raw = true;
  } else {
    enc_in(v, o, c.tag, c.scr, lane);
  }
}

__device__ __forceinline__ void send_plain(const PushCtx& c, int t, const u32x4 (&d)[4], int lane) {
  const int u = t - T_W0;
  const bool wr = task_is_w(t);
  const RowSet rs = row_set(task_set(t));
  const unsigned plain = c.par + lane * 16 +
      (wr ? (unsigned)XS_BYTES + (unsigned)((u >> 4) * (512 << 10) + (c.rank * 8 + ((u >> 1) & 7)) * 8192 + (u & 1) * 1024)
          : (unsigned)(rs.base * (128 << 10) + c.rank * rs.r * 16384 + (t - rs.task0) * 4096));
#pragma unroll
  for (int q = 0; q < 4; ++q) st16_sys(c.dst, plain + q * (wr ? 2048 : 1024), d[q]);
}

__device__ __forceinline__ int task_at(const PushCtx& c, int pos) { return min(pos < c.skip0 ? pos : pos + c.skipn, NTASK); }

struct PushState {
  int tp, tc, tn, tl;
  bool gone;
  u32x4 line;
  InTask o;
  u32x4 nx[4];
};

template <bool LDW>
__device__ __forceinline__ void push_step(const PushCtx& c, PushState& s, int lane) {
  unsigned w[16], tk;
  u32x4 x[4];
  const bool draw = s.tl < NTASK;
  asm volatile(SGPR_PAD "s_mov_b64 exec, 1\n\tglobal_atomic_add %0, %1, %2, %3 sc0\n\ts_mov_b64 exec, -1"
               : "=&v"(tk) : "v"(0u), "v"(draw ? 1u : 0u), "s"(c.lk) : "memory");
  if (s.tc == OPEN_AT) st1_agent(c.lk + 32, c.call);
  if constexpr (LDW) {
    const int u = s.tl - T_W0;
    ld_w(c.w, c.wl + (unsigned)((((u >> 1) & 7) * 64 + (u & 1) * 16) * GN + block_col(c.d, u >> 4)), w);
  } else {
    ld_x(c, s.tl, x);
  }
  const unsigned is0 = c.par + (unsigned)(XS_BYTES + WT_BYTES) + (unsigned)(c.rank * NTASK) * ITASK + lane * 16;
  if (s.tp >= 0) st16_sys_m(c.dst, is0 + (unsigned)s.tp * ITASK, s.line, 15ull);
  if (s.tc < NTASK) {
    if (!s.o.raw) {
      const unsigned slot = is0 + (unsigned)s.tc * ITASK;
      asm volatile(SGPR_PAD "global_store_dwordx4 %0, %1, %4 offset:256 sc0 sc1\n\tglobal_store_dwordx4 %0, %2, %4 offset:1280 sc0 sc1\n\t"
                   "global_store_dwordx4 %0, %3, %4 offset:2304 sc0 sc1\n\ts_nop 1"
                   :: "v"(slot), "v"(s.o.d[0]), "v"(s.o.d[1]), "v"(s.o.d[2]), "s"(c.dst) : "memory");
      if (s.o.mask != 0) st16_sys_m(c.dst, slot, s.o.d[3], s.o.mask);
      s.line = s.o.d[3];
    } else {
      if (!s.gone) send_plain(c, s.tc, s.o.d, lane);
      s.line = u32x4{c.tag | IHDR_RAW << 16, 0u, 0u, 0u};
    }
  }
  pack_task(c, s.nx, s.o, lane, c.local || s.tn >= NTASK);
  if constexpr (LDW) {
    asm volatile("s_waitcnt vmcnt(0)" : "+v"(tk), "+v"(w[0]), "+v"(w[1]), "+v"(w[2]), "+v"(w[3]), "+v"(w[4]), "+v"(w[5]), "+v"(w[6]), "+v"(w[7]),
                 "+v"(w[8]), "+v"(w[9]), "+v"(w[10]), "+v"(w[11]), "+v"(w[12]), "+v"(w[13]), "+v"(w[14]), "+v"(w[15]) :: "memory");
    transpose_w(w, s.nx);
  } else {
    asm volatile("s_waitcnt vmcnt(0)" : "+v"(tk), "+v"(x[0]), "+v"(x[1]), "+v"(x[2]), "+v"(x[3]) :: "memory");
#pragma unroll
    for (int q = 0; q < 4; ++q) s.nx[q] = x[q];
  }
  s.tp = s.tc; s.tc = s.tn; s.tn = s.tl; s.gone = false;
  s.tl = draw ? task_at(c, min(__builtin_amdgcn_readfirstlane(tk) - c.tick0, (unsigned)NTASK)) : NTASK;
}

#define GLDS_BITS " sc0 sc1"
__device__ __forceinline__ void glds2(const uint8_t* gbase, unsigned voff, unsigned lds) {
  unsigned keep;
  asm volatile("s_mov_b32 %0, m0\n\ts_mov_b32 m0, %3\n\ts_nop 2\n\t"
               "global_load_lds_dwordx4 %1, %2" GLDS_BITS "\n\t"
               "global_load_lds_dwordx4 %1, %2 offset:1024" GLDS_BITS "\n\t"
               "s_mov_b32 m0, %0"
               : "=&s"(keep) : "v"(voff), "s"(gbase), "s"(lds));
}
__device__ __forceinline__ void glds2o(const uint8_t* gbase, unsigned voff, unsigned lds) {
  unsigned keep;
  asm volatile("s_mov_b32 %0, m0\n\ts_mov_b32 m0, %3\n\ts_nop 2\n\t"
               "global_load_lds_dwordx4 %1, %2 offset:2048" GLDS_BITS "\n\t"
               "global_load_lds_dwordx4 %1, %2 offset:3072" GLDS_BITS "\n\t"
               "s_mov_b32 m0, %0"
               : "=&s"(keep) : "v"(voff), "s"(gbase), "s"(lds));
}
__device__ __forceinline__ void glds_half(const uint8_t* gbase, unsigned voff, unsigned lds) {
  unsigned keep;
  asm volatile("s_mov_b32 %0, m0\n\ts_mov_b32 m0, %3\n\ts_mov_b32 exec_hi, 0\n\ts_nop 1\n\t"
               "global_load_lds_dwordx4 %1, %2" GLDS_BITS "\n\t"
               "s_mov_b32 exec_hi, -1\n\ts_mov_b32 m0, %0"
               : "=&s"(keep) : "v"(voff), "s"(gbase), "s"(lds));
}

struct Frags { i32x8 a[2], b[4], a9, b9; };

__device__ __forceinline__ i32x8 ld_frag(const uint8_t* q) {
  const i32x4 lo = *(const i32x4*)q, hi = *(const i32x4*)(q + 1024);
  return __builtin_shufflevector(lo, hi, 0, 1, 2, 3, 4, 5, 6, 7);
}

#define MFMA(mi, j) acc[mi][j] = __builtin_amdgcn_mfma_scale_f32_32x32x64_f8f6f4(cur.a[mi], cur.b[j], acc[mi][j], 1, 1, 0, 0, 0, 0)
#define SB() __builtin_amdgcn_sched_barrier(0)

struct Ring {
  const uint8_t* ga;
  const uint8_t* gb;
  unsigned astep, ahop;
  int togo;
  unsigned va, vb, v9;
  unsigned wr, wr_end;
  unsigned wb, w9;
  unsigned rd;
  const uint8_t* qa;
  const uint8_t* qb;
  const uint8_t* q9;
  const uint8_t* la;
  const uint8_t* lb;
  const uint8_t* l9;
};

template <int MODE, int LEFT, bool DMA, bool RD>
__device__ __forceinline__ void kstep(f32x16 (&acc)[2][4], f32x16& acc9, const Frags& cur, Frags& nxt, Ring& g) {
  constexpr bool NINE = MODE == 9, TWO = MODE != 7;
  constexpr int STAGE = NINE ? 26624 : 24576;
  MFMA(0, 0); SB();
  asm volatile("s_waitcnt vmcnt(%0) lgkmcnt(0)\n\ts_barrier" :: "n"(LEFT * (NINE ? 7 : 6)) : "memory");
  SB();
  MFMA(0, 1); SB();
  if constexpr (RD) { nxt.a[0] = ld_frag(g.qa); nxt.b[0] = ld_frag(g.qb); }
  unsigned sa = 0, sb = 0;
  if constexpr (DMA) { const bool hop = --g.togo == 0; sa = hop ? g.ahop : g.astep; sb = hop ? 9 * 8192 : 8192; asm volatile("" : "+s"(sa), "+s"(sb)); }
  SB();
  MFMA(0, 2); SB();
  if constexpr (DMA) glds2(g.ga, g.va, g.wr);
  SB();
  MFMA(0, 3); SB();
  if constexpr (RD) { nxt.b[1] = ld_frag(g.qb + 2048); nxt.b[2] = ld_frag(g.qb + 4096); }
  SB();
  if constexpr (TWO) MFMA(1, 0);
  SB();
  if constexpr (DMA) glds2(g.gb, g.vb, g.wr + g.wb);
  SB();
  if constexpr (TWO) MFMA(1, 1);
  SB();
  if constexpr (RD) { nxt.b[3] = ld_frag(g.qb + 6144); if constexpr (TWO) nxt.a[1] = ld_frag(g.qa + 2048); }
  g.rd = g.rd == (NSTAGE - 1) * STAGE ? 0 : g.rd + STAGE;
  g.qa = g.la + g.rd;
  if constexpr (!NINE) g.qb = g.lb + g.rd;
  SB();
  if constexpr (TWO) MFMA(1, 2);
  SB();
  if constexpr (DMA) glds2o(g.ga, g.va, g.wr);
  SB();
  if constexpr (NINE) {
    if constexpr (TWO) MFMA(1, 3);
    SB();
    if constexpr (RD) { nxt.a9 = ld_frag(g.qb - 2048); nxt.b9 = ld_frag(g.q9); }
    g.qb = g.lb + g.rd; g.q9 = g.l9 + g.rd;
    if constexpr (DMA) g.gb += sb;
    SB();
    acc9 = __builtin_amdgcn_mfma_scale_f32_32x32x64_f8f6f4(cur.a9, cur.b9, acc9, 1, 1, 0, 0, 0, 0); SB();
    if constexpr (DMA) glds_half(g.ga, g.v9, g.wr + g.w9);
  } else {
    if constexpr (TWO) MFMA(1, 3);
    SB();
    if constexpr (DMA) g.gb += sb;
  }
  if constexpr (DMA) {
    g.ga += sa;
    g.wr += STAGE; g.wr = g.wr == g.wr_end ? g.wr - NSTAGE * STAGE : g.wr;
  }
  SB();
}

template <int MODE>
__device__ __forceinline__ void gemm_tile(const uint8_t* ga, unsigned astep, const uint8_t* gb, int skip, uint8_t* lds, int wave, int lane) {
  constexpr bool NINE = MODE == 9, TWO = MODE != 7;
  constexpr int STAGE = NINE ? 26624 : 24576, BOFF = STAGE - 8192;
  f32x16 acc[2][4], acc9;
  Ring g;
  g.ga = ga + (skip == 0 ? 8 * astep : 0); g.gb = gb + (skip == 0 ? 8 * 8192 : 0); g.astep = astep; g.ahop = 9 * astep;
  g.togo = skip > 0 ? 8 * skip - NSTAGE : -1;
  g.va = lane * 16 + wave * 4096; g.vb = lane * 16 + wave * 2048; g.v9 = 16384 + lane * 16 + wave * 512;
  g.wr = (unsigned)(uintptr_t)lds + wave * 4096; g.wr_end = g.wr + NSTAGE * STAGE;
  g.wb = BOFF - wave * 2048; g.w9 = 16384 - wave * 3584;
#pragma unroll
  for (int s = 0; s < NSTAGE; ++s) {
    glds2(g.ga, g.va, g.wr + s * STAGE);
    glds2(g.gb, g.vb, g.wr + s * STAGE + g.wb);
    glds2o(g.ga, g.va, g.wr + s * STAGE);
    if constexpr (NINE) glds_half(g.ga, g.v9, g.wr + s * STAGE + g.w9);
    g.ga += astep; g.gb += 8192;
  }
#pragma unroll
  for (int r = 0; r < 16; ++r) {
    acc9[r] = 0.f;
#pragma unroll
    for (int j = 0; j < 4; ++j) { acc[0][j][r] = 0.f; acc[1][j][r] = 0.f; }
  }
  g.la = lds + lane * 16 + wave * 4096; g.lb = lds + BOFF + lane * 16; g.l9 = g.lb + wave * 2048;
  asm volatile("s_waitcnt vmcnt(%0)\n\ts_barrier" :: "n"(5 * (NINE ? 7 : 6)) : "memory");
  Frags f0, f1;
  f0.a[0] = ld_frag(g.la); f0.b[0] = ld_frag(g.lb); f0.b[1] = ld_frag(g.lb + 2048);
  f0.b[2] = ld_frag(g.lb + 4096); f0.b[3] = ld_frag(g.lb + 6144);
  if constexpr (TWO) f0.a[1] = ld_frag(g.la + 2048);
  if constexpr (NINE) { f0.a9 = ld_frag(g.lb - 2048); f0.b9 = ld_frag(g.l9); }
  g.rd = STAGE; g.qa = g.la + STAGE; g.qb = g.lb + STAGE; g.q9 = g.l9 + STAGE;
#pragma unroll 1
  for (int s = skip < 0 ? 0 : 8; s < KSTEPS - NSTAGE; s += 2) {
    kstep<MODE, 4, true, true>(acc, acc9, f0, f1, g);
    kstep<MODE, 4, true, true>(acc, acc9, f1, f0, g);
  }
  kstep<MODE, 4, false, true>(acc, acc9, f0, f1, g);
  kstep<MODE, 3, false, true>(acc, acc9, f1, f0, g);
  kstep<MODE, 2, false, true>(acc, acc9, f0, f1, g);
  kstep<MODE, 1, false, true>(acc, acc9, f1, f0, g);
  kstep<MODE, 0, false, true>(acc, acc9, f0, f1, g);
  auto park = [&](const f32x16& a, int i) {
#pragma unroll
    for (int q = 0; q < 4; ++q)
      *(f32x4*)(lds + wave * 32768 + (i * 4 + q) * 1024 + lane * 16) = f32x4{a[4 * q], a[4 * q + 1], a[4 * q + 2], a[4 * q + 3]};
  };
  {
    const Frags& cur = f1;
    MFMA(0, 0); SB();
    asm volatile("s_waitcnt lgkmcnt(0)\n\ts_barrier" ::: "memory");
    SB();
    MFMA(0, 1); SB();
    MFMA(0, 2); SB();
    park(acc[0][0], 0); SB();
    MFMA(0, 3); SB();
    park(acc[0][1], 1); SB();
    if constexpr (TWO) MFMA(1, 0);
    SB();
    park(acc[0][2], 2); SB();
    if constexpr (TWO) MFMA(1, 1);
    SB();
    park(acc[0][3], 3); SB();
    if constexpr (TWO) {
      MFMA(1, 2); SB();
      park(acc[1][0], 4); SB();
      MFMA(1, 3); SB();
      park(acc[1][1], 5); SB();
      if constexpr (NINE) { acc9 = __builtin_amdgcn_mfma_scale_f32_32x32x64_f8f6f4(cur.a9, cur.b9, acc9, 1, 1, 0, 0, 0, 0); SB(); }
      park(acc[1][2], 6);
      park(acc[1][3], 7);
    }
  }
  if constexpr (NINE) {
#pragma unroll
    for (int q = 0; q < 4; ++q)
      *(f32x4*)(lds + 8 * 16384 + wave * 4096 + q * 1024 + lane * 16) = f32x4{acc9[4 * q], acc9[4 * q + 1], acc9[4 * q + 2], acc9[4 * q + 3]};
    __syncthreads();
  }
  __builtin_amdgcn_fence(__ATOMIC_ACQ_REL, "wavefront");
}

constexpr unsigned REC_CAP = 510, HDR_B = 0x40000000u;

__device__ __forceinline__ void put_records4(const unsigned (&q)[4], unsigned tag, unsigned& cnt, uint8_t* scr, unsigned dump) {
  unsigned m = 0;
#pragma unroll
  for (int k = 0; k < 4; ++k)
    m |= ((((q[k] & 0x77777777u) + 0x77777777u) | q[k]) & 0x88888888u) >> (3 - k);
  for (unsigned long long act = __ballot(m != 0); act != 0 && cnt <= REC_CAP; act = __ballot(m != 0)) {
    const unsigned bit = (unsigned)__builtin_ctz(m | 0x80000000u), sh = bit & 28u;
    const unsigned m1 = (unsigned)__builtin_amdgcn_sbfe((int)bit, 0, 1), m2 = (unsigned)__builtin_amdgcn_sbfe((int)bit, 1, 1);
    const unsigned w = bfi(m2, bfi(m1, q[3], q[2]), bfi(m1, q[1], q[0]));
    const unsigned idx = cnt + __builtin_amdgcn_mbcnt_hi((unsigned)(act >> 32), __builtin_amdgcn_mbcnt_lo((unsigned)act, 0u));
    const unsigned at = (m != 0 && idx < REC_CAP) ? 4 + 2 * idx : dump;
    *(unsigned short*)(scr + at) = (unsigned short)(tag | (bit & 3u) << 7 | sh << 2 | ((w >> sh) & 15u));
    m &= m - 1;
    cnt += __popcll(act);
  }
}

template <bool MODEB>
__device__ __forceinline__ bool enc_body(const float (&v)[4][16], float up, unsigned& cnt, uint8_t* blk, int lane) {
  unsigned bad = 0, low = 0;
#pragma unroll
  for (int h = 0; h < 2; ++h) {
    unsigned G[32];
#pragma unroll
    for (int j = 0; j < 4; ++j)
#pragma unroll
      for (int r = 0; r < 8; ++r) {
        const float x = v[j][8 * h + r] * up;
        unsigned g = (unsigned)(int)x;
        bad = __builtin_amdgcn_bitop3_b32(__builtin_bit_cast(unsigned, (float)(int)g), __builtin_bit_cast(unsigned, x), bad, 0xbe);
        low |= g;
        if (MODEB) g = g << 1 | (((g ^ g << 1) >> 27) & 16u);
        G[4 * r + j] = g;
      }
    unsigned q[4];
#pragma unroll
    for (int k = 0; k < 4; ++k) {
      const unsigned* g = G + 8 * k;
      const unsigned b0 = __builtin_amdgcn_perm(__builtin_amdgcn_perm(g[3], g[2], 0x0c0c0400u), __builtin_amdgcn_perm(g[1], g[0], 0x0c0c0400u), 0x05040100u);
      const unsigned b1 = __builtin_amdgcn_perm(__builtin_amdgcn_perm(g[7], g[6], 0x0c0c0400u), __builtin_amdgcn_perm(g[5], g[4], 0x0c0c0400u), 0x05040100u);
      q[k] = (b0 >> 4) | b1;
    }
#pragma unroll
    for (int j = 0; j < 2; ++j) {
      unsigned d[12];
#pragma unroll
      for (int g = 0; g < 4; ++g) {
        const unsigned* x = G + 16 * j + 4 * g;
        d[3 * g] = __builtin_amdgcn_perm(x[1], x[0], 0x05030201u);
        d[3 * g + 1] = __builtin_amdgcn_perm(x[2], x[1], 0x06050302u);
        d[3 * g + 2] = __builtin_amdgcn_perm(x[3], x[2], 0x07060503u);
      }
#pragma unroll
      for (int i = 0; i < 3; ++i)
        *(u32x4*)(blk + (1 + 6 * h + 3 * j + i) * 1024 + lane * 16) = u32x4{d[4 * i], d[4 * i + 1], d[4 * i + 2], d[4 * i + 3]};
    }
    put_records4(q, (unsigned)(lane << 10 | h << 9), cnt, blk, 13 * 1024 + 2 * lane);
  }
  return __ballot((bad | (low & 15u)) != 0) == 0 && cnt <= REC_CAP;
}

__device__ __forceinline__ unsigned enc_out(const float (&v)[4][16], uint8_t* blk, int lane) {
  float m = 0.f;
#pragma unroll
  for (int j = 0; j < 4; ++j)
#pragma unroll
    for (int r = 0; r < 16; r += 2) asm("v_max3_f32 %0, |%1|, |%2|, %0" : "+v"(m) : "v"(v[j][r]), "v"(v[j][r + 1]));
  const int e = max((int)(wave_max_u32(__builtin_bit_cast(unsigned, m)) >> 23) - 157, -126);
  const float up = __builtin_bit_cast(float, (unsigned)(127 - min(e, 0)) << 23);
  int ntop = 0, nlow = 0;
#pragma unroll
  for (int r = 0; r < 8; ++r) {
    const unsigned g = (unsigned)(int)(v[r & 3][2 * r] * up);
    ntop += __popcll(__ballot((int)(g ^ (g << 1)) < 0));
    nlow += __popcll(__ballot((g & 128u) != 0));
  }
  unsigned cnt = 0;
  const bool modeb = ntop < nlow;
  const bool fits = e <= 0 && (modeb ? enc_body<true>(v, up, cnt, blk, lane) : enc_body<false>(v, up, cnt, blk, lane));
  if (!fits) {
#pragma unroll
    for (int i = 0; i < 16; ++i) {
      const float* t = &v[i >> 2][(i & 3) * 4];
      *(f32x4*)(blk + i * 1024 + lane * 16) = f32x4{t[0], t[1], t[2], t[3]};
    }
    return 0;
  }
  *(unsigned*)(blk + (lane == 0 ? 0 : 13 * 1024 + 4 * lane)) = cnt | (unsigned)(e + 127) << 16 | (modeb ? HDR_B : 0u);
  return ((4 + 2 * cnt + 63) >> 6) * 4;
}

__device__ __forceinline__ void st16_out(void* base, unsigned off, f32x4 v) { st16_loc(base, off, __builtin_bit_cast(u32x4, v)); }

__device__ __forceinline__ constexpr unsigned row_off(int r) { return (unsigned)((r & 3) + 8 * (r >> 2)) * (GN * 4); }

__device__ __forceinline__ void send_out(const uint8_t* blk, unsigned nl, bool dead, uint8_t* slot, int lane, unsigned* next,
                                         unsigned call, bool last, int depth, unsigned* flag, unsigned half) {
  const bool raw = nl == 0 || dead;
  const u32x4 h = *(const u32x4*)(blk + lane * 16);
  st16_sys_m(slot, lane * 16, raw ? u32x4{HDR_RAW, 0u, 0u, 0u} : h, raw ? 15ull : nl >= 64 ? ~0ull : (1ull << nl) - 1);
  const int n = raw ? 16 : 12;
  const uint8_t* const src = blk + (raw ? 0 : 1024) + lane * 16;
  const unsigned nan = dead ? 0x7fc00000u : 0u;
#pragma unroll 1
  for (int i = 0; i < n; ++i) {
    const u32x4 v = *(const u32x4*)(src + i * 1024) | nan;
    if (depth == OUT_D) wait_vm<OUT_D - 1>(); else if (depth == OUT_D3) wait_vm<OUT_D3 - 1>(); else wait_vm<DEEP_D - 1>();
    if (last && i == n - OUT_REL) st1_agent(next, call);
    if (half != 0 && i == 6) { wait_vm<0>(); st1_sys(flag, half); }
    st16_sys(slot, (unsigned)(1024 + i * 1024 + lane * 16), v);
  }
}

template <int H>
__device__ __forceinline__ void dec_rows(const u32x4 (&y)[6], const u32x4& qq, bool modeb, float down, const f32x4 (&h)[16], float* ob,
                                         unsigned off, float sc) {
#pragma unroll
  for (int g = 0; g < 8; ++g) {
    const unsigned d0 = y[(3 * g) >> 2][(3 * g) & 3], d1 = y[(3 * g + 1) >> 2][(3 * g + 1) & 3], d2 = y[(3 * g + 2) >> 2][(3 * g + 2) & 3];
    const unsigned q = ((g & 1) ? qq[g >> 1] : qq[g >> 1] << 4) & 0xf0f0f0f0u;
    unsigned G[4] = {__builtin_amdgcn_perm(d0, q, 0x06050400u), __builtin_amdgcn_perm(d1, d0, 0x0504030cu) | ((q >> 8) & 255u),
                     __builtin_amdgcn_perm(d2, d1, 0x0403020cu) | ((q >> 16) & 255u), __builtin_amdgcn_perm(d2, q, 0x07060503u)};
    f32x4 v;
#pragma unroll
    for (int j = 0; j < 4; ++j) {
      if (modeb) G[j] = ((unsigned)((int)G[j] >> 1) & ~8u) ^ (G[j] << 27);
      v[j] = (__fmul_rn((float)(int)G[j], down) + h[4 * j + 2 * H + (g >> 2)][g & 3]) * sc;
    }
    st16_out(ob, off + row_off(8 * H + g), v);
  }
}

__device__ __forceinline__ void ld6_sys(const uint8_t* base, unsigned voff, u32x4 (&y)[6]) {
  asm volatile(SGPR_PAD "global_load_dwordx4 %0, %6, %7 offset:-3072 sc0 sc1\n\tglobal_load_dwordx4 %1, %6, %7 offset:-2048 sc0 sc1\n\t"
               "global_load_dwordx4 %2, %6, %7 offset:-1024 sc0 sc1\n\tglobal_load_dwordx4 %3, %6, %7 sc0 sc1\n\t"
               "global_load_dwordx4 %4, %6, %7 offset:1024 sc0 sc1\n\tglobal_load_dwordx4 %5, %6, %7 offset:2048 sc0 sc1\n\ts_waitcnt vmcnt(0)"
               : "=&v"(y[0]), "=&v"(y[1]), "=&v"(y[2]), "=&v"(y[3]), "=&v"(y[4]), "=&v"(y[5]) : "v"(voff + 3072), "s"(base) : "memory");
}

template <class F>
__device__ __forceinline__ void dec_out(const uint8_t* slot, const f32x4 (&h)[16], float* ob, unsigned off, float sc, unsigned* tmo, unsigned call,
                                        uint8_t* scr, int lane, bool halves, F rest) {
  u32x4 rec, y[6], z[6];
  unsigned tmv;
  asm volatile(SGPR_PAD "global_load_dword %13, %16, %18 sc1\n\t"
               "global_load_dwordx4 %0, %14, %17 offset:-4096 sc0 sc1\n\tglobal_load_dwordx4 %1, %14, %17 offset:-3072 sc0 sc1\n\t"
               "global_load_dwordx4 %2, %14, %17 offset:-2048 sc0 sc1\n\tglobal_load_dwordx4 %3, %14, %17 offset:-1024 sc0 sc1\n\t"
               "global_load_dwordx4 %4, %14, %17 sc0 sc1\n\tglobal_load_dwordx4 %5, %14, %17 offset:1024 sc0 sc1\n\t"
               "global_load_dwordx4 %6, %14, %17 offset:2048 sc0 sc1\n\tglobal_load_dwordx4 %7, %14, %17 offset:3072 sc0 sc1\n\t"
               "global_load_dwordx4 %8, %15, %17 offset:-4096 sc0 sc1\n\tglobal_load_dwordx4 %9, %15, %17 offset:-3072 sc0 sc1\n\t"
               "global_load_dwordx4 %10, %15, %17 offset:-2048 sc0 sc1\n\tglobal_load_dwordx4 %11, %15, %17 offset:-1024 sc0 sc1\n\t"
               "global_load_dwordx4 %12, %15, %17 sc0 sc1\n\ts_waitcnt vmcnt(0)"
               : "=&v"(rec), "=&v"(y[0]), "=&v"(y[1]), "=&v"(y[2]), "=&v"(y[3]), "=&v"(y[4]), "=&v"(y[5]), "=&v"(z[0]),
                 "=&v"(z[1]), "=&v"(z[2]), "=&v"(z[3]), "=&v"(z[4]), "=&v"(z[5]), "=&v"(tmv)
               : "v"(4096 + lane * 16), "v"(12288 + lane * 16), "v"(0u), "s"(slot), "s"(tmo) : "memory");
  const unsigned hdr = __builtin_amdgcn_readfirstlane(rec[0]);
  bool ok = __builtin_amdgcn_readfirstlane(tmv) != call;
  if (ok && !(hdr & HDR_RAW)) {
    const unsigned cnt = hdr & 0xffffu;
    const float down = __builtin_bit_cast(float, ((hdr >> 16) & 255u) << 23);
    *(u32x4*)(scr + lane * 32) = u32x4{0u, 0u, 0u, 0u};
    *(u32x4*)(scr + lane * 32 + 16) = u32x4{0u, 0u, 0u, 0u};
    __builtin_amdgcn_fence(__ATOMIC_ACQ_REL, "wavefront");
#pragma unroll
    for (int i = 0; i < 8; ++i) {
      const unsigned r16 = (rec[i >> 1] >> (16 * (i & 1))) & 0xffffu;
      const unsigned val = (unsigned)(lane * 8 + i - 2) < cnt ? (r16 & 15u) << ((r16 >> 2) & 28u) : 0u;
      __hip_atomic_fetch_or((unsigned*)(scr + (r16 >> 7) * 4), val, __ATOMIC_RELAXED, __HIP_MEMORY_SCOPE_WORKGROUP);
    }
    __builtin_amdgcn_fence(__ATOMIC_ACQ_REL, "wavefront");
    const u32x4 q0 = *(const u32x4*)(scr + lane * 32), q1 = *(const u32x4*)(scr + lane * 32 + 16);
    __builtin_amdgcn_fence(__ATOMIC_ACQ_REL, "wavefront");
    const bool modeb = (hdr & HDR_B) != 0;
    dec_rows<0>(y, q0, modeb, down, h, ob, off, sc);
    if (halves) {
      ok = rest();
      ld6_sys(slot, 7168 + lane * 16, z);
    }
    if (ok) dec_rows<1>(z, q1, modeb, down, h, ob, off, sc);
  } else if (ok) {
    if (halves) {
      ok = rest();
      ld6_sys(slot, 7168 + lane * 16, z);
    }
    u32x4 w[6];
    ld6_sys(slot, 11264 + lane * 16, w);
#pragma unroll
    for (int r = 0; r < 16; ++r) {
      f32x4 t;
#pragma unroll
      for (int j = 0; j < 4; ++j) {
        const int i = 4 * j + (r >> 2);
        t[j] = (__builtin_bit_cast(float, i < 6 ? y[i][r & 3] : i < 12 ? z[i - 6][r & 3] : w[i - 10][r & 3]) + h[i][r & 3]) * sc;
      }
      st16_out(ob, off + row_off(r), t);
    }
  }
  if (!ok) {
    const float n = __builtin_nanf("");
#pragma unroll 1
    for (int r = 0; r < 16; ++r) st16_out(ob, off + row_off(r), f32x4{n, n, n, n});
  }
}

__device__ __forceinline__ int wave_blocks(int r, int wave) { return wave < 3 ? 2 : r - 6; }
__device__ __forceinline__ unsigned out_off(int row0, int col, int wave, int lane) {
  return (unsigned)(((row0 + wave * 64 + 4 * (lane >> 5)) * GN + col + 4 * (lane & 31)) * 4);
}

template <bool NINE>
__device__ __forceinline__ void home_tile(const Params& p, int g, int src, int i, uint8_t* lds, int wave, int lane) {
  const int col = block_col(src, tile_block(g, i));
  const uint8_t* const xg = p.x + (size_t)(p.rank * 512 + (lane & 31)) * KS + (lane >> 5) * 32;
  auto xa = [&](int rho, int s) { return *(const i32x8*)(xg + rho * 32 * KS + s * 64); };
  i32x8 a0[8], a1[8], a9[8];
  {
    unsigned w[4][16];
    const unsigned o = (unsigned)((wave * 128 + (lane >> 5) * 32) * GN + col + (lane & 31) * 4);
#pragma unroll
    for (int t = 0; t < 4; ++t) ld_w(p.w, o + (unsigned)((t >> 1) * 64 + (t & 1) * 16) * GN, w[t]);
#pragma unroll
    for (int s = 0; s < 8; ++s) {
      a0[s] = xa((NINE ? 7 : 0) + wave * 2, s);
      a1[s] = xa((NINE ? 8 : 1) + wave * 2, s);
    }
    asm volatile("s_waitcnt vmcnt(0)" : "+v"(w[0][0]), "+v"(w[0][1]), "+v"(w[0][2]), "+v"(w[0][3]), "+v"(w[0][4]), "+v"(w[0][5]), "+v"(w[0][6]), "+v"(w[0][7]), "+v"(w[0][8]), "+v"(w[0][9]), "+v"(w[0][10]), "+v"(w[0][11]), "+v"(w[0][12]), "+v"(w[0][13]), "+v"(w[0][14]), "+v"(w[0][15]) :: "memory");
    asm volatile("" : "+v"(w[1][0]), "+v"(w[1][1]), "+v"(w[1][2]), "+v"(w[1][3]), "+v"(w[1][4]), "+v"(w[1][5]), "+v"(w[1][6]), "+v"(w[1][7]), "+v"(w[1][8]), "+v"(w[1][9]), "+v"(w[1][10]), "+v"(w[1][11]), "+v"(w[1][12]), "+v"(w[1][13]), "+v"(w[1][14]), "+v"(w[1][15]) :: "memory");
    asm volatile("" : "+v"(w[2][0]), "+v"(w[2][1]), "+v"(w[2][2]), "+v"(w[2][3]), "+v"(w[2][4]), "+v"(w[2][5]), "+v"(w[2][6]), "+v"(w[2][7]), "+v"(w[2][8]), "+v"(w[2][9]), "+v"(w[2][10]), "+v"(w[2][11]), "+v"(w[2][12]), "+v"(w[2][13]), "+v"(w[2][14]), "+v"(w[2][15]) :: "memory");
    asm volatile("" : "+v"(w[3][0]), "+v"(w[3][1]), "+v"(w[3][2]), "+v"(w[3][3]), "+v"(w[3][4]), "+v"(w[3][5]), "+v"(w[3][6]), "+v"(w[3][7]), "+v"(w[3][8]), "+v"(w[3][9]), "+v"(w[3][10]), "+v"(w[3][11]), "+v"(w[3][12]), "+v"(w[3][13]), "+v"(w[3][14]), "+v"(w[3][15]) :: "memory");
#pragma unroll
    for (int t = 0; t < 4; ++t) {
      u32x4 v[4];
      transpose_w(w[t], v);
#pragma unroll
      for (int j = 0; j < 4; ++j) *(u32x4*)(lds + (wave * 2 + (t >> 1)) * 8192 + j * 2048 + (t & 1) * 1024 + lane * 16) = v[j];
    }
  }
  if constexpr (NINE) {
#pragma unroll
    for (int s = 0; s < 8; ++s) a9[s] = xa(15, s);
  }
  __syncthreads();
  f32x16 h[2][4], h9;
#pragma unroll
  for (int r = 0; r < 16; ++r) {
    h9[r] = 0.f;
#pragma unroll
    for (int j = 0; j < 4; ++j) { h[0][j][r] = 0.f; h[1][j][r] = 0.f; }
  }
  i32x8 bn = ld_frag(lds + lane * 16);
#pragma unroll
  for (int t = 0; t < 32; ++t) {
    const i32x8 bc = bn;
    if (t < 31) bn = ld_frag(lds + (t + 1) * 2048 + lane * 16);
    i32x8 b9;
    if (NINE && (t & 3) == 0) b9 = ld_frag(lds + (t + wave) * 2048 + lane * 16);
    SB();
    h[0][t & 3] = __builtin_amdgcn_mfma_scale_f32_32x32x64_f8f6f4(a0[t >> 2], bc, h[0][t & 3], 1, 1, 0, 0, 0, 0);
    h[1][t & 3] = __builtin_amdgcn_mfma_scale_f32_32x32x64_f8f6f4(a1[t >> 2], bc, h[1][t & 3], 1, 1, 0, 0, 0, 0);
    if (NINE && (t & 3) == 0) h9 = __builtin_amdgcn_mfma_scale_f32_32x32x64_f8f6f4(a9[t >> 2], b9, h9, 1, 1, 0, 0, 0, 0);
    SB();
  }
  uint8_t* const pb = p.myws + WS_PB + (size_t)((g * 32 + i) * 9) * 16384;
#pragma unroll
  for (int q = 0; q < (NINE ? 36 : 32); ++q) {
    const f32x16& t = q < 32 ? h[q >> 4][(q >> 2) & 3] : h9;
    st16_out(pb, (unsigned)((q < 32 ? wave * 32 + q : 128 - 32 + wave * 4 + q) * 1024 + lane * 16),
             f32x4{t[4 * (q & 3)], t[4 * (q & 3) + 1], t[4 * (q & 3) + 2], t[4 * (q & 3) + 3]});
  }
  __syncthreads();
}

__device__ __forceinline__ void recv_tile(uint8_t* myws, float* out, int g, int src, int i, int k0, int k1, int wave, int lane,
                                          unsigned call, unsigned* tmo, float sc, uint8_t* scr) {
  const unsigned off = out_off(g < 4 ? 0 : 224, block_col(src, tile_block(g, i)), wave, lane);
  const uint8_t* const fl = myws + WS_DONE + (size_t)((src * 32 + i) * 4 + wave) * 128;
  const int nb = wave_blocks(g < 4 ? 7 : 9, wave);
  if (k0 >= min(nb, k1)) return;
  const uint8_t* const fh = myws + WS_HOME + (size_t)((g * 32 + i) * 4 + (lane & 3)) * 4;
  spin<RX_SLEEP>([&] { return __all(ld_agent(fh) >= call) != 0; }, tmo, call);
#pragma unroll 1
  for (int k = k0; k < min(nb, k1); ++k) {
    f32x4 h[16];
    asm volatile(SGPR_PAD "global_load_dwordx4 %0, %16, %18 offset:-4096 sc1\n\t""global_load_dwordx4 %1, %16, %18 offset:-3072 sc1\n\t"
                 "global_load_dwordx4 %2, %16, %18 offset:-2048 sc1\n\t""global_load_dwordx4 %3, %16, %18 offset:-1024 sc1\n\t"
                 "global_load_dwordx4 %4, %16, %18 offset:0 sc1\n\t""global_load_dwordx4 %5, %16, %18 offset:1024 sc1\n\t"
                 "global_load_dwordx4 %6, %16, %18 offset:2048 sc1\n\t""global_load_dwordx4 %7, %16, %18 offset:3072 sc1\n\t"
                 "global_load_dwordx4 %8, %17, %18 offset:-4096 sc1\n\t""global_load_dwordx4 %9, %17, %18 offset:-3072 sc1\n\t"
                 "global_load_dwordx4 %10, %17, %18 offset:-2048 sc1\n\t""global_load_dwordx4 %11, %17, %18 offset:-1024 sc1\n\t"
                 "global_load_dwordx4 %12, %17, %18 offset:0 sc1\n\t""global_load_dwordx4 %13, %17, %18 offset:1024 sc1\n\t"
                 "global_load_dwordx4 %14, %17, %18 offset:2048 sc1\n\t""global_load_dwordx4 %15, %17, %18 offset:3072 sc1"
                 : "=&v"(h[0]), "=&v"(h[1]), "=&v"(h[2]), "=&v"(h[3]), "=&v"(h[4]), "=&v"(h[5]), "=&v"(h[6]), "=&v"(h[7]),
                   "=&v"(h[8]), "=&v"(h[9]), "=&v"(h[10]), "=&v"(h[11]), "=&v"(h[12]), "=&v"(h[13]), "=&v"(h[14]), "=&v"(h[15])
                 : "v"(4096 + lane * 16), "v"(12288 + lane * 16), "s"(myws + WS_PB + (size_t)((g * 32 + i) * 9 + wave * 2 + k) * 16384) : "memory");
    const bool halves = i >= (g < 4 ? 24 : 16) && k == nb - 1;
    spin<RX_SLEEP>([&] { return __builtin_amdgcn_readfirstlane(ld_sys(fl)) >= 8 * call + 2 * k + 2 - halves; }, tmo, call);
    asm volatile("s_waitcnt vmcnt(0)" : "+v"(h[0]), "+v"(h[1]), "+v"(h[2]), "+v"(h[3]), "+v"(h[4]), "+v"(h[5]), "+v"(h[6]), "+v"(h[7]), "+v"(h[8]), "+v"(h[9]), "+v"(h[10]), "+v"(h[11]), "+v"(h[12]), "+v"(h[13]), "+v"(h[14]), "+v"(h[15]) :: "memory");
    dec_out(myws + WS_OS + (size_t)((src * 32 + i) * 9 + wave * 2 + k) * OBLK, h, out, off + (unsigned)k * 32 * (GN * 4), sc, tmo, call, scr, lane,
            halves, [&] { return spin<LAST_SLEEP>([&] { return __builtin_amdgcn_readfirstlane(ld_sys(fl)) >= 8 * call + 2 * k + 2; }, tmo, call); });
  }
}

}

__global__ void __launch_bounds__(256, 1) gemm_rs_fused(const Params p) {
  __shared__ __attribute__((aligned(16))) uint8_t lds[LDS_BYTES];
  const int b = blockIdx.x;
  const int lane = threadIdx.x & 63;
  const int wave = __builtin_amdgcn_readfirstlane(threadIdx.x >> 6);
  const int rank = p.rank;
  uint8_t* const myws = p.myws;
  asm volatile("" :: "s"(myws), "s"(rank), "s"(p.x), "s"(p.w));

  const int grp = b & 7, dst = grp ^ rank, ti = 31 - (b >> 3);
  const bool local = grp == 0 || (grp >= 4 && ti >= 24);
  const int set = local ? 7 + (grp != 0) : grp - 1, cb = tile_block(grp, ti);
  const RowSet rs = row_set(set);
  unsigned* const callp = (unsigned*)(myws + WS_CALL + (size_t)b * 128);
  unsigned* const tmo = (unsigned*)(myws + WS_TMO);
  unsigned* const lk = (unsigned*)(myws + WS_LINK) + grp * 64;
  uint8_t* const scr = lds + LDS_PK + wave * 2048;

  PushCtx c;
  c.x = p.x; c.d = dst; c.local = grp == 0;
  c.xl = (unsigned)((lane & 31) * KS + (lane >> 5) * 32);
  const RowSet hs = row_set(grp - 1);
  c.skip0 = grp == 0 ? NTASK : hs.task0; c.skipn = grp == 0 ? 0 : 4 * hs.r;

  unsigned call;
  if (ti >= 24) {
    const int w = ((b >> 3) * 4 + wave) & 31;
    c.w = p.w; c.dst = p.ws[dst]; c.lk = lk; c.rank = rank;
    c.wl = (unsigned)((lane >> 5) * 32 * GN + (lane & 31) * 4);
    c.scr = scr;
    PushState s;
    u32x4 x0[4];
    unsigned cv;
    s.tp = -1; s.tc = task_at(c, w); s.tn = task_at(c, w + 32); s.tl = task_at(c, w + 64); s.gone = true;
    asm volatile(SGPR_PAD "global_load_dword %0, %1, %2 sc1" : "=v"(cv) : "v"(0u), "s"(callp) : "memory");
    ld_x(c, s.tc, x0);
    __builtin_amdgcn_sched_barrier(0);
    ld_x(c, s.tn, s.nx);
    const RowSet fs = row_set(task_set(s.tc));
    unsigned first = (unsigned)((fs.base * 128 + rank * fs.r * 16 + (s.tc - fs.task0) * 4) << 10) + lane * 16;
    asm volatile("" : "+v"(first));
    asm volatile("s_waitcnt vmcnt(4)" : "+v"(cv), "+v"(x0[0]), "+v"(x0[1]), "+v"(x0[2]), "+v"(x0[3]) :: "memory");
    call = __builtin_amdgcn_readfirstlane(cv) + 1;
    c.par = (unsigned)(WS_DATA + (call & 1) * PAR_BYTES);
#pragma unroll
    for (int q = 0; q < 4; ++q) st16_sys(c.dst, c.par + first + q * 1024, x0[q]);
    asm volatile("s_waitcnt vmcnt(4)" : "+v"(s.nx[0]), "+v"(s.nx[1]), "+v"(s.nx[2]), "+v"(s.nx[3]) :: "memory");
    c.tick0 = (call - 1) * (NTASK - c.skipn - 64) - 96;
    c.tag = call & 0xffffu;
    c.call = call;
    s.line = u32x4{0u, 0u, 0u, 0u};
    pack_task(c, x0, s.o, lane, true);
#pragma unroll 1
    while (s.tp < NTASK) {
      if (task_is_w(s.tl)) push_step<true>(c, s, lane);
      else push_step<false>(c, s, lane);
    }
  } else {
    call = __builtin_amdgcn_readfirstlane(ld_agent(callp)) + 1;
  }
  uint8_t* const par = myws + WS_DATA + (size_t)(call & 1) * PAR_BYTES;

#pragma unroll 1
  for (int job = 0; job < 4; ++job) {
    if (job == 1 && grp != 0 && ti < 24) {
      __syncthreads();
      if (grp >= 4) {
        home_tile<true>(p, grp, dst, ti, lds, wave, lane);
      } else {
#pragma unroll 1
        for (int i = ti; i < 32; i += ti >= 16 ? 8 : 32) home_tile<false>(p, grp, dst, i, lds, wave, lane);
      }
    }
    const bool jw = grp == 0 ? job < 3 : job == 1;
    const int beta = grp == 0 ? (ti * 4 + wave) & 63 : (23 - ti) * 4 + wave;
    if (grp == 0 ? (job == 2 && ti >= 24) : (ti >= 24 || job >= 2 || (job == 0 && beta >= 8 * rs.r))) continue;
    const int n = job == 2 ? 24 + (ti & 7) : cb;
    const RowSet js = row_set(grp == 0 ? 7 + (ti >> 4) : set);
    const int bs = div_r(beta, js.r), bu = beta - bs * js.r;
    const int src = !jw ? bs : grp != 0 ? dst : job == 0 ? rank : job == 1 ? rank ^ (4 + (ti & 3)) : rank ^ (1 + (ti >> 3));
    const int t0 = jw ? T_W0 + n * 16 + wave * 4 : js.task0 + bu * 4;
    const uint8_t* const slot = par + XS_BYTES + WT_BYTES + (size_t)(src * NTASK + t0) * ITASK;
    const bool absent = !jw && grp != 0 && src == dst;
    if (!absent && !wait_tasks(slot, tmo, call, lane)) continue;
    if (src != rank && !absent) {
      const unsigned base = jw ? (unsigned)XS_BYTES + (unsigned)(n * (512 << 10) + (src * 8 + wave * 2) * 8192)
                               : (unsigned)(js.base * (128 << 10) + (src * js.r + bu) * 16384);
      unsigned off[4];
#pragma unroll
      for (int t = 0; t < 4; ++t) off[t] = base + (jw ? (t >> 1) * 8192 + (t & 1) * 1024 : t * 4096);
      dec_in4(slot, par, off, jw ? 2048 : 1024, lds + wave * 4096, lane);
      wait_vm<0>();
    }
    st1_agent(jw ? myws + WS_DW + (size_t)(n * 32 + src * 4 + wave) * 4
                 : myws + WS_DX + (size_t)((grp == 0 ? 7 + (ti >> 4) : set) * 128 + beta) * 4, call);
  }
  if (grp != 0 && ti < 24) {
    wait_vm<0>();
    st1_agent(myws + WS_HOME + (size_t)((grp * 32 + ti) * 4 + wave) * 4, call);
    if (grp < 4 && ti >= 16) st1_agent(myws + WS_HOME + (size_t)((grp * 32 + ti + 8) * 4 + wave) * 4, call);
  }

  unsigned* const ctrl = (unsigned*)(lds + LDS_CTRL);
  if (wave == 0) {
    const uint8_t* const fw = myws + WS_DW + (size_t)(cb * 32 + (lane & 31)) * 4;
    const uint8_t* const fx = myws + WS_DX + (size_t)(set * 128 + min(lane, 8 * rs.r - 1)) * 4;
    const uint8_t* const fy = myws + WS_DX + (size_t)(set * 128 + min(64 + lane, 8 * rs.r - 1)) * 4;
    const bool ok = spin([&] { const unsigned a = ld_agent(fw), c = ld_agent(fx), d = ld_agent(fy); return __all(min(min(a, c), d) >= call) != 0; },
                         tmo, call);
    if (lane == 0) ctrl[0] = !ok;
  }
  __syncthreads();
  const bool dead0 = __builtin_amdgcn_readfirstlane(ctrl[0]) != 0;

  if (wave == 0) st1_agent(callp, call);
  const float sc = p.sx[0] * p.sw[0];
  const unsigned off = out_off(rs.row0, block_col(rank, cb), wave, lane);
  float* const ob = p.out[rank];
  uint8_t* const blk = lds + wave * 32768;
  uint8_t* const oslot = p.ws[dst] + WS_OS + (size_t)((rank * 32 + ti) * 9 + wave * 2) * OBLK;
  unsigned* const flag = (unsigned*)(p.ws[dst] + WS_DONE + (size_t)((rank * 32 + ti) * 4 + wave) * 128);
  const int nb = wave_blocks(rs.r, wave);
  const unsigned* const turn = ti < NTURN ? lk + 32 : (const unsigned*)(myws + WS_TURN + (size_t)((grp * 32 + ti) * 4 + wave) * 128);
  unsigned* const next = (unsigned*)(myws + WS_TURN + (size_t)((grp * 32 + ((ti + NTURN) & 31)) * 4 + wave) * 128);
  const int depth = ti < DEEP_T ? DEEP_D : nb == 3 ? OUT_D3 : OUT_D;
  unsigned nl0 = 0, nl1 = 0, nl2 = 0;
  int sent = 0;

  if (!dead0) {
    const uint8_t* const ga = par + (size_t)rs.base * (128 << 10);
    const uint8_t* const gb = par + XS_BYTES + (size_t)cb * (512 << 10);
    if (rs.r == 9) gemm_tile<9>(ga, 9 * 2048, gb, dst, lds, wave, lane);
    else if (nb == 1) gemm_tile<7>(ga, 7 * 2048, gb, dst, lds, wave, lane);
    else gemm_tile<8>(ga, rs.r * 2048, gb, local ? -1 : dst, lds, wave, lane);
    unsigned tnv, tmv;
    asm volatile(SGPR_PAD "global_load_dword %0, %2, %3 sc1\n\tglobal_load_dword %1, %2, %4 sc1"
                 : "=&a"(tnv), "=&a"(tmv) : "v"(0u), "s"(turn), "s"(tmo) : "memory");
    auto landed = [&](unsigned& s_tn, unsigned& s_tm) {
      unsigned t0, t1;
      asm volatile("s_waitcnt vmcnt(0)\n\tv_accvgpr_read_b32 %2, %4\n\tv_accvgpr_read_b32 %3, %5\n\ts_nop 1\n\t"
                   "v_readfirstlane_b32 %0, %2\n\tv_readfirstlane_b32 %1, %3\n\ts_nop 1"
                   : "=&s"(s_tn), "=&s"(s_tm), "=&v"(t0), "=&v"(t1) : "a"(tnv), "a"(tmv) : "memory");
    };
    float s0 = sc;
#pragma unroll 1
    for (int k = 0; k < nb; ++k) {
      float v[4][16];
#pragma unroll
      for (int i = 0; i < 16; ++i) {
        const f32x4 t = *(const f32x4*)(blk + k * 16384 + i * 1024 + lane * 16);
#pragma unroll
        for (int q = 0; q < 4; ++q) v[i >> 2][(i & 3) * 4 + q] = t[q];
      }
      __builtin_amdgcn_fence(__ATOMIC_ACQ_REL, "wavefront");
      if (local) {
        if (k == 0) {
          unsigned tn, t;
          landed(tn, t);
          if (t == call) s0 = __builtin_nanf("");
        }
#pragma unroll
        for (int r = 0; r < 16; ++r)
          st16_out(ob, off + k * 32 * (GN * 4) + row_off(r), f32x4{v[0][r] * s0, v[1][r] * s0, v[2][r] * s0, v[3][r] * s0});
        continue;
      }
      const unsigned nl = enc_out(v, blk + k * 16384, lane);
      __builtin_amdgcn_fence(__ATOMIC_ACQ_REL, "wavefront");
      if (k == 0) nl0 = nl; else if (k == 1) nl1 = nl; else nl2 = nl;
      if (k == 0) {
        unsigned tn, t;
        landed(tn, t);
        if (ti < 24 && nb > 1 && t != call && (tn >= call || __builtin_amdgcn_readfirstlane(ld_agent(turn)) >= call)) {
          send_out(blk, nl, false, oslot, lane, next, call, false, depth, flag, 0);
          sent = 1;
        }
      }
    }
  } else if (local) {
    const float n = __builtin_nanf("");
#pragma unroll 1
    for (int t = 0; t < 32; ++t) st16_out(ob, off + (t >> 4) * 32 * (GN * 4) + row_off(t & 15), f32x4{n, n, n, n});
  }
  if (grp == 0) return;

  const int last8 = grp < 4 ? 24 : 16;
  if (ti >= 24) recv_tile(myws, p.out[rank], grp, dst, ti - 24, 0, 3, wave, lane, call, tmo, sc, scr);
  if (!local) {
    if (sent) {
      wait_vm<0>();
      st1_sys(flag, 8 * call + 2);
    } else {
      spin([&] { return __builtin_amdgcn_readfirstlane(ld_agent(turn)) >= call; }, tmo, call);
    }
    const bool dead = dead0 || __builtin_amdgcn_readfirstlane(ld_agent(tmo)) == call;
#pragma unroll 1
    for (int k = sent; k < nb; ++k) {
      send_out(blk + k * 16384, k == 0 ? nl0 : k == 1 ? nl1 : nl2, dead, oslot + k * OBLK, lane, next, call, k == nb - 1, depth, flag,
               ti >= last8 && k == nb - 1 ? 8 * call + 2 * k + 1 : 0);
      wait_vm<0>();
      st1_sys(flag, 8 * call + 2 * k + 2);
    }
  }
#pragma unroll 1
  for (int duty = 0; duty < 2; ++duty) {
    int i, k0, k1;
    if (duty == 0) {
      if (ti >= last8) continue;
      i = ti + 8; k0 = 0; k1 = i >= last8 ? 1 : 3;
    } else if (ti < 8) {
      i = last8 + ti; k0 = grp < 4 ? 1 : 2; k1 = 3;
    } else if (local) {
      i = ti - 8; k0 = 1; k1 = 2;
    } else {
      continue;
    }
    recv_tile(myws, p.out[rank], grp, dst, i, k0, k1, wave, lane, call, tmo, sc, scr);
  }
}

extern "C" void kernel_launch(int rank, int nranks, ncclComm_t comm, hipStream_t stream,
                              void* const* d_in, const int* in_sizes, int n_in,
                              void* d_out, int out_size, const int* in_dtypes, int out_dtype,
                              void* const* d_out_all, void* const* d_ws_all, size_t ws_bytes) {
  (void)comm; (void)in_sizes; (void)in_dtypes; (void)out_dtype; (void)d_out;
  if (nranks != NR || (unsigned)rank >= (unsigned)NR || n_in != 4 || ws_bytes < WS_NEED || out_size != 512 * GN) {
    fprintf(stderr, "kernel_launch: unexpected problem shape\n");
    return;
  }
  Params p;
  p.x = (const uint8_t*)d_in[0];
  p.w = (const uint8_t*)d_in[1];
  p.sx = (const float*)d_in[2];
  p.sw = (const float*)d_in[3];
  for (int i = 0; i < NR; ++i) {
    p.ws[i] = (uint8_t*)d_ws_all[i];
    p.out[i] = (float*)d_out_all[i];
  }
  p.myws = p.ws[rank];
  p.rank = rank;
  p.pad = 0;
  hipLaunchKernelGGL(gemm_rs_fused, dim3(NWG), dim3(256), 0, stream, p);
  HIP_CHECK(hipGetLastError());
}
```

```cpp
#include <hip/hip_runtime.h>
#include <rccl/rccl.h>
#include <cstdio>
#include <cstdint>

#define HIP_CHECK(expr) do { hipError_t _e = (expr); if (_e != hipSuccess) { \
    fprintf(stderr, "HIP error %s at %s:%d\n", hipGetErrorString(_e), __FILE__, __LINE__); \
    return; } } while (0)

typedef unsigned u32x4 __attribute__((ext_vector_type(4)));
typedef int i32x4 __attribute__((ext_vector_type(4)));
typedef int i32x8 __attribute__((ext_vector_type(8)));
typedef float f32x4 __attribute__((ext_vector_type(4)));
typedef float f32x16 __attribute__((ext_vector_type(16)));

#define SGPR_PAD "s_nop 4\n\t"

namespace {

constexpr int NR = 8;
constexpr int GN = 16384;
constexpr int KS = 512;
constexpr int NWG = 256;
constexpr int KSTEPS = 64;
constexpr int NSTAGE = 6;
constexpr int LDS_CTRL = NSTAGE * 26624;
constexpr int LDS_BYTES = LDS_CTRL + 64;
constexpr int LDS_PK = 9 * 16384;

constexpr int T_W0 = 228, T_OWN = T_W0 + 512, NTASK = T_OWN + 64;

constexpr size_t WS_CALL = 0;
constexpr size_t WS_TMO = 32768;
constexpr size_t WS_DONE = 896u << 10;
constexpr size_t WS_LINK = WS_TMO + 1024;
constexpr size_t WS_DX = WS_LINK + 4096;
constexpr size_t WS_DW = WS_DX + 8192;
constexpr size_t WS_HOME = WS_DW + 4096;
constexpr size_t WS_TURN = 768u << 10;
constexpr size_t WS_DATA = 1u << 20;
constexpr size_t XS_BYTES = 10u << 20;
constexpr size_t WT_BYTES = 16u << 20;
constexpr unsigned ITASK = 3328;
constexpr size_t IS_BYTES = 21u << 20;
constexpr size_t PAR_BYTES = XS_BYTES + WT_BYTES + IS_BYTES;
static_assert((size_t)8 * NTASK * ITASK <= IS_BYTES, "");
constexpr size_t OBLK = 17 * 1024;
constexpr size_t WS_OS = WS_DATA + 2 * PAR_BYTES;
constexpr size_t WS_PB = WS_OS + 8 * 32 * 9 * OBLK;
constexpr size_t WS_NEED = WS_PB + 8 * 32 * 9 * 16384;

constexpr unsigned long long DEADLINE_TICKS = 50000000ull;

constexpr int OPEN_AT = 790, NTURN = 24, OUT_REL = 3, OUT_D = 2, OUT_D3 = 3, DEEP_D = 4, DEEP_T = 8;
constexpr int POLL_SLEEP = 6, RX_SLEEP = 16, LAST_SLEEP = 8;

struct Params {
  const uint8_t* x;
  const uint8_t* w;
  const float* sx;
  const float* sw;
  uint8_t* ws[NR];
  float* out[NR];
  uint8_t* myws;
  int rank;
  int pad;
};

__device__ __forceinline__ unsigned ld_sys(const void* ptr) {
  return __hip_atomic_load((const unsigned*)ptr, __ATOMIC_RELAXED, __HIP_MEMORY_SCOPE_SYSTEM);
}
__device__ __forceinline__ unsigned ld_agent(const void* ptr) {
  return __hip_atomic_load((const unsigned*)ptr, __ATOMIC_RELAXED, __HIP_MEMORY_SCOPE_AGENT);
}
__device__ __forceinline__ void st_agent(void* ptr, unsigned v) {
  __hip_atomic_store((unsigned*)ptr, v, __ATOMIC_RELAXED, __HIP_MEMORY_SCOPE_AGENT);
}

template <int SLEEP = POLL_SLEEP, class F>
__device__ __forceinline__ bool spin(F ready, unsigned* tmo, unsigned call) {
  const unsigned long long t0 = __builtin_amdgcn_s_memrealtime();
  for (unsigned n = 1;; ++n) {
    if (ready()) return true;
    if ((n & 3) == 0 && (__builtin_amdgcn_readfirstlane(ld_agent(tmo)) == call || __builtin_amdgcn_s_memrealtime() - t0 > DEADLINE_TICKS)) {
      st_agent(tmo, call);
      return false;
    }
    __builtin_amdgcn_s_sleep(SLEEP);
  }
}

__device__ __forceinline__ void st16_sys(void* base, unsigned off, u32x4 v) {
  asm volatile(SGPR_PAD "global_store_dwordx4 %0, %1, %2 sc0 sc1\n\ts_nop 1" :: "v"(off), "v"(v), "s"(base) : "memory");
}

__device__ __forceinline__ void st16_loc(void* base, unsigned off, u32x4 v) {
  asm volatile(SGPR_PAD "global_store_dwordx4 %0, %1, %2 sc1\n\ts_nop 1" :: "v"(off), "v"(v), "s"(base) : "memory");
}

__device__ __forceinline__ void st1_agent(void* base, unsigned v) {
  asm volatile(SGPR_PAD "s_mov_b64 exec, 1\n\tglobal_store_dword %0, %1, %2 sc1\n\ts_mov_b64 exec, -1"
               :: "v"(0u), "v"(v), "s"(base) : "memory");
}

__device__ __forceinline__ void st1_sys(void* base, unsigned v) {
  asm volatile(SGPR_PAD "s_mov_b64 exec, 1\n\tglobal_store_dword %0, %1, %2 sc0 sc1\n\ts_mov_b64 exec, -1"
               :: "v"(0u), "v"(v), "s"(base) : "memory");
}
__device__ __forceinline__ void st16_sys_m(void* base, unsigned off, u32x4 v, unsigned long long mask) {
  asm volatile(SGPR_PAD "s_mov_b64 exec, %3\n\tglobal_store_dwordx4 %0, %1, %2 sc0 sc1\n\ts_mov_b64 exec, -1\n\ts_nop 0"
               :: "v"(off), "v"(v), "s"(base), "s"(mask) : "memory");
}

template <int N> __device__ __forceinline__ void wait_vm() {
  asm volatile("s_waitcnt vmcnt(%0)" :: "n"(N) : "memory");
}

__device__ __forceinline__ unsigned wave_max_u32(unsigned v) {
  v = max(v, (unsigned)__builtin_amdgcn_update_dpp(0, (int)v, 0x111, 0xf, 0xf, false));
  v = max(v, (unsigned)__builtin_amdgcn_update_dpp(0, (int)v, 0x112, 0xf, 0xf, false));
  v = max(v, (unsigned)__builtin_amdgcn_update_dpp(0, (int)v, 0x114, 0xf, 0xf, false));
  v = max(v, (unsigned)__builtin_amdgcn_update_dpp(0, (int)v, 0x118, 0xf, 0xf, false));
  return max(max((unsigned)__builtin_amdgcn_readlane((int)v, 15), (unsigned)__builtin_amdgcn_readlane((int)v, 31)),
             max((unsigned)__builtin_amdgcn_readlane((int)v, 47), (unsigned)__builtin_amdgcn_readlane((int)v, 63)));
}

__device__ __forceinline__ unsigned irec_slot(unsigned idx) { return idx + 3 + min((__umul24(idx, 37u) + 74u) >> 8, 3u); }

__device__ __forceinline__ unsigned wave_sum_u32(unsigned v) {
  v += (unsigned)__builtin_amdgcn_update_dpp(0, (int)v, 0x111, 0xf, 0xf, false);
  v += (unsigned)__builtin_amdgcn_update_dpp(0, (int)v, 0x112, 0xf, 0xf, false);
  v += (unsigned)__builtin_amdgcn_update_dpp(0, (int)v, 0x114, 0xf, 0xf, false);
  v += (unsigned)__builtin_amdgcn_update_dpp(0, (int)v, 0x118, 0xf, 0xf, false);
  return (unsigned)(__builtin_amdgcn_readlane((int)v, 15) + __builtin_amdgcn_readlane((int)v, 31) +
                    __builtin_amdgcn_readlane((int)v, 47) + __builtin_amdgcn_readlane((int)v, 63));
}
__device__ __forceinline__ unsigned bfi(unsigned m, unsigned a, unsigned b) { return (a & m) | (b & ~m); }
constexpr unsigned HDR_RAW = 0x80000000u;

constexpr unsigned IREC_CAP = 122, IHDR_RAW = 0x8000u;

struct InTask {
  u32x4 d[4];
  unsigned long long mask;
  bool raw;
};

__device__ __forceinline__ void put_records2(unsigned q0, unsigned q1, unsigned tag, unsigned& cnt, uint8_t* scr, unsigned dump) {
  unsigned m = bfi(0x55555555u, q0 | q0 >> 1, q1 | q1 << 1);
  for (unsigned long long act = __ballot(m != 0); act != 0 && cnt <= IREC_CAP; act = __ballot(m != 0)) {
    const unsigned bit = (unsigned)__builtin_ctz(m | 0x80000000u), sh = bit & 30u;
    const unsigned idx = cnt + __builtin_amdgcn_mbcnt_hi((unsigned)(act >> 32), __builtin_amdgcn_mbcnt_lo((unsigned)act, 0u));
    const unsigned sl = 2 * irec_slot(idx), at = ((m != 0) & (idx < IREC_CAP)) ? sl : dump;
    *(unsigned short*)(scr + at) = (unsigned short)(tag | (bit & 1u) << 6 | sh << 1 | ((((bit & 1u) ? q1 : q0) >> sh) & 3u));
    m &= m - 1;
    cnt += __popcll(act);
  }
}

__device__ __forceinline__ void enc_in(const u32x4 (&v)[4], InTask& o, unsigned tag, uint8_t* scr, int lane) {
  unsigned sum = 0;
#pragma unroll
  for (int q = 0; q < 4; ++q) sum = __builtin_amdgcn_udot4(v[q][0] & 0x7f7f7f7fu, 0x01010101u, sum, false);
  const unsigned e0 = (unsigned)min(max((int)((wave_sum_u32(sum) + 10445u) >> 12) - 7, 0), 24);
  const unsigned add = 0x80808080u - e0 * 0x04040404u;
  unsigned pay[12], qq[4], cnt = 0;
#pragma unroll
  for (int q = 0; q < 4; ++q) {
    unsigned R[4], S[4];
#pragma unroll
    for (int k = 0; k < 4; ++k) {
      R[k] = (v[q][k] & 0x7f7f7f7fu) + add;
      S[k] = bfi(0x80808080u, v[q][k], R[k]);
    }
    pay[3 * q] = bfi(0x60606060u, S[3] << 5, S[0]);
    pay[3 * q + 1] = bfi(0x60606060u, S[3] << 3, S[1]);
    pay[3 * q + 2] = bfi(0x20202020u, S[3] << 1, bfi(0x40404040u, S[3] >> 1, S[2]));
    qq[q] = bfi(0xc0c0c0c0u, R[3] << 1, bfi(0x30303030u, R[2] >> 1, bfi(0x0c0c0c0cu, R[1] >> 3, (R[0] >> 5) & 0x03030303u)));
  }
  put_records2(qq[0], qq[1], (unsigned)(lane << 8), cnt, scr, 256 + 2 * lane);
  put_records2(qq[2], qq[3], (unsigned)(lane << 8 | 128), cnt, scr, 256 + 2 * lane);
  *(unsigned short*)(scr + (lane < 4 ? lane * 16 : lane < 6 ? lane * 2 - 6 : 256 + 2 * lane)) =
      (unsigned short)(lane < 4 ? tag : lane == 4 ? cnt : e0);
  __builtin_amdgcn_fence(__ATOMIC_ACQ_REL, "wavefront");
  const u32x4 rec = *(const u32x4*)(scr + (lane & 15) * 16);
  __builtin_amdgcn_fence(__ATOMIC_ACQ_REL, "wavefront");
  o.raw = cnt > IREC_CAP;
  if (o.raw) {
#pragma unroll
    for (int q = 0; q < 4; ++q) o.d[q] = v[q];
    o.mask = ~0ull;
  } else {
#pragma unroll
    for (int q = 0; q < 3; ++q) o.d[q] = u32x4{pay[4 * q], pay[4 * q + 1], pay[4 * q + 2], pay[4 * q + 3]};
    o.d[3] = rec;
    o.mask = (1ull << (((irec_slot(cnt) + 31) >> 5) * 4)) - 16;
  }
}

__device__ __forceinline__ void dec_in4(const uint8_t* slot, uint8_t* out, const unsigned (&off)[4], unsigned stride, uint8_t* scr, int lane) {
  u32x4 y[4][3];
  unsigned rec[4];
  asm volatile(SGPR_PAD "global_load_dword %12, %16, %20 sc0 sc1\n\tglobal_load_dword %13, %16, %20 offset:3328 sc0 sc1\n\t"
               "global_load_dword %14, %17, %20 sc0 sc1\n\tglobal_load_dword %15, %17, %20 offset:3328 sc0 sc1\n\t"
               "global_load_dwordx4 %0, %18, %20 offset:-3072 sc0 sc1\n\tglobal_load_dwordx4 %1, %18, %20 offset:-2048 sc0 sc1\n\t"
               "global_load_dwordx4 %2, %18, %20 offset:-1024 sc0 sc1\n\t"
               "global_load_dwordx4 %3, %18, %20 offset:256 sc0 sc1\n\tglobal_load_dwordx4 %4, %18, %20 offset:1280 sc0 sc1\n\t"
               "global_load_dwordx4 %5, %18, %20 offset:2304 sc0 sc1\n\t"
               "global_load_dwordx4 %6, %19, %20 offset:-3072 sc0 sc1\n\tglobal_load_dwordx4 %7, %19, %20 offset:-2048 sc0 sc1\n\t"
               "global_load_dwordx4 %8, %19, %20 offset:-1024 sc0 sc1\n\t"
               "global_load_dwordx4 %9, %19, %20 offset:256 sc0 sc1\n\tglobal_load_dwordx4 %10, %19, %20 offset:1280 sc0 sc1\n\t"
               "global_load_dwordx4 %11, %19, %20 offset:2304 sc0 sc1\n\ts_waitcnt vmcnt(0)"
               : "=&v"(y[0][0]), "=&v"(y[0][1]), "=&v"(y[0][2]), "=&v"(y[1][0]), "=&v"(y[1][1]), "=&v"(y[1][2]),
                 "=&v"(y[2][0]), "=&v"(y[2][1]), "=&v"(y[2][2]), "=&v"(y[3][0]), "=&v"(y[3][1]), "=&v"(y[3][2]),
                 "=&v"(rec[0]), "=&v"(rec[1]), "=&v"(rec[2]), "=&v"(rec[3])
               : "v"(lane * 4), "v"(2 * ITASK + lane * 4), "v"(ITASK + lane * 16), "v"(3 * ITASK + lane * 16), "s"(slot) : "memory");
#pragma unroll
  for (int t = 0; t < 4; ++t) *(u32x4*)(scr + t * 1024 + lane * 16) = u32x4{0u, 0u, 0u, 0u};
  __builtin_amdgcn_fence(__ATOMIC_ACQ_REL, "wavefront");
  unsigned hdr[4];
#pragma unroll
  for (int t = 0; t < 4; ++t) {
    hdr[t] = (unsigned)__builtin_amdgcn_readlane((int)rec[t], 0) >> 16 | (unsigned)__builtin_amdgcn_readlane((int)rec[t], 1) << 16;
    const unsigned end = irec_slot(hdr[t] & 0x7fffu);
#pragma unroll
    for (int i = 0; i < 2; ++i) {
      const unsigned r16 = rec[t] >> (16 * i);
      const bool isrec = i == 0 ? (lane >= 16 || (lane & 3) != 0) && lane != 1 : lane != 0;
      const unsigned val = isrec && (unsigned)(lane * 2 + i) < end ? (r16 & 3u) << ((r16 >> 1) & 30u) : 0u;
      __hip_atomic_fetch_or((unsigned*)(scr + t * 1024 + ((r16 >> 6) & 255u) * 4), val, __ATOMIC_RELAXED, __HIP_MEMORY_SCOPE_WORKGROUP);
    }
  }
  __builtin_amdgcn_fence(__ATOMIC_ACQ_REL, "wavefront");
#pragma unroll
  for (int t = 0; t < 4; ++t) {
    if (hdr[t] & IHDR_RAW) continue;
    const u32x4 qq = *(const u32x4*)(scr + t * 1024 + lane * 16);
    const unsigned add = ((hdr[t] >> 16) & 31u) * 0x04040404u;
#pragma unroll
    for (int q = 0; q < 4; ++q) {
      const unsigned d0 = y[t][(3 * q) >> 2][(3 * q) & 3], d1 = y[t][(3 * q + 1) >> 2][(3 * q + 1) & 3], d2 = y[t][(3 * q + 2) >> 2][(3 * q + 2) & 3];
      const unsigned d3 = bfi(0x80808080u, d2 << 1, bfi(0x10101010u, d2 >> 1, bfi(0x0c0c0c0cu, d1 >> 3, d0 >> 5)));
      const unsigned X[4] = {bfi(0x60606060u, qq[q] << 5, d0), bfi(0x60606060u, qq[q] << 3, d1), bfi(0x60606060u, qq[q] << 1, d2),
                             bfi(0x60606060u, qq[q] >> 1, d3)};
      u32x4 r;
#pragma unroll
      for (int k = 0; k < 4; ++k) r[k] = bfi(0x80808080u, X[k], (X[k] & 0x7f7f7f7fu) + add);
      asm volatile(SGPR_PAD "global_store_dwordx4 %0, %1, %2 sc1\n\ts_nop 1" :: "v"(off[t] + q * stride + lane * 16), "v"(r), "s"(out) : "memory");
    }
  }
  __builtin_amdgcn_fence(__ATOMIC_ACQ_REL, "wavefront");
}

__device__ __forceinline__ bool wait_tasks(const uint8_t* slot, unsigned* tmo, unsigned call, int lane) {
  const uint8_t* q = slot + ((lane >> 2) & 3) * ITASK + (lane & 3) * 16;
  return spin([&] { return __all((ld_sys(q) & 0xffffu) == (call & 0xffffu)) != 0; }, tmo, call);
}

struct PushCtx {
  const uint8_t* x;
  const uint8_t* w;
  uint8_t* dst;
  unsigned* lk;
  uint8_t* scr;
  unsigned xl, wl;
  unsigned par, tick0, tag, call;
  int skip0, skipn;
  int rank, d;
  bool local;
};

struct RowSet { int r, row0, base, task0; };
__device__ __forceinline__ RowSet row_set(int set) {
  const int a = min(set, 3), o = max(set - 7, 0), b = set - a - o;
  const int own = (set + 1) >> 3, mid = ((set + 5) >> 3) - own;
  return {7 + 2 * mid + own, 224 * mid + 256 * o, 7 * a + 9 * b + 8 * o, 28 * a + 36 * b + 512 * own + 32 * o};
}
__device__ __forceinline__ int block_col(int q, int n) { return ((n & 3) * 4 + (q < 4 ? q : (q ^ n) & 3)) * 1024 + (n >> 2) * 128; }
__device__ __forceinline__ int tile_block(int g, int i) {
  const int c = (i * 43) >> 7, m = i - 3 * c;
  return g < 4 ? i : i >= 24 ? (g - 4) * 8 + i - 24 : 4 * c + m + (m >= g - 4);
}
__device__ __forceinline__ bool task_is_w(int t) { return t >= T_W0 && t < T_OWN; }
__device__ __forceinline__ int task_set(int t) {
  const int lo = min((t * 2341) >> 16, ((t + 24) * 1821) >> 16), hi = 7 + ((t - T_OWN) >> 5);
  return t < T_OWN ? lo : hi;
}

__device__ __forceinline__ int div_r(int v, int r) { return (v * (0x728093 >> (8 * (r - 7)) & 255)) >> 10; }

__device__ __forceinline__ void ld_x(const PushCtx& c, int t, u32x4 (&v)[4]) {
  t = min(t, NTASK - 1);
  const int set = task_set(t);
  const RowSet rs = row_set(set);
  const int p0 = 2 * (t - rs.task0), ks = div_r(p0, rs.r), rg = p0 - ks * rs.r;
  const unsigned o0 = c.xl + (unsigned)((((set < 7 ? set + 1 : 0) ^ c.d) * 512 + rs.row0 + rg * 32) * KS + ks * 64);
  const unsigned o1 = o0 + (unsigned)(rg + 1 == rs.r ? 64 - rg * 32 * KS : 32 * KS);
  asm volatile(SGPR_PAD "global_load_dwordx4 %0, %4, %6\n\tglobal_load_dwordx4 %1, %4, %6 offset:16\n\t"
               "global_load_dwordx4 %2, %5, %6\n\tglobal_load_dwordx4 %3, %5, %6 offset:16"
               : "=&v"(v[0]), "=&v"(v[1]), "=&v"(v[2]), "=&v"(v[3]) : "v"(o0), "v"(o1), "s"(c.x) : "memory");
}

__device__ __forceinline__ void ld_w(const uint8_t* base, unsigned o, unsigned (&w)[16]) {
    asm volatile(SGPR_PAD
                 "global_load_dword %0, %16, %17\n\tv_add_u32 %16, 0x4000, %16\n\t"
                 "global_load_dword %1, %16, %17\n\tv_add_u32 %16, 0x4000, %16\n\t"
                 "global_load_dword %2, %16, %17\n\tv_add_u32 %16, 0x4000, %16\n\t"
                 "global_load_dword %3, %16, %17\n\tv_add_u32 %16, 0x4000, %16\n\t"
                 "global_load_dword %4, %16, %17\n\tv_add_u32 %16, 0x4000, %16\n\t"
                 "global_load_dword %5, %16, %17\n\tv_add_u32 %16, 0x4000, %16\n\t"
                 "global_load_dword %6, %16, %17\n\tv_add_u32 %16, 0x4000, %16\n\t"
                 "global_load_dword %7, %16, %17\n\tv_add_u32 %16, 0x4000, %16\n\t"
                 "global_load_dword %8, %16, %17\n\tv_add_u32 %16, 0x4000, %16\n\t"
                 "global_load_dword %9, %16, %17\n\tv_add_u32 %16, 0x4000, %16\n\t"
                 "global_load_dword %10, %16, %17\n\tv_add_u32 %16, 0x4000, %16\n\t"
                 "global_load_dword %11, %16, %17\n\tv_add_u32 %16, 0x4000, %16\n\t"
                 "global_load_dword %12, %16, %17\n\tv_add_u32 %16, 0x4000, %16\n\t"
                 "global_load_dword %13, %16, %17\n\tv_add_u32 %16, 0x4000, %16\n\t"
                 "global_load_dword %14, %16, %17\n\tv_add_u32 %16, 0x4000, %16\n\t"
                 "global_load_dword %15, %16, %17\n\t"
                 : "=&v"(w[0]), "=&v"(w[1]), "=&v"(w[2]), "=&v"(w[3]), "=&v"(w[4]), "=&v"(w[5]), "=&v"(w[6]), "=&v"(w[7]),
                   "=&v"(w[8]), "=&v"(w[9]), "=&v"(w[10]), "=&v"(w[11]), "=&v"(w[12]), "=&v"(w[13]), "=&v"(w[14]), "=&v"(w[15]), "+v"(o) : "s"(base) : "memory");
    static_assert(GN == 0x4000, "");
}
__device__ __forceinline__ void transpose_w(const unsigned (&w)[16], u32x4 (&v)[4]) {
#pragma unroll
  for (int q = 0; q < 4; ++q) {
    const unsigned d0 = w[4 * q], d1 = w[4 * q + 1], d2 = w[4 * q + 2], d3 = w[4 * q + 3];
    const unsigned t0 = __builtin_amdgcn_perm(d1, d0, 0x05010400u), t1 = __builtin_amdgcn_perm(d1, d0, 0x07030602u);
    const unsigned t2 = __builtin_amdgcn_perm(d3, d2, 0x05010400u), t3 = __builtin_amdgcn_perm(d3, d2, 0x07030602u);
    v[0][q] = __builtin_amdgcn_perm(t2, t0, 0x05040100u);
    v[1][q] = __builtin_amdgcn_perm(t2, t0, 0x07060302u);
    v[2][q] = __builtin_amdgcn_perm(t3, t1, 0x05040100u);
    v[3][q] = __builtin_amdgcn_perm(t3, t1, 0x07060302u);
  }
}

__device__ __forceinline__ void pack_task(const PushCtx& c, const u32x4 (&v)[4], InTask& o, int lane, bool plain) {
  if (plain) {
#pragma unroll
    for (int q = 0; q < 4; ++q) o.d[q] = v[q];
    o.mask = ~0ull;
    o.raw = true;
  } else {
    enc_in(v, o, c.tag, c.scr, lane);
  }
}

__device__ __forceinline__ void send_plain(const PushCtx& c, int t, const u32x4 (&d)[4], int lane) {
  const int u = t - T_W0;
  const bool wr = task_is_w(t);
  const RowSet rs = row_set(task_set(t));
  const unsigned plain = c.par + lane * 16 +
      (wr ? (unsigned)XS_BYTES + (unsigned)((u >> 4) * (512 << 10) + (c.rank * 8 + ((u >> 1) & 7)) * 8192 + (u & 1) * 1024)
          : (unsigned)(rs.base * (128 << 10) + c.rank * rs.r * 16384 + (t - rs.task0) * 4096));
#pragma unroll
  for (int q = 0; q < 4; ++q) st16_sys(c.dst, plain + q * (wr ? 2048 : 1024), d[q]);
}

__device__ __forceinline__ int task_at(const PushCtx& c, int pos) { return min(pos < c.skip0 ? pos : pos + c.skipn, NTASK); }

struct PushState {
  int tp, tc, tn, tl;
  bool gone;
  u32x4 line;
  InTask o;
  u32x4 nx[4];
};

template <bool LDW>
__device__ __forceinline__ void push_step(const PushCtx& c, PushState& s, int lane) {
  unsigned w[16], tk;
  u32x4 x[4];
  const bool draw = s.tl < NTASK;
  asm volatile(SGPR_PAD "s_mov_b64 exec, 1\n\tglobal_atomic_add %0, %1, %2, %3 sc0\n\ts_mov_b64 exec, -1"
               : "=&v"(tk) : "v"(0u), "v"(draw ? 1u : 0u), "s"(c.lk) : "memory");
  if (s.tc == OPEN_AT) st1_agent(c.lk + 32, c.call);
  if constexpr (LDW) {
    const int u = s.tl - T_W0;
    ld_w(c.w, c.wl + (unsigned)((((u >> 1) & 7) * 64 + (u & 1) * 16) * GN + block_col(c.d, u >> 4)), w);
  } else {
    ld_x(c, s.tl, x);
  }
  const unsigned is0 = c.par + (unsigned)(XS_BYTES + WT_BYTES) + (unsigned)(c.rank * NTASK) * ITASK + lane * 16;
  if (s.tp >= 0) st16_sys_m(c.dst, is0 + (unsigned)s.tp * ITASK, s.line, 15ull);
  if (s.tc < NTASK) {
    if (!s.o.raw) {
      const unsigned slot = is0 + (unsigned)s.tc * ITASK;
      asm volatile(SGPR_PAD "global_store_dwordx4 %0, %1, %4 offset:256 sc0 sc1\n\tglobal_store_dwordx4 %0, %2, %4 offset:1280 sc0 sc1\n\t"
                   "global_store_dwordx4 %0, %3, %4 offset:2304 sc0 sc1\n\ts_nop 1"
                   :: "v"(slot), "v"(s.o.d[0]), "v"(s.o.d[1]), "v"(s.o.d[2]), "s"(c.dst) : "memory");
      if (s.o.mask != 0) st16_sys_m(c.dst, slot, s.o.d[3], s.o.mask);
      s.line = s.o.d[3];
    } else {
      if (!s.gone) send_plain(c, s.tc, s.o.d, lane);
      s.line = u32x4{c.tag | IHDR_RAW << 16, 0u, 0u, 0u};
    }
  }
  pack_task(c, s.nx, s.o, lane, c.local || s.tn >= NTASK);
  if constexpr (LDW) {
    asm volatile("s_waitcnt vmcnt(0)" : "+v"(tk), "+v"(w[0]), "+v"(w[1]), "+v"(w[2]), "+v"(w[3]), "+v"(w[4]), "+v"(w[5]), "+v"(w[6]), "+v"(w[7]),
                 "+v"(w[8]), "+v"(w[9]), "+v"(w[10]), "+v"(w[11]), "+v"(w[12]), "+v"(w[13]), "+v"(w[14]), "+v"(w[15]) :: "memory");
    transpose_w(w, s.nx);
  } else {
    asm volatile("s_waitcnt vmcnt(0)" : "+v"(tk), "+v"(x[0]), "+v"(x[1]), "+v"(x[2]), "+v"(x[3]) :: "memory");
#pragma unroll
    for (int q = 0; q < 4; ++q) s.nx[q] = x[q];
  }
  s.tp = s.tc; s.tc = s.tn; s.tn = s.tl; s.gone = false;
  s.tl = draw ? task_at(c, min(__builtin_amdgcn_readfirstlane(tk) - c.tick0, (unsigned)NTASK)) : NTASK;
}

#define GLDS_BITS " sc0 sc1"
__device__ __forceinline__ void glds2(const uint8_t* gbase, unsigned voff, unsigned lds) {
  unsigned keep;
  asm volatile("s_mov_b32 %0, m0\n\ts_mov_b32 m0, %3\n\ts_nop 2\n\t"
               "global_load_lds_dwordx4 %1, %2" GLDS_BITS "\n\t"
               "global_load_lds_dwordx4 %1, %2 offset:1024" GLDS_BITS "\n\t"
               "s_mov_b32 m0, %0"
               : "=&s"(keep) : "v"(voff), "s"(gbase), "s"(lds));
}
__device__ __forceinline__ void glds2o(const uint8_t* gbase, unsigned voff, unsigned lds) {
  unsigned keep;
  asm volatile("s_mov_b32 %0, m0\n\ts_mov_b32 m0, %3\n\ts_nop 2\n\t"
               "global_load_lds_dwordx4 %1, %2 offset:2048" GLDS_BITS "\n\t"
               "global_load_lds_dwordx4 %1, %2 offset:3072" GLDS_BITS "\n\t"
               "s_mov_b32 m0, %0"
               : "=&s"(keep) : "v"(voff), "s"(gbase), "s"(lds));
}
__device__ __forceinline__ void glds_half(const uint8_t* gbase, unsigned voff, unsigned lds) {
  unsigned keep;
  asm volatile("s_mov_b32 %0, m0\n\ts_mov_b32 m0, %3\n\ts_mov_b32 exec_hi, 0\n\ts_nop 1\n\t"
               "global_load_lds_dwordx4 %1, %2" GLDS_BITS "\n\t"
               "s_mov_b32 exec_hi, -1\n\ts_mov_b32 m0, %0"
               : "=&s"(keep) : "v"(voff), "s"(gbase), "s"(lds));
}

struct Frags { i32x8 a[2], b[4], a9, b9; };

__device__ __forceinline__ i32x8 ld_frag(const uint8_t* q) {
  const i32x4 lo = *(const i32x4*)q, hi = *(const i32x4*)(q + 1024);
  return __builtin_shufflevector(lo, hi, 0, 1, 2, 3, 4, 5, 6, 7);
}

#define MFMA(mi, j) acc[mi][j] = __builtin_amdgcn_mfma_scale_f32_32x32x64_f8f6f4(cur.a[mi], cur.b[j], acc[mi][j], 1, 1, 0, 0, 0, 0)
#define SB() __builtin_amdgcn_sched_barrier(0)

struct Ring {
  const uint8_t* ga;
  const uint8_t* gb;
  unsigned astep, ahop;
  int togo;
  unsigned va, vb, v9;
  unsigned wr, wr_end;
  unsigned wb, w9;
  unsigned rd;
  const uint8_t* qa;
  const uint8_t* qb;
  const uint8_t* q9;
  const uint8_t* la;
  const uint8_t* lb;
  const uint8_t* l9;
};

template <int MODE, int LEFT, bool DMA, bool RD>
__device__ __forceinline__ void kstep(f32x16 (&acc)[2][4], f32x16& acc9, const Frags& cur, Frags& nxt, Ring& g) {
  constexpr bool NINE = MODE == 9, TWO = MODE != 7;
  constexpr int STAGE = NINE ? 26624 : 24576;
  MFMA(0, 0); SB();
  asm volatile("s_waitcnt vmcnt(%0) lgkmcnt(0)\n\ts_barrier" :: "n"(LEFT * (NINE ? 7 : 6)) : "memory");
  SB();
  MFMA(0, 1); SB();
  if constexpr (RD) { nxt.a[0] = ld_frag(g.qa); nxt.b[0] = ld_frag(g.qb); }
  unsigned sa = 0, sb = 0;
  if constexpr (DMA) { const bool hop = --g.togo == 0; sa = hop ? g.ahop : g.astep; sb = hop ? 9 * 8192 : 8192; asm volatile("" : "+s"(sa), "+s"(sb)); }
  SB();
  MFMA(0, 2); SB();
  if constexpr (DMA) glds2(g.ga, g.va, g.wr);
  SB();
  MFMA(0, 3); SB();
  if constexpr (RD) { nxt.b[1] = ld_frag(g.qb + 2048); nxt.b[2] = ld_frag(g.qb + 4096); }
  SB();
  if constexpr (TWO) MFMA(1, 0);
  SB();
  if constexpr (DMA) glds2(g.gb, g.vb, g.wr + g.wb);
  SB();
  if constexpr (TWO) MFMA(1, 1);
  SB();
  if constexpr (RD) { nxt.b[3] = ld_frag(g.qb + 6144); if constexpr (TWO) nxt.a[1] = ld_frag(g.qa + 2048); }
  g.rd = g.rd == (NSTAGE - 1) * STAGE ? 0 : g.rd + STAGE;
  g.qa = g.la + g.rd;
  if constexpr (!NINE) g.qb = g.lb + g.rd;
  SB();
  if constexpr (TWO) MFMA(1, 2);
  SB();
  if constexpr (DMA) glds2o(g.ga, g.va, g.wr);
  SB();
  if constexpr (NINE) {
    if constexpr (TWO) MFMA(1, 3);
    SB();
    if constexpr (RD) { nxt.a9 = ld_frag(g.qb - 2048); nxt.b9 = ld_frag(g.q9); }
    g.qb = g.lb + g.rd; g.q9 = g.l9 + g.rd;
    if constexpr (DMA) g.gb += sb;
    SB();
    acc9 = __builtin_amdgcn_mfma_scale_f32_32x32x64_f8f6f4(cur.a9, cur.b9, acc9, 1, 1, 0, 0, 0, 0); SB();
    if constexpr (DMA) glds_half(g.ga, g.v9, g.wr + g.w9);
  } else {
    if constexpr (TWO) MFMA(1, 3);
    SB();
    if constexpr (DMA) g.gb += sb;
  }
  if constexpr (DMA) {
    g.ga += sa;
    g.wr += STAGE; g.wr = g.wr == g.wr_end ? g.wr - NSTAGE * STAGE : g.wr;
  }
  SB();
}

template <int MODE>
__device__ __forceinline__ void gemm_tile(const uint8_t* ga, unsigned astep, const uint8_t* gb, int skip, uint8_t* lds, int wave, int lane) {
  constexpr bool NINE = MODE == 9, TWO = MODE != 7;
  constexpr int STAGE = NINE ? 26624 : 24576, BOFF = STAGE - 8192;
  f32x16 acc[2][4], acc9;
  Ring g;
  g.ga = ga + (skip == 0 ? 8 * astep : 0); g.gb = gb + (skip == 0 ? 8 * 8192 : 0); g.astep = astep; g.ahop = 9 * astep;
  g.togo = skip > 0 ? 8 * skip - NSTAGE : -1;
  g.va = lane * 16 + wave * 4096; g.vb = lane * 16 + wave * 2048; g.v9 = 16384 + lane * 16 + wave * 512;
  g.wr = (unsigned)(uintptr_t)lds + wave * 4096; g.wr_end = g.wr + NSTAGE * STAGE;
  g.wb = BOFF - wave * 2048; g.w9 = 16384 - wave * 3584;
#pragma unroll
  for (int s = 0; s < NSTAGE; ++s) {
    glds2(g.ga, g.va, g.wr + s * STAGE);
    glds2(g.gb, g.vb, g.wr + s * STAGE + g.wb);
    glds2o(g.ga, g.va, g.wr + s * STAGE);
    if constexpr (NINE) glds_half(g.ga, g.v9, g.wr + s * STAGE + g.w9);
    g.ga += astep; g.gb += 8192;
  }
#pragma unroll
  for (int r = 0; r < 16; ++r) {
    acc9[r] = 0.f;
#pragma unroll
    for (int j = 0; j < 4; ++j) { acc[0][j][r] = 0.f; acc[1][j][r] = 0.f; }
  }
  g.la = lds + lane * 16 + wave * 4096; g.lb = lds + BOFF + lane * 16; g.l9 = g.lb + wave * 2048;
  asm volatile("s_waitcnt vmcnt(%0)\n\ts_barrier" :: "n"(5 * (NINE ? 7 : 6)) : "memory");
  Frags f0, f1;
  f0.a[0] = ld_frag(g.la); f0.b[0] = ld_frag(g.lb); f0.b[1] = ld_frag(g.lb + 2048);
  f0.b[2] = ld_frag(g.lb + 4096); f0.b[3] = ld_frag(g.lb + 6144);
  if constexpr (TWO) f0.a[1] = ld_frag(g.la + 2048);
  if constexpr (NINE) { f0.a9 = ld_frag(g.lb - 2048); f0.b9 = ld_frag(g.l9); }
  g.rd = STAGE; g.qa = g.la + STAGE; g.qb = g.lb + STAGE; g.q9 = g.l9 + STAGE;
#pragma unroll 1
  for (int s = skip < 0 ? 0 : 8; s < KSTEPS - NSTAGE; s += 2) {
    kstep<MODE, 4, true, true>(acc, acc9, f0, f1, g);
    kstep<MODE, 4, true, true>(acc, acc9, f1, f0, g);
  }
  kstep<MODE, 4, false, true>(acc, acc9, f0, f1, g);
  kstep<MODE, 3, false, true>(acc, acc9, f1, f0, g);
  kstep<MODE, 2, false, true>(acc, acc9, f0, f1, g);
  kstep<MODE, 1, false, true>(acc, acc9, f1, f0, g);
  kstep<MODE, 0, false, true>(acc, acc9, f0, f1, g);
  auto park = [&](const f32x16& a, int i) {
#pragma unroll
    for (int q = 0; q < 4; ++q)
      *(f32x4*)(lds + wave * 32768 + (i * 4 + q) * 1024 + lane * 16) = f32x4{a[4 * q], a[4 * q + 1], a[4 * q + 2], a[4 * q + 3]};
  };
  {
    const Frags& cur = f1;
    MFMA(0, 0); SB();
    asm volatile("s_waitcnt lgkmcnt(0)\n\ts_barrier" ::: "memory");
    SB();
    MFMA(0, 1); SB();
    MFMA(0, 2); SB();
    park(acc[0][0], 0); SB();
    MFMA(0, 3); SB();
    park(acc[0][1], 1); SB();
    if constexpr (TWO) MFMA(1, 0);
    SB();
    park(acc[0][2], 2); SB();
    if constexpr (TWO) MFMA(1, 1);
    SB();
    park(acc[0][3], 3); SB();
    if constexpr (TWO) {
      MFMA(1, 2); SB();
      park(acc[1][0], 4); SB();
      MFMA(1, 3); SB();
      park(acc[1][1], 5); SB();
      if constexpr (NINE) { acc9 = __builtin_amdgcn_mfma_scale_f32_32x32x64_f8f6f4(cur.a9, cur.b9, acc9, 1, 1, 0, 0, 0, 0); SB(); }
      park(acc[1][2], 6);
      park(acc[1][3], 7);
    }
  }
  if constexpr (NINE) {
#pragma unroll
    for (int q = 0; q < 4; ++q)
      *(f32x4*)(lds + 8 * 16384 + wave * 4096 + q * 1024 + lane * 16) = f32x4{acc9[4 * q], acc9[4 * q + 1], acc9[4 * q + 2], acc9[4 * q + 3]};
    __syncthreads();
  }
  __builtin_amdgcn_fence(__ATOMIC_ACQ_REL, "wavefront");
}

constexpr unsigned REC_CAP = 510, HDR_B = 0x40000000u;

__device__ __forceinline__ void put_records4(const unsigned (&q)[4], unsigned tag, unsigned& cnt, uint8_t* scr, unsigned dump) {
  unsigned m = 0;
#pragma unroll
  for (int k = 0; k < 4; ++k)
    m |= ((((q[k] & 0x77777777u) + 0x77777777u) | q[k]) & 0x88888888u) >> (3 - k);
  for (unsigned long long act = __ballot(m != 0); act != 0 && cnt <= REC_CAP; act = __ballot(m != 0)) {
    const unsigned bit = (unsigned)__builtin_ctz(m | 0x80000000u), sh = bit & 28u;
    const unsigned m1 = (unsigned)__builtin_amdgcn_sbfe((int)bit, 0, 1), m2 = (unsigned)__builtin_amdgcn_sbfe((int)bit, 1, 1);
    const unsigned w = bfi(m2, bfi(m1, q[3], q[2]), bfi(m1, q[1], q[0]));
    const unsigned idx = cnt + __builtin_amdgcn_mbcnt_hi((unsigned)(act >> 32), __builtin_amdgcn_mbcnt_lo((unsigned)act, 0u));
    const unsigned at = (m != 0 && idx < REC_CAP) ? 4 + 2 * idx : dump;
    *(unsigned short*)(scr + at) = (unsigned short)(tag | (bit & 3u) << 7 | sh << 2 | ((w >> sh) & 15u));
    m &= m - 1;
    cnt += __popcll(act);
  }
}

template <bool MODEB>
__device__ __forceinline__ bool enc_body(const float (&v)[4][16], float up, unsigned& cnt, uint8_t* blk, int lane) {
  unsigned bad = 0, low = 0;
#pragma unroll
  for (int h = 0; h < 2; ++h) {
    unsigned G[32];
#pragma unroll
    for (int j = 0; j < 4; ++j)
#pragma unroll
      for (int r = 0; r < 8; ++r) {
        const float x = v[j][8 * h + r] * up;
        unsigned g = (unsigned)(int)x;
        bad = __builtin_amdgcn_bitop3_b32(__builtin_bit_cast(unsigned, (float)(int)g), __builtin_bit_cast(unsigned, x), bad, 0xbe);
        low |= g;
        if (MODEB) g = g << 1 | (((g ^ g << 1) >> 27) & 16u);
        G[4 * r + j] = g;
      }
    unsigned q[4];
#pragma unroll
    for (int k = 0; k < 4; ++k) {
      const unsigned* g = G + 8 * k;
      const unsigned b0 = __builtin_amdgcn_perm(__builtin_amdgcn_perm(g[3], g[2], 0x0c0c0400u), __builtin_amdgcn_perm(g[1], g[0], 0x0c0c0400u), 0x05040100u);
      const unsigned b1 = __builtin_amdgcn_perm(__builtin_amdgcn_perm(g[7], g[6], 0x0c0c0400u), __builtin_amdgcn_perm(g[5], g[4], 0x0c0c0400u), 0x05040100u);
      q[k] = (b0 >> 4) | b1;
    }
#pragma unroll
    for (int j = 0; j < 2; ++j) {
      unsigned d[12];
#pragma unroll
      for (int g = 0; g < 4; ++g) {
        const unsigned* x = G + 16 * j + 4 * g;
        d[3 * g] = __builtin_amdgcn_perm(x[1], x[0], 0x05030201u);
        d[3 * g + 1] = __builtin_amdgcn_perm(x[2], x[1], 0x06050302u);
        d[3 * g + 2] = __builtin_amdgcn_perm(x[3], x[2], 0x07060503u);
      }
#pragma unroll
      for (int i = 0; i < 3; ++i)
        *(u32x4*)(blk + (1 + 6 * h + 3 * j + i) * 1024 + lane * 16) = u32x4{d[4 * i], d[4 * i + 1], d[4 * i + 2], d[4 * i + 3]};
    }
    put_records4(q, (unsigned)(lane << 10 | h << 9), cnt, blk, 13 * 1024 + 2 * lane);
  }
  return __ballot((bad | (low & 15u)) != 0) == 0 && cnt <= REC_CAP;
}

__device__ __forceinline__ unsigned enc_out(const float (&v)[4][16], uint8_t* blk, int lane) {
  float m = 0.f;
#pragma unroll
  for (int j = 0; j < 4; ++j)
#pragma unroll
    for (int r = 0; r < 16; r += 2) asm("v_max3_f32 %0, |%1|, |%2|, %0" : "+v"(m) : "v"(v[j][r]), "v"(v[j][r + 1]));
  const int e = max((int)(wave_max_u32(__builtin_bit_cast(unsigned, m)) >> 23) - 157, -126);
  const float up = __builtin_bit_cast(float, (unsigned)(127 - min(e, 0)) << 23);
  int ntop = 0, nlow = 0;
#pragma unroll
  for (int r = 0; r < 8; ++r) {
    const unsigned g = (unsigned)(int)(v[r & 3][2 * r] * up);
    ntop += __popcll(__ballot((int)(g ^ (g << 1)) < 0));
    nlow += __popcll(__ballot((g & 128u) != 0));
  }
  unsigned cnt = 0;
  const bool modeb = ntop < nlow;
  const bool fits = e <= 0 && (modeb ? enc_body<true>(v, up, cnt, blk, lane) : enc_body<false>(v, up, cnt, blk, lane));
  if (!fits) {
#pragma unroll
    for (int i = 0; i < 16; ++i) {
      const float* t = &v[i >> 2][(i & 3) * 4];
      *(f32x4*)(blk + i * 1024 + lane * 16) = f32x4{t[0], t[1], t[2], t[3]};
    }
    return 0;
  }
  *(unsigned*)(blk + (lane == 0 ? 0 : 13 * 1024 + 4 * lane)) = cnt | (unsigned)(e + 127) << 16 | (modeb ? HDR_B : 0u);
  return ((4 + 2 * cnt + 63) >> 6) * 4;
}

__device__ __forceinline__ void st16_out(void* base, unsigned off, f32x4 v) { st16_loc(base, off, __builtin_bit_cast(u32x4, v)); }

__device__ __forceinline__ constexpr unsigned row_off(int r) { return (unsigned)((r & 3) + 8 * (r >> 2)) * (GN * 4); }

__device__ __forceinline__ void send_out(const uint8_t* blk, unsigned nl, bool dead, uint8_t* slot, int lane, unsigned* next,
                                         unsigned call, bool last, int depth, unsigned* flag, unsigned half) {
  const bool raw = nl == 0 || dead;
  const u32x4 h = *(const u32x4*)(blk + lane * 16);
  st16_sys_m(slot, lane * 16, raw ? u32x4{HDR_RAW, 0u, 0u, 0u} : h, raw ? 15ull : nl >= 64 ? ~0ull : (1ull << nl) - 1);
  const int n = raw ? 16 : 12;
  const uint8_t* const src = blk + (raw ? 0 : 1024) + lane * 16;
  const unsigned nan = dead ? 0x7fc00000u : 0u;
#pragma unroll 1
  for (int i = 0; i < n; ++i) {
    const u32x4 v = *(const u32x4*)(src + i * 1024) | nan;
    if (depth == OUT_D) wait_vm<OUT_D - 1>(); else if (depth == OUT_D3) wait_vm<OUT_D3 - 1>(); else wait_vm<DEEP_D - 1>();
    if (last && i == n - OUT_REL) st1_agent(next, call);
    if (half != 0 && i == 6) { wait_vm<0>(); st1_sys(flag, half); }
    st16_sys(slot, (unsigned)(1024 + i * 1024 + lane * 16), v);
  }
}

template <int H>
__device__ __forceinline__ void dec_rows(const u32x4 (&y)[6], const u32x4& qq, bool modeb, float down, const f32x4 (&h)[16], float* ob,
                                         unsigned off, float sc) {
#pragma unroll
  for (int g = 0; g < 8; ++g) {
    const unsigned d0 = y[(3 * g) >> 2][(3 * g) & 3], d1 = y[(3 * g + 1) >> 2][(3 * g + 1) & 3], d2 = y[(3 * g + 2) >> 2][(3 * g + 2) & 3];
    const unsigned q = ((g & 1) ? qq[g >> 1] : qq[g >> 1] << 4) & 0xf0f0f0f0u;
    unsigned G[4] = {__builtin_amdgcn_perm(d0, q, 0x06050400u), __builtin_amdgcn_perm(d1, d0, 0x0504030cu) | ((q >> 8) & 255u),
                     __builtin_amdgcn_perm(d2, d1, 0x0403020cu) | ((q >> 16) & 255u), __builtin_amdgcn_perm(d2, q, 0x07060503u)};
    f32x4 v;
#pragma unroll
    for (int j = 0; j < 4; ++j) {
      if (modeb) G[j] = ((unsigned)((int)G[j] >> 1) & ~8u) ^ (G[j] << 27);
      v[j] = (__fmul_rn((float)(int)G[j], down) + h[4 * j + 2 * H + (g >> 2)][g & 3]) * sc;
    }
    st16_out(ob, off + row_off(8 * H + g), v);
  }
}

__device__ __forceinline__ void ld6_sys(const uint8_t* base, unsigned voff, u32x4 (&y)[6]) {
  asm volatile(SGPR_PAD "global_load_dwordx4 %0, %6, %7 offset:-3072 sc0 sc1\n\tglobal_load_dwordx4 %1, %6, %7 offset:-2048 sc0 sc1\n\t"
               "global_load_dwordx4 %2, %6, %7 offset:-1024 sc0 sc1\n\tglobal_load_dwordx4 %3, %6, %7 sc0 sc1\n\t"
               "global_load_dwordx4 %4, %6, %7 offset:1024 sc0 sc1\n\tglobal_load_dwordx4 %5, %6, %7 offset:2048 sc0 sc1\n\ts_waitcnt vmcnt(0)"
               : "=&v"(y[0]), "=&v"(y[1]), "=&v"(y[2]), "=&v"(y[3]), "=&v"(y[4]), "=&v"(y[5]) : "v"(voff + 3072), "s"(base) : "memory");
}

template <class F>
__device__ __forceinline__ void dec_out(const uint8_t* slot, const f32x4 (&h)[16], float* ob, unsigned off, float sc, unsigned* tmo, unsigned call,
                                        uint8_t* scr, int lane, bool halves, F rest) {
  u32x4 rec, y[6], z[6];
  unsigned tmv;
  asm volatile(SGPR_PAD "global_load_dword %13, %16, %18 sc1\n\t"
               "global_load_dwordx4 %0, %14, %17 offset:-4096 sc0 sc1\n\tglobal_load_dwordx4 %1, %14, %17 offset:-3072 sc0 sc1\n\t"
               "global_load_dwordx4 %2, %14, %17 offset:-2048 sc0 sc1\n\tglobal_load_dwordx4 %3, %14, %17 offset:-1024 sc0 sc1\n\t"
               "global_load_dwordx4 %4, %14, %17 sc0 sc1\n\tglobal_load_dwordx4 %5, %14, %17 offset:1024 sc0 sc1\n\t"
               "global_load_dwordx4 %6, %14, %17 offset:2048 sc0 sc1\n\tglobal_load_dwordx4 %7, %14, %17 offset:3072 sc0 sc1\n\t"
               "global_load_dwordx4 %8, %15, %17 offset:-4096 sc0 sc1\n\tglobal_load_dwordx4 %9, %15, %17 offset:-3072 sc0 sc1\n\t"
               "global_load_dwordx4 %10, %15, %17 offset:-2048 sc0 sc1\n\tglobal_load_dwordx4 %11, %15, %17 offset:-1024 sc0 sc1\n\t"
               "global_load_dwordx4 %12, %15, %17 sc0 sc1\n\ts_waitcnt vmcnt(0)"
               : "=&v"(rec), "=&v"(y[0]), "=&v"(y[1]), "=&v"(y[2]), "=&v"(y[3]), "=&v"(y[4]), "=&v"(y[5]), "=&v"(z[0]),
                 "=&v"(z[1]), "=&v"(z[2]), "=&v"(z[3]), "=&v"(z[4]), "=&v"(z[5]), "=&v"(tmv)
               : "v"(4096 + lane * 16), "v"(12288 + lane * 16), "v"(0u), "s"(slot), "s"(tmo) : "memory");
  const unsigned hdr = __builtin_amdgcn_readfirstlane(rec[0]);
  bool ok = __builtin_amdgcn_readfirstlane(tmv) != call;
  if (ok && !(hdr & HDR_RAW)) {
    const unsigned cnt = hdr & 0xffffu;
    const float down = __builtin_bit_cast(float, ((hdr >> 16) & 255u) << 23);
    *(u32x4*)(scr + lane * 32) = u32x4{0u, 0u, 0u, 0u};
    *(u32x4*)(scr + lane * 32 + 16) = u32x4{0u, 0u, 0u, 0u};
    __builtin_amdgcn_fence(__ATOMIC_ACQ_REL, "wavefront");
#pragma unroll
    for (int i = 0; i < 8; ++i) {
      const unsigned r16 = (rec[i >> 1] >> (16 * (i & 1))) & 0xffffu;
      const unsigned val = (unsigned)(lane * 8 + i - 2) < cnt ? (r16 & 15u) << ((r16 >> 2) & 28u) : 0u;
      __hip_atomic_fetch_or((unsigned*)(scr + (r16 >> 7) * 4), val, __ATOMIC_RELAXED, __HIP_MEMORY_SCOPE_WORKGROUP);
    }
    __builtin_amdgcn_fence(__ATOMIC_ACQ_REL, "wavefront");
    const u32x4 q0 = *(const u32x4*)(scr + lane * 32), q1 = *(const u32x4*)(scr + lane * 32 + 16);
    __builtin_amdgcn_fence(__ATOMIC_ACQ_REL, "wavefront");
    const bool modeb = (hdr & HDR_B) != 0;
    dec_rows<0>(y, q0, modeb, down, h, ob, off, sc);
    if (halves) {
      ok = rest();
      ld6_sys(slot, 7168 + lane * 16, z);
    }
    if (ok) dec_rows<1>(z, q1, modeb, down, h, ob, off, sc);
  } else if (ok) {
    if (halves) {
      ok = rest();
      ld6_sys(slot, 7168 + lane * 16, z);
    }
    u32x4 w[6];
    ld6_sys(slot, 11264 + lane * 16, w);
#pragma unroll
    for (int r = 0; r < 16; ++r) {
      f32x4 t;
#pragma unroll
      for (int j = 0; j < 4; ++j) {
        const int i = 4 * j + (r >> 2);
        t[j] = (__builtin_bit_cast(float, i < 6 ? y[i][r & 3] : i < 12 ? z[i - 6][r & 3] : w[i - 10][r & 3]) + h[i][r & 3]) * sc;
      }
      st16_out(ob, off + row_off(r), t);
    }
  }
  if (!ok) {
    const float n = __builtin_nanf("");
#pragma unroll 1
    for (int r = 0; r < 16; ++r) st16_out(ob, off + row_off(r), f32x4{n, n, n, n});
  }
}

__device__ __forceinline__ int wave_blocks(int r, int wave) { return wave < 3 ? 2 : r - 6; }
__device__ __forceinline__ unsigned out_off(int row0, int col, int wave, int lane) {
  return (unsigned)(((row0 + wave * 64 + 4 * (lane >> 5)) * GN + col + 4 * (lane & 31)) * 4);
}

template <bool NINE>
__device__ __forceinline__ void home_tile(const Params& p, int g, int src, int i, uint8_t* lds, int wave, int lane) {
  const int col = block_col(src, tile_block(g, i));
  const uint8_t* const xg = p.x + (size_t)(p.rank * 512 + (lane & 31)) * KS + (lane >> 5) * 32;
  auto xa = [&](int rho, int s) { return *(const i32x8*)(xg + rho * 32 * KS + s * 64); };
  i32x8 a0[8], a1[8], a9[8];
  {
    unsigned w[4][16];
    const unsigned o = (unsigned)((wave * 128 + (lane >> 5) * 32) * GN + col + (lane & 31) * 4);
#pragma unroll
    for (int t = 0; t < 4; ++t) ld_w(p.w, o + (unsigned)((t >> 1) * 64 + (t & 1) * 16) * GN, w[t]);
#pragma unroll
    for (int s = 0; s < 8; ++s) {
      a0[s] = xa((NINE ? 7 : 0) + wave * 2, s);
      a1[s] = xa((NINE ? 8 : 1) + wave * 2, s);
    }
    asm volatile("s_waitcnt vmcnt(0)" : "+v"(w[0][0]), "+v"(w[0][1]), "+v"(w[0][2]), "+v"(w[0][3]), "+v"(w[0][4]), "+v"(w[0][5]), "+v"(w[0][6]), "+v"(w[0][7]), "+v"(w[0][8]), "+v"(w[0][9]), "+v"(w[0][10]), "+v"(w[0][11]), "+v"(w[0][12]), "+v"(w[0][13]), "+v"(w[0][14]), "+v"(w[0][15]) :: "memory");
    asm volatile("" : "+v"(w[1][0]), "+v"(w[1][1]), "+v"(w[1][2]), "+v"(w[1][3]), "+v"(w[1][4]), "+v"(w[1][5]), "+v"(w[1][6]), "+v"(w[1][7]), "+v"(w[1][8]), "+v"(w[1][9]), "+v"(w[1][10]), "+v"(w[1][11]), "+v"(w[1][12]), "+v"(w[1][13]), "+v"(w[1][14]), "+v"(w[1][15]) :: "memory");
    asm volatile("" : "+v"(w[2][0]), "+v"(w[2][1]), "+v"(w[2][2]), "+v"(w[2][3]), "+v"(w[2][4]), "+v"(w[2][5]), "+v"(w[2][6]), "+v"(w[2][7]), "+v"(w[2][8]), "+v"(w[2][9]), "+v"(w[2][10]), "+v"(w[2][11]), "+v"(w[2][12]), "+v"(w[2][13]), "+v"(w[2][14]), "+v"(w[2][15]) :: "memory");
    asm volatile("" : "+v"(w[3][0]), "+v"(w[3][1]), "+v"(w[3][2]), "+v"(w[3][3]), "+v"(w[3][4]), "+v"(w[3][5]), "+v"(w[3][6]), "+v"(w[3][7]), "+v"(w[3][8]), "+v"(w[3][9]), "+v"(w[3][10]), "+v"(w[3][11]), "+v"(w[3][12]), "+v"(w[3][13]), "+v"(w[3][14]), "+v"(w[3][15]) :: "memory");
#pragma unroll
    for (int t = 0; t < 4; ++t) {
      u32x4 v[4];
      transpose_w(w[t], v);
#pragma unroll
      for (int j = 0; j < 4; ++j) *(u32x4*)(lds + (wave * 2 + (t >> 1)) * 8192 + j * 2048 + (t & 1) * 1024 + lane * 16) = v[j];
    }
  }
  if constexpr (NINE) {
#pragma unroll
    for (int s = 0; s < 8; ++s) a9[s] = xa(15, s);
  }
  __syncthreads();
  f32x16 h[2][4], h9;
#pragma unroll
  for (int r = 0; r < 16; ++r) {
    h9[r] = 0.f;
#pragma unroll
    for (int j = 0; j < 4; ++j) { h[0][j][r] = 0.f; h[1][j][r] = 0.f; }
  }
  i32x8 bn = ld_frag(lds + lane * 16);
#pragma unroll
  for (int t = 0; t < 32; ++t) {
    const i32x8 bc = bn;
    if (t < 31) bn = ld_frag(lds + (t + 1) * 2048 + lane * 16);
    i32x8 b9;
    if (NINE && (t & 3) == 0) b9 = ld_frag(lds + (t + wave) * 2048 + lane * 16);
    SB();
    h[0][t & 3] = __builtin_amdgcn_mfma_scale_f32_32x32x64_f8f6f4(a0[t >> 2], bc, h[0][t & 3], 1, 1, 0, 0, 0, 0);
    h[1][t & 3] = __builtin_amdgcn_mfma_scale_f32_32x32x64_f8f6f4(a1[t >> 2], bc, h[1][t & 3], 1, 1, 0, 0, 0, 0);
    if (NINE && (t & 3) == 0) h9 = __builtin_amdgcn_mfma_scale_f32_32x32x64_f8f6f4(a9[t >> 2], b9, h9, 1, 1, 0, 0, 0, 0);
    SB();
  }
  uint8_t* const pb = p.myws + WS_PB + (size_t)((g * 32 + i) * 9) * 16384;
#pragma unroll
  for (int q = 0; q < (NINE ? 36 : 32); ++q) {
    const f32x16& t = q < 32 ? h[q >> 4][(q >> 2) & 3] : h9;
    st16_out(pb, (unsigned)((q < 32 ? wave * 32 + q : 128 - 32 + wave * 4 + q) * 1024 + lane * 16),
             f32x4{t[4 * (q & 3)], t[4 * (q & 3) + 1], t[4 * (q & 3) + 2], t[4 * (q & 3) + 3]});
  }
  __syncthreads();
}

__device__ __forceinline__ void recv_tile(uint8_t* myws, float* out, int g, int src, int i, int k0, int k1, int wave, int lane,
                                          unsigned call, unsigned* tmo, float sc, uint8_t* scr) {
  const unsigned off = out_off(g < 4 ? 0 : 224, block_col(src, tile_block(g, i)), wave, lane);
  const uint8_t* const fl = myws + WS_DONE + (size_t)((src * 32 + i) * 4 + wave) * 128;
  const int nb = wave_blocks(g < 4 ? 7 : 9, wave);
  if (k0 >= min(nb, k1)) return;
  const uint8_t* const fh = myws + WS_HOME + (size_t)((g * 32 + i) * 4 + (lane & 3)) * 4;
  spin<RX_SLEEP>([&] { return __all(ld_agent(fh) >= call) != 0; }, tmo, call);
#pragma unroll 1
  for (int k = k0; k < min(nb, k1); ++k) {
    f32x4 h[16];
    asm volatile(SGPR_PAD "global_load_dwordx4 %0, %16, %18 offset:-4096 sc1\n\t""global_load_dwordx4 %1, %16, %18 offset:-3072 sc1\n\t"
                 "global_load_dwordx4 %2, %16, %18 offset:-2048 sc1\n\t""global_load_dwordx4 %3, %16, %18 offset:-1024 sc1\n\t"
                 "global_load_dwordx4 %4, %16, %18 offset:0 sc1\n\t""global_load_dwordx4 %5, %16, %18 offset:1024 sc1\n\t"
                 "global_load_dwordx4 %6, %16, %18 offset:2048 sc1\n\t""global_load_dwordx4 %7, %16, %18 offset:3072 sc1\n\t"
                 "global_load_dwordx4 %8, %17, %18 offset:-4096 sc1\n\t""global_load_dwordx4 %9, %17, %18 offset:-3072 sc1\n\t"
                 "global_load_dwordx4 %10, %17, %18 offset:-2048 sc1\n\t""global_load_dwordx4 %11, %17, %18 offset:-1024 sc1\n\t"
                 "global_load_dwordx4 %12, %17, %18 offset:0 sc1\n\t""global_load_dwordx4 %13, %17, %18 offset:1024 sc1\n\t"
                 "global_load_dwordx4 %14, %17, %18 offset:2048 sc1\n\t""global_load_dwordx4 %15, %17, %18 offset:3072 sc1"
                 : "=&v"(h[0]), "=&v"(h[1]), "=&v"(h[2]), "=&v"(h[3]), "=&v"(h[4]), "=&v"(h[5]), "=&v"(h[6]), "=&v"(h[7]),
                   "=&v"(h[8]), "=&v"(h[9]), "=&v"(h[10]), "=&v"(h[11]), "=&v"(h[12]), "=&v"(h[13]), "=&v"(h[14]), "=&v"(h[15])
                 : "v"(4096 + lane * 16), "v"(12288 + lane * 16), "s"(myws + WS_PB + (size_t)((g * 32 + i) * 9 + wave * 2 + k) * 16384) : "memory");
    const bool halves = i >= (g < 4 ? 24 : 16) && k == nb - 1;
    spin<RX_SLEEP>([&] { return __builtin_amdgcn_readfirstlane(ld_sys(fl)) >= 8 * call + 2 * k + 2 - halves; }, tmo, call);
    asm volatile("s_waitcnt vmcnt(0)" : "+v"(h[0]), "+v"(h[1]), "+v"(h[2]), "+v"(h[3]), "+v"(h[4]), "+v"(h[5]), "+v"(h[6]), "+v"(h[7]), "+v"(h[8]), "+v"(h[9]), "+v"(h[10]), "+v"(h[11]), "+v"(h[12]), "+v"(h[13]), "+v"(h[14]), "+v"(h[15]) :: "memory");
    dec_out(myws + WS_OS + (size_t)((src * 32 + i) * 9 + wave * 2 + k) * OBLK, h, out, off + (unsigned)k * 32 * (GN * 4), sc, tmo, call, scr, lane,
            halves, [&] { return spin<LAST_SLEEP>([&] { return __builtin_amdgcn_readfirstlane(ld_sys(fl)) >= 8 * call + 2 * k + 2; }, tmo, call); });
  }
}

}

__global__ void __launch_bounds__(256, 1) gemm_rs_fused(const Params p) {
  __shared__ __attribute__((aligned(16))) uint8_t lds[LDS_BYTES];
  const int b = blockIdx.x;
  const int lane = threadIdx.x & 63;
  const int wave = __builtin_amdgcn_readfirstlane(threadIdx.x >> 6);
  const int rank = p.rank;
  uint8_t* const myws = p.myws;
  asm volatile("" :: "s"(myws), "s"(rank), "s"(p.x), "s"(p.w));

  const int grp = b & 7, dst = grp ^ rank, ti = 31 - (b >> 3);
  const bool local = grp == 0 || (grp >= 4 && ti >= 24);
  const int set = local ? 7 + (grp != 0) : grp - 1, cb = tile_block(grp, ti);
  const RowSet rs = row_set(set);
  unsigned* const callp = (unsigned*)(myws + WS_CALL + (size_t)b * 128);
  unsigned* const tmo = (unsigned*)(myws + WS_TMO);
  unsigned* const lk = (unsigned*)(myws + WS_LINK) + grp * 64;
  uint8_t* const scr = lds + LDS_PK + wave * 2048;

  PushCtx c;
  c.x = p.x; c.d = dst; c.local = grp == 0;
  c.xl = (unsigned)((lane & 31) * KS + (lane >> 5) * 32);
  const RowSet hs = row_set(grp - 1);
  c.skip0 = grp == 0 ? NTASK : hs.task0; c.skipn = grp == 0 ? 0 : 4 * hs.r;

  unsigned call;
  if (ti >= (grp == 0 ? 8 : 24)) {
    const int w = (b >> 3) * 4 + wave, npw = grp == 0 ? 96 : 32;
    c.w = p.w; c.dst = p.ws[dst]; c.lk = lk; c.rank = rank;
    c.wl = (unsigned)((lane >> 5) * 32 * GN + (lane & 31) * 4);
    c.scr = scr;
    PushState s;
    u32x4 x0[4];
    unsigned cv;
    s.tp = -1; s.tc = task_at(c, w); s.tn = task_at(c, w + npw); s.tl = task_at(c, w + 2 * npw); s.gone = true;
    asm volatile(SGPR_PAD "global_load_dword %0, %1, %2 sc1" : "=v"(cv) : "v"(0u), "s"(callp) : "memory");
    ld_x(c, s.tc, x0);
    __builtin_amdgcn_sched_barrier(0);
    ld_x(c, s.tn, s.nx);
    const RowSet fs = row_set(task_set(s.tc));
    unsigned first = (unsigned)((fs.base * 128 + rank * fs.r * 16 + (s.tc - fs.task0) * 4) << 10) + lane * 16;
    asm volatile("" : "+v"(first));
    asm volatile("s_waitcnt vmcnt(4)" : "+v"(cv), "+v"(x0[0]), "+v"(x0[1]), "+v"(x0[2]), "+v"(x0[3]) :: "memory");
    call = __builtin_amdgcn_readfirstlane(cv) + 1;
    c.par = (unsigned)(WS_DATA + (call & 1) * PAR_BYTES);
#pragma unroll
    for (int q = 0; q < 4; ++q) st16_sys(c.dst, c.par + first + q * 1024, x0[q]);
    asm volatile("s_waitcnt vmcnt(4)" : "+v"(s.nx[0]), "+v"(s.nx[1]), "+v"(s.nx[2]), "+v"(s.nx[3]) :: "memory");
    c.tick0 = (call - 1) * (NTASK - c.skipn - 2 * npw) - 3 * npw;
    c.tag = call & 0xffffu;
    c.call = call;
    s.line = u32x4{0u, 0u, 0u, 0u};
    pack_task(c, x0, s.o, lane, true);
#pragma unroll 1
    while (s.tp < NTASK) {
      if (task_is_w(s.tl)) push_step<true>(c, s, lane);
      else push_step<false>(c, s, lane);
    }
  } else {
    call = __builtin_amdgcn_readfirstlane(ld_agent(callp)) + 1;
  }
  uint8_t* const par = myws + WS_DATA + (size_t)(call & 1) * PAR_BYTES;

#pragma unroll 1
  for (int job = 0; job < 4; ++job) {
    if (job == 1 && grp != 0 && ti < 24) {
      __syncthreads();
      if (grp >= 4) {
        home_tile<true>(p, grp, dst, ti, lds, wave, lane);
      } else {
#pragma unroll 1
        for (int i = ti; i < 32; i += ti >= 16 ? 8 : 32) home_tile<false>(p, grp, dst, i, lds, wave, lane);
      }
    }
    const bool jw = grp == 0 ? job < 3 : job == 1;
    const int beta = grp == 0 ? (ti * 4 + wave) & 63 : (23 - ti) * 4 + wave;
    if (grp == 0 ? (job == 2 && ti >= 24) : (ti >= 24 || job >= 2 || (job == 0 && beta >= 8 * rs.r))) continue;
    const int n = job == 2 ? 24 + (ti & 7) : cb;
    const RowSet js = row_set(grp == 0 ? 7 + (ti >> 4) : set);
    const int bs = div_r(beta, js.r), bu = beta - bs * js.r;
    const int src = !jw ? bs : grp != 0 ? dst : job == 0 ? rank : job == 1 ? rank ^ (4 + (ti & 3)) : rank ^ (1 + (ti >> 3));
    const int t0 = jw ? T_W0 + n * 16 + wave * 4 : js.task0 + bu * 4;
    const uint8_t* const slot = par + XS_BYTES + WT_BYTES + (size_t)(src * NTASK + t0) * ITASK;
    const bool absent = !jw && grp != 0 && src == dst;
    if (!absent && !wait_tasks(slot, tmo, call, lane)) continue;
    if (src != rank && !absent) {
      const unsigned base = jw ? (unsigned)XS_BYTES + (unsigned)(n * (512 << 10) + (src * 8 + wave * 2) * 8192)
                               : (unsigned)(js.base * (128 << 10) + (src * js.r + bu) * 16384);
      unsigned off[4];
#pragma unroll
      for (int t = 0; t < 4; ++t) off[t] = base + (jw ? (t >> 1) * 8192 + (t & 1) * 1024 : t * 4096);
      dec_in4(slot, par, off, jw ? 2048 : 1024, lds + wave * 4096, lane);
      wait_vm<0>();
    }
    st1_agent(jw ? myws + WS_DW + (size_t)(n * 32 + src * 4 + wave) * 4
                 : myws + WS_DX + (size_t)((grp == 0 ? 7 + (ti >> 4) : set) * 128 + beta) * 4, call);
  }
  if (grp != 0 && ti < 24) {
    wait_vm<0>();
    st1_agent(myws + WS_HOME + (size_t)((grp * 32 + ti) * 4 + wave) * 4, call);
    if (grp < 4 && ti >= 16) st1_agent(myws + WS_HOME + (size_t)((grp * 32 + ti + 8) * 4 + wave) * 4, call);
  }

  unsigned* const ctrl = (unsigned*)(lds + LDS_CTRL);
  if (wave == 0) {
    const uint8_t* const fw = myws + WS_DW + (size_t)(cb * 32 + (lane & 31)) * 4;
    const uint8_t* const fx = myws + WS_DX + (size_t)(set * 128 + min(lane, 8 * rs.r - 1)) * 4;
    const uint8_t* const fy = myws + WS_DX + (size_t)(set * 128 + min(64 + lane, 8 * rs.r - 1)) * 4;
    const bool ok = spin([&] { const unsigned a = ld_agent(fw), c = ld_agent(fx), d = ld_agent(fy); return __all(min(min(a, c), d) >= call) != 0; },
                         tmo, call);
    if (lane == 0) ctrl[0] = !ok;
  }
  __syncthreads();
  const bool dead0 = __builtin_amdgcn_readfirstlane(ctrl[0]) != 0;

  if (wave == 0) st1_agent(callp, call);
  const float sc = p.sx[0] * p.sw[0];
  const unsigned off = out_off(rs.row0, block_col(rank, cb), wave, lane);
  float* const ob = p.out[rank];
  uint8_t* const blk = lds + wave * 32768;
  uint8_t* const oslot = p.ws[dst] + WS_OS + (size_t)((rank * 32 + ti) * 9 + wave * 2) * OBLK;
  unsigned* const flag = (unsigned*)(p.ws[dst] + WS_DONE + (size_t)((rank * 32 + ti) * 4 + wave) * 128);
  const int nb = wave_blocks(rs.r, wave);
  const unsigned* const turn = ti < NTURN ? lk + 32 : (const unsigned*)(myws + WS_TURN + (size_t)((grp * 32 + ti) * 4 + wave) * 128);
  unsigned* const next = (unsigned*)(myws + WS_TURN + (size_t)((grp * 32 + ((ti + NTURN) & 31)) * 4 + wave) * 128);
  const int depth = ti < DEEP_T ? DEEP_D : nb == 3 ? OUT_D3 : OUT_D;
  unsigned nl0 = 0, nl1 = 0, nl2 = 0;
  int sent = 0;

  if (!dead0) {
    const uint8_t* const ga = par + (size_t)rs.base * (128 << 10);
    const uint8_t* const gb = par + XS_BYTES + (size_t)cb * (512 << 10);
    if (rs.r == 9) gemm_tile<9>(ga, 9 * 2048, gb, dst, lds, wave, lane);
    else if (nb == 1) gemm_tile<7>(ga, 7 * 2048, gb, dst, lds, wave, lane);
    else gemm_tile<8>(ga, rs.r * 2048, gb, local ? -1 : dst, lds, wave, lane);
    unsigned tnv, tmv;
    asm volatile(SGPR_PAD "global_load_dword %0, %2, %3 sc1\n\tglobal_load_dword %1, %2, %4 sc1"
                 : "=&a"(tnv), "=&a"(tmv) : "v"(0u), "s"(turn), "s"(tmo) : "memory");
    auto landed = [&](unsigned& s_tn, unsigned& s_tm) {
      unsigned t0, t1;
      asm volatile("s_waitcnt vmcnt(0)\n\tv_accvgpr_read_b32 %2, %4\n\tv_accvgpr_read_b32 %3, %5\n\ts_nop 1\n\t"
                   "v_readfirstlane_b32 %0, %2\n\tv_readfirstlane_b32 %1, %3\n\ts_nop 1"
                   : "=&s"(s_tn), "=&s"(s_tm), "=&v"(t0), "=&v"(t1) : "a"(tnv), "a"(tmv) : "memory");
    };
    float s0 = sc;
#pragma unroll 1
    for (int k = 0; k < nb; ++k) {
      float v[4][16];
#pragma unroll
      for (int i = 0; i < 16; ++i) {
        const f32x4 t = *(const f32x4*)(blk + k * 16384 + i * 1024 + lane * 16);
#pragma unroll
        for (int q = 0; q < 4; ++q) v[i >> 2][(i & 3) * 4 + q] = t[q];
      }
      __builtin_amdgcn_fence(__ATOMIC_ACQ_REL, "wavefront");
      if (local) {
        if (k == 0) {
          unsigned tn, t;
          landed(tn, t);
          if (t == call) s0 = __builtin_nanf("");
        }
#pragma unroll
        for (int r = 0; r < 16; ++r)
          st16_out(ob, off + k * 32 * (GN * 4) + row_off(r), f32x4{v[0][r] * s0, v[1][r] * s0, v[2][r] * s0, v[3][r] * s0});
        continue;
      }
      const unsigned nl = enc_out(v, blk + k * 16384, lane);
      __builtin_amdgcn_fence(__ATOMIC_ACQ_REL, "wavefront");
      if (k == 0) nl0 = nl; else if (k == 1) nl1 = nl; else nl2 = nl;
      if (k == 0) {
        unsigned tn, t;
        landed(tn, t);
        if (ti < 24 && nb > 1 && t != call && (tn >= call || __builtin_amdgcn_readfirstlane(ld_agent(turn)) >= call)) {
          send_out(blk, nl, false, oslot, lane, next, call, false, depth, flag, 0);
          sent = 1;
        }
      }
    }
  } else if (local) {
    const float n = __builtin_nanf("");
#pragma unroll 1
    for (int t = 0; t < 32; ++t) st16_out(ob, off + (t >> 4) * 32 * (GN * 4) + row_off(t & 15), f32x4{n, n, n, n});
  }
  if (grp == 0) return;

  const int last8 = grp < 4 ? 24 : 16;
  if (ti >= 24) recv_tile(myws, p.out[rank], grp, dst, ti - 24, 0, 3, wave, lane, call, tmo, sc, scr);
  if (!local) {
    if (sent) {
      wait_vm<0>();
      st1_sys(flag, 8 * call + 2);
    } else {
      spin([&] { return __builtin_amdgcn_readfirstlane(ld_agent(turn)) >= call; }, tmo, call);
    }
    const bool dead = dead0 || __builtin_amdgcn_readfirstlane(ld_agent(tmo)) == call;
#pragma unroll 1
    for (int k = sent; k < nb; ++k) {
      send_out(blk + k * 16384, k == 0 ? nl0 : k == 1 ? nl1 : nl2, dead, oslot + k * OBLK, lane, next, call, k == nb - 1, depth, flag,
               ti >= last8 && k == nb - 1 ? 8 * call + 2 * k + 1 : 0);
      wait_vm<0>();
      st1_sys(flag, 8 * call + 2 * k + 2);
    }
  }
#pragma unroll 1
  for (int duty = 0; duty < 2; ++duty) {
    int i, k0, k1;
    if (duty == 0) {
      if (ti >= last8) continue;
      i = ti + 8; k0 = 0; k1 = i >= last8 ? 1 : 3;
    } else if (ti < 8) {
      i = last8 + ti; k0 = grp < 4 ? 1 : 2; k1 = 3;
    } else if (local) {
      i = ti - 8; k0 = 1; k1 = 2;
    } else {
      continue;
    }
    recv_tile(myws, p.out[rank], grp, dst, i, k0, k1, wave, lane, call, tmo, sc, scr);
  }
}

extern "C" void kernel_launch(int rank, int nranks, ncclComm_t comm, hipStream_t stream,
                              void* const* d_in, const int* in_sizes, int n_in,
                              void* d_out, int out_size, const int* in_dtypes, int out_dtype,
                              void* const* d_out_all, void* const* d_ws_all, size_t ws_bytes) {
  (void)comm; (void)in_sizes; (void)in_dtypes; (void)out_dtype; (void)d_out;
  if (nranks != NR || (unsigned)rank >= (unsigned)NR || n_in != 4 || ws_bytes < WS_NEED || out_size != 512 * GN) {
    fprintf(stderr, "kernel_launch: unexpected problem shape\n");
    return;
  }
  Params p;
  p.x = (const uint8_t*)d_in[0];
  p.w = (const uint8_t*)d_in[1];
  p.sx = (const float*)d_in[2];
  p.sw = (const float*)d_in[3];
  for (int i = 0; i < NR; ++i) {
    p.ws[i] = (uint8_t*)d_ws_all[i];
    p.out[i] = (float*)d_out_all[i];
  }
  p.myws = p.ws[rank];
  p.rank = rank;
  p.pad = 0;
  hipLaunchKernelGGL(gemm_rs_fused, dim3(NWG), dim3(256), 0, stream, p);
  HIP_CHECK(hipGetLastError());
}
```

```cpp
#include <hip/hip_runtime.h>
#include <rccl/rccl.h>
#include <cstdio>
#include <cstdint>

#define HIP_CHECK(expr) do { hipError_t _e = (expr); if (_e != hipSuccess) { \
    fprintf(stderr, "HIP error %s at %s:%d\n", hipGetErrorString(_e), __FILE__, __LINE__); \
    return; } } while (0)

typedef unsigned u32x4 __attribute__((ext_vector_type(4)));
typedef int i32x4 __attribute__((ext_vector_type(4)));
typedef int i32x8 __attribute__((ext_vector_type(8)));
typedef float f32x4 __attribute__((ext_vector_type(4)));
typedef float f32x16 __attribute__((ext_vector_type(16)));

#define SGPR_PAD "s_nop 4\n\t"

namespace {

constexpr int NR = 8;
constexpr int GN = 16384;
constexpr int KS = 512;
constexpr int NWG = 256;
constexpr int KSTEPS = 64;
constexpr int NSTAGE = 6;
constexpr int LDS_CTRL = NSTAGE * 26624;
constexpr int LDS_BYTES = LDS_CTRL + 64;
constexpr int LDS_PK = 9 * 16384;

constexpr int T_W0 = 228, T_OWN = T_W0 + 512, NTASK = T_OWN + 64;

constexpr size_t WS_CALL = 0;
constexpr size_t WS_TMO = 32768;
constexpr size_t WS_DONE = 896u << 10;
constexpr size_t WS_LINK = WS_TMO + 1024;
constexpr size_t WS_DX = WS_LINK + 4096;
constexpr size_t WS_DW = WS_DX + 8192;
constexpr size_t WS_HOME = WS_DW + 4096;
constexpr size_t WS_DATA = 1u << 20;
constexpr size_t XS_BYTES = 10u << 20;
constexpr size_t WT_BYTES = 16u << 20;
constexpr unsigned ITASK = 3328;
constexpr size_t IS_BYTES = 21u << 20;
constexpr size_t PAR_BYTES = XS_BYTES + WT_BYTES + IS_BYTES;
static_assert((size_t)8 * NTASK * ITASK <= IS_BYTES, "");
constexpr size_t OBLK = 17 * 1024;
constexpr size_t WS_OS = WS_DATA + 2 * PAR_BYTES;
constexpr size_t WS_PB = WS_OS + 8 * 32 * 9 * OBLK;
constexpr size_t WS_NEED = WS_PB + 8 * 32 * 9 * 16384;

constexpr unsigned long long DEADLINE_TICKS = 50000000ull;

constexpr int OPEN_AT = 790, OUT_D = 2, OUT_D3 = 3, DEEP_D = 4, DEEP_T = 8;
constexpr int POLL_SLEEP = 6, RX_SLEEP = 16, LAST_SLEEP = 8;

struct Params {
  const uint8_t* x;
  const uint8_t* w;
  const float* sx;
  const float* sw;
  uint8_t* ws[NR];
  float* out[NR];
  uint8_t* myws;
  int rank;
  int pad;
};

__device__ __forceinline__ unsigned ld_sys(const void* ptr) {
  return __hip_atomic_load((const unsigned*)ptr, __ATOMIC_RELAXED, __HIP_MEMORY_SCOPE_SYSTEM);
}
__device__ __forceinline__ unsigned ld_agent(const void* ptr) {
  return __hip_atomic_load((const unsigned*)ptr, __ATOMIC_RELAXED, __HIP_MEMORY_SCOPE_AGENT);
}
__device__ __forceinline__ void st_agent(void* ptr, unsigned v) {
  __hip_atomic_store((unsigned*)ptr, v, __ATOMIC_RELAXED, __HIP_MEMORY_SCOPE_AGENT);
}

template <int SLEEP = POLL_SLEEP, class F>
__device__ __forceinline__ bool spin(F ready, unsigned* tmo, unsigned call) {
  const unsigned long long t0 = __builtin_amdgcn_s_memrealtime();
  for (unsigned n = 1;; ++n) {
    if (ready()) return true;
    if ((n & 3) == 0 && (__builtin_amdgcn_readfirstlane(ld_agent(tmo)) == call || __builtin_amdgcn_s_memrealtime() - t0 > DEADLINE_TICKS)) {
      st_agent(tmo, call);
      return false;
    }
    __builtin_amdgcn_s_sleep(SLEEP);
  }
}

__device__ __forceinline__ void st16_sys(void* base, unsigned off, u32x4 v) {
  asm volatile(SGPR_PAD "global_store_dwordx4 %0, %1, %2 sc0 sc1\n\ts_nop 1" :: "v"(off), "v"(v), "s"(base) : "memory");
}

__device__ __forceinline__ void st16_loc(void* base, unsigned off, u32x4 v) {
  asm volatile(SGPR_PAD "global_store_dwordx4 %0, %1, %2 sc1\n\ts_nop 1" :: "v"(off), "v"(v), "s"(base) : "memory");
}

__device__ __forceinline__ void st1_agent(void* base, unsigned v) {
  asm volatile(SGPR_PAD "s_mov_b64 exec, 1\n\tglobal_store_dword %0, %1, %2 sc1\n\ts_mov_b64 exec, -1"
               :: "v"(0u), "v"(v), "s"(base) : "memory");
}

__device__ __forceinline__ void st1_sys(void* base, unsigned v) {
  asm volatile(SGPR_PAD "s_mov_b64 exec, 1\n\tglobal_store_dword %0, %1, %2 sc0 sc1\n\ts_mov_b64 exec, -1"
               :: "v"(0u), "v"(v), "s"(base) : "memory");
}
__device__ __forceinline__ void st16_sys_m(void* base, unsigned off, u32x4 v, unsigned long long mask) {
  asm volatile(SGPR_PAD "s_mov_b64 exec, %3\n\tglobal_store_dwordx4 %0, %1, %2 sc0 sc1\n\ts_mov_b64 exec, -1\n\ts_nop 0"
               :: "v"(off), "v"(v), "s"(base), "s"(mask) : "memory");
}

template <int N> __device__ __forceinline__ void wait_vm() {
  asm volatile("s_waitcnt vmcnt(%0)" :: "n"(N) : "memory");
}

__device__ __forceinline__ unsigned wave_max_u32(unsigned v) {
  v = max(v, (unsigned)__builtin_amdgcn_update_dpp(0, (int)v, 0x111, 0xf, 0xf, false));
  v = max(v, (unsigned)__builtin_amdgcn_update_dpp(0, (int)v, 0x112, 0xf, 0xf, false));
  v = max(v, (unsigned)__builtin_amdgcn_update_dpp(0, (int)v, 0x114, 0xf, 0xf, false));
  v = max(v, (unsigned)__builtin_amdgcn_update_dpp(0, (int)v, 0x118, 0xf, 0xf, false));
  return max(max((unsigned)__builtin_amdgcn_readlane((int)v, 15), (unsigned)__builtin_amdgcn_readlane((int)v, 31)),
             max((unsigned)__builtin_amdgcn_readlane((int)v, 47), (unsigned)__builtin_amdgcn_readlane((int)v, 63)));
}

__device__ __forceinline__ unsigned irec_slot(unsigned idx) { return idx + 3 + min((__umul24(idx, 37u) + 74u) >> 8, 3u); }

__device__ __forceinline__ unsigned wave_sum_u32(unsigned v) {
  v += (unsigned)__builtin_amdgcn_update_dpp(0, (int)v, 0x111, 0xf, 0xf, false);
  v += (unsigned)__builtin_amdgcn_update_dpp(0, (int)v, 0x112, 0xf, 0xf, false);
  v += (unsigned)__builtin_amdgcn_update_dpp(0, (int)v, 0x114, 0xf, 0xf, false);
  v += (unsigned)__builtin_amdgcn_update_dpp(0, (int)v, 0x118, 0xf, 0xf, false);
  return (unsigned)(__builtin_amdgcn_readlane((int)v, 15) + __builtin_amdgcn_readlane((int)v, 31) +
                    __builtin_amdgcn_readlane((int)v, 47) + __builtin_amdgcn_readlane((int)v, 63));
}
__device__ __forceinline__ unsigned bfi(unsigned m, unsigned a, unsigned b) { return (a & m) | (b & ~m); }
constexpr unsigned HDR_RAW = 0x80000000u;

constexpr unsigned IREC_CAP = 122, IHDR_RAW = 0x8000u;

struct InTask {
  u32x4 d[4];
  unsigned long long mask;
  bool raw;
};

__device__ __forceinline__ void put_records2(unsigned q0, unsigned q1, unsigned tag, unsigned& cnt, uint8_t* scr, unsigned dump) {
  unsigned m = bfi(0x55555555u, q0 | q0 >> 1, q1 | q1 << 1);
  for (unsigned long long act = __ballot(m != 0); act != 0 && cnt <= IREC_CAP; act = __ballot(m != 0)) {
    const unsigned bit = (unsigned)__builtin_ctz(m | 0x80000000u), sh = bit & 30u;
    const unsigned idx = cnt + __builtin_amdgcn_mbcnt_hi((unsigned)(act >> 32), __builtin_amdgcn_mbcnt_lo((unsigned)act, 0u));
    const unsigned sl = 2 * irec_slot(idx), at = ((m != 0) & (idx < IREC_CAP)) ? sl : dump;
    *(unsigned short*)(scr + at) = (unsigned short)(tag | (bit & 1u) << 6 | sh << 1 | ((((bit & 1u) ? q1 : q0) >> sh) & 3u));
    m &= m - 1;
    cnt += __popcll(act);
  }
}

__device__ __forceinline__ void enc_in(const u32x4 (&v)[4], InTask& o, unsigned tag, uint8_t* scr, int lane) {
  unsigned sum = 0;
#pragma unroll
  for (int q = 0; q < 4; ++q) sum = __builtin_amdgcn_udot4(v[q][0] & 0x7f7f7f7fu, 0x01010101u, sum, false);
  const unsigned e0 = (unsigned)min(max((int)((wave_sum_u32(sum) + 10445u) >> 12) - 7, 0), 24);
  const unsigned add = 0x80808080u - e0 * 0x04040404u;
  unsigned pay[12], qq[4], cnt = 0;
#pragma unroll
  for (int q = 0; q < 4; ++q) {
    unsigned R[4], S[4];
#pragma unroll
    for (int k = 0; k < 4; ++k) {
      R[k] = (v[q][k] & 0x7f7f7f7fu) + add;
      S[k] = bfi(0x80808080u, v[q][k], R[k]);
    }
    pay[3 * q] = bfi(0x60606060u, S[3] << 5, S[0]);
    pay[3 * q + 1] = bfi(0x60606060u, S[3] << 3, S[1]);
    pay[3 * q + 2] = bfi(0x20202020u, S[3] << 1, bfi(0x40404040u, S[3] >> 1, S[2]));
    qq[q] = bfi(0xc0c0c0c0u, R[3] << 1, bfi(0x30303030u, R[2] >> 1, bfi(0x0c0c0c0cu, R[1] >> 3, (R[0] >> 5) & 0x03030303u)));
  }
  put_records2(qq[0], qq[1], (unsigned)(lane << 8), cnt, scr, 256 + 2 * lane);
  put_records2(qq[2], qq[3], (unsigned)(lane << 8 | 128), cnt, scr, 256 + 2 * lane);
  *(unsigned short*)(scr + (lane < 4 ? lane * 16 : lane < 6 ? lane * 2 - 6 : 256 + 2 * lane)) =
      (unsigned short)(lane < 4 ? tag : lane == 4 ? cnt : e0);
  __builtin_amdgcn_fence(__ATOMIC_ACQ_REL, "wavefront");
  const u32x4 rec = *(const u32x4*)(scr + (lane & 15) * 16);
  __builtin_amdgcn_fence(__ATOMIC_ACQ_REL, "wavefront");
  o.raw = cnt > IREC_CAP;
  if (o.raw) {
#pragma unroll
    for (int q = 0; q < 4; ++q) o.d[q] = v[q];
    o.mask = ~0ull;
  } else {
#pragma unroll
    for (int q = 0; q < 3; ++q) o.d[q] = u32x4{pay[4 * q], pay[4 * q + 1], pay[4 * q + 2], pay[4 * q + 3]};
    o.d[3] = rec;
    o.mask = (1ull << (((irec_slot(cnt) + 31) >> 5) * 4)) - 16;
  }
}

__device__ __forceinline__ void dec_in4(const uint8_t* slot, uint8_t* out, const unsigned (&off)[4], unsigned stride, uint8_t* scr, int lane) {
  u32x4 y[4][3];
  unsigned rec[4];
  asm volatile(SGPR_PAD "global_load_dword %12, %16, %20 sc0 sc1\n\tglobal_load_dword %13, %16, %20 offset:3328 sc0 sc1\n\t"
               "global_load_dword %14, %17, %20 sc0 sc1\n\tglobal_load_dword %15, %17, %20 offset:3328 sc0 sc1\n\t"
               "global_load_dwordx4 %0, %18, %20 offset:-3072 sc0 sc1\n\tglobal_load_dwordx4 %1, %18, %20 offset:-2048 sc0 sc1\n\t"
               "global_load_dwordx4 %2, %18, %20 offset:-1024 sc0 sc1\n\t"
               "global_load_dwordx4 %3, %18, %20 offset:256 sc0 sc1\n\tglobal_load_dwordx4 %4, %18, %20 offset:1280 sc0 sc1\n\t"
               "global_load_dwordx4 %5, %18, %20 offset:2304 sc0 sc1\n\t"
               "global_load_dwordx4 %6, %19, %20 offset:-3072 sc0 sc1\n\tglobal_load_dwordx4 %7, %19, %20 offset:-2048 sc0 sc1\n\t"
               "global_load_dwordx4 %8, %19, %20 offset:-1024 sc0 sc1\n\t"
               "global_load_dwordx4 %9, %19, %20 offset:256 sc0 sc1\n\tglobal_load_dwordx4 %10, %19, %20 offset:1280 sc0 sc1\n\t"
               "global_load_dwordx4 %11, %19, %20 offset:2304 sc0 sc1\n\ts_waitcnt vmcnt(0)"
               : "=&v"(y[0][0]), "=&v"(y[0][1]), "=&v"(y[0][2]), "=&v"(y[1][0]), "=&v"(y[1][1]), "=&v"(y[1][2]),
                 "=&v"(y[2][0]), "=&v"(y[2][1]), "=&v"(y[2][2]), "=&v"(y[3][0]), "=&v"(y[3][1]), "=&v"(y[3][2]),
                 "=&v"(rec[0]), "=&v"(rec[1]), "=&v"(rec[2]), "=&v"(rec[3])
               : "v"(lane * 4), "v"(2 * ITASK + lane * 4), "v"(ITASK + lane * 16), "v"(3 * ITASK + lane * 16), "s"(slot) : "memory");
#pragma unroll
  for (int t = 0; t < 4; ++t) *(u32x4*)(scr + t * 1024 + lane * 16) = u32x4{0u, 0u, 0u, 0u};
  __builtin_amdgcn_fence(__ATOMIC_ACQ_REL, "wavefront");
  unsigned hdr[4];
#pragma unroll
  for (int t = 0; t < 4; ++t) {
    hdr[t] = (unsigned)__builtin_amdgcn_readlane((int)rec[t], 0) >> 16 | (unsigned)__builtin_amdgcn_readlane((int)rec[t], 1) << 16;
    const unsigned end = irec_slot(hdr[t] & 0x7fffu);
#pragma unroll
    for (int i = 0; i < 2; ++i) {
      const unsigned r16 = rec[t] >> (16 * i);
      const bool isrec = i == 0 ? (lane >= 16 || (lane & 3) != 0) && lane != 1 : lane != 0;
      const unsigned val = isrec && (unsigned)(lane * 2 + i) < end ? (r16 & 3u) << ((r16 >> 1) & 30u) : 0u;
      __hip_atomic_fetch_or((unsigned*)(scr + t * 1024 + ((r16 >> 6) & 255u) * 4), val, __ATOMIC_RELAXED, __HIP_MEMORY_SCOPE_WORKGROUP);
    }
  }
  __builtin_amdgcn_fence(__ATOMIC_ACQ_REL, "wavefront");
#pragma unroll
  for (int t = 0; t < 4; ++t) {
    if (hdr[t] & IHDR_RAW) continue;
    const u32x4 qq = *(const u32x4*)(scr + t * 1024 + lane * 16);
    const unsigned add = ((hdr[t] >> 16) & 31u) * 0x04040404u;
#pragma unroll
    for (int q = 0; q < 4; ++q) {
      const unsigned d0 = y[t][(3 * q) >> 2][(3 * q) & 3], d1 = y[t][(3 * q + 1) >> 2][(3 * q + 1) & 3], d2 = y[t][(3 * q + 2) >> 2][(3 * q + 2) & 3];
      const unsigned d3 = bfi(0x80808080u, d2 << 1, bfi(0x10101010u, d2 >> 1, bfi(0x0c0c0c0cu, d1 >> 3, d0 >> 5)));
      const unsigned X[4] = {bfi(0x60606060u, qq[q] << 5, d0), bfi(0x60606060u, qq[q] << 3, d1), bfi(0x60606060u, qq[q] << 1, d2),
                             bfi(0x60606060u, qq[q] >> 1, d3)};
      u32x4 r;
#pragma unroll
      for (int k = 0; k < 4; ++k) r[k] = bfi(0x80808080u, X[k], (X[k] & 0x7f7f7f7fu) + add);
      asm volatile(SGPR_PAD "global_store_dwordx4 %0, %1, %2 sc1\n\ts_nop 1" :: "v"(off[t] + q * stride + lane * 16), "v"(r), "s"(out) : "memory");
    }
  }
  __builtin_amdgcn_fence(__ATOMIC_ACQ_REL, "wavefront");
}

__device__ __forceinline__ bool wait_tasks(const uint8_t* slot, unsigned* tmo, unsigned call, int lane) {
  const uint8_t* q = slot + ((lane >> 2) & 3) * ITASK + (lane & 3) * 16;
  return spin([&] { return __all((ld_sys(q) & 0xffffu) == (call & 0xffffu)) != 0; }, tmo, call);
}

struct PushCtx {
  const uint8_t* x;
  const uint8_t* w;
  uint8_t* dst;
  unsigned* lk;
  uint8_t* scr;
  unsigned xl, wl;
  unsigned par, tick0, tag, call;
  int skip0, skipn;
  int rank, d;
  bool local;
};

struct RowSet { int r, row0, base, task0; };
__device__ __forceinline__ RowSet row_set(int set) {
  const int a = min(set, 3), o = max(set - 7, 0), b = set - a - o;
  const int own = (set + 1) >> 3, mid = ((set + 5) >> 3) - own;
  return {7 + 2 * mid + own, 224 * mid + 256 * o, 7 * a + 9 * b + 8 * o, 28 * a + 36 * b + 512 * own + 32 * o};
}
__device__ __forceinline__ int block_col(int q, int n) { return ((n & 3) * 4 + (q < 4 ? q : (q ^ n) & 3)) * 1024 + (n >> 2) * 128; }
__device__ __forceinline__ int tile_block(int g, int i) {
  const int c = (i * 43) >> 7, m = i - 3 * c;
  return g < 4 ? i : i >= 24 ? (g - 4) * 8 + i - 24 : 4 * c + m + (m >= g - 4);
}
__device__ __forceinline__ bool task_is_w(int t) { return t >= T_W0 && t < T_OWN; }
__device__ __forceinline__ int task_set(int t) {
  const int lo = min((t * 2341) >> 16, ((t + 24) * 1821) >> 16), hi = 7 + ((t - T_OWN) >> 5);
  return t < T_OWN ? lo : hi;
}

__device__ __forceinline__ int div_r(int v, int r) { return (v * (0x728093 >> (8 * (r - 7)) & 255)) >> 10; }

__device__ __forceinline__ void ld_x(const PushCtx& c, int t, u32x4 (&v)[4]) {
  t = min(t, NTASK - 1);
  const int set = task_set(t);
  const RowSet rs = row_set(set);
  const int p0 = 2 * (t - rs.task0), ks = div_r(p0, rs.r), rg = p0 - ks * rs.r;
  const unsigned o0 = c.xl + (unsigned)((((set < 7 ? set + 1 : 0) ^ c.d) * 512 + rs.row0 + rg * 32) * KS + ks * 64);
  const unsigned o1 = o0 + (unsigned)(rg + 1 == rs.r ? 64 - rg * 32 * KS : 32 * KS);
  asm volatile(SGPR_PAD "global_load_dwordx4 %0, %4, %6\n\tglobal_load_dwordx4 %1, %4, %6 offset:16\n\t"
               "global_load_dwordx4 %2, %5, %6\n\tglobal_load_dwordx4 %3, %5, %6 offset:16"
               : "=&v"(v[0]), "=&v"(v[1]), "=&v"(v[2]), "=&v"(v[3]) : "v"(o0), "v"(o1), "s"(c.x) : "memory");
}

__device__ __forceinline__ void ld_w(const uint8_t* base, unsigned o, unsigned (&w)[16]) {
    asm volatile(SGPR_PAD
                 "global_load_dword %0, %16, %17\n\tv_add_u32 %16, 0x4000, %16\n\t"
                 "global_load_dword %1, %16, %17\n\tv_add_u32 %16, 0x4000, %16\n\t"
                 "global_load_dword %2, %16, %17\n\tv_add_u32 %16, 0x4000, %16\n\t"
                 "global_load_dword %3, %16, %17\n\tv_add_u32 %16, 0x4000, %16\n\t"
                 "global_load_dword %4, %16, %17\n\tv_add_u32 %16, 0x4000, %16\n\t"
                 "global_load_dword %5, %16, %17\n\tv_add_u32 %16, 0x4000, %16\n\t"
                 "global_load_dword %6, %16, %17\n\tv_add_u32 %16, 0x4000, %16\n\t"
                 "global_load_dword %7, %16, %17\n\tv_add_u32 %16, 0x4000, %16\n\t"
                 "global_load_dword %8, %16, %17\n\tv_add_u32 %16, 0x4000, %16\n\t"
                 "global_load_dword %9, %16, %17\n\tv_add_u32 %16, 0x4000, %16\n\t"
                 "global_load_dword %10, %16, %17\n\tv_add_u32 %16, 0x4000, %16\n\t"
                 "global_load_dword %11, %16, %17\n\tv_add_u32 %16, 0x4000, %16\n\t"
                 "global_load_dword %12, %16, %17\n\tv_add_u32 %16, 0x4000, %16\n\t"
                 "global_load_dword %13, %16, %17\n\tv_add_u32 %16, 0x4000, %16\n\t"
                 "global_load_dword %14, %16, %17\n\tv_add_u32 %16, 0x4000, %16\n\t"
                 "global_load_dword %15, %16, %17\n\t"
                 : "=&v"(w[0]), "=&v"(w[1]), "=&v"(w[2]), "=&v"(w[3]), "=&v"(w[4]), "=&v"(w[5]), "=&v"(w[6]), "=&v"(w[7]),
                   "=&v"(w[8]), "=&v"(w[9]), "=&v"(w[10]), "=&v"(w[11]), "=&v"(w[12]), "=&v"(w[13]), "=&v"(w[14]), "=&v"(w[15]), "+v"(o) : "s"(base) : "memory");
    static_assert(GN == 0x4000, "");
}
__device__ __forceinline__ void transpose_w(const unsigned (&w)[16], u32x4 (&v)[4]) {
#pragma unroll
  for (int q = 0; q < 4; ++q) {
    const unsigned d0 = w[4 * q], d1 = w[4 * q + 1], d2 = w[4 * q + 2], d3 = w[4 * q + 3];
    const unsigned t0 = __builtin_amdgcn_perm(d1, d0, 0x05010400u), t1 = __builtin_amdgcn_perm(d1, d0, 0x07030602u);
    const unsigned t2 = __builtin_amdgcn_perm(d3, d2, 0x05010400u), t3 = __builtin_amdgcn_perm(d3, d2, 0x07030602u);
    v[0][q] = __builtin_amdgcn_perm(t2, t0, 0x05040100u);
    v[1][q] = __builtin_amdgcn_perm(t2, t0, 0x07060302u);
    v[2][q] = __builtin_amdgcn_perm(t3, t1, 0x05040100u);
    v[3][q] = __builtin_amdgcn_perm(t3, t1, 0x07060302u);
  }
}

__device__ __forceinline__ void pack_task(const PushCtx& c, const u32x4 (&v)[4], InTask& o, int lane, bool plain) {
  if (plain) {
#pragma unroll
    for (int q = 0; q < 4; ++q) o.d[q] = v[q];
    o.mask = ~0ull;
    o.raw = true;
  } else {
    enc_in(v, o, c.tag, c.scr, lane);
  }
}

__device__ __forceinline__ void send_plain(const PushCtx& c, int t, const u32x4 (&d)[4], int lane) {
  const int u = t - T_W0;
  const bool wr = task_is_w(t);
  const RowSet rs = row_set(task_set(t));
  const unsigned plain = c.par + lane * 16 +
      (wr ? (unsigned)XS_BYTES + (unsigned)((u >> 4) * (512 << 10) + (c.rank * 8 + ((u >> 1) & 7)) * 8192 + (u & 1) * 1024)
          : (unsigned)(rs.base * (128 << 10) + c.rank * rs.r * 16384 + (t - rs.task0) * 4096));
#pragma unroll
  for (int q = 0; q < 4; ++q) st16_sys(c.dst, plain + q * (wr ? 2048 : 1024), d[q]);
}

__device__ __forceinline__ int task_at(const PushCtx& c, int pos) { return min(pos < c.skip0 ? pos : pos + c.skipn, NTASK); }

struct PushState {
  int tp, tc, tn, tl;
  bool gone;
  u32x4 line;
  InTask o;
  u32x4 nx[4];
};

template <bool LDW>
__device__ __forceinline__ void push_step(const PushCtx& c, PushState& s, int lane) {
  unsigned w[16], tk;
  u32x4 x[4];
  const bool draw = s.tl < NTASK;
  asm volatile(SGPR_PAD "s_mov_b64 exec, 1\n\tglobal_atomic_add %0, %1, %2, %3 sc0\n\ts_mov_b64 exec, -1"
               : "=&v"(tk) : "v"(0u), "v"(draw ? 1u : 0u), "s"(c.lk) : "memory");
  if (s.tc == OPEN_AT) st1_agent(c.lk + 32, c.call);
  if constexpr (LDW) {
    const int u = s.tl - T_W0;
    ld_w(c.w, c.wl + (unsigned)((((u >> 1) & 7) * 64 + (u & 1) * 16) * GN + block_col(c.d, u >> 4)), w);
  } else {
    ld_x(c, s.tl, x);
  }
  const unsigned is0 = c.par + (unsigned)(XS_BYTES + WT_BYTES) + (unsigned)(c.rank * NTASK) * ITASK + lane * 16;
  if (s.tp >= 0) st16_sys_m(c.dst, is0 + (unsigned)s.tp * ITASK, s.line, 15ull);
  if (s.tc < NTASK) {
    if (!s.o.raw) {
      const unsigned slot = is0 + (unsigned)s.tc * ITASK;
      asm volatile(SGPR_PAD "global_store_dwordx4 %0, %1, %4 offset:256 sc0 sc1\n\tglobal_store_dwordx4 %0, %2, %4 offset:1280 sc0 sc1\n\t"
                   "global_store_dwordx4 %0, %3, %4 offset:2304 sc0 sc1\n\ts_nop 1"
                   :: "v"(slot), "v"(s.o.d[0]), "v"(s.o.d[1]), "v"(s.o.d[2]), "s"(c.dst) : "memory");
      if (s.o.mask != 0) st16_sys_m(c.dst, slot, s.o.d[3], s.o.mask);
      s.line = s.o.d[3];
    } else {
      if (!s.gone) send_plain(c, s.tc, s.o.d, lane);
      s.line = u32x4{c.tag | IHDR_RAW << 16, 0u, 0u, 0u};
    }
  }
  pack_task(c, s.nx, s.o, lane, c.local || s.tn >= NTASK);
  if constexpr (LDW) {
    asm volatile("s_waitcnt vmcnt(0)" : "+v"(tk), "+v"(w[0]), "+v"(w[1]), "+v"(w[2]), "+v"(w[3]), "+v"(w[4]), "+v"(w[5]), "+v"(w[6]), "+v"(w[7]),
                 "+v"(w[8]), "+v"(w[9]), "+v"(w[10]), "+v"(w[11]), "+v"(w[12]), "+v"(w[13]), "+v"(w[14]), "+v"(w[15]) :: "memory");
    transpose_w(w, s.nx);
  } else {
    asm volatile("s_waitcnt vmcnt(0)" : "+v"(tk), "+v"(x[0]), "+v"(x[1]), "+v"(x[2]), "+v"(x[3]) :: "memory");
#pragma unroll
    for (int q = 0; q < 4; ++q) s.nx[q] = x[q];
  }
  s.tp = s.tc; s.tc = s.tn; s.tn = s.tl; s.gone = false;
  s.tl = draw ? task_at(c, min(__builtin_amdgcn_readfirstlane(tk) - c.tick0, (unsigned)NTASK)) : NTASK;
}

#define GLDS_BITS " sc0 sc1"
__device__ __forceinline__ void glds2(const uint8_t* gbase, unsigned voff, unsigned lds) {
  unsigned keep;
  asm volatile("s_mov_b32 %0, m0\n\ts_mov_b32 m0, %3\n\ts_nop 2\n\t"
               "global_load_lds_dwordx4 %1, %2" GLDS_BITS "\n\t"
               "global_load_lds_dwordx4 %1, %2 offset:1024" GLDS_BITS "\n\t"
               "s_mov_b32 m0, %0"
               : "=&s"(keep) : "v"(voff), "s"(gbase), "s"(lds));
}
__device__ __forceinline__ void glds2o(const uint8_t* gbase, unsigned voff, unsigned lds) {
  unsigned keep;
  asm volatile("s_mov_b32 %0, m0\n\ts_mov_b32 m0, %3\n\ts_nop 2\n\t"
               "global_load_lds_dwordx4 %1, %2 offset:2048" GLDS_BITS "\n\t"
               "global_load_lds_dwordx4 %1, %2 offset:3072" GLDS_BITS "\n\t"
               "s_mov_b32 m0, %0"
               : "=&s"(keep) : "v"(voff), "s"(gbase), "s"(lds));
}
__device__ __forceinline__ void glds_half(const uint8_t* gbase, unsigned voff, unsigned lds) {
  unsigned keep;
  asm volatile("s_mov_b32 %0, m0\n\ts_mov_b32 m0, %3\n\ts_mov_b32 exec_hi, 0\n\ts_nop 1\n\t"
               "global_load_lds_dwordx4 %1, %2" GLDS_BITS "\n\t"
               "s_mov_b32 exec_hi, -1\n\ts_mov_b32 m0, %0"
               : "=&s"(keep) : "v"(voff), "s"(gbase), "s"(lds));
}

struct Frags { i32x8 a[2], b[4], a9, b9; };

__device__ __forceinline__ i32x8 ld_frag(const uint8_t* q) {
  const i32x4 lo = *(const i32x4*)q, hi = *(const i32x4*)(q + 1024);
  return __builtin_shufflevector(lo, hi, 0, 1, 2, 3, 4, 5, 6, 7);
}

#define MFMA(mi, j) acc[mi][j] = __builtin_amdgcn_mfma_scale_f32_32x32x64_f8f6f4(cur.a[mi], cur.b[j], acc[mi][j], 1, 1, 0, 0, 0, 0)
#define SB() __builtin_amdgcn_sched_barrier(0)

struct Ring {
  const uint8_t* ga;
  const uint8_t* gb;
  unsigned astep, ahop;
  int togo;
  unsigned va, vb, v9;
  unsigned wr, wr_end;
  unsigned wb, w9;
  unsigned rd;
  const uint8_t* qa;
  const uint8_t* qb;
  const uint8_t* q9;
  const uint8_t* la;
  const uint8_t* lb;
  const uint8_t* l9;
};

template <int MODE, int LEFT, bool DMA, bool RD>
__device__ __forceinline__ void kstep(f32x16 (&acc)[2][4], f32x16& acc9, const Frags& cur, Frags& nxt, Ring& g) {
  constexpr bool NINE = MODE == 9, TWO = MODE != 7;
  constexpr int STAGE = NINE ? 26624 : 24576;
  MFMA(0, 0); SB();
  asm volatile("s_waitcnt vmcnt(%0) lgkmcnt(0)\n\ts_barrier" :: "n"(LEFT * (NINE ? 7 : 6)) : "memory");
  SB();
  MFMA(0, 1); SB();
  if constexpr (RD) { nxt.a[0] = ld_frag(g.qa); nxt.b[0] = ld_frag(g.qb); }
  unsigned sa = 0, sb = 0;
  if constexpr (DMA) { const bool hop = --g.togo == 0; sa = hop ? g.ahop : g.astep; sb = hop ? 9 * 8192 : 8192; asm volatile("" : "+s"(sa), "+s"(sb)); }
  SB();
  MFMA(0, 2); SB();
  if constexpr (DMA) glds2(g.ga, g.va, g.wr);
  SB();
  MFMA(0, 3); SB();
  if constexpr (RD) { nxt.b[1] = ld_frag(g.qb + 2048); nxt.b[2] = ld_frag(g.qb + 4096); }
  SB();
  if constexpr (TWO) MFMA(1, 0);
  SB();
  if constexpr (DMA) glds2(g.gb, g.vb, g.wr + g.wb);
  SB();
  if constexpr (TWO) MFMA(1, 1);
  SB();
  if constexpr (RD) { nxt.b[3] = ld_frag(g.qb + 6144); if constexpr (TWO) nxt.a[1] = ld_frag(g.qa + 2048); }
  g.rd = g.rd == (NSTAGE - 1) * STAGE ? 0 : g.rd + STAGE;
  g.qa = g.la + g.rd;
  if constexpr (!NINE) g.qb = g.lb + g.rd;
  SB();
  if constexpr (TWO) MFMA(1, 2);
  SB();
  if constexpr (DMA) glds2o(g.ga, g.va, g.wr);
  SB();
  if constexpr (NINE) {
    if constexpr (TWO) MFMA(1, 3);
    SB();
    if constexpr (RD) { nxt.a9 = ld_frag(g.qb - 2048); nxt.b9 = ld_frag(g.q9); }
    g.qb = g.lb + g.rd; g.q9 = g.l9 + g.rd;
    if constexpr (DMA) g.gb += sb;
    SB();
    acc9 = __builtin_amdgcn_mfma_scale_f32_32x32x64_f8f6f4(cur.a9, cur.b9, acc9, 1, 1, 0, 0, 0, 0); SB();
    if constexpr (DMA) glds_half(g.ga, g.v9, g.wr + g.w9);
  } else {
    if constexpr (TWO) MFMA(1, 3);
    SB();
    if constexpr (DMA) g.gb += sb;
  }
  if constexpr (DMA) {
    g.ga += sa;
    g.wr += STAGE; g.wr = g.wr == g.wr_end ? g.wr - NSTAGE * STAGE : g.wr;
  }
  SB();
}

template <int MODE>
__device__ __forceinline__ void gemm_tile(const uint8_t* ga, unsigned astep, const uint8_t* gb, int skip, uint8_t* lds, int wave, int lane) {
  constexpr bool NINE = MODE == 9, TWO = MODE != 7;
  constexpr int STAGE = NINE ? 26624 : 24576, BOFF = STAGE - 8192;
  f32x16 acc[2][4], acc9;
  Ring g;
  g.ga = ga + (skip == 0 ? 8 * astep : 0); g.gb = gb + (skip == 0 ? 8 * 8192 : 0); g.astep = astep; g.ahop = 9 * astep;
  g.togo = skip > 0 ? 8 * skip - NSTAGE : -1;
  g.va = lane * 16 + wave * 4096; g.vb = lane * 16 + wave * 2048; g.v9 = 16384 + lane * 16 + wave * 512;
  g.wr = (unsigned)(uintptr_t)lds + wave * 4096; g.wr_end = g.wr + NSTAGE * STAGE;
  g.wb = BOFF - wave * 2048; g.w9 = 16384 - wave * 3584;
#pragma unroll
  for (int s = 0; s < NSTAGE; ++s) {
    glds2(g.ga, g.va, g.wr + s * STAGE);
    glds2(g.gb, g.vb, g.wr + s * STAGE + g.wb);
    glds2o(g.ga, g.va, g.wr + s * STAGE);
    if constexpr (NINE) glds_half(g.ga, g.v9, g.wr + s * STAGE + g.w9);
    g.ga += astep; g.gb += 8192;
  }
#pragma unroll
  for (int r = 0; r < 16; ++r) {
    acc9[r] = 0.f;
#pragma unroll
    for (int j = 0; j < 4; ++j) { acc[0][j][r] = 0.f; acc[1][j][r] = 0.f; }
  }
  g.la = lds + lane * 16 + wave * 4096; g.lb = lds + BOFF + lane * 16; g.l9 = g.lb + wave * 2048;
  asm volatile("s_waitcnt vmcnt(%0)\n\ts_barrier" :: "n"(5 * (NINE ? 7 : 6)) : "memory");
  Frags f0, f1;
  f0.a[0] = ld_frag(g.la); f0.b[0] = ld_frag(g.lb); f0.b[1] = ld_frag(g.lb + 2048);
  f0.b[2] = ld_frag(g.lb + 4096); f0.b[3] = ld_frag(g.lb + 6144);
  if constexpr (TWO) f0.a[1] = ld_frag(g.la + 2048);
  if constexpr (NINE) { f0.a9 = ld_frag(g.lb - 2048); f0.b9 = ld_frag(g.l9); }
  g.rd = STAGE; g.qa = g.la + STAGE; g.qb = g.lb + STAGE; g.q9 = g.l9 + STAGE;
#pragma unroll 1
  for (int s = skip < 0 ? 0 : 8; s < KSTEPS - NSTAGE; s += 2) {
    kstep<MODE, 4, true, true>(acc, acc9, f0, f1, g);
    kstep<MODE, 4, true, true>(acc, acc9, f1, f0, g);
  }
  kstep<MODE, 4, false, true>(acc, acc9, f0, f1, g);
  kstep<MODE, 3, false, true>(acc, acc9, f1, f0, g);
  kstep<MODE, 2, false, true>(acc, acc9, f0, f1, g);
  kstep<MODE, 1, false, true>(acc, acc9, f1, f0, g);
  kstep<MODE, 0, false, true>(acc, acc9, f0, f1, g);
  auto park = [&](const f32x16& a, int i) {
#pragma unroll
    for (int q = 0; q < 4; ++q)
      *(f32x4*)(lds + wave * 32768 + (i * 4 + q) * 1024 + lane * 16) = f32x4{a[4 * q], a[4 * q + 1], a[4 * q + 2], a[4 * q + 3]};
  };
  {
    const Frags& cur = f1;
    MFMA(0, 0); SB();
    asm volatile("s_waitcnt lgkmcnt(0)\n\ts_barrier" ::: "memory");
    SB();
    MFMA(0, 1); SB();
    MFMA(0, 2); SB();
    park(acc[0][0], 0); SB();
    MFMA(0, 3); SB();
    park(acc[0][1], 1); SB();
    if constexpr (TWO) MFMA(1, 0);
    SB();
    park(acc[0][2], 2); SB();
    if constexpr (TWO) MFMA(1, 1);
    SB();
    park(acc[0][3], 3); SB();
    if constexpr (TWO) {
      MFMA(1, 2); SB();
      park(acc[1][0], 4); SB();
      MFMA(1, 3); SB();
      park(acc[1][1], 5); SB();
      if constexpr (NINE) { acc9 = __builtin_amdgcn_mfma_scale_f32_32x32x64_f8f6f4(cur.a9, cur.b9, acc9, 1, 1, 0, 0, 0, 0); SB(); }
      park(acc[1][2], 6);
      park(acc[1][3], 7);
    }
  }
  if constexpr (NINE) {
#pragma unroll
    for (int q = 0; q < 4; ++q)
      *(f32x4*)(lds + 8 * 16384 + wave * 4096 + q * 1024 + lane * 16) = f32x4{acc9[4 * q], acc9[4 * q + 1], acc9[4 * q + 2], acc9[4 * q + 3]};
    __syncthreads();
  }
  __builtin_amdgcn_fence(__ATOMIC_ACQ_REL, "wavefront");
}

constexpr unsigned REC_CAP = 510, HDR_B = 0x40000000u;

__device__ __forceinline__ void put_records4(const unsigned (&q)[4], unsigned tag, unsigned& cnt, uint8_t* scr, unsigned dump) {
  unsigned m = 0;
#pragma unroll
  for (int k = 0; k < 4; ++k)
    m |= ((((q[k] & 0x77777777u) + 0x77777777u) | q[k]) & 0x88888888u) >> (3 - k);
  for (unsigned long long act = __ballot(m != 0); act != 0 && cnt <= REC_CAP; act = __ballot(m != 0)) {
    const unsigned bit = (unsigned)__builtin_ctz(m | 0x80000000u), sh = bit & 28u;
    const unsigned m1 = (unsigned)__builtin_amdgcn_sbfe((int)bit, 0, 1), m2 = (unsigned)__builtin_amdgcn_sbfe((int)bit, 1, 1);
    const unsigned w = bfi(m2, bfi(m1, q[3], q[2]), bfi(m1, q[1], q[0]));
    const unsigned idx = cnt + __builtin_amdgcn_mbcnt_hi((unsigned)(act >> 32), __builtin_amdgcn_mbcnt_lo((unsigned)act, 0u));
    const unsigned at = (m != 0 && idx < REC_CAP) ? 4 + 2 * idx : dump;
    *(unsigned short*)(scr + at) = (unsigned short)(tag | (bit & 3u) << 7 | sh << 2 | ((w >> sh) & 15u));
    m &= m - 1;
    cnt += __popcll(act);
  }
}

template <bool MODEB>
__device__ __forceinline__ bool enc_body(const float (&v)[4][16], float up, unsigned& cnt, uint8_t* blk, int lane) {
  unsigned bad = 0, low = 0;
#pragma unroll
  for (int h = 0; h < 2; ++h) {
    unsigned G[32];
#pragma unroll
    for (int j = 0; j < 4; ++j)
#pragma unroll
      for (int r = 0; r < 8; ++r) {
        const float x = v[j][8 * h + r] * up;
        unsigned g = (unsigned)(int)x;
        bad = __builtin_amdgcn_bitop3_b32(__builtin_bit_cast(unsigned, (float)(int)g), __builtin_bit_cast(unsigned, x), bad, 0xbe);
        low |= g;
        if (MODEB) g = g << 1 | (((g ^ g << 1) >> 27) & 16u);
        G[4 * r + j] = g;
      }
    unsigned q[4];
#pragma unroll
    for (int k = 0; k < 4; ++k) {
      const unsigned* g = G + 8 * k;
      const unsigned b0 = __builtin_amdgcn_perm(__builtin_amdgcn_perm(g[3], g[2], 0x0c0c0400u), __builtin_amdgcn_perm(g[1], g[0], 0x0c0c0400u), 0x05040100u);
      const unsigned b1 = __builtin_amdgcn_perm(__builtin_amdgcn_perm(g[7], g[6], 0x0c0c0400u), __builtin_amdgcn_perm(g[5], g[4], 0x0c0c0400u), 0x05040100u);
      q[k] = (b0 >> 4) | b1;
    }
#pragma unroll
    for (int j = 0; j < 2; ++j) {
      unsigned d[12];
#pragma unroll
      for (int g = 0; g < 4; ++g) {
        const unsigned* x = G + 16 * j + 4 * g;
        d[3 * g] = __builtin_amdgcn_perm(x[1], x[0], 0x05030201u);
        d[3 * g + 1] = __builtin_amdgcn_perm(x[2], x[1], 0x06050302u);
        d[3 * g + 2] = __builtin_amdgcn_perm(x[3], x[2], 0x07060503u);
      }
#pragma unroll
      for (int i = 0; i < 3; ++i)
        *(u32x4*)(blk + (1 + 6 * h + 3 * j + i) * 1024 + lane * 16) = u32x4{d[4 * i], d[4 * i + 1], d[4 * i + 2], d[4 * i + 3]};
    }
    put_records4(q, (unsigned)(lane << 10 | h << 9), cnt, blk, 13 * 1024 + 2 * lane);
  }
  return __ballot((bad | (low & 15u)) != 0) == 0 && cnt <= REC_CAP;
}

__device__ __forceinline__ unsigned enc_out(const float (&v)[4][16], uint8_t* blk, int lane) {
  float m = 0.f;
#pragma unroll
  for (int j = 0; j < 4; ++j)
#pragma unroll
    for (int r = 0; r < 16; r += 2) asm("v_max3_f32 %0, |%1|, |%2|, %0" : "+v"(m) : "v"(v[j][r]), "v"(v[j][r + 1]));
  const int e = max((int)(wave_max_u32(__builtin_bit_cast(unsigned, m)) >> 23) - 157, -126);
  const float up = __builtin_bit_cast(float, (unsigned)(127 - min(e, 0)) << 23);
  int ntop = 0, nlow = 0;
#pragma unroll
  for (int r = 0; r < 8; ++r) {
    const unsigned g = (unsigned)(int)(v[r & 3][2 * r] * up);
    ntop += __popcll(__ballot((int)(g ^ (g << 1)) < 0));
    nlow += __popcll(__ballot((g & 128u) != 0));
  }
  unsigned cnt = 0;
  const bool modeb = ntop < nlow;
  const bool fits = e <= 0 && (modeb ? enc_body<true>(v, up, cnt, blk, lane) : enc_body<false>(v, up, cnt, blk, lane));
  if (!fits) {
#pragma unroll
    for (int i = 0; i < 16; ++i) {
      const float* t = &v[i >> 2][(i & 3) * 4];
      *(f32x4*)(blk + i * 1024 + lane * 16) = f32x4{t[0], t[1], t[2], t[3]};
    }
    return 0;
  }
  *(unsigned*)(blk + (lane == 0 ? 0 : 13 * 1024 + 4 * lane)) = cnt | (unsigned)(e + 127) << 16 | (modeb ? HDR_B : 0u);
  return ((4 + 2 * cnt + 63) >> 6) * 4;
}

__device__ __forceinline__ void st16_out(void* base, unsigned off, f32x4 v) { st16_loc(base, off, __builtin_bit_cast(u32x4, v)); }

__device__ __forceinline__ constexpr unsigned row_off(int r) { return (unsigned)((r & 3) + 8 * (r >> 2)) * (GN * 4); }

__device__ __forceinline__ void send_out(const uint8_t* blk, unsigned nl, bool dead, uint8_t* slot, int lane, int depth,
                                         unsigned* flag, unsigned half) {
  const bool raw = nl == 0 || dead;
  const u32x4 h = *(const u32x4*)(blk + lane * 16);
  st16_sys_m(slot, lane * 16, raw ? u32x4{HDR_RAW, 0u, 0u, 0u} : h, raw ? 15ull : nl >= 64 ? ~0ull : (1ull << nl) - 1);
  const int n = raw ? 16 : 12;
  const uint8_t* const src = blk + (raw ? 0 : 1024) + lane * 16;
  const unsigned nan = dead ? 0x7fc00000u : 0u;
#pragma unroll 1
  for (int i = 0; i < n; ++i) {
    const u32x4 v = *(const u32x4*)(src + i * 1024) | nan;
    if (depth == OUT_D) wait_vm<OUT_D - 1>(); else if (depth == OUT_D3) wait_vm<OUT_D3 - 1>(); else wait_vm<DEEP_D - 1>();
    if (half != 0 && i == 6) { wait_vm<0>(); st1_sys(flag, half); }
    st16_sys(slot, (unsigned)(1024 + i * 1024 + lane * 16), v);
  }
}

template <int H>
__device__ __forceinline__ void dec_rows(const u32x4 (&y)[6], const u32x4& qq, bool modeb, float down, const f32x4 (&h)[16], float* ob,
                                         unsigned off, float sc) {
#pragma unroll
  for (int g = 0; g < 8; ++g) {
    const unsigned d0 = y[(3 * g) >> 2][(3 * g) & 3], d1 = y[(3 * g + 1) >> 2][(3 * g + 1) & 3], d2 = y[(3 * g + 2) >> 2][(3 * g + 2) & 3];
    const unsigned q = ((g & 1) ? qq[g >> 1] : qq[g >> 1] << 4) & 0xf0f0f0f0u;
    unsigned G[4] = {__builtin_amdgcn_perm(d0, q, 0x06050400u), __builtin_amdgcn_perm(d1, d0, 0x0504030cu) | ((q >> 8) & 255u),
                     __builtin_amdgcn_perm(d2, d1, 0x0403020cu) | ((q >> 16) & 255u), __builtin_amdgcn_perm(d2, q, 0x07060503u)};
    f32x4 v;
#pragma unroll
    for (int j = 0; j < 4; ++j) {
      if (modeb) G[j] = ((unsigned)((int)G[j] >> 1) & ~8u) ^ (G[j] << 27);
      v[j] = (__fmul_rn((float)(int)G[j], down) + h[4 * j + 2 * H + (g >> 2)][g & 3]) * sc;
    }
    st16_out(ob, off + row_off(8 * H + g), v);
  }
}

__device__ __forceinline__ void ld6_sys(const uint8_t* base, unsigned voff, u32x4 (&y)[6]) {
  asm volatile(SGPR_PAD "global_load_dwordx4 %0, %6, %7 offset:-3072 sc0 sc1\n\tglobal_load_dwordx4 %1, %6, %7 offset:-2048 sc0 sc1\n\t"
               "global_load_dwordx4 %2, %6, %7 offset:-1024 sc0 sc1\n\tglobal_load_dwordx4 %3, %6, %7 sc0 sc1\n\t"
               "global_load_dwordx4 %4, %6, %7 offset:1024 sc0 sc1\n\tglobal_load_dwordx4 %5, %6, %7 offset:2048 sc0 sc1\n\ts_waitcnt vmcnt(0)"
               : "=&v"(y[0]), "=&v"(y[1]), "=&v"(y[2]), "=&v"(y[3]), "=&v"(y[4]), "=&v"(y[5]) : "v"(voff + 3072), "s"(base) : "memory");
}

template <class F>
__device__ __forceinline__ void dec_out(const uint8_t* slot, const f32x4 (&h)[16], float* ob, unsigned off, float sc, unsigned* tmo, unsigned call,
                                        uint8_t* scr, int lane, bool halves, F rest) {
  u32x4 rec, y[6], z[6];
  unsigned tmv;
  asm volatile(SGPR_PAD "global_load_dword %13, %16, %18 sc1\n\t"
               "global_load_dwordx4 %0, %14, %17 offset:-4096 sc0 sc1\n\tglobal_load_dwordx4 %1, %14, %17 offset:-3072 sc0 sc1\n\t"
               "global_load_dwordx4 %2, %14, %17 offset:-2048 sc0 sc1\n\tglobal_load_dwordx4 %3, %14, %17 offset:-1024 sc0 sc1\n\t"
               "global_load_dwordx4 %4, %14, %17 sc0 sc1\n\tglobal_load_dwordx4 %5, %14, %17 offset:1024 sc0 sc1\n\t"
               "global_load_dwordx4 %6, %14, %17 offset:2048 sc0 sc1\n\tglobal_load_dwordx4 %7, %14, %17 offset:3072 sc0 sc1\n\t"
               "global_load_dwordx4 %8, %15, %17 offset:-4096 sc0 sc1\n\tglobal_load_dwordx4 %9, %15, %17 offset:-3072 sc0 sc1\n\t"
               "global_load_dwordx4 %10, %15, %17 offset:-2048 sc0 sc1\n\tglobal_load_dwordx4 %11, %15, %17 offset:-1024 sc0 sc1\n\t"
               "global_load_dwordx4 %12, %15, %17 sc0 sc1\n\ts_waitcnt vmcnt(0)"
               : "=&v"(rec), "=&v"(y[0]), "=&v"(y[1]), "=&v"(y[2]), "=&v"(y[3]), "=&v"(y[4]), "=&v"(y[5]), "=&v"(z[0]),
                 "=&v"(z[1]), "=&v"(z[2]), "=&v"(z[3]), "=&v"(z[4]), "=&v"(z[5]), "=&v"(tmv)
               : "v"(4096 + lane * 16), "v"(12288 + lane * 16), "v"(0u), "s"(slot), "s"(tmo) : "memory");
  const unsigned hdr = __builtin_amdgcn_readfirstlane(rec[0]);
  bool ok = __builtin_amdgcn_readfirstlane(tmv) != call;
  if (ok && !(hdr & HDR_RAW)) {
    const unsigned cnt = hdr & 0xffffu;
    const float down = __builtin_bit_cast(float, ((hdr >> 16) & 255u) << 23);
    *(u32x4*)(scr + lane * 32) = u32x4{0u, 0u, 0u, 0u};
    *(u32x4*)(scr + lane * 32 + 16) = u32x4{0u, 0u, 0u, 0u};
    __builtin_amdgcn_fence(__ATOMIC_ACQ_REL, "wavefront");
#pragma unroll
    for (int i = 0; i < 8; ++i) {
      const unsigned r16 = (rec[i >> 1] >> (16 * (i & 1))) & 0xffffu;
      const unsigned val = (unsigned)(lane * 8 + i - 2) < cnt ? (r16 & 15u) << ((r16 >> 2) & 28u) : 0u;
      __hip_atomic_fetch_or((unsigned*)(scr + (r16 >> 7) * 4), val, __ATOMIC_RELAXED, __HIP_MEMORY_SCOPE_WORKGROUP);
    }
    __builtin_amdgcn_fence(__ATOMIC_ACQ_REL, "wavefront");
    const u32x4 q0 = *(const u32x4*)(scr + lane * 32), q1 = *(const u32x4*)(scr + lane * 32 + 16);
    __builtin_amdgcn_fence(__ATOMIC_ACQ_REL, "wavefront");
    const bool modeb = (hdr & HDR_B) != 0;
    dec_rows<0>(y, q0, modeb, down, h, ob, off, sc);
    if (halves) {
      ok = rest();
      ld6_sys(slot, 7168 + lane * 16, z);
    }
    if (ok) dec_rows<1>(z, q1, modeb, down, h, ob, off, sc);
  } else if (ok) {
    if (halves) {
      ok = rest();
      ld6_sys(slot, 7168 + lane * 16, z);
    }
    u32x4 w[6];
    ld6_sys(slot, 11264 + lane * 16, w);
#pragma unroll
    for (int r = 0; r < 16; ++r) {
      f32x4 t;
#pragma unroll
      for (int j = 0; j < 4; ++j) {
        const int i = 4 * j + (r >> 2);
        t[j] = (__builtin_bit_cast(float, i < 6 ? y[i][r & 3] : i < 12 ? z[i - 6][r & 3] : w[i - 10][r & 3]) + h[i][r & 3]) * sc;
      }
      st16_out(ob, off + row_off(r), t);
    }
  }
  if (!ok) {
    const float n = __builtin_nanf("");
#pragma unroll 1
    for (int r = 0; r < 16; ++r) st16_out(ob, off + row_off(r), f32x4{n, n, n, n});
  }
}

__device__ __forceinline__ int wave_blocks(int r, int wave) { return wave < 3 ? 2 : r - 6; }
__device__ __forceinline__ unsigned out_off(int row0, int col, int wave, int lane) {
  return (unsigned)(((row0 + wave * 64 + 4 * (lane >> 5)) * GN + col + 4 * (lane & 31)) * 4);
}

template <bool NINE>
__device__ __forceinline__ void home_tile(const Params& p, int g, int src, int i, uint8_t* lds, int wave, int lane) {
  const int col = block_col(src, tile_block(g, i));
  const uint8_t* const xg = p.x + (size_t)(p.rank * 512 + (lane & 31)) * KS + (lane >> 5) * 32;
  auto xa = [&](int rho, int s) { return *(const i32x8*)(xg + rho * 32 * KS + s * 64); };
  i32x8 a0[8], a1[8], a9[8];
  {
    unsigned w[4][16];
    const unsigned o = (unsigned)((wave * 128 + (lane >> 5) * 32) * GN + col + (lane & 31) * 4);
#pragma unroll
    for (int t = 0; t < 4; ++t) ld_w(p.w, o + (unsigned)((t >> 1) * 64 + (t & 1) * 16) * GN, w[t]);
#pragma unroll
    for (int s = 0; s < 8; ++s) {
      a0[s] = xa((NINE ? 7 : 0) + wave * 2, s);
      a1[s] = xa((NINE ? 8 : 1) + wave * 2, s);
    }
    asm volatile("s_waitcnt vmcnt(0)" : "+v"(w[0][0]), "+v"(w[0][1]), "+v"(w[0][2]), "+v"(w[0][3]), "+v"(w[0][4]), "+v"(w[0][5]), "+v"(w[0][6]), "+v"(w[0][7]), "+v"(w[0][8]), "+v"(w[0][9]), "+v"(w[0][10]), "+v"(w[0][11]), "+v"(w[0][12]), "+v"(w[0][13]), "+v"(w[0][14]), "+v"(w[0][15]) :: "memory");
    asm volatile("" : "+v"(w[1][0]), "+v"(w[1][1]), "+v"(w[1][2]), "+v"(w[1][3]), "+v"(w[1][4]), "+v"(w[1][5]), "+v"(w[1][6]), "+v"(w[1][7]), "+v"(w[1][8]), "+v"(w[1][9]), "+v"(w[1][10]), "+v"(w[1][11]), "+v"(w[1][12]), "+v"(w[1][13]), "+v"(w[1][14]), "+v"(w[1][15]) :: "memory");
    asm volatile("" : "+v"(w[2][0]), "+v"(w[2][1]), "+v"(w[2][2]), "+v"(w[2][3]), "+v"(w[2][4]), "+v"(w[2][5]), "+v"(w[2][6]), "+v"(w[2][7]), "+v"(w[2][8]), "+v"(w[2][9]), "+v"(w[2][10]), "+v"(w[2][11]), "+v"(w[2][12]), "+v"(w[2][13]), "+v"(w[2][14]), "+v"(w[2][15]) :: "memory");
    asm volatile("" : "+v"(w[3][0]), "+v"(w[3][1]), "+v"(w[3][2]), "+v"(w[3][3]), "+v"(w[3][4]), "+v"(w[3][5]), "+v"(w[3][6]), "+v"(w[3][7]), "+v"(w[3][8]), "+v"(w[3][9]), "+v"(w[3][10]), "+v"(w[3][11]), "+v"(w[3][12]), "+v"(w[3][13]), "+v"(w[3][14]), "+v"(w[3][15]) :: "memory");
#pragma unroll
    for (int t = 0; t < 4; ++t) {
      u32x4 v[4];
      transpose_w(w[t], v);
#pragma unroll
      for (int j = 0; j < 4; ++j) *(u32x4*)(lds + (wave * 2 + (t >> 1)) * 8192 + j * 2048 + (t & 1) * 1024 + lane * 16) = v[j];
    }
  }
  if constexpr (NINE) {
#pragma unroll
    for (int s = 0; s < 8; ++s) a9[s] = xa(15, s);
  }
  __syncthreads();
  f32x16 h[2][4], h9;
#pragma unroll
  for (int r = 0; r < 16; ++r) {
    h9[r] = 0.f;
#pragma unroll
    for (int j = 0; j < 4; ++j) { h[0][j][r] = 0.f; h[1][j][r] = 0.f; }
  }
  i32x8 bn = ld_frag(lds + lane * 16);
#pragma unroll
  for (int t = 0; t < 32; ++t) {
    const i32x8 bc = bn;
    if (t < 31) bn = ld_frag(lds + (t + 1) * 2048 + lane * 16);
    i32x8 b9;
    if (NINE && (t & 3) == 0) b9 = ld_frag(lds + (t + wave) * 2048 + lane * 16);
    SB();
    h[0][t & 3] = __builtin_amdgcn_mfma_scale_f32_32x32x64_f8f6f4(a0[t >> 2], bc, h[0][t & 3], 1, 1, 0, 0, 0, 0);
    h[1][t & 3] = __builtin_amdgcn_mfma_scale_f32_32x32x64_f8f6f4(a1[t >> 2], bc, h[1][t & 3], 1, 1, 0, 0, 0, 0);
    if (NINE && (t & 3) == 0) h9 = __builtin_amdgcn_mfma_scale_f32_32x32x64_f8f6f4(a9[t >> 2], b9, h9, 1, 1, 0, 0, 0, 0);
    SB();
  }
  uint8_t* const pb = p.myws + WS_PB + (size_t)((g * 32 + i) * 9) * 16384;
#pragma unroll
  for (int q = 0; q < (NINE ? 36 : 32); ++q) {
    const f32x16& t = q < 32 ? h[q >> 4][(q >> 2) & 3] : h9;
    st16_out(pb, (unsigned)((q < 32 ? wave * 32 + q : 128 - 32 + wave * 4 + q) * 1024 + lane * 16),
             f32x4{t[4 * (q & 3)], t[4 * (q & 3) + 1], t[4 * (q & 3) + 2], t[4 * (q & 3) + 3]});
  }
  __syncthreads();
}

__device__ __forceinline__ bool in_halves(int g, int i, int k, int nb) { return i >= (g < 4 ? 24 : 16) && k == nb - 1; }

__device__ __forceinline__ void block_at(int g, int t, int& i, int& w, int& k) {
  if (g < 4) {
    const int s = t < 56 ? 0 : t < 168 ? 1 : 2, u = t - (s == 0 ? 0 : s == 1 ? 56 : 168), n = s == 1 ? 16 : 8, i0 = s == 0 ? 0 : s == 1 ? 8 : 24;
    k = u >= 4 * n;
    const int v = u - 4 * n, q = (v * 43) >> 7;
    i = i0 + (k ? q : u >> 2); w = k ? v - 3 * q : u & 3;
  } else {
    k = t < 32 ? 0 : t < 64 ? 1 : t < 128 ? 0 : t < 136 ? 2 : t < 200 ? 1 : 2;
    i = t < 64 ? (t & 31) >> 2 : t < 128 ? 8 + ((t - 64) >> 2) : t < 136 ? t - 128 : t < 200 ? 8 + ((t - 136) >> 2) : t - 192;
    w = k == 2 ? 3 : t & 3;
  }
}

__device__ __forceinline__ void recv_block(uint8_t* myws, float* out, int g, int src, int i, int wave, int k, int lane,
                                           unsigned call, unsigned* tmo, float sc, uint8_t* scr) {
  const unsigned off = out_off(g < 4 ? 0 : 224, block_col(src, tile_block(g, i)), wave, lane);
  const uint8_t* const fl = myws + WS_DONE + (size_t)((src * 32 + i) * 4 + wave) * 128;
  const int nb = wave_blocks(g < 4 ? 7 : 9, wave);
  {
    f32x4 h[16];
    asm volatile(SGPR_PAD "global_load_dwordx4 %0, %16, %18 offset:-4096 sc1\n\t""global_load_dwordx4 %1, %16, %18 offset:-3072 sc1\n\t"
                 "global_load_dwordx4 %2, %16, %18 offset:-2048 sc1\n\t""global_load_dwordx4 %3, %16, %18 offset:-1024 sc1\n\t"
                 "global_load_dwordx4 %4, %16, %18 offset:0 sc1\n\t""global_load_dwordx4 %5, %16, %18 offset:1024 sc1\n\t"
                 "global_load_dwordx4 %6, %16, %18 offset:2048 sc1\n\t""global_load_dwordx4 %7, %16, %18 offset:3072 sc1\n\t"
                 "global_load_dwordx4 %8, %17, %18 offset:-4096 sc1\n\t""global_load_dwordx4 %9, %17, %18 offset:-3072 sc1\n\t"
                 "global_load_dwordx4 %10, %17, %18 offset:-2048 sc1\n\t""global_load_dwordx4 %11, %17, %18 offset:-1024 sc1\n\t"
                 "global_load_dwordx4 %12, %17, %18 offset:0 sc1\n\t""global_load_dwordx4 %13, %17, %18 offset:1024 sc1\n\t"
                 "global_load_dwordx4 %14, %17, %18 offset:2048 sc1\n\t""global_load_dwordx4 %15, %17, %18 offset:3072 sc1"
                 : "=&v"(h[0]), "=&v"(h[1]), "=&v"(h[2]), "=&v"(h[3]), "=&v"(h[4]), "=&v"(h[5]), "=&v"(h[6]), "=&v"(h[7]),
                   "=&v"(h[8]), "=&v"(h[9]), "=&v"(h[10]), "=&v"(h[11]), "=&v"(h[12]), "=&v"(h[13]), "=&v"(h[14]), "=&v"(h[15])
                 : "v"(4096 + lane * 16), "v"(12288 + lane * 16), "s"(myws + WS_PB + (size_t)((g * 32 + i) * 9 + wave * 2 + k) * 16384) : "memory");
    const bool halves = in_halves(g, i, k, nb);
    spin<RX_SLEEP>([&] { return __builtin_amdgcn_readfirstlane(ld_sys(fl)) >= 8 * call + 2 * k + 2 - halves; }, tmo, call);
    asm volatile("s_waitcnt vmcnt(0)" : "+v"(h[0]), "+v"(h[1]), "+v"(h[2]), "+v"(h[3]), "+v"(h[4]), "+v"(h[5]), "+v"(h[6]), "+v"(h[7]), "+v"(h[8]), "+v"(h[9]), "+v"(h[10]), "+v"(h[11]), "+v"(h[12]), "+v"(h[13]), "+v"(h[14]), "+v"(h[15]) :: "memory");
    dec_out(myws + WS_OS + (size_t)((src * 32 + i) * 9 + wave * 2 + k) * OBLK, h, out, off + (unsigned)k * 32 * (GN * 4), sc, tmo, call, scr, lane,
            halves, [&] { return spin<LAST_SLEEP>([&] { return __builtin_amdgcn_readfirstlane(ld_sys(fl)) >= 8 * call + 2 * k + 2; }, tmo, call); });
  }
}

}

__global__ void __launch_bounds__(256, 1) gemm_rs_fused(const Params p) {
  __shared__ __attribute__((aligned(16))) uint8_t lds[LDS_BYTES];
  const int b = blockIdx.x;
  const int lane = threadIdx.x & 63;
  const int wave = __builtin_amdgcn_readfirstlane(threadIdx.x >> 6);
  const int rank = p.rank;
  uint8_t* const myws = p.myws;
  asm volatile("" :: "s"(myws), "s"(rank), "s"(p.x), "s"(p.w));

  const int grp = b & 7, dst = grp ^ rank, ti = 31 - (b >> 3);
  const bool local = grp == 0 || (grp >= 4 && ti >= 24);
  const int set = local ? 7 + (grp != 0) : grp - 1, cb = tile_block(grp, ti);
  const RowSet rs = row_set(set);
  unsigned* const callp = (unsigned*)(myws + WS_CALL + (size_t)b * 128);
  unsigned* const tmo = (unsigned*)(myws + WS_TMO);
  unsigned* const lk = (unsigned*)(myws + WS_LINK) + grp * 64;
  uint8_t* const scr = lds + LDS_PK + wave * 2048;

  PushCtx c;
  c.x = p.x; c.d = dst; c.local = grp == 0;
  c.xl = (unsigned)((lane & 31) * KS + (lane >> 5) * 32);
  const RowSet hs = row_set(grp - 1);
  c.skip0 = grp == 0 ? NTASK : hs.task0; c.skipn = grp == 0 ? 0 : 4 * hs.r;

  unsigned call;
  if (ti >= (grp == 0 ? 8 : 24)) {
    const int w = (b >> 3) * 4 + wave, npw = grp == 0 ? 96 : 32;
    c.w = p.w; c.dst = p.ws[dst]; c.lk = lk; c.rank = rank;
    c.wl = (unsigned)((lane >> 5) * 32 * GN + (lane & 31) * 4);
    c.scr = scr;
    PushState s;
    u32x4 x0[4];
    unsigned cv;
    s.tp = -1; s.tc = task_at(c, w); s.tn = task_at(c, w + npw); s.tl = task_at(c, w + 2 * npw); s.gone = true;
    asm volatile(SGPR_PAD "global_load_dword %0, %1, %2 sc1" : "=v"(cv) : "v"(0u), "s"(callp) : "memory");
    ld_x(c, s.tc, x0);
    __builtin_amdgcn_sched_barrier(0);
    ld_x(c, s.tn, s.nx);
    const RowSet fs = row_set(task_set(s.tc));
    unsigned first = (unsigned)((fs.base * 128 + rank * fs.r * 16 + (s.tc - fs.task0) * 4) << 10) + lane * 16;
    asm volatile("" : "+v"(first));
    asm volatile("s_waitcnt vmcnt(4)" : "+v"(cv), "+v"(x0[0]), "+v"(x0[1]), "+v"(x0[2]), "+v"(x0[3]) :: "memory");
    call = __builtin_amdgcn_readfirstlane(cv) + 1;
    c.par = (unsigned)(WS_DATA + (call & 1) * PAR_BYTES);
#pragma unroll
    for (int q = 0; q < 4; ++q) st16_sys(c.dst, c.par + first + q * 1024, x0[q]);
    asm volatile("s_waitcnt vmcnt(4)" : "+v"(s.nx[0]), "+v"(s.nx[1]), "+v"(s.nx[2]), "+v"(s.nx[3]) :: "memory");
    c.tick0 = (call - 1) * (NTASK - c.skipn - 2 * npw) - 3 * npw;
    c.tag = call & 0xffffu;
    c.call = call;
    s.line = u32x4{0u, 0u, 0u, 0u};
    pack_task(c, x0, s.o, lane, true);
#pragma unroll 1
    while (s.tp < NTASK) {
      if (task_is_w(s.tl)) push_step<true>(c, s, lane);
      else push_step<false>(c, s, lane);
    }
  } else {
    call = __builtin_amdgcn_readfirstlane(ld_agent(callp)) + 1;
  }
  uint8_t* const par = myws + WS_DATA + (size_t)(call & 1) * PAR_BYTES;

#pragma unroll 1
  for (int job = 0; job < 4; ++job) {
    if (job == 1 && grp != 0 && ti < 24) {
      __syncthreads();
      if (grp >= 4) {
        home_tile<true>(p, grp, dst, ti, lds, wave, lane);
      } else {
#pragma unroll 1
        for (int i = ti; i < 32; i += ti >= 16 ? 8 : 32) home_tile<false>(p, grp, dst, i, lds, wave, lane);
      }
    }
    const bool jw = grp == 0 ? job < 3 : job == 1;
    const int beta = grp == 0 ? (ti * 4 + wave) & 63 : (23 - ti) * 4 + wave;
    if (grp == 0 ? (job == 2 && ti >= 24) : (ti >= 24 || job >= 2 || (job == 0 && beta >= 8 * rs.r))) continue;
    const int n = job == 2 ? 24 + (ti & 7) : cb;
    const RowSet js = row_set(grp == 0 ? 7 + (ti >> 4) : set);
    const int bs = div_r(beta, js.r), bu = beta - bs * js.r;
    const int src = !jw ? bs : grp != 0 ? dst : job == 0 ? rank : job == 1 ? rank ^ (4 + (ti & 3)) : rank ^ (1 + (ti >> 3));
    const int t0 = jw ? T_W0 + n * 16 + wave * 4 : js.task0 + bu * 4;
    const uint8_t* const slot = par + XS_BYTES + WT_BYTES + (size_t)(src * NTASK + t0) * ITASK;
    const bool absent = !jw && grp != 0 && src == dst;
    if (!absent && !wait_tasks(slot, tmo, call, lane)) continue;
    if (src != rank && !absent) {
      const unsigned base = jw ? (unsigned)XS_BYTES + (unsigned)(n * (512 << 10) + (src * 8 + wave * 2) * 8192)
                               : (unsigned)(js.base * (128 << 10) + (src * js.r + bu) * 16384);
      unsigned off[4];
#pragma unroll
      for (int t = 0; t < 4; ++t) off[t] = base + (jw ? (t >> 1) * 8192 + (t & 1) * 1024 : t * 4096);
      dec_in4(slot, par, off, jw ? 2048 : 1024, lds + wave * 4096, lane);
      wait_vm<0>();
    }
    st1_agent(jw ? myws + WS_DW + (size_t)(n * 32 + src * 4 + wave) * 4
                 : myws + WS_DX + (size_t)((grp == 0 ? 7 + (ti >> 4) : set) * 128 + beta) * 4, call);
  }
  if (grp != 0 && ti < 24) {
    wait_vm<0>();
    st1_agent(myws + WS_HOME + (size_t)((grp * 32 + ti) * 4 + wave) * 4, call);
    if (grp < 4 && ti >= 16) st1_agent(myws + WS_HOME + (size_t)((grp * 32 + ti + 8) * 4 + wave) * 4, call);
  }

  unsigned* const ctrl = (unsigned*)(lds + LDS_CTRL);
  if (wave == 0) {
    const uint8_t* const fw = myws + WS_DW + (size_t)(cb * 32 + (lane & 31)) * 4;
    const uint8_t* const fx = myws + WS_DX + (size_t)(set * 128 + min(lane, 8 * rs.r - 1)) * 4;
    const uint8_t* const fy = myws + WS_DX + (size_t)(set * 128 + min(64 + lane, 8 * rs.r - 1)) * 4;
    const bool ok = spin([&] { const unsigned a = ld_agent(fw), c = ld_agent(fx), d = ld_agent(fy); return __all(min(min(a, c), d) >= call) != 0; },
                         tmo, call);
    if (lane == 0) ctrl[0] = !ok;
  }
  __syncthreads();
  const bool dead0 = __builtin_amdgcn_readfirstlane(ctrl[0]) != 0;

  if (wave == 0) st1_agent(callp, call);
  const float sc = p.sx[0] * p.sw[0];
  const unsigned off = out_off(rs.row0, block_col(rank, cb), wave, lane);
  float* const ob = p.out[rank];
  uint8_t* const blk = lds + wave * 32768;
  uint8_t* const oslot = p.ws[dst] + WS_OS + (size_t)((rank * 32 + ti) * 9 + wave * 2) * OBLK;
  unsigned* const flag = (unsigned*)(p.ws[dst] + WS_DONE + (size_t)((rank * 32 + ti) * 4 + wave) * 128);
  const int nb = wave_blocks(rs.r, wave);
  const unsigned* const turn = lk + 32;
  const int depth = ti < DEEP_T ? DEEP_D : nb == 3 ? OUT_D3 : OUT_D;
  unsigned nl0 = 0, nl1 = 0, nl2 = 0;
  int sent = 0;

  if (!dead0) {
    const uint8_t* const ga = par + (size_t)rs.base * (128 << 10);
    const uint8_t* const gb = par + XS_BYTES + (size_t)cb * (512 << 10);
    if (rs.r == 9) gemm_tile<9>(ga, 9 * 2048, gb, dst, lds, wave, lane);
    else if (nb == 1) gemm_tile<7>(ga, 7 * 2048, gb, dst, lds, wave, lane);
    else gemm_tile<8>(ga, rs.r * 2048, gb, local ? -1 : dst, lds, wave, lane);
    unsigned tnv, tmv;
    asm volatile(SGPR_PAD "global_load_dword %0, %2, %3 sc1\n\tglobal_load_dword %1, %2, %4 sc1"
                 : "=&a"(tnv), "=&a"(tmv) : "v"(0u), "s"(turn), "s"(tmo) : "memory");
    auto landed = [&](unsigned& s_tn, unsigned& s_tm) {
      unsigned t0, t1;
      asm volatile("s_waitcnt vmcnt(0)\n\tv_accvgpr_read_b32 %2, %4\n\tv_accvgpr_read_b32 %3, %5\n\ts_nop 1\n\t"
                   "v_readfirstlane_b32 %0, %2\n\tv_readfirstlane_b32 %1, %3\n\ts_nop 1"
                   : "=&s"(s_tn), "=&s"(s_tm), "=&v"(t0), "=&v"(t1) : "a"(tnv), "a"(tmv) : "memory");
    };
    float s0 = sc;
#pragma unroll 1
    for (int k = 0; k < nb; ++k) {
      float v[4][16];
#pragma unroll
      for (int i = 0; i < 16; ++i) {
        const f32x4 t = *(const f32x4*)(blk + k * 16384 + i * 1024 + lane * 16);
#pragma unroll
        for (int q = 0; q < 4; ++q) v[i >> 2][(i & 3) * 4 + q] = t[q];
      }
      __builtin_amdgcn_fence(__ATOMIC_ACQ_REL, "wavefront");
      if (local) {
        if (k == 0) {
          unsigned tn, t;
          landed(tn, t);
          if (t == call) s0 = __builtin_nanf("");
        }
#pragma unroll
        for (int r = 0; r < 16; ++r)
          st16_out(ob, off + k * 32 * (GN * 4) + row_off(r), f32x4{v[0][r] * s0, v[1][r] * s0, v[2][r] * s0, v[3][r] * s0});
        continue;
      }
      const unsigned nl = enc_out(v, blk + k * 16384, lane);
      __builtin_amdgcn_fence(__ATOMIC_ACQ_REL, "wavefront");
      if (k == 0) nl0 = nl; else if (k == 1) nl1 = nl; else nl2 = nl;
      if (k == 0) {
        unsigned tn, t;
        landed(tn, t);
        if (nb > 1 && t != call && (tn >= call || __builtin_amdgcn_readfirstlane(ld_agent(turn)) >= call)) {
          send_out(blk, nl, false, oslot, lane, depth, flag, 0);
          sent = 1;
        }
      }
    }
  } else if (local) {
    const float n = __builtin_nanf("");
#pragma unroll 1
    for (int t = 0; t < 32; ++t) st16_out(ob, off + (t >> 4) * 32 * (GN * 4) + row_off(t & 15), f32x4{n, n, n, n});
  }
  if (grp == 0) return;

  if (!local) {
    if (sent) {
      wait_vm<0>();
      st1_sys(flag, 8 * call + 2);
    } else {
      spin([&] { return __builtin_amdgcn_readfirstlane(ld_agent(turn)) >= call; }, tmo, call);
    }
    const bool dead = dead0 || __builtin_amdgcn_readfirstlane(ld_agent(tmo)) == call;
#pragma unroll 1
    for (int k = sent; k < nb; ++k) {
      send_out(blk + k * 16384, k == 0 ? nl0 : k == 1 ? nl1 : nl2, dead, oslot + k * OBLK, lane, depth, flag,
               in_halves(grp, ti, k, nb) ? 8 * call + 2 * k + 1 : 0);
      wait_vm<0>();
      st1_sys(flag, 8 * call + 2 * k + 2);
    }
  }
  {
    const uint8_t* const fa = myws + WS_HOME + (size_t)(grp * 128 + min(lane, grp < 4 ? 127 : 95)) * 4;
    const uint8_t* const fb = myws + WS_HOME + (size_t)(grp * 128 + min(64 + lane, grp < 4 ? 127 : 95)) * 4;
    spin<RX_SLEEP>([&] { const unsigned a = ld_agent(fa), c = ld_agent(fb); return __all(min(a, c) >= call) != 0; }, tmo, call);
  }
  const unsigned nblk = grp < 4 ? 224 : 216;
#pragma unroll 1
  for (;;) {
    unsigned t = 0;
    if (lane == 0) t = __hip_atomic_fetch_add(lk + 512, 1u, __ATOMIC_RELAXED, __HIP_MEMORY_SCOPE_AGENT);
    t = __builtin_amdgcn_readfirstlane(t) - (call - 1) * (nblk + 128);
    if (t >= nblk) break;
    int i, w, k;
    block_at(grp, (int)t, i, w, k);
    recv_block(myws, p.out[rank], grp, dst, i, w, k, lane, call, tmo, sc, scr);
  }
}

extern "C" void kernel_launch(int rank, int nranks, ncclComm_t comm, hipStream_t stream,
                              void* const* d_in, const int* in_sizes, int n_in,
                              void* d_out, int out_size, const int* in_dtypes, int out_dtype,
                              void* const* d_out_all, void* const* d_ws_all, size_t ws_bytes) {
  (void)comm; (void)in_sizes; (void)in_dtypes; (void)out_dtype; (void)d_out;
  if (nranks != NR || (unsigned)rank >= (unsigned)NR || n_in != 4 || ws_bytes < WS_NEED || out_size != 512 * GN) {
    fprintf(stderr, "kernel_launch: unexpected problem shape\n");
    return;
  }
  Params p;
  p.x = (const uint8_t*)d_in[0];
  p.w = (const uint8_t*)d_in[1];
  p.sx = (const float*)d_in[2];
  p.sw = (const float*)d_in[3];
  for (int i = 0; i < NR; ++i) {
    p.ws[i] = (uint8_t*)d_ws_all[i];
    p.out[i] = (float*)d_out_all[i];
  }
  p.myws = p.ws[rank];
  p.rank = rank;
  p.pad = 0;
  hipLaunchKernelGGL(gemm_rs_fused, dim3(NWG), dim3(256), 0, stream, p);
  HIP_CHECK(hipGetLastError());
}
```

```cpp
#include <hip/hip_runtime.h>
#include <rccl/rccl.h>
#include <cstdio>
#include <cstdint>

#define HIP_CHECK(expr) do { hipError_t _e = (expr); if (_e != hipSuccess) { \
    fprintf(stderr, "HIP error %s at %s:%d\n", hipGetErrorString(_e), __FILE__, __LINE__); \
    return; } } while (0)

typedef unsigned u32x4 __attribute__((ext_vector_type(4)));
typedef int i32x4 __attribute__((ext_vector_type(4)));
typedef int i32x8 __attribute__((ext_vector_type(8)));
typedef float f32x4 __attribute__((ext_vector_type(4)));
typedef float f32x16 __attribute__((ext_vector_type(16)));

#define SGPR_PAD "s_nop 4\n\t"

namespace {

constexpr int NR = 8;
constexpr int GN = 16384;
constexpr int KS = 512;
constexpr int NWG = 256;
constexpr int KSTEPS = 64;
constexpr int NSTAGE = 6;
constexpr int LDS_CTRL = NSTAGE * 26624;
constexpr int LDS_BYTES = LDS_CTRL + 64;
constexpr int LDS_PK = 9 * 16384;

constexpr int T_W0 = 228, T_OWN = T_W0 + 512, NTASK = T_OWN + 64;

constexpr size_t WS_CALL = 0;
constexpr size_t WS_TMO = 32768;
constexpr size_t WS_DONE = 896u << 10;
constexpr size_t WS_LINK = WS_TMO + 1024;
constexpr size_t WS_DX = WS_LINK + 4096;
constexpr size_t WS_DW = WS_DX + 8192;
constexpr size_t WS_HOME = WS_DW + 4096;
constexpr size_t WS_DATA = 1u << 20;
constexpr size_t XS_BYTES = 10u << 20;
constexpr size_t WT_BYTES = 16u << 20;
constexpr unsigned ITASK = 3328;
constexpr size_t IS_BYTES = 21u << 20;
constexpr size_t PAR_BYTES = XS_BYTES + WT_BYTES + IS_BYTES;
static_assert((size_t)8 * NTASK * ITASK <= IS_BYTES, "");
constexpr size_t OBLK = 17 * 1024;
constexpr size_t WS_OS = WS_DATA + 2 * PAR_BYTES;
constexpr size_t WS_PB = WS_OS + 8 * 32 * 9 * OBLK;
constexpr size_t WS_NEED = WS_PB + 8 * 32 * 9 * 16384;

constexpr unsigned long long DEADLINE_TICKS = 50000000ull;

constexpr int OPEN_AT = 790, OUT_D = 2, DEEP_D = 4, DEEP_D3 = 6, DEEP_T = 8;
constexpr int POLL_SLEEP = 6, RX_SLEEP = 16, LAST_SLEEP = 8;

struct Params {
  const uint8_t* x;
  const uint8_t* w;
  const float* sx;
  const float* sw;
  uint8_t* ws[NR];
  float* out[NR];
  uint8_t* myws;
  int rank;
  int pad;
};

__device__ __forceinline__ unsigned ld_sys(const void* ptr) {
  return __hip_atomic_load((const unsigned*)ptr, __ATOMIC_RELAXED, __HIP_MEMORY_SCOPE_SYSTEM);
}
__device__ __forceinline__ unsigned ld_agent(const void* ptr) {
  return __hip_atomic_load((const unsigned*)ptr, __ATOMIC_RELAXED, __HIP_MEMORY_SCOPE_AGENT);
}
__device__ __forceinline__ void st_agent(void* ptr, unsigned v) {
  __hip_atomic_store((unsigned*)ptr, v, __ATOMIC_RELAXED, __HIP_MEMORY_SCOPE_AGENT);
}

template <int SLEEP = POLL_SLEEP, class F>
__device__ __forceinline__ bool spin(F ready, unsigned* tmo, unsigned call) {
  const unsigned long long t0 = __builtin_amdgcn_s_memrealtime();
  for (unsigned n = 1;; ++n) {
    if (ready()) return true;
    if ((n & 3) == 0 && (__builtin_amdgcn_readfirstlane(ld_agent(tmo)) == call || __builtin_amdgcn_s_memrealtime() - t0 > DEADLINE_TICKS)) {
      st_agent(tmo, call);
      return false;
    }
    __builtin_amdgcn_s_sleep(SLEEP);
  }
}

__device__ __forceinline__ void st16_sys(void* base, unsigned off, u32x4 v) {
  asm volatile(SGPR_PAD "global_store_dwordx4 %0, %1, %2 sc0 sc1\n\ts_nop 1" :: "v"(off), "v"(v), "s"(base) : "memory");
}

__device__ __forceinline__ void st16_loc(void* base, unsigned off, u32x4 v) {
  asm volatile(SGPR_PAD "global_store_dwordx4 %0, %1, %2 sc1\n\ts_nop 1" :: "v"(off), "v"(v), "s"(base) : "memory");
}

__device__ __forceinline__ void st1_agent(void* base, unsigned v) {
  asm volatile(SGPR_PAD "s_mov_b64 exec, 1\n\tglobal_store_dword %0, %1, %2 sc1\n\ts_mov_b64 exec, -1"
               :: "v"(0u), "v"(v), "s"(base) : "memory");
}

__device__ __forceinline__ void st1_sys(void* base, unsigned v) {
  asm volatile(SGPR_PAD "s_mov_b64 exec, 1\n\tglobal_store_dword %0, %1, %2 sc0 sc1\n\ts_mov_b64 exec, -1"
               :: "v"(0u), "v"(v), "s"(base) : "memory");
}
__device__ __forceinline__ void st16_sys_m(void* base, unsigned off, u32x4 v, unsigned long long mask) {
  asm volatile(SGPR_PAD "s_mov_b64 exec, %3\n\tglobal_store_dwordx4 %0, %1, %2 sc0 sc1\n\ts_mov_b64 exec, -1\n\ts_nop 0"
               :: "v"(off), "v"(v), "s"(base), "s"(mask) : "memory");
}

template <int N> __device__ __forceinline__ void wait_vm() {
  asm volatile("s_waitcnt vmcnt(%0)" :: "n"(N) : "memory");
}

__device__ __forceinline__ unsigned wave_max_u32(unsigned v) {
  v = max(v, (unsigned)__builtin_amdgcn_update_dpp(0, (int)v, 0x111, 0xf, 0xf, false));
  v = max(v, (unsigned)__builtin_amdgcn_update_dpp(0, (int)v, 0x112, 0xf, 0xf, false));
  v = max(v, (unsigned)__builtin_amdgcn_update_dpp(0, (int)v, 0x114, 0xf, 0xf, false));
  v = max(v, (unsigned)__builtin_amdgcn_update_dpp(0, (int)v, 0x118, 0xf, 0xf, false));
  return max(max((unsigned)__builtin_amdgcn_readlane((int)v, 15), (unsigned)__builtin_amdgcn_readlane((int)v, 31)),
             max((unsigned)__builtin_amdgcn_readlane((int)v, 47), (unsigned)__builtin_amdgcn_readlane((int)v, 63)));
}

__device__ __forceinline__ unsigned irec_slot(unsigned idx) { return idx + 3 + min((__umul24(idx, 37u) + 74u) >> 8, 3u); }

__device__ __forceinline__ unsigned wave_sum_u32(unsigned v) {
  v += (unsigned)__builtin_amdgcn_update_dpp(0, (int)v, 0x111, 0xf, 0xf, false);
  v += (unsigned)__builtin_amdgcn_update_dpp(0, (int)v, 0x112, 0xf, 0xf, false);
  v += (unsigned)__builtin_amdgcn_update_dpp(0, (int)v, 0x114, 0xf, 0xf, false);
  v += (unsigned)__builtin_amdgcn_update_dpp(0, (int)v, 0x118, 0xf, 0xf, false);
  return (unsigned)(__builtin_amdgcn_readlane((int)v, 15) + __builtin_amdgcn_readlane((int)v, 31) +
                    __builtin_amdgcn_readlane((int)v, 47) + __builtin_amdgcn_readlane((int)v, 63));
}
__device__ __forceinline__ unsigned bfi(unsigned m, unsigned a, unsigned b) { return (a & m) | (b & ~m); }
constexpr unsigned HDR_RAW = 0x80000000u;

constexpr unsigned IREC_CAP = 122, IHDR_RAW = 0x8000u;

struct InTask {
  u32x4 d[4];
  unsigned long long mask;
  bool raw;
};

__device__ __forceinline__ void put_records2(unsigned q0, unsigned q1, unsigned tag, unsigned& cnt, uint8_t* scr, unsigned dump) {
  unsigned m = bfi(0x55555555u, q0 | q0 >> 1, q1 | q1 << 1);
  for (unsigned long long act = __ballot(m != 0); act != 0 && cnt <= IREC_CAP; act = __ballot(m != 0)) {
    const unsigned bit = (unsigned)__builtin_ctz(m | 0x80000000u), sh = bit & 30u;
    const unsigned idx = cnt + __builtin_amdgcn_mbcnt_hi((unsigned)(act >> 32), __builtin_amdgcn_mbcnt_lo((unsigned)act, 0u));
    const unsigned sl = 2 * irec_slot(idx), at = ((m != 0) & (idx < IREC_CAP)) ? sl : dump;
    *(unsigned short*)(scr + at) = (unsigned short)(tag | (bit & 1u) << 6 | sh << 1 | ((((bit & 1u) ? q1 : q0) >> sh) & 3u));
    m &= m - 1;
    cnt += __popcll(act);
  }
}

__device__ __forceinline__ void enc_in(const u32x4 (&v)[4], InTask& o, unsigned tag, uint8_t* scr, int lane) {
  unsigned sum = 0;
#pragma unroll
  for (int q = 0; q < 4; ++q) sum = __builtin_amdgcn_udot4(v[q][0] & 0x7f7f7f7fu, 0x01010101u, sum, false);
  const unsigned e0 = (unsigned)min(max((int)((wave_sum_u32(sum) + 10445u) >> 12) - 7, 0), 24);
  const unsigned add = 0x80808080u - e0 * 0x04040404u;
  unsigned pay[12], qq[4], cnt = 0;
#pragma unroll
  for (int q = 0; q < 4; ++q) {
    unsigned R[4], S[4];
#pragma unroll
    for (int k = 0; k < 4; ++k) {
      R[k] = (v[q][k] & 0x7f7f7f7fu) + add;
      S[k] = bfi(0x80808080u, v[q][k], R[k]);
    }
    pay[3 * q] = bfi(0x60606060u, S[3] << 5, S[0]);
    pay[3 * q + 1] = bfi(0x60606060u, S[3] << 3, S[1]);
    pay[3 * q + 2] = bfi(0x20202020u, S[3] << 1, bfi(0x40404040u, S[3] >> 1, S[2]));
    qq[q] = bfi(0xc0c0c0c0u, R[3] << 1, bfi(0x30303030u, R[2] >> 1, bfi(0x0c0c0c0cu, R[1] >> 3, (R[0] >> 5) & 0x03030303u)));
  }
  put_records2(qq[0], qq[1], (unsigned)(lane << 8), cnt, scr, 256 + 2 * lane);
  put_records2(qq[2], qq[3], (unsigned)(lane << 8 | 128), cnt, scr, 256 + 2 * lane);
  *(unsigned short*)(scr + (lane < 4 ? lane * 16 : lane < 6 ? lane * 2 - 6 : 256 + 2 * lane)) =
      (unsigned short)(lane < 4 ? tag : lane == 4 ? cnt : e0);
  __builtin_amdgcn_fence(__ATOMIC_ACQ_REL, "wavefront");
  const u32x4 rec = *(const u32x4*)(scr + (lane & 15) * 16);
  __builtin_amdgcn_fence(__ATOMIC_ACQ_REL, "wavefront");
  o.raw = cnt > IREC_CAP;
  if (o.raw) {
#pragma unroll
    for (int q = 0; q < 4; ++q) o.d[q] = v[q];
    o.mask = ~0ull;
  } else {
#pragma unroll
    for (int q = 0; q < 3; ++q) o.d[q] = u32x4{pay[4 * q], pay[4 * q + 1], pay[4 * q + 2], pay[4 * q + 3]};
    o.d[3] = rec;
    o.mask = (1ull << max(((irec_slot(cnt) + 15) >> 4) * 2, 4u)) - 16;
  }
}

__device__ __forceinline__ void dec_in4(const uint8_t* slot, uint8_t* out, const unsigned (&off)[4], unsigned stride, uint8_t* scr, int lane) {
  u32x4 y[4][3];
  unsigned rec[4];
  asm volatile(SGPR_PAD "global_load_dword %12, %16, %20 sc0 sc1\n\tglobal_load_dword %13, %16, %20 offset:3328 sc0 sc1\n\t"
               "global_load_dword %14, %17, %20 sc0 sc1\n\tglobal_load_dword %15, %17, %20 offset:3328 sc0 sc1\n\t"
               "global_load_dwordx4 %0, %18, %20 offset:-3072 sc0 sc1\n\tglobal_load_dwordx4 %1, %18, %20 offset:-2048 sc0 sc1\n\t"
               "global_load_dwordx4 %2, %18, %20 offset:-1024 sc0 sc1\n\t"
               "global_load_dwordx4 %3, %18, %20 offset:256 sc0 sc1\n\tglobal_load_dwordx4 %4, %18, %20 offset:1280 sc0 sc1\n\t"
               "global_load_dwordx4 %5, %18, %20 offset:2304 sc0 sc1\n\t"
               "global_load_dwordx4 %6, %19, %20 offset:-3072 sc0 sc1\n\tglobal_load_dwordx4 %7, %19, %20 offset:-2048 sc0 sc1\n\t"
               "global_load_dwordx4 %8, %19, %20 offset:-1024 sc0 sc1\n\t"
               "global_load_dwordx4 %9, %19, %20 offset:256 sc0 sc1\n\tglobal_load_dwordx4 %10, %19, %20 offset:1280 sc0 sc1\n\t"
               "global_load_dwordx4 %11, %19, %20 offset:2304 sc0 sc1\n\ts_waitcnt vmcnt(0)"
               : "=&v"(y[0][0]), "=&v"(y[0][1]), "=&v"(y[0][2]), "=&v"(y[1][0]), "=&v"(y[1][1]), "=&v"(y[1][2]),
                 "=&v"(y[2][0]), "=&v"(y[2][1]), "=&v"(y[2][2]), "=&v"(y[3][0]), "=&v"(y[3][1]), "=&v"(y[3][2]),
                 "=&v"(rec[0]), "=&v"(rec[1]), "=&v"(rec[2]), "=&v"(rec[3])
               : "v"(lane * 4), "v"(2 * ITASK + lane * 4), "v"(ITASK + lane * 16), "v"(3 * ITASK + lane * 16), "s"(slot) : "memory");
#pragma unroll
  for (int t = 0; t < 4; ++t) *(u32x4*)(scr + t * 1024 + lane * 16) = u32x4{0u, 0u, 0u, 0u};
  __builtin_amdgcn_fence(__ATOMIC_ACQ_REL, "wavefront");
  unsigned hdr[4];
#pragma unroll
  for (int t = 0; t < 4; ++t) {
    hdr[t] = (unsigned)__builtin_amdgcn_readlane((int)rec[t], 0) >> 16 | (unsigned)__builtin_amdgcn_readlane((int)rec[t], 1) << 16;
    const unsigned end = irec_slot(hdr[t] & 0x7fffu);
#pragma unroll
    for (int i = 0; i < 2; ++i) {
      const unsigned r16 = rec[t] >> (16 * i);
      const bool isrec = i == 0 ? (lane >= 16 || (lane & 3) != 0) && lane != 1 : lane != 0;
      const unsigned val = isrec && (unsigned)(lane * 2 + i) < end ? (r16 & 3u) << ((r16 >> 1) & 30u) : 0u;
      __hip_atomic_fetch_or((unsigned*)(scr + t * 1024 + ((r16 >> 6) & 255u) * 4), val, __ATOMIC_RELAXED, __HIP_MEMORY_SCOPE_WORKGROUP);
    }
  }
  __builtin_amdgcn_fence(__ATOMIC_ACQ_REL, "wavefront");
#pragma unroll
  for (int t = 0; t < 4; ++t) {
    if (hdr[t] & IHDR_RAW) continue;
    const u32x4 qq = *(const u32x4*)(scr + t * 1024 + lane * 16);
    const unsigned add = ((hdr[t] >> 16) & 31u) * 0x04040404u;
#pragma unroll
    for (int q = 0; q < 4; ++q) {
      const unsigned d0 = y[t][(3 * q) >> 2][(3 * q) & 3], d1 = y[t][(3 * q + 1) >> 2][(3 * q + 1) & 3], d2 = y[t][(3 * q + 2) >> 2][(3 * q + 2) & 3];
      const unsigned d3 = bfi(0x80808080u, d2 << 1, bfi(0x10101010u, d2 >> 1, bfi(0x0c0c0c0cu, d1 >> 3, d0 >> 5)));
      const unsigned X[4] = {bfi(0x60606060u, qq[q] << 5, d0), bfi(0x60606060u, qq[q] << 3, d1), bfi(0x60606060u, qq[q] << 1, d2),
                             bfi(0x60606060u, qq[q] >> 1, d3)};
      u32x4 r;
#pragma unroll
      for (int k = 0; k < 4; ++k) r[k] = bfi(0x80808080u, X[k], (X[k] & 0x7f7f7f7fu) + add);
      asm volatile(SGPR_PAD "global_store_dwordx4 %0, %1, %2 sc1\n\ts_nop 1" :: "v"(off[t] + q * stride + lane * 16), "v"(r), "s"(out) : "memory");
    }
  }
  __builtin_amdgcn_fence(__ATOMIC_ACQ_REL, "wavefront");
}

__device__ __forceinline__ bool wait_tasks(const uint8_t* slot, unsigned* tmo, unsigned call, int lane) {
  const uint8_t* q = slot + ((lane >> 2) & 3) * ITASK + (lane & 3) * 16;
  return spin([&] { return __all((ld_sys(q) & 0xffffu) == (call & 0xffffu)) != 0; }, tmo, call);
}

struct PushCtx {
  const uint8_t* x;
  const uint8_t* w;
  uint8_t* dst;
  unsigned* lk;
  uint8_t* scr;
  unsigned xl, wl;
  unsigned par, tick0, tag, call;
  int skip0, skipn;
  int rank, d;
  bool local;
};

struct RowSet { int r, row0, base, task0; };
__device__ __forceinline__ RowSet row_set(int set) {
  const int a = min(set, 3), o = max(set - 7, 0), b = set - a - o;
  const int own = (set + 1) >> 3, mid = ((set + 5) >> 3) - own;
  return {7 + 2 * mid + own, 224 * mid + 256 * o, 7 * a + 9 * b + 8 * o, 28 * a + 36 * b + 512 * own + 32 * o};
}
__device__ __forceinline__ int block_col(int q, int n) { return ((n & 3) * 4 + (q < 4 ? q : (q ^ n) & 3)) * 1024 + (n >> 2) * 128; }
__device__ __forceinline__ int tile_block(int g, int i) {
  const int c = (i * 43) >> 7, m = i - 3 * c;
  return g < 4 ? i : i >= 24 ? (g - 4) * 8 + i - 24 : 4 * c + m + (m >= g - 4);
}
__device__ __forceinline__ bool task_is_w(int t) { return t >= T_W0 && t < T_OWN; }
__device__ __forceinline__ int task_set(int t) {
  const int lo = min((t * 2341) >> 16, ((t + 24) * 1821) >> 16), hi = 7 + ((t - T_OWN) >> 5);
  return t < T_OWN ? lo : hi;
}

__device__ __forceinline__ int div_r(int v, int r) { return (v * (0x728093 >> (8 * (r - 7)) & 255)) >> 10; }

__device__ __forceinline__ void ld_x(const PushCtx& c, int t, u32x4 (&v)[4]) {
  t = min(t, NTASK - 1);
  const int set = task_set(t);
  const RowSet rs = row_set(set);
  const int p0 = 2 * (t - rs.task0), ks = div_r(p0, rs.r), rg = p0 - ks * rs.r;
  const unsigned o0 = c.xl + (unsigned)((((set < 7 ? set + 1 : 0) ^ c.d) * 512 + rs.row0 + rg * 32) * KS + ks * 64);
  const unsigned o1 = o0 + (unsigned)(rg + 1 == rs.r ? 64 - rg * 32 * KS : 32 * KS);
  asm volatile(SGPR_PAD "global_load_dwordx4 %0, %4, %6\n\tglobal_load_dwordx4 %1, %4, %6 offset:16\n\t"
               "global_load_dwordx4 %2, %5, %6\n\tglobal_load_dwordx4 %3, %5, %6 offset:16"
               : "=&v"(v[0]), "=&v"(v[1]), "=&v"(v[2]), "=&v"(v[3]) : "v"(o0), "v"(o1), "s"(c.x) : "memory");
}

__device__ __forceinline__ void ld_w(const uint8_t* base, unsigned o, unsigned (&w)[16]) {
    asm volatile(SGPR_PAD
                 "global_load_dword %0, %16, %17\n\tv_add_u32 %16, 0x4000, %16\n\t"
                 "global_load_dword %1, %16, %17\n\tv_add_u32 %16, 0x4000, %16\n\t"
                 "global_load_dword %2, %16, %17\n\tv_add_u32 %16, 0x4000, %16\n\t"
                 "global_load_dword %3, %16, %17\n\tv_add_u32 %16, 0x4000, %16\n\t"
                 "global_load_dword %4, %16, %17\n\tv_add_u32 %16, 0x4000, %16\n\t"
                 "global_load_dword %5, %16, %17\n\tv_add_u32 %16, 0x4000, %16\n\t"
                 "global_load_dword %6, %16, %17\n\tv_add_u32 %16, 0x4000, %16\n\t"
                 "global_load_dword %7, %16, %17\n\tv_add_u32 %16, 0x4000, %16\n\t"
                 "global_load_dword %8, %16, %17\n\tv_add_u32 %16, 0x4000, %16\n\t"
                 "global_load_dword %9, %16, %17\n\tv_add_u32 %16, 0x4000, %16\n\t"
                 "global_load_dword %10, %16, %17\n\tv_add_u32 %16, 0x4000, %16\n\t"
                 "global_load_dword %11, %16, %17\n\tv_add_u32 %16, 0x4000, %16\n\t"
                 "global_load_dword %12, %16, %17\n\tv_add_u32 %16, 0x4000, %16\n\t"
                 "global_load_dword %13, %16, %17\n\tv_add_u32 %16, 0x4000, %16\n\t"
                 "global_load_dword %14, %16, %17\n\tv_add_u32 %16, 0x4000, %16\n\t"
                 "global_load_dword %15, %16, %17\n\t"
                 : "=&v"(w[0]), "=&v"(w[1]), "=&v"(w[2]), "=&v"(w[3]), "=&v"(w[4]), "=&v"(w[5]), "=&v"(w[6]), "=&v"(w[7]),
                   "=&v"(w[8]), "=&v"(w[9]), "=&v"(w[10]), "=&v"(w[11]), "=&v"(w[12]), "=&v"(w[13]), "=&v"(w[14]), "=&v"(w[15]), "+v"(o) : "s"(base) : "memory");
    static_assert(GN == 0x4000, "");
}
__device__ __forceinline__ void transpose_w(const unsigned (&w)[16], u32x4 (&v)[4]) {
#pragma unroll
  for (int q = 0; q < 4; ++q) {
    const unsigned d0 = w[4 * q], d1 = w[4 * q + 1], d2 = w[4 * q + 2], d3 = w[4 * q + 3];
    const unsigned t0 = __builtin_amdgcn_perm(d1, d0, 0x05010400u), t1 = __builtin_amdgcn_perm(d1, d0, 0x07030602u);
    const unsigned t2 = __builtin_amdgcn_perm(d3, d2, 0x05010400u), t3 = __builtin_amdgcn_perm(d3, d2, 0x07030602u);
    v[0][q] = __builtin_amdgcn_perm(t2, t0, 0x05040100u);
    v[1][q] = __builtin_amdgcn_perm(t2, t0, 0x07060302u);
    v[2][q] = __builtin_amdgcn_perm(t3, t1, 0x05040100u);
    v[3][q] = __builtin_amdgcn_perm(t3, t1, 0x07060302u);
  }
}

__device__ __forceinline__ void pack_task(const PushCtx& c, const u32x4 (&v)[4], InTask& o, int lane, bool plain) {
  if (plain) {
#pragma unroll
    for (int q = 0; q < 4; ++q) o.d[q] = v[q];
    o.mask = ~0ull;
    o.raw = true;
  } else {
    enc_in(v, o, c.tag, c.scr, lane);
  }
}

__device__ __forceinline__ void send_plain(const PushCtx& c, int t, const u32x4 (&d)[4], int lane) {
  const int u = t - T_W0;
  const bool wr = task_is_w(t);
  const RowSet rs = row_set(task_set(t));
  const unsigned plain = c.par + lane * 16 +
      (wr ? (unsigned)XS_BYTES + (unsigned)((u >> 4) * (512 << 10) + (c.rank * 8 + ((u >> 1) & 7)) * 8192 + (u & 1) * 1024)
          : (unsigned)(rs.base * (128 << 10) + c.rank * rs.r * 16384 + (t - rs.task0) * 4096));
#pragma unroll
  for (int q = 0; q < 4; ++q) st16_sys(c.dst, plain + q * (wr ? 2048 : 1024), d[q]);
}

__device__ __forceinline__ int task_at(const PushCtx& c, int pos) { return min(pos < c.skip0 ? pos : pos + c.skipn, NTASK); }

struct PushState {
  int tp, tc, tn, tl;
  bool gone;
  u32x4 line;
  InTask o;
  u32x4 nx[4];
};

template <bool LDW>
__device__ __forceinline__ void push_step(const PushCtx& c, PushState& s, int lane) {
  unsigned w[16], tk;
  u32x4 x[4];
  const bool draw = s.tl < NTASK;
  asm volatile(SGPR_PAD "s_mov_b64 exec, 1\n\tglobal_atomic_add %0, %1, %2, %3 sc0\n\ts_mov_b64 exec, -1"
               : "=&v"(tk) : "v"(0u), "v"(draw ? 1u : 0u), "s"(c.lk) : "memory");
  if (s.tc == OPEN_AT) st1_agent(c.lk + 32, c.call);
  if constexpr (LDW) {
    const int u = s.tl - T_W0;
    ld_w(c.w, c.wl + (unsigned)((((u >> 1) & 7) * 64 + (u & 1) * 16) * GN + block_col(c.d, u >> 4)), w);
  } else {
    ld_x(c, s.tl, x);
  }
  const unsigned is0 = c.par + (unsigned)(XS_BYTES + WT_BYTES) + (unsigned)(c.rank * NTASK) * ITASK + lane * 16;
  if (s.tp >= 0) st16_sys_m(c.dst, is0 + (unsigned)s.tp * ITASK, s.line, 15ull);
  if (s.tc < NTASK) {
    if (!s.o.raw) {
      const unsigned slot = is0 + (unsigned)s.tc * ITASK;
      asm volatile(SGPR_PAD "global_store_dwordx4 %0, %1, %4 offset:256 sc0 sc1\n\tglobal_store_dwordx4 %0, %2, %4 offset:1280 sc0 sc1\n\t"
                   "global_store_dwordx4 %0, %3, %4 offset:2304 sc0 sc1\n\ts_nop 1"
                   :: "v"(slot), "v"(s.o.d[0]), "v"(s.o.d[1]), "v"(s.o.d[2]), "s"(c.dst) : "memory");
      if (s.o.mask != 0) st16_sys_m(c.dst, slot, s.o.d[3], s.o.mask);
      s.line = s.o.d[3];
    } else {
      if (!s.gone) send_plain(c, s.tc, s.o.d, lane);
      s.line = u32x4{c.tag | IHDR_RAW << 16, 0u, 0u, 0u};
    }
  }
  pack_task(c, s.nx, s.o, lane, c.local || s.tn >= NTASK);
  if constexpr (LDW) {
    asm volatile("s_waitcnt vmcnt(0)" : "+v"(tk), "+v"(w[0]), "+v"(w[1]), "+v"(w[2]), "+v"(w[3]), "+v"(w[4]), "+v"(w[5]), "+v"(w[6]), "+v"(w[7]),
                 "+v"(w[8]), "+v"(w[9]), "+v"(w[10]), "+v"(w[11]), "+v"(w[12]), "+v"(w[13]), "+v"(w[14]), "+v"(w[15]) :: "memory");
    transpose_w(w, s.nx);
  } else {
    asm volatile("s_waitcnt vmcnt(0)" : "+v"(tk), "+v"(x[0]), "+v"(x[1]), "+v"(x[2]), "+v"(x[3]) :: "memory");
#pragma unroll
    for (int q = 0; q < 4; ++q) s.nx[q] = x[q];
  }
  s.tp = s.tc; s.tc = s.tn; s.tn = s.tl; s.gone = false;
  s.tl = draw ? task_at(c, min(__builtin_amdgcn_readfirstlane(tk) - c.tick0, (unsigned)NTASK)) : NTASK;
}

#define GLDS_BITS " sc0 sc1"
__device__ __forceinline__ void glds2(const uint8_t* gbase, unsigned voff, unsigned lds) {
  unsigned keep;
  asm volatile("s_mov_b32 %0, m0\n\ts_mov_b32 m0, %3\n\ts_nop 2\n\t"
               "global_load_lds_dwordx4 %1, %2" GLDS_BITS "\n\t"
               "global_load_lds_dwordx4 %1, %2 offset:1024" GLDS_BITS "\n\t"
               "s_mov_b32 m0, %0"
               : "=&s"(keep) : "v"(voff), "s"(gbase), "s"(lds));
}
__device__ __forceinline__ void glds2o(const uint8_t* gbase, unsigned voff, unsigned lds) {
  unsigned keep;
  asm volatile("s_mov_b32 %0, m0\n\ts_mov_b32 m0, %3\n\ts_nop 2\n\t"
               "global_load_lds_dwordx4 %1, %2 offset:2048" GLDS_BITS "\n\t"
               "global_load_lds_dwordx4 %1, %2 offset:3072" GLDS_BITS "\n\t"
               "s_mov_b32 m0, %0"
               : "=&s"(keep) : "v"(voff), "s"(gbase), "s"(lds));
}
__device__ __forceinline__ void glds_half(const uint8_t* gbase, unsigned voff, unsigned lds) {
  unsigned keep;
  asm volatile("s_mov_b32 %0, m0\n\ts_mov_b32 m0, %3\n\ts_mov_b32 exec_hi, 0\n\ts_nop 1\n\t"
               "global_load_lds_dwordx4 %1, %2" GLDS_BITS "\n\t"
               "s_mov_b32 exec_hi, -1\n\ts_mov_b32 m0, %0"
               : "=&s"(keep) : "v"(voff), "s"(gbase), "s"(lds));
}

struct Frags { i32x8 a[2], b[4], a9, b9; };

__device__ __forceinline__ i32x8 ld_frag(const uint8_t* q) {
  const i32x4 lo = *(const i32x4*)q, hi = *(const i32x4*)(q + 1024);
  return __builtin_shufflevector(lo, hi, 0, 1, 2, 3, 4, 5, 6, 7);
}

#define MFMA(mi, j) acc[mi][j] = __builtin_amdgcn_mfma_scale_f32_32x32x64_f8f6f4(cur.a[mi], cur.b[j], acc[mi][j], 1, 1, 0, 0, 0, 0)
#define SB() __builtin_amdgcn_sched_barrier(0)

struct Ring {
  const uint8_t* ga;
  const uint8_t* gb;
  unsigned astep, ahop;
  int togo;
  unsigned va, vb, v9;
  unsigned wr, wr_end;
  unsigned wb, w9;
  unsigned rd;
  const uint8_t* qa;
  const uint8_t* qb;
  const uint8_t* q9;
  const uint8_t* la;
  const uint8_t* lb;
  const uint8_t* l9;
};

template <int MODE, int LEFT, bool DMA, bool RD>
__device__ __forceinline__ void kstep(f32x16 (&acc)[2][4], f32x16& acc9, const Frags& cur, Frags& nxt, Ring& g) {
  constexpr bool NINE = MODE == 9, TWO = MODE != 7;
  constexpr int STAGE = NINE ? 26624 : 24576;
  MFMA(0, 0); SB();
  asm volatile("s_waitcnt vmcnt(%0) lgkmcnt(0)\n\ts_barrier" :: "n"(LEFT * (NINE ? 7 : 6)) : "memory");
  SB();
  MFMA(0, 1); SB();
  if constexpr (RD) { nxt.a[0] = ld_frag(g.qa); nxt.b[0] = ld_frag(g.qb); }
  unsigned sa = 0, sb = 0;
  if constexpr (DMA) { const bool hop = --g.togo == 0; sa = hop ? g.ahop : g.astep; sb = hop ? 9 * 8192 : 8192; asm volatile("" : "+s"(sa), "+s"(sb)); }
  SB();
  MFMA(0, 2); SB();
  if constexpr (DMA) glds2(g.ga, g.va, g.wr);
  SB();
  MFMA(0, 3); SB();
  if constexpr (RD) { nxt.b[1] = ld_frag(g.qb + 2048); nxt.b[2] = ld_frag(g.qb + 4096); }
  SB();
  if constexpr (TWO) MFMA(1, 0);
  SB();
  if constexpr (DMA) glds2(g.gb, g.vb, g.wr + g.wb);
  SB();
  if constexpr (TWO) MFMA(1, 1);
  SB();
  if constexpr (RD) { nxt.b[3] = ld_frag(g.qb + 6144); if constexpr (TWO) nxt.a[1] = ld_frag(g.qa + 2048); }
  g.rd = g.rd == (NSTAGE - 1) * STAGE ? 0 : g.rd + STAGE;
  g.qa = g.la + g.rd;
  if constexpr (!NINE) g.qb = g.lb + g.rd;
  SB();
  if constexpr (TWO) MFMA(1, 2);
  SB();
  if constexpr (DMA) glds2o(g.ga, g.va, g.wr);
  SB();
  if constexpr (NINE) {
    if constexpr (TWO) MFMA(1, 3);
    SB();
    if constexpr (RD) { nxt.a9 = ld_frag(g.qb - 2048); nxt.b9 = ld_frag(g.q9); }
    g.qb = g.lb + g.rd; g.q9 = g.l9 + g.rd;
    if constexpr (DMA) g.gb += sb;
    SB();
    acc9 = __builtin_amdgcn_mfma_scale_f32_32x32x64_f8f6f4(cur.a9, cur.b9, acc9, 1, 1, 0, 0, 0, 0); SB();
    if constexpr (DMA) glds_half(g.ga, g.v9, g.wr + g.w9);
  } else {
    if constexpr (TWO) MFMA(1, 3);
    SB();
    if constexpr (DMA) g.gb += sb;
  }
  if constexpr (DMA) {
    g.ga += sa;
    g.wr += STAGE; g.wr = g.wr == g.wr_end ? g.wr - NSTAGE * STAGE : g.wr;
  }
  SB();
}

template <int MODE>
__device__ __forceinline__ void gemm_tile(const uint8_t* ga, unsigned astep, const uint8_t* gb, int skip, uint8_t* lds, int wave, int lane) {
  constexpr bool NINE = MODE == 9, TWO = MODE != 7;
  constexpr int STAGE = NINE ? 26624 : 24576, BOFF = STAGE - 8192;
  f32x16 acc[2][4], acc9;
  Ring g;
  g.ga = ga + (skip == 0 ? 8 * astep : 0); g.gb = gb + (skip == 0 ? 8 * 8192 : 0); g.astep = astep; g.ahop = 9 * astep;
  g.togo = skip > 0 ? 8 * skip - NSTAGE : -1;
  g.va = lane * 16 + wave * 4096; g.vb = lane * 16 + wave * 2048; g.v9 = 16384 + lane * 16 + wave * 512;
  g.wr = (unsigned)(uintptr_t)lds + wave * 4096; g.wr_end = g.wr + NSTAGE * STAGE;
  g.wb = BOFF - wave * 2048; g.w9 = 16384 - wave * 3584;
#pragma unroll
  for (int s = 0; s < NSTAGE; ++s) {
    glds2(g.ga, g.va, g.wr + s * STAGE);
    glds2(g.gb, g.vb, g.wr + s * STAGE + g.wb);
    glds2o(g.ga, g.va, g.wr + s * STAGE);
    if constexpr (NINE) glds_half(g.ga, g.v9, g.wr + s * STAGE + g.w9);
    g.ga += astep; g.gb += 8192;
  }
#pragma unroll
  for (int r = 0; r < 16; ++r) {
    acc9[r] = 0.f;
#pragma unroll
    for (int j = 0; j < 4; ++j) { acc[0][j][r] = 0.f; acc[1][j][r] = 0.f; }
  }
  g.la = lds + lane * 16 + wave * 4096; g.lb = lds + BOFF + lane * 16; g.l9 = g.lb + wave * 2048;
  asm volatile("s_waitcnt vmcnt(%0)\n\ts_barrier" :: "n"(5 * (NINE ? 7 : 6)) : "memory");
  Frags f0, f1;
  f0.a[0] = ld_frag(g.la); f0.b[0] = ld_frag(g.lb); f0.b[1] = ld_frag(g.lb + 2048);
  f0.b[2] = ld_frag(g.lb + 4096); f0.b[3] = ld_frag(g.lb + 6144);
  if constexpr (TWO) f0.a[1] = ld_frag(g.la + 2048);
  if constexpr (NINE) { f0.a9 = ld_frag(g.lb - 2048); f0.b9 = ld_frag(g.l9); }
  g.rd = STAGE; g.qa = g.la + STAGE; g.qb = g.lb + STAGE; g.q9 = g.l9 + STAGE;
#pragma unroll 1
  for (int s = skip < 0 ? 0 : 8; s < KSTEPS - NSTAGE; s += 2) {
    kstep<MODE, 4, true, true>(acc, acc9, f0, f1, g);
    kstep<MODE, 4, true, true>(acc, acc9, f1, f0, g);
  }
  kstep<MODE, 4, false, true>(acc, acc9, f0, f1, g);
  kstep<MODE, 3, false, true>(acc, acc9, f1, f0, g);
  kstep<MODE, 2, false, true>(acc, acc9, f0, f1, g);
  kstep<MODE, 1, false, true>(acc, acc9, f1, f0, g);
  kstep<MODE, 0, false, true>(acc, acc9, f0, f1, g);
  auto park = [&](const f32x16& a, int i) {
#pragma unroll
    for (int q = 0; q < 4; ++q)
      *(f32x4*)(lds + wave * 32768 + (i * 4 + q) * 1024 + lane * 16) = f32x4{a[4 * q], a[4 * q + 1], a[4 * q + 2], a[4 * q + 3]};
  };
  {
    const Frags& cur = f1;
    MFMA(0, 0); SB();
    asm volatile("s_waitcnt lgkmcnt(0)\n\ts_barrier" ::: "memory");
    SB();
    MFMA(0, 1); SB();
    MFMA(0, 2); SB();
    park(acc[0][0], 0); SB();
    MFMA(0, 3); SB();
    park(acc[0][1], 1); SB();
    if constexpr (TWO) MFMA(1, 0);
    SB();
    park(acc[0][2], 2); SB();
    if constexpr (TWO) MFMA(1, 1);
    SB();
    park(acc[0][3], 3); SB();
    if constexpr (TWO) {
      MFMA(1, 2); SB();
      park(acc[1][0], 4); SB();
      MFMA(1, 3); SB();
      park(acc[1][1], 5); SB();
      if constexpr (NINE) { acc9 = __builtin_amdgcn_mfma_scale_f32_32x32x64_f8f6f4(cur.a9, cur.b9, acc9, 1, 1, 0, 0, 0, 0); SB(); }
      park(acc[1][2], 6);
      park(acc[1][3], 7);
    }
  }
  if constexpr (NINE) {
#pragma unroll
    for (int q = 0; q < 4; ++q)
      *(f32x4*)(lds + 8 * 16384 + wave * 4096 + q * 1024 + lane * 16) = f32x4{acc9[4 * q], acc9[4 * q + 1], acc9[4 * q + 2], acc9[4 * q + 3]};
    __syncthreads();
  }
  __builtin_amdgcn_fence(__ATOMIC_ACQ_REL, "wavefront");
}

constexpr unsigned REC_CAP = 510, HDR_B = 0x40000000u;

__device__ __forceinline__ void put_records4(const unsigned (&q)[4], unsigned tag, unsigned& cnt, uint8_t* scr, unsigned dump) {
  unsigned m = 0;
#pragma unroll
  for (int k = 0; k < 4; ++k)
    m |= ((((q[k] & 0x77777777u) + 0x77777777u) | q[k]) & 0x88888888u) >> (3 - k);
  for (unsigned long long act = __ballot(m != 0); act != 0 && cnt <= REC_CAP; act = __ballot(m != 0)) {
    const unsigned bit = (unsigned)__builtin_ctz(m | 0x80000000u), sh = bit & 28u;
    const unsigned m1 = (unsigned)__builtin_amdgcn_sbfe((int)bit, 0, 1), m2 = (unsigned)__builtin_amdgcn_sbfe((int)bit, 1, 1);
    const unsigned w = bfi(m2, bfi(m1, q[3], q[2]), bfi(m1, q[1], q[0]));
    const unsigned idx = cnt + __builtin_amdgcn_mbcnt_hi((unsigned)(act >> 32), __builtin_amdgcn_mbcnt_lo((unsigned)act, 0u));
    const unsigned at = (m != 0 && idx < REC_CAP) ? 4 + 2 * idx : dump;
    *(unsigned short*)(scr + at) = (unsigned short)(tag | (bit & 3u) << 7 | sh << 2 | ((w >> sh) & 15u));
    m &= m - 1;
    cnt += __popcll(act);
  }
}

template <bool MODEB>
__device__ __forceinline__ bool enc_body(const float (&v)[4][16], float up, unsigned& cnt, uint8_t* blk, int lane) {
  unsigned bad = 0, low = 0;
#pragma unroll
  for (int h = 0; h < 2; ++h) {
    unsigned G[32];
#pragma unroll
    for (int j = 0; j < 4; ++j)
#pragma unroll
      for (int r = 0; r < 8; ++r) {
        const float x = v[j][8 * h + r] * up;
        unsigned g = (unsigned)(int)x;
        bad = __builtin_amdgcn_bitop3_b32(__builtin_bit_cast(unsigned, (float)(int)g), __builtin_bit_cast(unsigned, x), bad, 0xbe);
        low |= g;
        if (MODEB) g = g << 1 | (((g ^ g << 1) >> 27) & 16u);
        G[4 * r + j] = g;
      }
    unsigned q[4];
#pragma unroll
    for (int k = 0; k < 4; ++k) {
      const unsigned* g = G + 8 * k;
      const unsigned b0 = __builtin_amdgcn_perm(__builtin_amdgcn_perm(g[3], g[2], 0x0c0c0400u), __builtin_amdgcn_perm(g[1], g[0], 0x0c0c0400u), 0x05040100u);
      const unsigned b1 = __builtin_amdgcn_perm(__builtin_amdgcn_perm(g[7], g[6], 0x0c0c0400u), __builtin_amdgcn_perm(g[5], g[4], 0x0c0c0400u), 0x05040100u);
      q[k] = (b0 >> 4) | b1;
    }
#pragma unroll
    for (int j = 0; j < 2; ++j) {
      unsigned d[12];
#pragma unroll
      for (int g = 0; g < 4; ++g) {
        const unsigned* x = G + 16 * j + 4 * g;
        d[3 * g] = __builtin_amdgcn_perm(x[1], x[0], 0x05030201u);
        d[3 * g + 1] = __builtin_amdgcn_perm(x[2], x[1], 0x06050302u);
        d[3 * g + 2] = __builtin_amdgcn_perm(x[3], x[2], 0x07060503u);
      }
#pragma unroll
      for (int i = 0; i < 3; ++i)
        *(u32x4*)(blk + (1 + 6 * h + 3 * j + i) * 1024 + lane * 16) = u32x4{d[4 * i], d[4 * i + 1], d[4 * i + 2], d[4 * i + 3]};
    }
    put_records4(q, (unsigned)(lane << 10 | h << 9), cnt, blk, 13 * 1024 + 2 * lane);
  }
  return __ballot((bad | (low & 15u)) != 0) == 0 && cnt <= REC_CAP;
}

__device__ __forceinline__ unsigned enc_out(const float (&v)[4][16], uint8_t* blk, int lane) {
  float m = 0.f;
#pragma unroll
  for (int j = 0; j < 4; ++j)
#pragma unroll
    for (int r = 0; r < 16; r += 2) asm("v_max3_f32 %0, |%1|, |%2|, %0" : "+v"(m) : "v"(v[j][r]), "v"(v[j][r + 1]));
  const int e = max((int)(wave_max_u32(__builtin_bit_cast(unsigned, m)) >> 23) - 157, -126);
  const float up = __builtin_bit_cast(float, (unsigned)(127 - min(e, 0)) << 23);
  int ntop = 0, nlow = 0;
#pragma unroll
  for (int r = 0; r < 8; ++r) {
    const unsigned g = (unsigned)(int)(v[r & 3][2 * r] * up);
    ntop += __popcll(__ballot((int)(g ^ (g << 1)) < 0));
    nlow += __popcll(__ballot((g & 128u) != 0));
  }
  unsigned cnt = 0;
  const bool modeb = ntop < nlow;
  const bool fits = e <= 0 && (modeb ? enc_body<true>(v, up, cnt, blk, lane) : enc_body<false>(v, up, cnt, blk, lane));
  if (!fits) {
#pragma unroll
    for (int i = 0; i < 16; ++i) {
      const float* t = &v[i >> 2][(i & 3) * 4];
      *(f32x4*)(blk + i * 1024 + lane * 16) = f32x4{t[0], t[1], t[2], t[3]};
    }
    return 0;
  }
  *(unsigned*)(blk + (lane == 0 ? 0 : 13 * 1024 + 4 * lane)) = cnt | (unsigned)(e + 127) << 16 | (modeb ? HDR_B : 0u);
  return ((4 + 2 * cnt + 31) >> 5) * 2;
}

__device__ __forceinline__ void st16_out(void* base, unsigned off, f32x4 v) { st16_loc(base, off, __builtin_bit_cast(u32x4, v)); }

__device__ __forceinline__ constexpr unsigned row_off(int r) { return (unsigned)((r & 3) + 8 * (r >> 2)) * (GN * 4); }

__device__ __forceinline__ void send_out(const uint8_t* blk, unsigned nl, bool dead, uint8_t* slot, int lane, int depth,
                                         unsigned* flag, unsigned half) {
  const bool raw = nl == 0 || dead;
  const u32x4 h = *(const u32x4*)(blk + lane * 16);
  st16_sys_m(slot, lane * 16, raw ? u32x4{HDR_RAW, 0u, 0u, 0u} : h, raw ? 15ull : nl >= 64 ? ~0ull : (1ull << nl) - 1);
  const int n = raw ? 16 : 12;
  const uint8_t* const src = blk + (raw ? 0 : 1024) + lane * 16;
  const unsigned nan = dead ? 0x7fc00000u : 0u;
#pragma unroll 1
  for (int i = 0; i < n; ++i) {
    const u32x4 v = *(const u32x4*)(src + i * 1024) | nan;
    if (depth == 0) wait_vm<OUT_D - 1>(); else if (depth == 1) wait_vm<DEEP_D - 1>(); else wait_vm<DEEP_D3 - 1>();
    if (half != 0 && i == 6) { wait_vm<0>(); st1_sys(flag, half); }
    st16_sys(slot, (unsigned)(1024 + i * 1024 + lane * 16), v);
  }
}

template <int H>
__device__ __forceinline__ void dec_rows(const u32x4 (&y)[6], const u32x4& qq, bool modeb, float down, const f32x4 (&h)[16], float* ob,
                                         unsigned off, float sc) {
#pragma unroll
  for (int g = 0; g < 8; ++g) {
    const unsigned d0 = y[(3 * g) >> 2][(3 * g) & 3], d1 = y[(3 * g + 1) >> 2][(3 * g + 1) & 3], d2 = y[(3 * g + 2) >> 2][(3 * g + 2) & 3];
    const unsigned q = ((g & 1) ? qq[g >> 1] : qq[g >> 1] << 4) & 0xf0f0f0f0u;
    unsigned G[4] = {__builtin_amdgcn_perm(d0, q, 0x06050400u), __builtin_amdgcn_perm(d1, d0, 0x0504030cu) | ((q >> 8) & 255u),
                     __builtin_amdgcn_perm(d2, d1, 0x0403020cu) | ((q >> 16) & 255u), __builtin_amdgcn_perm(d2, q, 0x07060503u)};
    f32x4 v;
#pragma unroll
    for (int j = 0; j < 4; ++j) {
      if (modeb) G[j] = ((unsigned)((int)G[j] >> 1) & ~8u) ^ (G[j] << 27);
      v[j] = (__fmul_rn((float)(int)G[j], down) + h[4 * j + 2 * H + (g >> 2)][g & 3]) * sc;
    }
    st16_out(ob, off + row_off(8 * H + g), v);
  }
}

__device__ __forceinline__ void ld6_sys(const uint8_t* base, unsigned voff, u32x4 (&y)[6]) {
  asm volatile(SGPR_PAD "global_load_dwordx4 %0, %6, %7 offset:-3072 sc0 sc1\n\tglobal_load_dwordx4 %1, %6, %7 offset:-2048 sc0 sc1\n\t"
               "global_load_dwordx4 %2, %6, %7 offset:-1024 sc0 sc1\n\tglobal_load_dwordx4 %3, %6, %7 sc0 sc1\n\t"
               "global_load_dwordx4 %4, %6, %7 offset:1024 sc0 sc1\n\tglobal_load_dwordx4 %5, %6, %7 offset:2048 sc0 sc1\n\ts_waitcnt vmcnt(0)"
               : "=&v"(y[0]), "=&v"(y[1]), "=&v"(y[2]), "=&v"(y[3]), "=&v"(y[4]), "=&v"(y[5]) : "v"(voff + 3072), "s"(base) : "memory");
}

template <class F>
__device__ __forceinline__ void dec_out(const uint8_t* slot, const f32x4 (&h)[16], float* ob, unsigned off, float sc, unsigned* tmo, unsigned call,
                                        uint8_t* scr, int lane, bool halves, F rest) {
  u32x4 rec, y[6], z[6];
  unsigned tmv;
  asm volatile(SGPR_PAD "global_load_dword %13, %16, %18 sc1\n\t"
               "global_load_dwordx4 %0, %14, %17 offset:-4096 sc0 sc1\n\tglobal_load_dwordx4 %1, %14, %17 offset:-3072 sc0 sc1\n\t"
               "global_load_dwordx4 %2, %14, %17 offset:-2048 sc0 sc1\n\tglobal_load_dwordx4 %3, %14, %17 offset:-1024 sc0 sc1\n\t"
               "global_load_dwordx4 %4, %14, %17 sc0 sc1\n\tglobal_load_dwordx4 %5, %14, %17 offset:1024 sc0 sc1\n\t"
               "global_load_dwordx4 %6, %14, %17 offset:2048 sc0 sc1\n\tglobal_load_dwordx4 %7, %14, %17 offset:3072 sc0 sc1\n\t"
               "global_load_dwordx4 %8, %15, %17 offset:-4096 sc0 sc1\n\tglobal_load_dwordx4 %9, %15, %17 offset:-3072 sc0 sc1\n\t"
               "global_load_dwordx4 %10, %15, %17 offset:-2048 sc0 sc1\n\tglobal_load_dwordx4 %11, %15, %17 offset:-1024 sc0 sc1\n\t"
               "global_load_dwordx4 %12, %15, %17 sc0 sc1\n\ts_waitcnt vmcnt(0)"
               : "=&v"(rec), "=&v"(y[0]), "=&v"(y[1]), "=&v"(y[2]), "=&v"(y[3]), "=&v"(y[4]), "=&v"(y[5]), "=&v"(z[0]),
                 "=&v"(z[1]), "=&v"(z[2]), "=&v"(z[3]), "=&v"(z[4]), "=&v"(z[5]), "=&v"(tmv)
               : "v"(4096 + lane * 16), "v"(12288 + lane * 16), "v"(0u), "s"(slot), "s"(tmo) : "memory");
  const unsigned hdr = __builtin_amdgcn_readfirstlane(rec[0]);
  bool ok = __builtin_amdgcn_readfirstlane(tmv) != call;
  if (ok && !(hdr & HDR_RAW)) {
    const unsigned cnt = hdr & 0xffffu;
    const float down = __builtin_bit_cast(float, ((hdr >> 16) & 255u) << 23);
    *(u32x4*)(scr + lane * 32) = u32x4{0u, 0u, 0u, 0u};
    *(u32x4*)(scr + lane * 32 + 16) = u32x4{0u, 0u, 0u, 0u};
    __builtin_amdgcn_fence(__ATOMIC_ACQ_REL, "wavefront");
#pragma unroll
    for (int i = 0; i < 8; ++i) {
      const unsigned r16 = (rec[i >> 1] >> (16 * (i & 1))) & 0xffffu;
      const unsigned val = (unsigned)(lane * 8 + i - 2) < cnt ? (r16 & 15u) << ((r16 >> 2) & 28u) : 0u;
      __hip_atomic_fetch_or((unsigned*)(scr + (r16 >> 7) * 4), val, __ATOMIC_RELAXED, __HIP_MEMORY_SCOPE_WORKGROUP);
    }
    __builtin_amdgcn_fence(__ATOMIC_ACQ_REL, "wavefront");
    const u32x4 q0 = *(const u32x4*)(scr + lane * 32), q1 = *(const u32x4*)(scr + lane * 32 + 16);
    __builtin_amdgcn_fence(__ATOMIC_ACQ_REL, "wavefront");
    const bool modeb = (hdr & HDR_B) != 0;
    dec_rows<0>(y, q0, modeb, down, h, ob, off, sc);
    if (halves) {
      ok = rest();
      ld6_sys(slot, 7168 + lane * 16, z);
    }
    if (ok) dec_rows<1>(z, q1, modeb, down, h, ob, off, sc);
  } else if (ok) {
    if (halves) {
      ok = rest();
      ld6_sys(slot, 7168 + lane * 16, z);
    }
    u32x4 w[6];
    ld6_sys(slot, 11264 + lane * 16, w);
#pragma unroll
    for (int r = 0; r < 16; ++r) {
      f32x4 t;
#pragma unroll
      for (int j = 0; j < 4; ++j) {
        const int i = 4 * j + (r >> 2);
        t[j] = (__builtin_bit_cast(float, i < 6 ? y[i][r & 3] : i < 12 ? z[i - 6][r & 3] : w[i - 10][r & 3]) + h[i][r & 3]) * sc;
      }
      st16_out(ob, off + row_off(r), t);
    }
  }
  if (!ok) {
    const float n = __builtin_nanf("");
#pragma unroll 1
    for (int r = 0; r < 16; ++r) st16_out(ob, off + row_off(r), f32x4{n, n, n, n});
  }
}

__device__ __forceinline__ int wave_blocks(int r, int wave) { return wave < 3 ? 2 : r - 6; }
__device__ __forceinline__ unsigned out_off(int row0, int col, int wave, int lane) {
  return (unsigned)(((row0 + wave * 64 + 4 * (lane >> 5)) * GN + col + 4 * (lane & 31)) * 4);
}

template <bool NINE>
__device__ __forceinline__ void home_tile(const Params& p, int g, int src, int i, uint8_t* lds, int wave, int lane) {
  const int col = block_col(src, tile_block(g, i));
  const uint8_t* const xg = p.x + (size_t)(p.rank * 512 + (lane & 31)) * KS + (lane >> 5) * 32;
  auto xa = [&](int rho, int s) { return *(const i32x8*)(xg + rho * 32 * KS + s * 64); };
  i32x8 a0[8], a1[8], a9[8];
  {
    unsigned w[4][16];
    const unsigned o = (unsigned)((wave * 128 + (lane >> 5) * 32) * GN + col + (lane & 31) * 4);
#pragma unroll
    for (int t = 0; t < 4; ++t) ld_w(p.w, o + (unsigned)((t >> 1) * 64 + (t & 1) * 16) * GN, w[t]);
#pragma unroll
    for (int s = 0; s < 8; ++s) {
      a0[s] = xa((NINE ? 7 : 0) + wave * 2, s);
      a1[s] = xa((NINE ? 8 : 1) + wave * 2, s);
    }
    asm volatile("s_waitcnt vmcnt(0)" : "+v"(w[0][0]), "+v"(w[0][1]), "+v"(w[0][2]), "+v"(w[0][3]), "+v"(w[0][4]), "+v"(w[0][5]), "+v"(w[0][6]), "+v"(w[0][7]), "+v"(w[0][8]), "+v"(w[0][9]), "+v"(w[0][10]), "+v"(w[0][11]), "+v"(w[0][12]), "+v"(w[0][13]), "+v"(w[0][14]), "+v"(w[0][15]) :: "memory");
    asm volatile("" : "+v"(w[1][0]), "+v"(w[1][1]), "+v"(w[1][2]), "+v"(w[1][3]), "+v"(w[1][4]), "+v"(w[1][5]), "+v"(w[1][6]), "+v"(w[1][7]), "+v"(w[1][8]), "+v"(w[1][9]), "+v"(w[1][10]), "+v"(w[1][11]), "+v"(w[1][12]), "+v"(w[1][13]), "+v"(w[1][14]), "+v"(w[1][15]) :: "memory");
    asm volatile("" : "+v"(w[2][0]), "+v"(w[2][1]), "+v"(w[2][2]), "+v"(w[2][3]), "+v"(w[2][4]), "+v"(w[2][5]), "+v"(w[2][6]), "+v"(w[2][7]), "+v"(w[2][8]), "+v"(w[2][9]), "+v"(w[2][10]), "+v"(w[2][11]), "+v"(w[2][12]), "+v"(w[2][13]), "+v"(w[2][14]), "+v"(w[2][15]) :: "memory");
    asm volatile("" : "+v"(w[3][0]), "+v"(w[3][1]), "+v"(w[3][2]), "+v"(w[3][3]), "+v"(w[3][4]), "+v"(w[3][5]), "+v"(w[3][6]), "+v"(w[3][7]), "+v"(w[3][8]), "+v"(w[3][9]), "+v"(w[3][10]), "+v"(w[3][11]), "+v"(w[3][12]), "+v"(w[3][13]), "+v"(w[3][14]), "+v"(w[3][15]) :: "memory");
#pragma unroll
    for (int t = 0; t < 4; ++t) {
      u32x4 v[4];
      transpose_w(w[t], v);
#pragma unroll
      for (int j = 0; j < 4; ++j) *(u32x4*)(lds + (wave * 2 + (t >> 1)) * 8192 + j * 2048 + (t & 1) * 1024 + lane * 16) = v[j];
    }
  }
  if constexpr (NINE) {
#pragma unroll
    for (int s = 0; s < 8; ++s) a9[s] = xa(15, s);
  }
  __syncthreads();
  f32x16 h[2][4], h9;
#pragma unroll
  for (int r = 0; r < 16; ++r) {
    h9[r] = 0.f;
#pragma unroll
    for (int j = 0; j < 4; ++j) { h[0][j][r] = 0.f; h[1][j][r] = 0.f; }
  }
  i32x8 bn = ld_frag(lds + lane * 16);
#pragma unroll
  for (int t = 0; t < 32; ++t) {
    const i32x8 bc = bn;
    if (t < 31) bn = ld_frag(lds + (t + 1) * 2048 + lane * 16);
    i32x8 b9;
    if (NINE && (t & 3) == 0) b9 = ld_frag(lds + (t + wave) * 2048 + lane * 16);
    SB();
    h[0][t & 3] = __builtin_amdgcn_mfma_scale_f32_32x32x64_f8f6f4(a0[t >> 2], bc, h[0][t & 3], 1, 1, 0, 0, 0, 0);
    h[1][t & 3] = __builtin_amdgcn_mfma_scale_f32_32x32x64_f8f6f4(a1[t >> 2], bc, h[1][t & 3], 1, 1, 0, 0, 0, 0);
    if (NINE && (t & 3) == 0) h9 = __builtin_amdgcn_mfma_scale_f32_32x32x64_f8f6f4(a9[t >> 2], b9, h9, 1, 1, 0, 0, 0, 0);
    SB();
  }
  uint8_t* const pb = p.myws + WS_PB + (size_t)((g * 32 + i) * 9) * 16384;
#pragma unroll
  for (int q = 0; q < (NINE ? 36 : 32); ++q) {
    const f32x16& t = q < 32 ? h[q >> 4][(q >> 2) & 3] : h9;
    st16_out(pb, (unsigned)((q < 32 ? wave * 32 + q : 128 - 32 + wave * 4 + q) * 1024 + lane * 16),
             f32x4{t[4 * (q & 3)], t[4 * (q & 3) + 1], t[4 * (q & 3) + 2], t[4 * (q & 3) + 3]});
  }
  __syncthreads();
}

__device__ __forceinline__ bool in_halves(int g, int i, int k, int nb) { return i >= (g < 4 ? 24 : 16) && k == nb - 1; }

__device__ __forceinline__ void block_at(int g, int t, int& i, int& w, int& k) {
  if (g < 4) {
    const int s = t < 56 ? 0 : t < 168 ? 1 : 2, u = t - (s == 0 ? 0 : s == 1 ? 56 : 168), n = s == 1 ? 16 : 8, i0 = s == 0 ? 0 : s == 1 ? 8 : 24;
    k = u >= 4 * n;
    const int v = u - 4 * n, q = (v * 43) >> 7;
    i = i0 + (k ? q : u >> 2); w = k ? v - 3 * q : u & 3;
  } else {
    k = t < 32 ? 0 : t < 64 ? 1 : t < 128 ? 0 : t < 136 ? 2 : t < 200 ? 1 : 2;
    i = t < 64 ? (t & 31) >> 2 : t < 128 ? 8 + ((t - 64) >> 2) : t < 136 ? t - 128 : t < 200 ? 8 + ((t - 136) >> 2) : t - 192;
    w = k == 2 ? 3 : t & 3;
  }
}

__device__ __forceinline__ void recv_block(uint8_t* myws, float* out, int g, int src, int i, int wave, int k, int lane,
                                           unsigned call, unsigned* tmo, float sc, uint8_t* scr) {
  const unsigned off = out_off(g < 4 ? 0 : 224, block_col(src, tile_block(g, i)), wave, lane);
  const uint8_t* const fl = myws + WS_DONE + (size_t)((src * 32 + i) * 4 + wave) * 128;
  const int nb = wave_blocks(g < 4 ? 7 : 9, wave);
  {
    f32x4 h[16];
    asm volatile(SGPR_PAD "global_load_dwordx4 %0, %16, %18 offset:-4096 sc1\n\t""global_load_dwordx4 %1, %16, %18 offset:-3072 sc1\n\t"
                 "global_load_dwordx4 %2, %16, %18 offset:-2048 sc1\n\t""global_load_dwordx4 %3, %16, %18 offset:-1024 sc1\n\t"
                 "global_load_dwordx4 %4, %16, %18 offset:0 sc1\n\t""global_load_dwordx4 %5, %16, %18 offset:1024 sc1\n\t"
                 "global_load_dwordx4 %6, %16, %18 offset:2048 sc1\n\t""global_load_dwordx4 %7, %16, %18 offset:3072 sc1\n\t"
                 "global_load_dwordx4 %8, %17, %18 offset:-4096 sc1\n\t""global_load_dwordx4 %9, %17, %18 offset:-3072 sc1\n\t"
                 "global_load_dwordx4 %10, %17, %18 offset:-2048 sc1\n\t""global_load_dwordx4 %11, %17, %18 offset:-1024 sc1\n\t"
                 "global_load_dwordx4 %12, %17, %18 offset:0 sc1\n\t""global_load_dwordx4 %13, %17, %18 offset:1024 sc1\n\t"
                 "global_load_dwordx4 %14, %17, %18 offset:2048 sc1\n\t""global_load_dwordx4 %15, %17, %18 offset:3072 sc1"
                 : "=&v"(h[0]), "=&v"(h[1]), "=&v"(h[2]), "=&v"(h[3]), "=&v"(h[4]), "=&v"(h[5]), "=&v"(h[6]), "=&v"(h[7]),
                   "=&v"(h[8]), "=&v"(h[9]), "=&v"(h[10]), "=&v"(h[11]), "=&v"(h[12]), "=&v"(h[13]), "=&v"(h[14]), "=&v"(h[15])
                 : "v"(4096 + lane * 16), "v"(12288 + lane * 16), "s"(myws + WS_PB + (size_t)((g * 32 + i) * 9 + wave * 2 + k) * 16384) : "memory");
    const bool halves = in_halves(g, i, k, nb);
    spin<RX_SLEEP>([&] { return __builtin_amdgcn_readfirstlane(ld_sys(fl)) >= 8 * call + 2 * k + 2 - halves; }, tmo, call);
    asm volatile("s_waitcnt vmcnt(0)" : "+v"(h[0]), "+v"(h[1]), "+v"(h[2]), "+v"(h[3]), "+v"(h[4]), "+v"(h[5]), "+v"(h[6]), "+v"(h[7]), "+v"(h[8]), "+v"(h[9]), "+v"(h[10]), "+v"(h[11]), "+v"(h[12]), "+v"(h[13]), "+v"(h[14]), "+v"(h[15]) :: "memory");
    dec_out(myws + WS_OS + (size_t)((src * 32 + i) * 9 + wave * 2 + k) * OBLK, h, out, off + (unsigned)k * 32 * (GN * 4), sc, tmo, call, scr, lane,
            halves, [&] { return spin<LAST_SLEEP>([&] { return __builtin_amdgcn_readfirstlane(ld_sys(fl)) >= 8 * call + 2 * k + 2; }, tmo, call); });
  }
}

}

__global__ void __launch_bounds__(256, 1) gemm_rs_fused(const Params p) {
  __shared__ __attribute__((aligned(16))) uint8_t lds[LDS_BYTES];
  const int b = blockIdx.x;
  const int lane = threadIdx.x & 63;
  const int wave = __builtin_amdgcn_readfirstlane(threadIdx.x >> 6);
  const int rank = p.rank;
  uint8_t* const myws = p.myws;
  asm volatile("" :: "s"(myws), "s"(rank), "s"(p.x), "s"(p.w));

  const int grp = b & 7, dst = grp ^ rank, ti = 31 - (b >> 3);
  const bool local = grp == 0 || (grp >= 4 && ti >= 24);
  const int set = local ? 7 + (grp != 0) : grp - 1, cb = tile_block(grp, ti);
  const RowSet rs = row_set(set);
  unsigned* const callp = (unsigned*)(myws + WS_CALL + (size_t)b * 128);
  unsigned* const tmo = (unsigned*)(myws + WS_TMO);
  unsigned* const lk = (unsigned*)(myws + WS_LINK) + grp * 64;
  uint8_t* const scr = lds + LDS_PK + wave * 2048;

  PushCtx c;
  c.x = p.x; c.d = dst; c.local = grp == 0;
  c.xl = (unsigned)((lane & 31) * KS + (lane >> 5) * 32);
  const RowSet hs = row_set(grp - 1);
  c.skip0 = grp == 0 ? NTASK : hs.task0; c.skipn = grp == 0 ? 0 : 4 * hs.r;

  unsigned call;
  if (ti >= (grp == 0 ? 8 : 24)) {
    const int w = (b >> 3) * 4 + wave, npw = grp == 0 ? 96 : 32;
    c.w = p.w; c.dst = p.ws[dst]; c.lk = lk; c.rank = rank;
    c.wl = (unsigned)((lane >> 5) * 32 * GN + (lane & 31) * 4);
    c.scr = scr;
    PushState s;
    u32x4 x0[4];
    unsigned cv;
    s.tp = -1; s.tc = task_at(c, w); s.tn = task_at(c, w + npw); s.tl = task_at(c, w + 2 * npw); s.gone = true;
    asm volatile(SGPR_PAD "global_load_dword %0, %1, %2 sc1" : "=v"(cv) : "v"(0u), "s"(callp) : "memory");
    ld_x(c, s.tc, x0);
    __builtin_amdgcn_sched_barrier(0);
    ld_x(c, s.tn, s.nx);
    const RowSet fs = row_set(task_set(s.tc));
    unsigned first = (unsigned)((fs.base * 128 + rank * fs.r * 16 + (s.tc - fs.task0) * 4) << 10) + lane * 16;
    asm volatile("" : "+v"(first));
    asm volatile("s_waitcnt vmcnt(4)" : "+v"(cv), "+v"(x0[0]), "+v"(x0[1]), "+v"(x0[2]), "+v"(x0[3]) :: "memory");
    call = __builtin_amdgcn_readfirstlane(cv) + 1;
    c.par = (unsigned)(WS_DATA + (call & 1) * PAR_BYTES);
#pragma unroll
    for (int q = 0; q < 4; ++q) st16_sys(c.dst, c.par + first + q * 1024, x0[q]);
    asm volatile("s_waitcnt vmcnt(4)" : "+v"(s.nx[0]), "+v"(s.nx[1]), "+v"(s.nx[2]), "+v"(s.nx[3]) :: "memory");
    c.tick0 = (call - 1) * (NTASK - c.skipn - 2 * npw) - 3 * npw;
    c.tag = call & 0xffffu;
    c.call = call;
    s.line = u32x4{0u, 0u, 0u, 0u};
    pack_task(c, x0, s.o, lane, true);
#pragma unroll 1
    while (s.tp < NTASK) {
      if (task_is_w(s.tl)) push_step<true>(c, s, lane);
      else push_step<false>(c, s, lane);
    }
  } else {
    call = __builtin_amdgcn_readfirstlane(ld_agent(callp)) + 1;
  }
  uint8_t* const par = myws + WS_DATA + (size_t)(call & 1) * PAR_BYTES;

#pragma unroll 1
  for (int job = 0; job < 4; ++job) {
    if (job == 1 && grp != 0 && ti < 24) {
      __syncthreads();
      if (grp >= 4) {
        home_tile<true>(p, grp, dst, ti, lds, wave, lane);
      } else {
#pragma unroll 1
        for (int i = ti; i < 32; i += ti >= 16 ? 8 : 32) home_tile<false>(p, grp, dst, i, lds, wave, lane);
      }
    }
    const bool jw = grp == 0 ? job < 3 : job == 1;
    const int beta = grp == 0 ? (ti * 4 + wave) & 63 : (23 - ti) * 4 + wave;
    if (grp == 0 ? (job == 2 && ti >= 24) : (ti >= 24 || job >= 2 || (job == 0 && beta >= 8 * rs.r))) continue;
    const int n = job == 2 ? 24 + (ti & 7) : cb;
    const RowSet js = row_set(grp == 0 ? 7 + (ti >> 4) : set);
    const int bs = div_r(beta, js.r), bu = beta - bs * js.r;
    const int src = !jw ? bs : grp != 0 ? dst : job == 0 ? rank : job == 1 ? rank ^ (4 + (ti & 3)) : rank ^ (1 + (ti >> 3));
    const int t0 = jw ? T_W0 + n * 16 + wave * 4 : js.task0 + bu * 4;
    const uint8_t* const slot = par + XS_BYTES + WT_BYTES + (size_t)(src * NTASK + t0) * ITASK;
    const bool absent = !jw && grp != 0 && src == dst;
    if (!absent && !wait_tasks(slot, tmo, call, lane)) continue;
    if (src != rank && !absent) {
      const unsigned base = jw ? (unsigned)XS_BYTES + (unsigned)(n * (512 << 10) + (src * 8 + wave * 2) * 8192)
                               : (unsigned)(js.base * (128 << 10) + (src * js.r + bu) * 16384);
      unsigned off[4];
#pragma unroll
      for (int t = 0; t < 4; ++t) off[t] = base + (jw ? (t >> 1) * 8192 + (t & 1) * 1024 : t * 4096);
      dec_in4(slot, par, off, jw ? 2048 : 1024, lds + wave * 4096, lane);
      wait_vm<0>();
    }
    st1_agent(jw ? myws + WS_DW + (size_t)(n * 32 + src * 4 + wave) * 4
                 : myws + WS_DX + (size_t)((grp == 0 ? 7 + (ti >> 4) : set) * 128 + beta) * 4, call);
  }
  if (grp != 0 && ti < 24) {
    wait_vm<0>();
    st1_agent(myws + WS_HOME + (size_t)((grp * 32 + ti) * 4 + wave) * 4, call);
    if (grp < 4 && ti >= 16) st1_agent(myws + WS_HOME + (size_t)((grp * 32 + ti + 8) * 4 + wave) * 4, call);
  }

  unsigned* const ctrl = (unsigned*)(lds + LDS_CTRL);
  if (wave == 0) {
    const uint8_t* const fw = myws + WS_DW + (size_t)(cb * 32 + (lane & 31)) * 4;
    const uint8_t* const fx = myws + WS_DX + (size_t)(set * 128 + min(lane, 8 * rs.r - 1)) * 4;
    const uint8_t* const fy = myws + WS_DX + (size_t)(set * 128 + min(64 + lane, 8 * rs.r - 1)) * 4;
    const bool ok = spin([&] { const unsigned a = ld_agent(fw), c = ld_agent(fx), d = ld_agent(fy); return __all(min(min(a, c), d) >= call) != 0; },
                         tmo, call);
    if (lane == 0) ctrl[0] = !ok;
  }
  __syncthreads();
  const bool dead0 = __builtin_amdgcn_readfirstlane(ctrl[0]) != 0;

  if (wave == 0) st1_agent(callp, call);
  const float sc = p.sx[0] * p.sw[0];
  const unsigned off = out_off(rs.row0, block_col(rank, cb), wave, lane);
  float* const ob = p.out[rank];
  uint8_t* const blk = lds + wave * 32768;
  uint8_t* const oslot = p.ws[dst] + WS_OS + (size_t)((rank * 32 + ti) * 9 + wave * 2) * OBLK;
  unsigned* const flag = (unsigned*)(p.ws[dst] + WS_DONE + (size_t)((rank * 32 + ti) * 4 + wave) * 128);
  const int nb = wave_blocks(rs.r, wave);
  const unsigned* const turn = lk + 32;
  const int depth = (ti < DEEP_T) + (nb == 3);
  unsigned nl0 = 0, nl1 = 0, nl2 = 0;
  int sent = 0;

  if (!dead0) {
    const uint8_t* const ga = par + (size_t)rs.base * (128 << 10);
    const uint8_t* const gb = par + XS_BYTES + (size_t)cb * (512 << 10);
    if (rs.r == 9) gemm_tile<9>(ga, 9 * 2048, gb, dst, lds, wave, lane);
    else if (nb == 1) gemm_tile<7>(ga, 7 * 2048, gb, dst, lds, wave, lane);
    else gemm_tile<8>(ga, rs.r * 2048, gb, local ? -1 : dst, lds, wave, lane);
    unsigned tnv, tmv;
    asm volatile(SGPR_PAD "global_load_dword %0, %2, %3 sc1\n\tglobal_load_dword %1, %2, %4 sc1"
                 : "=&a"(tnv), "=&a"(tmv) : "v"(0u), "s"(turn), "s"(tmo) : "memory");
    auto landed = [&](unsigned& s_tn, unsigned& s_tm) {
      unsigned t0, t1;
      asm volatile("s_waitcnt vmcnt(0)\n\tv_accvgpr_read_b32 %2, %4\n\tv_accvgpr_read_b32 %3, %5\n\ts_nop 1\n\t"
                   "v_readfirstlane_b32 %0, %2\n\tv_readfirstlane_b32 %1, %3\n\ts_nop 1"
                   : "=&s"(s_tn), "=&s"(s_tm), "=&v"(t0), "=&v"(t1) : "a"(tnv), "a"(tmv) : "memory");
    };
    float s0 = sc;
#pragma unroll 1
    for (int k = 0; k < nb; ++k) {
      float v[4][16];
#pragma unroll
      for (int i = 0; i < 16; ++i) {
        const f32x4 t = *(const f32x4*)(blk + k * 16384 + i * 1024 + lane * 16);
#pragma unroll
        for (int q = 0; q < 4; ++q) v[i >> 2][(i & 3) * 4 + q] = t[q];
      }
      __builtin_amdgcn_fence(__ATOMIC_ACQ_REL, "wavefront");
      if (local) {
        if (k == 0) {
          unsigned tn, t;
          landed(tn, t);
          if (t == call) s0 = __builtin_nanf("");
        }
#pragma unroll
        for (int r = 0; r < 16; ++r)
          st16_out(ob, off + k * 32 * (GN * 4) + row_off(r), f32x4{v[0][r] * s0, v[1][r] * s0, v[2][r] * s0, v[3][r] * s0});
        continue;
      }
      const unsigned nl = enc_out(v, blk + k * 16384, lane);
      __builtin_amdgcn_fence(__ATOMIC_ACQ_REL, "wavefront");
      if (k == 0) nl0 = nl; else if (k == 1) nl1 = nl; else nl2 = nl;
      if (k == 0) {
        unsigned tn, t;
        landed(tn, t);
        if (nb > 1 && t != call && (tn >= call || __builtin_amdgcn_readfirstlane(ld_agent(turn)) >= call)) {
          send_out(blk, nl, false, oslot, lane, depth, flag, 0);
          sent = 1;
        }
      }
    }
  } else if (local) {
    const float n = __builtin_nanf("");
#pragma unroll 1
    for (int t = 0; t < 32; ++t) st16_out(ob, off + (t >> 4) * 32 * (GN * 4) + row_off(t & 15), f32x4{n, n, n, n});
  }
  if (grp == 0) return;

  if (!local) {
    if (sent) {
      wait_vm<0>();
      st1_sys(flag, 8 * call + 2);
    } else {
      spin([&] { return __builtin_amdgcn_readfirstlane(ld_agent(turn)) >= call; }, tmo, call);
    }
    const bool dead = dead0 || __builtin_amdgcn_readfirstlane(ld_agent(tmo)) == call;
#pragma unroll 1
    for (int k = sent; k < nb; ++k) {
      send_out(blk + k * 16384, k == 0 ? nl0 : k == 1 ? nl1 : nl2, dead, oslot + k * OBLK, lane, depth, flag,
               in_halves(grp, ti, k, nb) ? 8 * call + 2 * k + 1 : 0);
      wait_vm<0>();
      st1_sys(flag, 8 * call + 2 * k + 2);
    }
  }
  {
    const uint8_t* const fa = myws + WS_HOME + (size_t)(grp * 128 + min(lane, grp < 4 ? 127 : 95)) * 4;
    const uint8_t* const fb = myws + WS_HOME + (size_t)(grp * 128 + min(64 + lane, grp < 4 ? 127 : 95)) * 4;
    spin<RX_SLEEP>([&] { const unsigned a = ld_agent(fa), c = ld_agent(fb); return __all(min(a, c) >= call) != 0; }, tmo, call);
  }
  const unsigned nblk = grp < 4 ? 224 : 216;
#pragma unroll 1
  for (;;) {
    unsigned t = 0;
    if (lane == 0) t = __hip_atomic_fetch_add(lk + 512, 1u, __ATOMIC_RELAXED, __HIP_MEMORY_SCOPE_AGENT);
    t = __builtin_amdgcn_readfirstlane(t) - (call - 1) * (nblk + 128);
    if (t >= nblk) break;
    int i, w, k;
    block_at(grp, (int)t, i, w, k);
    recv_block(myws, p.out[rank], grp, dst, i, w, k, lane, call, tmo, sc, scr);
  }
}

extern "C" void kernel_launch(int rank, int nranks, ncclComm_t comm, hipStream_t stream,
                              void* const* d_in, const int* in_sizes, int n_in,
                              void* d_out, int out_size, const int* in_dtypes, int out_dtype,
                              void* const* d_out_all, void* const* d_ws_all, size_t ws_bytes) {
  (void)comm; (void)in_sizes; (void)in_dtypes; (void)out_dtype; (void)d_out;
  if (nranks != NR || (unsigned)rank >= (unsigned)NR || n_in != 4 || ws_bytes < WS_NEED || out_size != 512 * GN) {
    fprintf(stderr, "kernel_launch: unexpected problem shape\n");
    return;
  }
  Params p;
  p.x = (const uint8_t*)d_in[0];
  p.w = (const uint8_t*)d_in[1];
  p.sx = (const float*)d_in[2];
  p.sw = (const float*)d_in[3];
  for (int i = 0; i < NR; ++i) {
    p.ws[i] = (uint8_t*)d_ws_all[i];
    p.out[i] = (float*)d_out_all[i];
  }
  p.myws = p.ws[rank];
  p.rank = rank;
  p.pad = 0;
  hipLaunchKernelGGL(gemm_rs_fused, dim3(NWG), dim3(256), 0, stream, p);
  HIP_CHECK(hipGetLastError());
}
```
